```python
import numpy as np

kernel_name = "gpu_ag_gemm_m4096_k4096_n16384_bf16_relu"


ABSMAX_RTOL = 0.008

COMPUTE_DTYPE = "bfloat16"
EPILOGUE = "relu"
IN_QUANT = None
OUT_QUANT = None
IN_DTYPES = {"x": "bfloat16", "w_mat": "bfloat16"}
WORLD_SIZE = 8

def _to_bf16(x):
    u = np.ascontiguousarray(x, dtype=np.float32).view(np.uint32)
    return ((u + 0x7FFF + ((u >> 16) & 1)) & 0xFFFF0000).view(np.float32)

def setup_inputs():

    x = _to_bf16(np.random.randn(4096, 4096).astype(np.float32))
    w_mat = _to_bf16((np.random.randn(4096, 16384) / np.sqrt(4096)).astype(np.float32))
    return {"x": x, "w_mat": w_mat}

def _epilogue(y):
    return np.maximum(y, 0.0).astype(np.float32)

def reference(x, w_mat):
    y = (x @ w_mat).astype(np.float32)
    return _epilogue(y)
```

```cpp
#include <hip/hip_runtime.h>
#include <rccl/rccl.h>
#include <cstdio>

#define HIP_CHECK(expr) do { hipError_t _e = (expr); if (_e != hipSuccess) { \
    fprintf(stderr, "HIP error %s at %s:%d\n", hipGetErrorString(_e), __FILE__, __LINE__); \
    return; } } while (0)

namespace {

#define LAS __attribute__((address_space(3)))
#define DI __device__ __forceinline__
typedef short    s16x4  __attribute__((ext_vector_type(4)));
typedef short    s16x8  __attribute__((ext_vector_type(8)));
typedef __bf16   bf16x8 __attribute__((ext_vector_type(8)));
typedef __bf16   bf16x2 __attribute__((ext_vector_type(2)));
typedef float    f32x2  __attribute__((ext_vector_type(2)));
typedef float    f32x4  __attribute__((ext_vector_type(4)));
typedef unsigned u32x4  __attribute__((ext_vector_type(4)));
typedef unsigned short u16x2 __attribute__((ext_vector_type(2)));

constexpr int NR = 8;
constexpr int KD = 4096, NL = 2048, ML = 512, MG = NR * ML;
constexpr int NSTEP = KD / 32;
constexpr int NRG = ML / 64;
constexpr int NUNIT = NSTEP * NRG;
constexpr int GRID = 256, BLOCK = 512;

#define NSET 6
#define NGRP 3
constexpr int RING = NSET * NGRP;
#define WPF 3
constexpr int WN = 4;
static_assert(WPF < WN, "");
constexpr int INFL = 6;
#define NPULL 252
#define TCYC 188
#define EARLYPUB 30
#define POLLSLEEP 32
#define XAUX 17

constexpr int L_FREE = RING * 8192, L_XRDY = L_FREE + 1024, L_WFULL = L_FREE + 2048, L_MISC = L_FREE + 3072;
constexpr int LDS_BYTES = L_MISC + 64;
static_assert(LDS_BYTES <= 160 * 1024 && LDS_BYTES > 80 * 1024, "");
constexpr unsigned INF = 0x7fffffffu;

constexpr size_t XT_BYTES = (size_t)MG * KD * 2;
constexpr size_t XT_OFF = 0;
constexpr size_t PK_BYTES = (size_t)NUNIT * 3072;
constexpr size_t PK_OFF = XT_OFF + 2 * XT_BYTES;
constexpr size_t F1_BYTES = (size_t)NR * NUNIT * 128;
constexpr size_t F1_OFF = PK_OFF + 2 * PK_BYTES;
constexpr size_t F2_OFF = F1_OFF + 2 * F1_BYTES;
constexpr size_t F2_BYTES = (size_t)NR * NUNIT * 4;
constexpr size_t CT_OFF = F2_OFF + F2_BYTES;
constexpr size_t CC_OFF = CT_OFF + 4096;
constexpr size_t DBG_OFF = CC_OFF + GRID * 128;
constexpr size_t DBG_BYTES = 1 << 20;
constexpr size_t WS_NEED = DBG_OFF + DBG_BYTES;
constexpr unsigned long long DEADLINE_TICKS = 20000000ull;

struct Params { const void* x; const void* w; void* out; char* ws[NR]; int rank; int pad; };

DI unsigned rfl(unsigned v) { return __builtin_amdgcn_readfirstlane(v); }
DI unsigned long long rt() { return __builtin_amdgcn_s_memrealtime(); }
DI void st_sys(void* p, u32x4 v) { asm volatile("global_store_dwordx4 %0, %1, off sc0 sc1\n\ts_nop 1" :: "v"(p), "v"(v) : "memory"); }
DI void wait_vm0() { asm volatile("s_waitcnt vmcnt(0)" ::: "memory"); }
DI void lds_st(unsigned addr, unsigned v) { asm volatile("ds_write_b32 %0, %1" :: "v"(addr), "v"(v) : "memory"); }
DI unsigned lds_ld(unsigned addr) { unsigned v; asm volatile("ds_read_b32 %0, %1\n\ts_waitcnt lgkmcnt(0)" : "=&v"(v) : "v"(addr) : "memory"); return v; }
#define PIN() __builtin_amdgcn_sched_barrier(0)

#define STAMP(slot) do {} while (0)
#define STAMPC(slot) do {} while (0)

struct CW {
  f32x4 acc[4][8];
  u32x4 xs[NSET][4];
  s16x4 wf[WN][2];
  unsigned wa[4];
  unsigned ctl;
  unsigned xvo;
  unsigned freev;
  unsigned ready;
  unsigned need;
  unsigned xso;
  unsigned grp;
  __amdgpu_buffer_rsrc_t xr;
};

DI void wait_ready(CW& c) {
  asm volatile(
      "s_cmp_ge_u32 %0, %2\n\t"
      "s_cbranch_scc1 2f\n"
      "1:\n\t"
      "ds_read_b32 v254, %1 offset:1024\n\t"
      "ds_read_b32 v255, %1 offset:2048\n\t"
      "s_waitcnt lgkmcnt(0)\n\t"
      "v_min_u32 v254, v254, v255\n\t"
      "s_nop 1\n\t"
      "v_readfirstlane_b32 %0, v254\n\t"
      "s_nop 3\n\t"
      "s_cmp_ge_u32 %0, %2\n\t"
      "s_cbranch_scc1 2f\n\t"
      "s_sleep 1\n\t"
      "s_branch 1b\n"
      "2:"
      : "+s"(c.ready) : "v"(c.ctl), "s"(c.need) : "scc", "memory", "v254", "v255");
}

template <int SET, int M> DI void xload(CW& c) {
  c.xs[SET][M] = __builtin_amdgcn_raw_buffer_load_b128(c.xr, c.xvo + M * 1024, c.xso, XAUX);
}
template <int SL, int J, int T> DI void wread(CW& c) {
  c.wf[J % WN][T] = __builtin_amdgcn_ds_read_tr16_b64_v4i16((LAS s16x4*)((LAS char*)0 + c.wa[J & 3] + (SL * 8192 + (J >> 2) * 4096 + T * 1024)));
}
template <int S, int J, int M> DI void mma(CW& c) {
  const s16x8 w = __builtin_shufflevector(c.wf[J % WN][0], c.wf[J % WN][1], 0, 1, 2, 3, 4, 5, 6, 7);
  c.acc[M][J] = __builtin_amdgcn_mfma_f32_16x16x32_bf16(__builtin_bit_cast(bf16x8, w), __builtin_bit_cast(bf16x8, c.xs[S][M]), c.acc[M][J], 0, 0, 0);
}
DI void next_group(CW& c) {
  const bool wrap = (c.grp == NGRP - 1);
  const unsigned d = wrap ? (unsigned)(-(NGRP - 1) * NSET * 8192) : (unsigned)(NSET * 8192);
  c.grp = wrap ? 0u : c.grp + 1u;
#pragma unroll
  for (int i = 0; i < 4; ++i) c.wa[i] += d;
}
template <int S, int J> DI void frag(CW& c) {
  constexpr int JP = (J + WPF) & 7;
  constexpr bool NXT = (J + WPF) >= 8;
  constexpr int SL = NXT ? (S + 1) % NSET : S;
  constexpr int LS = (S + NSET - 1) % NSET;
  if constexpr (NXT && JP == 0) {
    c.freev += 1; lds_st(c.ctl, c.freev);
    if constexpr (S == NSET - 1) next_group(c);
    PIN();
  }
  mma<S, J, 0>(c); PIN();
  wread<SL, JP, 0>(c); PIN();
  mma<S, J, 1>(c); PIN();
  wread<SL, JP, 1>(c); PIN();
  mma<S, J, 2>(c); PIN();
  if constexpr (J < 4) { xload<LS, J>(c); PIN(); }
  mma<S, J, 3>(c); PIN();
}
template <int S> DI void step(CW& c) {
  wait_ready(c); c.need += 1; PIN();
  frag<S, 0>(c); frag<S, 1>(c); frag<S, 2>(c); frag<S, 3>(c);
  c.xso += 64 * 4096;
  frag<S, 4>(c); frag<S, 5>(c); frag<S, 6>(c); frag<S, 7>(c);
}
template <int S> DI void pro_x(CW& c) {
  c.need = S + 1; wait_ready(c); PIN();
  xload<S, 0>(c); xload<S, 1>(c); xload<S, 2>(c); xload<S, 3>(c); c.xso += 64 * 4096; PIN();
}

DI unsigned cvt_pk(float lo, float hi) { f32x2 v = {lo, hi}; return __builtin_bit_cast(unsigned, __builtin_convertvector(v, bf16x2)); }
DI float relu(float v) { return __builtin_elementwise_maximum(v, 0.0f); }

template <int S, int E> DI void steps(CW& c) { if constexpr (S < E) { step<S>(c); steps<S + 1, E>(c); } }
template <int S, int E> DI void pros(CW& c) { if constexpr (S < E) { pro_x<S>(c); pros<S + 1, E>(c); } }
template <int J, int E> DI void prow(CW& c) { if constexpr (J < E) { wread<0, J, 0>(c); wread<0, J, 1>(c); prow<J + 1, E>(c); } }

DI unsigned dpp_max(unsigned v, const int ctrl) {
  unsigned o;
  switch (ctrl) {
    case 1: o = __builtin_amdgcn_update_dpp(0u, v, 0x111, 0xf, 0xf, true); break;
    case 2: o = __builtin_amdgcn_update_dpp(0u, v, 0x112, 0xf, 0xf, true); break;
    case 4: o = __builtin_amdgcn_update_dpp(0u, v, 0x114, 0xf, 0xf, true); break;
    default: o = __builtin_amdgcn_update_dpp(0u, v, 0x118, 0xf, 0xf, true); break;
  }
  return o > v ? o : v;
}

__global__ __launch_bounds__(BLOCK, 2) void ag_gemm(Params p) {
  extern __shared__ __attribute__((aligned(16))) char lds[];
  const int lane = threadIdx.x & 63;
  const int wave = rfl(threadIdx.x >> 6);
  const unsigned lb = (unsigned)(size_t)(LAS char*)lds;
  const int b = blockIdx.x, rank = p.rank & (NR - 1);
  char* const my = p.ws[rank];
  const int xcd = b & 7, bi = b >> 3;
  const int tm = (xcd >> 1) * 4 + (bi >> 3), tn = (xcd & 1) * 8 + (bi & 7);
  const int src = tm >> 1;
  unsigned* const tmo = (unsigned*)(my + CT_OFF + 128);
  if (wave == 0) STAMP(0);

  const int pstep = 2 * (b >> 2) + (wave & 1), prg = 2 * (b & 3) + ((wave >> 1) & 1);
  u32x4 pv[4];
  if (wave < 4) {
    const char* xp = (const char*)p.x + ((size_t)(prg * 64 + (lane & 15)) * KD + pstep * 32 + (lane >> 4) * 8) * 2;
#pragma unroll
    for (int m = 0; m < 4; ++m) pv[m] = *(const u32x4*)(xp + (size_t)m * 16 * KD * 2);
  } else if (wave == 7) {
    unsigned t = 0;
    if (lane == 0) t = __hip_atomic_fetch_add((unsigned*)(my + CC_OFF + b * 128), 1u, __ATOMIC_RELAXED, __HIP_MEMORY_SCOPE_AGENT);
    t = rfl(t) + 1;
    if (lane < 16) lds_st(lb + L_MISC + lane * 4, lane == 0 ? t : 0u);
  } else {
#pragma unroll
    for (int i = 0; i < 4; ++i) lds_st(lb + L_FREE + (wave - 4) * 1024 + i * 256 + lane * 4, 0u);
  }
  asm volatile("s_waitcnt lgkmcnt(0)" ::: "memory");
  __builtin_amdgcn_s_barrier();
  asm volatile("" ::: "memory");
  const unsigned call = rfl(lds_ld(lb + L_MISC));
  const unsigned par = call & 1u;

  if (wave < 4) {
    {
      u16x2 mx = {0, 0};
#pragma unroll
      for (int m = 0; m < 4; ++m)
#pragma unroll
        for (int i = 0; i < 4; ++i) mx = __builtin_elementwise_max(mx, __builtin_bit_cast(u16x2, pv[m][i] & 0x7fff7fffu));
      unsigned v = mx[0] > mx[1] ? mx[0] : mx[1];
      v = dpp_max(v, 1); v = dpp_max(v, 2); v = dpp_max(v, 4); v = dpp_max(v, 8);
      const unsigned a0 = __builtin_amdgcn_readlane(v, 15), a1 = __builtin_amdgcn_readlane(v, 31), a2 = __builtin_amdgcn_readlane(v, 47), a3 = __builtin_amdgcn_readlane(v, 63);
      const unsigned top = max(max(a0, a1), max(a2, a3)) >> 8;
      const unsigned base = top > 7u ? top - 7u : 0u, base4 = base * 0x01010101u;
      u32x4 lo01, lo23, hh; unsigned esc = 0;
#pragma unroll
      for (int m = 0; m < 4; ++m) {
        const unsigned l0 = __builtin_amdgcn_perm(pv[m][1], pv[m][0], 0x06040200u), l1 = __builtin_amdgcn_perm(pv[m][3], pv[m][2], 0x06040200u);
        const unsigned h0 = __builtin_amdgcn_perm(pv[m][1], pv[m][0], 0x07050301u), h1 = __builtin_amdgcn_perm(pv[m][3], pv[m][2], 0x07050301u);
        const unsigned t0 = (h0 | 0x80808080u) - base4, t1 = (h1 | 0x80808080u) - base4;
        const unsigned c0 = (h0 & 0x80808080u) | (t0 & 0x07070707u), c1 = (h1 & 0x80808080u) | (t1 & 0x07070707u);
        unsigned h = c0 | __builtin_rotateleft32(c1, 4);
        const bool fits = ((t0 & 0xf8f8f8f8u) == 0x80808080u) && ((t1 & 0xf8f8f8f8u) == 0x80808080u) && h != 0u;
        if (!fits) { h = 0u; esc |= 1u << m; }
        hh[m] = h;
        if (m < 2) { lo01[2 * m] = l0; lo01[2 * m + 1] = l1; } else { lo23[2 * m - 4] = l0; lo23[2 * m - 3] = l1; }
      }
      const int u = pstep * NRG + prg;
      if (wave == 0) STAMP(7);
      char* pk = my + PK_OFF + par * PK_BYTES + (size_t)u * 3072 + lane * 16;
      st_sys(pk, lo01); st_sys(pk + 1024, lo23); st_sys(pk + 2048, hh);
      const size_t xo = XT_OFF + par * XT_BYTES + (size_t)(pstep * 64 + rank * NRG + prg) * 4096 + lane * 16;
#pragma unroll
      for (int m = 0; m < 4; ++m) st_sys(my + xo + m * 1024, pv[m]);
      if (esc) {
        for (int i = 1; i < NR; ++i) {
          char* d = p.ws[(rank + i) & (NR - 1)] + xo;
#pragma unroll
          for (int m = 0; m < 4; ++m) if (esc >> m & 1u) st_sys(d + m * 1024, pv[m]);
        }
      }
      wait_vm0();
      if (lane < NR && lane != rank)
        __hip_atomic_store((unsigned*)(p.ws[lane] + F1_OFF + par * F1_BYTES + (size_t)(rank * NUNIT + u) * 128), (call << 8) | base, __ATOMIC_RELAXED, __HIP_MEMORY_SCOPE_SYSTEM);
      if (lane == rank) __hip_atomic_store((unsigned*)(my + F2_OFF) + (rank * NRG + prg) * NSTEP + pstep, call, __ATOMIC_RELAXED, __HIP_MEMORY_SCOPE_AGENT);
    }
    if (wave == 0) STAMP(1);
    CW c;
    const int g = lane >> 4, q = (lane >> 2) & 3, pp = lane & 3;
#pragma unroll
    for (int m = 0; m < 4; ++m)
#pragma unroll
      for (int j = 0; j < 8; ++j) c.acc[m][j] = (f32x4){0.f, 0.f, 0.f, 0.f};
#pragma unroll
    for (int i = 0; i < 4; ++i) c.wa[i] = lb + (g >> 1) * 2048 + q * 256 + (g & 1) * 128 + ((i ^ q) * 32) + pp * 8;
    c.ctl = lb + L_FREE + wave * 256 + lane * 4;
    c.freev = 0; c.ready = 0; c.grp = 0;
    const int rg = (tm & 1) * 4 + wave;
    c.xr = __builtin_amdgcn_make_buffer_rsrc((void*)(my + XT_OFF + par * XT_BYTES), 0, (unsigned)XT_BYTES, 0x00020000);
    c.xvo = lane * 16; c.xso = (src * NRG + rg) * 4096;
    PIN();
    pros<0, NSET - 1>(c);
    prow<0, WPF>(c);
    c.need = NSET;
    PIN();
    if (wave == 0) { STAMP(2); STAMPC(16); }
#pragma unroll 1
    for (int it = 0; it < NSTEP / NSET; ++it) steps<0, NSET>(c);
    steps<0, NSTEP % NSET>(c);
    if (wave == 0) { STAMP(3); STAMPC(17); }
    unsigned verdict;
    while ((verdict = rfl(lds_ld(lb + L_MISC + 4))) == 0u) __builtin_amdgcn_s_sleep(1);
    int tz = threadIdx.x; asm volatile("" : "+v"(tz));
    const int l2 = tz & 63, g2 = l2 >> 4;
    char* op = (char*)p.out + ((size_t)(tm * 256 + wave * 64 + (l2 & 15)) * NL + tn * 128 + (g2 & 1) * 16 + (g2 >> 1) * 8) * 2;
#pragma unroll
    for (int m = 0; m < 4; ++m)
#pragma unroll
      for (int j = 0; j < 8; j += 2) {
        const f32x4 a = c.acc[m][j], d = c.acc[m][j + 1];
        const unsigned a0 = cvt_pk(relu(a[0]), relu(a[1])), a1 = cvt_pk(relu(a[2]), relu(a[3]));
        const unsigned d0 = cvt_pk(relu(d[0]), relu(d[1])), d1 = cvt_pk(relu(d[2]), relu(d[3]));
        const auto s0 = __builtin_amdgcn_permlane16_swap(a0, d0, false, false), s1 = __builtin_amdgcn_permlane16_swap(a1, d1, false, false);
        u32x4 o = {s0[0], s1[0], s0[1], s1[1]};
        if (verdict != 1u) o = (u32x4){0x7fc07fc0u, 0x7fc07fc0u, 0x7fc07fc0u, 0x7fc07fc0u};
        *(u32x4*)(op + (size_t)m * 16 * NL * 2 + j * 32) = o;
      }
  } else if (wave == 4) {
    const auto wr = __builtin_amdgcn_make_buffer_rsrc((void*)p.w, 0, KD * NL * 2, 0x00020000);
    const int q = lane >> 4, gl = (lane >> 3) & 1, pos = (lane >> 1) & 3;
    const unsigned vo = (8 * gl + q) * (NL * 2) + ((pos ^ q) * 32) + (lane & 1) * 16;
    const unsigned pubaddr = lb + L_WFULL + (lane & 3) * 256 + (lane >> 2) * 4;
    int pub = 0, slot = 0; unsigned freec = 0;
    for (int s = 0; s < NSTEP; ++s) {
      if (s >= RING && freec < (unsigned)(s - RING + 1)) {
        if (pub < s) { wait_vm0(); pub = s; lds_st(pubaddr, pub + NSET - 2); }
        for (;;) {
          const unsigned f = lds_ld(lb + L_FREE + (lane & 3) * 256);
          freec = min(min(__builtin_amdgcn_readlane(f, 0), __builtin_amdgcn_readlane(f, 1)), min(__builtin_amdgcn_readlane(f, 2), __builtin_amdgcn_readlane(f, 3)));
          if (freec >= (unsigned)(s - RING + 1)) break;
          __builtin_amdgcn_s_sleep(2);
        }
      }
      const unsigned so = s * 32 * (NL * 2) + tn * 256;
#pragma unroll
      for (int i = 0; i < 8; ++i)
        __builtin_amdgcn_raw_ptr_buffer_load_lds(wr, (LAS void*)((LAS char*)lds + slot * 8192 + i * 1024), 16, vo,
                                                 so + (16 * ((i >> 1) & 1) + 4 * (i & 1)) * (NL * 2) + (i >> 2) * 128, 0, 0);
      slot = slot == RING - 1 ? 0 : slot + 1;
      if (s + 1 - pub > INFL) { asm volatile("s_waitcnt vmcnt(%0)" :: "i"(8 * INFL) : "memory"); pub = s + 1 - INFL; lds_st(pubaddr, pub + NSET - 2); }
    }
    wait_vm0(); lds_st(pubaddr, INF);
  } else if (wave == 5) {
    if (b < NPULL) {
      constexpr unsigned PW = NPULL / (NR - 1);
      const int qi = b % (NR - 1), s = (rank + 1 + qi) & (NR - 1);
      const char* ctl = my + F2_OFF;
      const unsigned ctro = (unsigned)(CT_OFF - F2_OFF) + (2 + qi) * 128, dummy = (unsigned)(CC_OFF - F2_OFF) + b * 128 + 64;
      const unsigned tbase = (call - 1u) * (unsigned)NUNIT - 2u * PW;
      const auto pr = __builtin_amdgcn_make_buffer_rsrc((void*)(p.ws[s] + PK_OFF + par * PK_BYTES), 0, (unsigned)PK_BYTES, 0x00020000);
      const char* f1 = my + F1_OFF + par * F1_BYTES + (size_t)s * NUNIT * 128;
      char* const xt = my + XT_OFF + par * XT_BYTES + (size_t)(s * NRG) * 4096 + lane * 16;
      const unsigned vo = lane * 16, one = 1u;
      unsigned u0 = b / (NR - 1), u1 = u0 + PW, f0 = 0, prev = dummy;
      bool dead = false;
      const unsigned phase = u0 * TCYC / PW;
      unsigned tnext = 0;
      while (u0 < (unsigned)NUNIT) {
        if ((f0 >> 8) < call) {
          wait_vm0();
          if (lane == 0) __hip_atomic_store((unsigned*)(ctl + prev), call, __ATOMIC_RELAXED, __HIP_MEMORY_SCOPE_AGENT);
          prev = dummy;
          const unsigned long long t0 = rt();
          unsigned n = 0;
          while (((f0 = rfl(__hip_atomic_load((const unsigned*)(f1 + (size_t)u0 * 128), __ATOMIC_RELAXED, __HIP_MEMORY_SCOPE_SYSTEM))) >> 8) < call) {
            if ((++n & 63u) == 0u && (rfl(__hip_atomic_load(tmo, __ATOMIC_RELAXED, __HIP_MEMORY_SCOPE_AGENT)) == call || rt() - t0 > DEADLINE_TICKS)) {
              __hip_atomic_store(tmo, call, __ATOMIC_RELAXED, __HIP_MEMORY_SCOPE_AGENT); dead = true; break; }
            __builtin_amdgcn_s_sleep(2);
          }
          if (dead) break;
          const unsigned now = (unsigned)rt();
          tnext = now + (TCYC - 1) - (now + (TCYC - 1) - phase) % TCYC;
        }
        while ((int)((unsigned)rt() - tnext) < 0) __builtin_amdgcn_s_sleep(1);
        tnext += TCYC;
        unsigned tk, fl; u32x4 lo01, lo23, hh;
        const unsigned fo = (u1 < (unsigned)NUNIT ? u1 : 0u) * 128u;
        asm volatile(
            "s_nop 4\n\t"
            "buffer_load_dwordx4 %2, %5, %6, %7 offen sc0 sc1\n\t"
            "buffer_load_dwordx4 %3, %5, %6, %7 offen offset:1024 sc0 sc1\n\t"
            "buffer_load_dwordx4 %4, %5, %6, %7 offen offset:2048 sc0 sc1\n\t"
            "s_mov_b64 exec, 1\n\t"
            "global_load_dword %1, %8, %9 sc0 sc1\n\t"
            "global_atomic_add %0, %10, %11, %12 sc0\n\t"
            "s_waitcnt vmcnt(5)\n\t"
            "global_store_dword %13, %14, %12 sc1\n\t"
            "s_mov_b64 exec, -1\n\t"
            "s_waitcnt vmcnt(1)"
            : "=&v"(tk), "=&v"(fl), "=&v"(lo01), "=&v"(lo23), "=&v"(hh)
            : "v"(vo), "s"(pr), "s"(u0 * 3072u), "v"(fo), "s"(f1), "v"(ctro), "v"(one), "s"(ctl), "v"(prev), "v"(call)
            : "memory");
        const unsigned stp = u0 >> 3, rg = u0 & 7u;
        const unsigned base4 = (f0 & 0xffu) * 0x01010101u;
        char* d = xt + (size_t)(stp * 64 + rg) * 4096;
#pragma unroll
        for (int m = 0; m < 4; ++m) {
          const unsigned l0 = m < 2 ? lo01[2 * m] : lo23[2 * m - 4], l1 = m < 2 ? lo01[2 * m + 1] : lo23[2 * m - 3], h = hh[m];
          const unsigned h0 = (h & 0x87878787u) + base4, h1 = (__builtin_rotateright32(h, 4) & 0x87878787u) + base4;
          const u32x4 o = {__builtin_amdgcn_perm(h0, l0, 0x05010400u), __builtin_amdgcn_perm(h0, l0, 0x07030602u),
                           __builtin_amdgcn_perm(h1, l1, 0x05010400u), __builtin_amdgcn_perm(h1, l1, 0x07030602u)};
          if (h != 0u) st_sys(d + m * 1024, o);
        }
        prev = ((s * NRG + rg) * NSTEP + stp) * 4u;
        u0 = u1; u1 = rfl(tk) - tbase; f0 = rfl(fl);
        if ((int)(tnext - (unsigned)rt()) > EARLYPUB) {
          wait_vm0();
          if (lane == 0) __hip_atomic_store((unsigned*)(ctl + prev), call, __ATOMIC_RELAXED, __HIP_MEMORY_SCOPE_AGENT);
          prev = dummy;
        }
      }
      wait_vm0();
      if (lane == 0) __hip_atomic_store((unsigned*)(ctl + prev), call, __ATOMIC_RELAXED, __HIP_MEMORY_SCOPE_AGENT);
      while (u0 < (unsigned)NUNIT) {
        unsigned t = 0;
        if (lane == 0) t = __hip_atomic_fetch_add((unsigned*)(ctl + ctro), 1u, __ATOMIC_RELAXED, __HIP_MEMORY_SCOPE_AGENT);
        u0 = u1; u1 = rfl(t) - tbase;
      }
    }
  } else if (wave == 6) {
    const int rgi = lane >> 4, i = lane & 15;
    const unsigned xaddr = lb + L_XRDY + rgi * 256 + i * 4;
    bool dead = false;
    {
      const unsigned* f2 = (const unsigned*)(my + F2_OFF) + (src * NRG + (tm & 1) * 4 + rgi) * NSTEP;
      unsigned base = 0;
      const unsigned long long t0 = rt();
      for (unsigned n = 1;; ++n) {
        const unsigned idx = base + i;
        unsigned v = call;
        if (idx < (unsigned)NSTEP) v = __hip_atomic_load(f2 + idx, __ATOMIC_RELAXED, __HIP_MEMORY_SCOPE_AGENT);
        const unsigned long long ok = __ballot(v >= call);
        const unsigned sub = (unsigned)(ok >> (16 * rgi)) | 0xffff0000u;
        const unsigned adv = sub == 0xffffffffu ? 16u : (unsigned)__builtin_ctz(~sub);
        base += adv;
        if (__all(base >= (unsigned)NSTEP)) {
          break; }
        lds_st(xaddr, base >= (unsigned)NSTEP ? INF : base);
        if ((n & 63u) == 0u) {
          if (rfl(__hip_atomic_load(tmo, __ATOMIC_RELAXED, __HIP_MEMORY_SCOPE_AGENT)) == call || rt() - t0 > DEADLINE_TICKS) {
            __hip_atomic_store(tmo, call, __ATOMIC_RELAXED, __HIP_MEMORY_SCOPE_AGENT); dead = true; break; }
        }
        __builtin_amdgcn_s_sleep(POLLSLEEP);
      }
    }
    lds_st(xaddr, INF);
    STAMP(6);
    if (rfl(__hip_atomic_load(tmo, __ATOMIC_RELAXED, __HIP_MEMORY_SCOPE_AGENT)) == call) dead = true;
    lds_st(lb + L_MISC + 4 + lane * 0, dead ? 2u : 1u);
  }
}

}

extern "C" void kernel_launch(int rank, int nranks, ncclComm_t comm, hipStream_t stream,
                              void* const* d_in, const int* in_sizes, int n_in,
                              void* d_out, int out_size, const int* in_dtypes, int out_dtype,
                              void* const* d_out_all, void* const* d_ws_all, size_t ws_bytes) {
  if (nranks != NR || (unsigned)rank >= (unsigned)NR || n_in != 2 || in_sizes[0] != ML * KD || in_sizes[1] != KD * NL ||
      out_size != MG * NL || ws_bytes < WS_NEED) { fprintf(stderr, "unexpected problem shape\n"); return; }
  static bool attr[NR] = {};
  if (!attr[rank]) { HIP_CHECK(hipFuncSetAttribute((const void*)ag_gemm, hipFuncAttributeMaxDynamicSharedMemorySize, LDS_BYTES)); attr[rank] = true; }
  Params p;
  p.x = d_in[0]; p.w = d_in[1]; p.out = d_out; p.rank = rank; p.pad = 0;
  for (int i = 0; i < NR; ++i) p.ws[i] = (char*)d_ws_all[i];
  ag_gemm<<<GRID, BLOCK, LDS_BYTES, stream>>>(p);
  HIP_CHECK(hipGetLastError());
}
```

```cpp
#include <hip/hip_runtime.h>
#include <rccl/rccl.h>
#include <cstdio>

#define HIP_CHECK(expr) do { hipError_t _e = (expr); if (_e != hipSuccess) { \
    fprintf(stderr, "HIP error %s at %s:%d\n", hipGetErrorString(_e), __FILE__, __LINE__); \
    return; } } while (0)

namespace {

#define LAS __attribute__((address_space(3)))
#define DI __device__ __forceinline__
typedef short    s16x4  __attribute__((ext_vector_type(4)));
typedef short    s16x8  __attribute__((ext_vector_type(8)));
typedef __bf16   bf16x8 __attribute__((ext_vector_type(8)));
typedef __bf16   bf16x2 __attribute__((ext_vector_type(2)));
typedef float    f32x2  __attribute__((ext_vector_type(2)));
typedef float    f32x4  __attribute__((ext_vector_type(4)));
typedef unsigned u32x4  __attribute__((ext_vector_type(4)));
typedef unsigned short u16x2 __attribute__((ext_vector_type(2)));

constexpr int NR = 8;
constexpr int KD = 4096, NL = 2048, ML = 512, MG = NR * ML;
constexpr int NSTEP = KD / 32;
constexpr int NRG = ML / 64;
constexpr int NUNIT = NSTEP * 2;
constexpr int UBYTES = 4 * 3072;
constexpr int GRID = 256, BLOCK = 512;

#define NSET 6
#define NGRP 3
constexpr int RING = NSET * NGRP;
#define WPF 3
constexpr int WN = 4;
static_assert(WPF < WN, "");
constexpr int INFL = 6;
constexpr int PW = 36;
#define TAU16 334
#define NRAW 0
#define RAWGAP 9
#define POLLSLEEP 8
#define POLLFAST 4
#define POLLEND 6
#define RSPACE 2
#define SBITS "sc0 sc1"
#define NB 2
#define NTAIL 0
#define POLLW 16
#define OWN0 100
#define OWNGAP 25
#define WGAP 25
#define GMIN 0
#define XAUX 17

constexpr int L_FREE = RING * 8192, L_XRDY = L_FREE + 1024, L_WFULL = L_FREE + 2048, L_MISC = L_FREE + 3072;
constexpr int LDS_BYTES = L_MISC + 64;
static_assert(LDS_BYTES <= 160 * 1024 && LDS_BYTES > 80 * 1024, "");
constexpr unsigned INF = 0x7fffffffu;

constexpr size_t XT_BYTES = (size_t)MG * KD * 2;
constexpr size_t XT_OFF = 0;
constexpr size_t PK_BYTES = (size_t)NUNIT * UBYTES;
constexpr size_t PK_OFF = XT_OFF + 2 * XT_BYTES;
constexpr size_t F1_BYTES = (size_t)NR * NUNIT * 128;
constexpr size_t F1_OFF = PK_OFF + 2 * PK_BYTES;
constexpr size_t F2_OFF = F1_OFF + 2 * F1_BYTES;
constexpr size_t F2_BYTES = (size_t)NR * NUNIT * 4;
constexpr size_t CT_OFF = F2_OFF + F2_BYTES;
constexpr size_t CC_OFF = CT_OFF + 4096;
constexpr size_t DBG_OFF = CC_OFF + GRID * 128;
constexpr size_t DBG_BYTES = 1 << 20;
constexpr size_t WS_NEED = DBG_OFF + DBG_BYTES;
constexpr unsigned long long DEADLINE_TICKS = 20000000ull;

struct Params { const void* x; const void* w; void* out; char* ws[NR]; int rank; int pad; };

DI unsigned rfl(unsigned v) { return __builtin_amdgcn_readfirstlane(v); }
DI unsigned long long rt() { return __builtin_amdgcn_s_memrealtime(); }
DI void st_sys(void* p, u32x4 v) { asm volatile("global_store_dwordx4 %0, %1, off sc0 sc1\n\ts_nop 1" :: "v"(p), "v"(v) : "memory"); }
DI void wait_vm0() { asm volatile("s_waitcnt vmcnt(0)" ::: "memory"); }
DI void lds_st(unsigned addr, unsigned v) { asm volatile("ds_write_b32 %0, %1" :: "v"(addr), "v"(v) : "memory"); }
DI unsigned lds_ld(unsigned addr) { unsigned v; asm volatile("ds_read_b32 %0, %1\n\ts_waitcnt lgkmcnt(0)" : "=&v"(v) : "v"(addr) : "memory"); return v; }
DI unsigned peek(const unsigned* p) { unsigned v; asm volatile("global_atomic_add %0, %1, %2, off sc0 sc1\n\ts_waitcnt vmcnt(0)" : "=&v"(v) : "v"(p), "v"(0u) : "memory"); return v; }
DI unsigned long long peek64(const void* p) { unsigned long long v; asm volatile("global_atomic_add_x2 %0, %1, %2, off sc0 sc1\n\ts_waitcnt vmcnt(0)" : "=&v"(v) : "v"(p), "v"(0ull) : "memory"); return v; }
DI void sleep_until(unsigned t) {
  for (int d; (d = (int)(t - (unsigned)rt())) > 0;) { if (d > 200) __builtin_amdgcn_s_sleep(48); else if (d > 40) __builtin_amdgcn_s_sleep(8); else __builtin_amdgcn_s_sleep(1); }
  asm volatile("" ::: "memory");
}
#define PIN() __builtin_amdgcn_sched_barrier(0)

#define STAMP(slot) do {} while (0)
#define STAMPC(slot) do {} while (0)

struct CW {
  f32x4 acc[4][8];
  u32x4 xs[NSET][4];
  s16x4 wf[WN][2];
  unsigned wa[4];
  unsigned ctl;
  unsigned xvo;
  unsigned freev;
  unsigned ready;
  unsigned need;
  unsigned xso;
  unsigned grp;
  __amdgpu_buffer_rsrc_t xr;
};

DI void wait_ready(CW& c) {
  asm volatile(
      "s_cmp_lt_u32 %0, %2\n\t"
      "s_cbranch_scc1 1f\n"
      "2:\n\t"
      ".subsection 1\n"
      "3:\n\t"
      "s_sleep 1\n"
      "1:\n\t"
      "ds_read_b32 v254, %1 offset:1024\n\t"
      "ds_read_b32 v255, %1 offset:2048\n\t"
      "s_waitcnt lgkmcnt(0)\n\t"
      "v_min_u32 v254, v254, v255\n\t"
      "s_nop 1\n\t"
      "v_readfirstlane_b32 %0, v254\n\t"
      "s_nop 3\n\t"
      "s_cmp_lt_u32 %0, %2\n\t"
      "s_cbranch_scc1 3b\n\t"
      "s_branch 2b\n\t"
      ".subsection 0"
      : "+s"(c.ready) : "v"(c.ctl), "s"(c.need) : "scc", "memory", "v254", "v255");
}

template <int SET, int M> DI void xload(CW& c) {
  c.xs[SET][M] = __builtin_amdgcn_raw_buffer_load_b128(c.xr, c.xvo + M * 1024, c.xso, XAUX);
}
template <int SL, int J, int T> DI void wread(CW& c) {
  c.wf[J % WN][T] = __builtin_amdgcn_ds_read_tr16_b64_v4i16((LAS s16x4*)((LAS char*)0 + c.wa[J & 3] + (SL * 8192 + (J >> 2) * 4096 + T * 1024)));
}
template <int S, int J, int M> DI void mma(CW& c) {
  const s16x8 w = __builtin_shufflevector(c.wf[J % WN][0], c.wf[J % WN][1], 0, 1, 2, 3, 4, 5, 6, 7);
  c.acc[M][J] = __builtin_amdgcn_mfma_f32_16x16x32_bf16(__builtin_bit_cast(bf16x8, w), __builtin_bit_cast(bf16x8, c.xs[S][M]), c.acc[M][J], 0, 0, 0);
}
DI void next_group(CW& c) {
  const bool wrap = (c.grp == NGRP - 1);
  const unsigned d = wrap ? (unsigned)(-(NGRP - 1) * NSET * 8192) : (unsigned)(NSET * 8192);
  c.grp = wrap ? 0u : c.grp + 1u;
#pragma unroll
  for (int i = 0; i < 4; ++i) c.wa[i] += d;
}
template <int S, int J> DI void frag(CW& c) {
  constexpr int JP = (J + WPF) & 7;
  constexpr bool NXT = (J + WPF) >= 8;
  constexpr int SL = NXT ? (S + 1) % NSET : S;
  constexpr int LS = (S + NSET - 1) % NSET;
  mma<S, J, 0>(c); PIN();
  wread<SL, JP, 0>(c); PIN();
  mma<S, J, 1>(c); PIN();
  wread<SL, JP, 1>(c); PIN();
  mma<S, J, 2>(c); PIN();
  if constexpr (J < 4) { xload<LS, J>(c); PIN(); }
  if constexpr (J == 7 - WPF) {
    c.freev += 1; lds_st(c.ctl, c.freev);
    if constexpr (S == NSET - 1) next_group(c);
    PIN();
  }
  if constexpr (J == 6) { c.need += 1; c.xso += 64 * 4096; asm volatile("" : "+s"(c.need), "+s"(c.xso)); PIN(); }
  if constexpr (J == 7) { wait_ready(c); PIN(); }
  mma<S, J, 3>(c); PIN();
}
template <int S> DI void step(CW& c) {
  frag<S, 0>(c); frag<S, 1>(c); frag<S, 2>(c); frag<S, 3>(c);
  frag<S, 4>(c); frag<S, 5>(c); frag<S, 6>(c); frag<S, 7>(c);
}
template <int S> DI void pro_x(CW& c) {
  c.need = S + 1; wait_ready(c); PIN();
  xload<S, 0>(c); xload<S, 1>(c); xload<S, 2>(c); xload<S, 3>(c); c.xso += 64 * 4096; PIN();
}

DI unsigned cvt_pk(float lo, float hi) { f32x2 v = {lo, hi}; return __builtin_bit_cast(unsigned, __builtin_convertvector(v, bf16x2)); }
DI float relu(float v) { return __builtin_elementwise_maximum(v, 0.0f); }

template <int S, int E> DI void steps(CW& c) { if constexpr (S < E) { step<S>(c); steps<S + 1, E>(c); } }
template <int S, int E> DI void pros(CW& c) { if constexpr (S < E) { pro_x<S>(c); pros<S + 1, E>(c); } }
template <int J, int E> DI void prow(CW& c) { if constexpr (J < E) { wread<0, J, 0>(c); wread<0, J, 1>(c); prow<J + 1, E>(c); } }

DI unsigned dpp_max(unsigned v, const int ctrl) {
  unsigned o;
  switch (ctrl) {
    case 1: o = __builtin_amdgcn_update_dpp(0u, v, 0x111, 0xf, 0xf, true); break;
    case 2: o = __builtin_amdgcn_update_dpp(0u, v, 0x112, 0xf, 0xf, true); break;
    case 4: o = __builtin_amdgcn_update_dpp(0u, v, 0x114, 0xf, 0xf, true); break;
    default: o = __builtin_amdgcn_update_dpp(0u, v, 0x118, 0xf, 0xf, true); break;
  }
  return o > v ? o : v;
}

DI bool wait_f1(const char* f, unsigned call, unsigned* tmo, unsigned& payload) {
  const unsigned long long t0 = rt();
  for (unsigned n = 1;; ++n) {
    unsigned long long v = 0;
    if (threadIdx.x % 64 == 0) v = peek64(f);
    if (rfl((unsigned)v) >= call) { payload = rfl((unsigned)(v >> 32)); return true; }
    if ((n & 63u) == 0u && (rfl(__hip_atomic_load(tmo, __ATOMIC_RELAXED, __HIP_MEMORY_SCOPE_AGENT)) == call || rt() - t0 > DEADLINE_TICKS)) {
      __hip_atomic_store(tmo, call, __ATOMIC_RELAXED, __HIP_MEMORY_SCOPE_AGENT); return false; }
    __builtin_amdgcn_s_sleep(4);
  }
}

struct Pull {
  __amdgpu_buffer_rsrc_t pr, xw; const char* f1; char* my; unsigned* tmo; unsigned call, t0; int s, lane;
  unsigned tfree;
  unsigned fcall = 0, fbases = 0;
  unsigned lsum = 0, lmax = 0, lcnt = 0;
};
template <int N> DI bool pull(Pull& q, unsigned k, unsigned knext) {
  const unsigned u = k >> 2, w0 = k & 3u;
  unsigned bases = q.fbases;
  if (q.fcall < q.call && !wait_f1(q.f1 + u * 128, q.call, q.tmo, bases)) return false;
  bases >>= 8 * w0;
  const unsigned tgo = q.t0 + (k - 8 * NRAW) * TAU16 / 64;
  sleep_until((int)(tgo - q.tfree) > 0 ? tgo : q.tfree);
  const unsigned long long fn = __hip_atomic_load((const unsigned long long*)(q.f1 + (knext < 4u * NUNIT ? knext >> 2 : 0u) * 128), __ATOMIC_RELAXED, __HIP_MEMORY_SCOPE_SYSTEM);
  PIN();
  u32x4 v[3 * N];
#pragma unroll
  for (int i = 0; i < 3 * N; ++i) {
    v[i] = __builtin_amdgcn_raw_buffer_load_b128(q.pr, q.lane * 16 + (i % 3) * 1024, (k + i / 3) * 3072, 17);
    if (RSPACE > 0 && i < 3 * N - 1) { PIN(); __builtin_amdgcn_s_sleep(RSPACE); PIN(); }
  }
  const unsigned d = ((u >> 1) * 64 + q.s * NRG + (k & 7u)) * 4096;
#pragma unroll
  for (int w = 0; w < N; ++w) {
    const unsigned base4 = ((bases >> (8 * w)) & 0xffu) * 0x01010101u;
#pragma unroll
    for (int m = 0; m < 4; ++m) {
      const unsigned l0 = v[3 * w + (m >> 1)][2 * (m & 1)], l1 = v[3 * w + (m >> 1)][2 * (m & 1) + 1], h = v[3 * w + 2][m];
      const unsigned h0 = (h & 0x87878787u) + base4, h1 = (__builtin_rotateright32(h, 4) & 0x87878787u) + base4;
      const u32x4 o = {__builtin_amdgcn_perm(h0, l0, 0x05010400u), __builtin_amdgcn_perm(h0, l0, 0x07030602u),
                       __builtin_amdgcn_perm(h1, l1, 0x05010400u), __builtin_amdgcn_perm(h1, l1, 0x07030602u)};
      asm volatile("buffer_store_dwordx4 %0, %1, %2, %3 offen offset:%4 " SBITS "\n\ts_nop 1"
                   :: "v"(o), "v"(h != 0u ? q.lane * 16u : 0x80000000u), "s"(q.xw), "s"(d + w * 4096), "i"(m * 1024) : "memory");
    }
  }
  wait_vm0();
  q.fcall = rfl((unsigned)fn); q.fbases = rfl((unsigned)(fn >> 32));
  q.tfree = (unsigned)rt() + GMIN;
  if (q.lane == 0) __hip_atomic_fetch_add((unsigned*)(q.my + F2_OFF) + (q.s * 2 + (u & 1)) * NSTEP + (u >> 1), (unsigned)N, __ATOMIC_RELAXED, __HIP_MEMORY_SCOPE_SYSTEM);
  return true;
}

__global__ __launch_bounds__(BLOCK, 2) void ag_gemm(Params p) {
  extern __shared__ __attribute__((aligned(16))) char lds[];
  const int lane = threadIdx.x & 63;
  const int wave = rfl(threadIdx.x >> 6);
  const unsigned lb = (unsigned)(size_t)(LAS char*)lds;
  const int b = blockIdx.x, rank = p.rank & (NR - 1);
  char* const my = p.ws[rank];
  const int xcd = b & 7, bi = b >> 3;
  const int tm = (xcd >> 1) * 4 + (bi >> 3), tn = (xcd & 1) * 8 + (bi & 7);
  unsigned* const tmo = (unsigned*)(my + CT_OFF);
  const unsigned tstart = (unsigned)rt();
  if (wave == 0) STAMP(0);

  const int ppair = (bi >> 1) * 8 + xcd;
  int pstep = 2 * (ppair >> 1) + (bi & 1), prg = (ppair & 1) * 4 + wave;
  constexpr int RAW0 = GRID - NRAW * NRG * (NR - 1);
  static_assert(RAW0 >= 0, "");
  const int rblk = (b - RAW0) / (NR - 1);
  const bool pusher = NRAW > 0 && wave == 7 && b >= RAW0;
  if (pusher) { pstep = rblk >> 3; prg = rblk & 7; }
  u32x4 pv[4];
  if (wave < 4 || pusher) {
    const char* xp = (const char*)p.x + ((size_t)(prg * 64 + (lane & 15)) * KD + pstep * 32 + (lane >> 4) * 8) * 2;
#pragma unroll
    for (int m = 0; m < 4; ++m) pv[m] = *(const u32x4*)(xp + (size_t)m * 16 * KD * 2);
  }
  if (wave == 6) {
    unsigned t = 0;
    if (lane == 0) t = __hip_atomic_fetch_add((unsigned*)(my + CC_OFF + b * 128), 1u, __ATOMIC_RELAXED, __HIP_MEMORY_SCOPE_AGENT);
    t = rfl(t) + 1;
    if (lane < 16) lds_st(lb + L_MISC + lane * 4, lane == 0 ? t : 0u);
  } else if (wave == 4 || wave == 5) {
#pragma unroll
    for (int i = 0; i < 6; ++i) lds_st(lb + L_FREE + (wave - 4) * 1536 + i * 256 + lane * 4, 0u);
  }
  asm volatile("s_waitcnt lgkmcnt(0)" ::: "memory");
  __builtin_amdgcn_s_barrier();
  asm volatile("" ::: "memory");
  const unsigned call = rfl(lds_ld(lb + L_MISC));
  const unsigned par = call & 1u;
  const size_t xo = XT_OFF + par * XT_BYTES + (size_t)(pstep * 64 + rank * NRG + prg) * 4096 + lane * 16;
  const size_t f2o = F2_OFF + ((rank * 2 + (prg >> 2)) * NSTEP + pstep) * 4;

  if (wave < 4) {
    if (pstep >= NRAW) {
      u16x2 mx = {0, 0};
#pragma unroll
      for (int m = 0; m < 4; ++m)
#pragma unroll
        for (int i = 0; i < 4; ++i) mx = __builtin_elementwise_max(mx, __builtin_bit_cast(u16x2, pv[m][i] & 0x7fff7fffu));
      unsigned v = mx[0] > mx[1] ? mx[0] : mx[1];
      v = dpp_max(v, 1); v = dpp_max(v, 2); v = dpp_max(v, 4); v = dpp_max(v, 8);
      const unsigned a0 = __builtin_amdgcn_readlane(v, 15), a1 = __builtin_amdgcn_readlane(v, 31), a2 = __builtin_amdgcn_readlane(v, 47), a3 = __builtin_amdgcn_readlane(v, 63);
      const unsigned top = max(max(a0, a1), max(a2, a3)) >> 8;
      const unsigned base = top > 7u ? top - 7u : 0u, base4 = base * 0x01010101u;
      u32x4 lo01, lo23, hh; unsigned esc = 0;
#pragma unroll
      for (int m = 0; m < 4; ++m) {
        const unsigned l0 = __builtin_amdgcn_perm(pv[m][1], pv[m][0], 0x06040200u), l1 = __builtin_amdgcn_perm(pv[m][3], pv[m][2], 0x06040200u);
        const unsigned h0 = __builtin_amdgcn_perm(pv[m][1], pv[m][0], 0x07050301u), h1 = __builtin_amdgcn_perm(pv[m][3], pv[m][2], 0x07050301u);
        const unsigned t0 = (h0 | 0x80808080u) - base4, t1 = (h1 | 0x80808080u) - base4;
        const unsigned c0 = (h0 & 0x80808080u) | (t0 & 0x07070707u), c1 = (h1 & 0x80808080u) | (t1 & 0x07070707u);
        unsigned h = c0 | __builtin_rotateleft32(c1, 4);
        const bool fits = ((t0 & 0xf8f8f8f8u) == 0x80808080u) && ((t1 & 0xf8f8f8f8u) == 0x80808080u) && h != 0u;
        if (!fits) { h = 0u; esc |= 1u << m; }
        hh[m] = h;
        if (m < 2) { lo01[2 * m] = l0; lo01[2 * m + 1] = l1; } else { lo23[2 * m - 4] = l0; lo23[2 * m - 3] = l1; }
      }
      if (wave == 0) STAMP(7);
      const int u = pstep * 2 + (prg >> 2);
      char* pk = my + PK_OFF + par * PK_BYTES + (size_t)u * UBYTES + wave * 3072 + lane * 16;
      st_sys(pk, lo01); st_sys(pk + 1024, lo23); st_sys(pk + 2048, hh);
      if (esc) {
        for (int i = 1; i < NR; ++i) {
          char* d = p.ws[(rank + i) & (NR - 1)] + xo;
#pragma unroll
          for (int m = 0; m < 4; ++m) if (esc >> m & 1u) st_sys(d + m * 1024, pv[m]);
        }
      }
      wait_vm0();
      unsigned old;
      asm volatile("s_mov_b64 exec, 1\n\tds_or_b32 %1, %2 offset:4\n\tds_add_rtn_u32 %0, %1, %3\n\ts_mov_b64 exec, -1\n\ts_waitcnt lgkmcnt(0)"
                   : "=&v"(old) : "v"(lb + L_MISC + 8), "v"(base << (8 * wave)), "v"(1u) : "memory");
      if (rfl(old) == 3u) {
        const unsigned bases = lds_ld(lb + L_MISC + 12);
        if (lane < NR && lane != rank)
          __hip_atomic_store((unsigned long long*)(p.ws[lane] + F1_OFF + par * F1_BYTES + (size_t)(rank * NUNIT + u) * 128),
                             ((unsigned long long)bases << 32) | call, __ATOMIC_RELAXED, __HIP_MEMORY_SCOPE_SYSTEM);
        STAMP(8);
      }
    } else {
      asm volatile("s_mov_b64 exec, 1\n\tds_add_u32 %0, %1\n\ts_mov_b64 exec, -1" :: "v"(lb + L_MISC + 8), "v"(1u) : "memory");
    }
    if (wave == 0) STAMP(1);
    CW c;
    const int g = lane >> 4, q = (lane >> 2) & 3, pp = lane & 3;
#pragma unroll
    for (int m = 0; m < 4; ++m)
#pragma unroll
      for (int j = 0; j < 8; ++j) c.acc[m][j] = (f32x4){0.f, 0.f, 0.f, 0.f};
#pragma unroll
    for (int i = 0; i < 4; ++i) c.wa[i] = lb + (g >> 1) * 2048 + q * 256 + (g & 1) * 128 + ((i ^ q) * 32) + pp * 8;
    c.ctl = lb + L_FREE + wave * 256 + lane * 4;
    c.freev = 0; c.ready = 0; c.grp = 0;
    c.xr = __builtin_amdgcn_make_buffer_rsrc((void*)(my + XT_OFF + par * XT_BYTES), 0, (unsigned)XT_BYTES, 0x00020000);
    c.xvo = lane * 16; c.xso = (tm * 4 + wave) * 4096;
    PIN();
    pros<0, NSET - 1>(c);
    prow<0, WPF>(c);
    c.need = NSET;
    wait_ready(c);
    PIN();
    if (wave == 0) { STAMP(2); STAMPC(16); }
#pragma unroll 1
    for (int it = 0; it < NSTEP / NSET; ++it) steps<0, NSET>(c);
    steps<0, NSTEP % NSET>(c);
    if (wave == 0) { STAMP(3); STAMPC(17); }
    unsigned verdict;
    while ((verdict = rfl(lds_ld(lb + L_MISC + 4))) == 0u) __builtin_amdgcn_s_sleep(1);
    int tz = threadIdx.x; asm volatile("" : "+v"(tz));
    const int l2 = tz & 63, g2 = l2 >> 4;
    char* op = (char*)p.out + ((size_t)(tm * 256 + wave * 64 + (l2 & 15)) * NL + tn * 128 + (g2 & 1) * 16 + (g2 >> 1) * 8) * 2;
#pragma unroll
    for (int m = 0; m < 4; ++m)
#pragma unroll
      for (int j = 0; j < 8; j += 2) {
        const f32x4 a = c.acc[m][j], d = c.acc[m][j + 1];
        const unsigned a0 = cvt_pk(relu(a[0]), relu(a[1])), a1 = cvt_pk(relu(a[2]), relu(a[3]));
        const unsigned d0 = cvt_pk(relu(d[0]), relu(d[1])), d1 = cvt_pk(relu(d[2]), relu(d[3]));
        const auto s0 = __builtin_amdgcn_permlane16_swap(a0, d0, false, false), s1 = __builtin_amdgcn_permlane16_swap(a1, d1, false, false);
        u32x4 o = {s0[0], s1[0], s0[1], s1[1]};
        if (verdict != 1u) o = (u32x4){0x7fc07fc0u, 0x7fc07fc0u, 0x7fc07fc0u, 0x7fc07fc0u};
        *(u32x4*)(op + (size_t)m * 16 * NL * 2 + j * 32) = o;
      }
  } else if (wave == 4) {
    while (rfl(lds_ld(lb + L_MISC + 8)) < 4u) __builtin_amdgcn_s_sleep(2);
    const auto wr = __builtin_amdgcn_make_buffer_rsrc((void*)p.w, 0, KD * NL * 2, 0x00020000);
    const int q = lane >> 4, gl = (lane >> 3) & 1, pos = (lane >> 1) & 3;
    const unsigned vo = (8 * gl + q) * (NL * 2) + ((pos ^ q) * 32) + (lane & 1) * 16;
    const unsigned pubaddr = lb + L_WFULL + (lane & 3) * 256 + (lane >> 2) * 4;
    int pub = 0, slot = 0; unsigned freec = 0;
    const unsigned tw = (unsigned)rt();
    for (int s = 0; s < NSTEP; ++s) {
      if (s >= RING && freec < (unsigned)(s - RING + 1)) {
        if (pub < s) { wait_vm0(); pub = s; lds_st(pubaddr, pub + NSET - 2); }
        for (;;) {
          const unsigned f = lds_ld(lb + L_FREE + (lane & 3) * 256);
          freec = min(min(__builtin_amdgcn_readlane(f, 0), __builtin_amdgcn_readlane(f, 1)), min(__builtin_amdgcn_readlane(f, 2), __builtin_amdgcn_readlane(f, 3)));
          if (freec >= (unsigned)(s - RING + 1)) break;
          __builtin_amdgcn_s_sleep(2);
        }
      }
      if (s < RING) {
        const unsigned tgo = tw + s * WGAP;
        if ((int)(tgo - (unsigned)rt()) > 0) { if (pub < s) { wait_vm0(); pub = s; lds_st(pubaddr, pub + NSET - 2); } sleep_until(tgo); }
      }
      const unsigned so = s * 32 * (NL * 2) + tn * 256;
#pragma unroll
      for (int i = 0; i < 8; ++i)
        __builtin_amdgcn_raw_ptr_buffer_load_lds(wr, (LAS void*)((LAS char*)lds + slot * 8192 + i * 1024), 16, vo,
                                                 so + (16 * ((i >> 1) & 1) + 4 * (i & 1)) * (NL * 2) + (i >> 2) * 128, 0, 0);
      slot = slot == RING - 1 ? 0 : slot + 1;
      if (s + 1 - pub > INFL) { asm volatile("s_waitcnt vmcnt(%0)" :: "i"(8 * INFL) : "memory"); pub = s + 1 - INFL; lds_st(pubaddr, pub + NSET - 2); }
    }
    wait_vm0(); lds_st(pubaddr, INF);
  } else if (wave == 6) {
    const unsigned xaddr = lb + L_XRDY + (lane & 3) * 256 + (lane >> 2) * 4;
    bool dead = false;
    {
      const unsigned* f2 = (const unsigned*)(my + F2_OFF) + tm * NSTEP;
      unsigned base = 0;
      const unsigned long long t0 = rt();
      for (unsigned n = 1;; ++n) {
        const unsigned idx = base + lane;
        unsigned v = 4u * call;
        if (idx < (unsigned)NSTEP && lane < POLLW) v = peek(f2 + idx);
        const unsigned nok = ~(unsigned)__ballot(v >= 4u * call);
        base += nok ? (unsigned)__builtin_ctz(nok) : (unsigned)POLLW;
        if (base >= (unsigned)NSTEP) {
          break; }
        lds_st(xaddr, base);
        if ((n & 63u) == 0u) {
          if (rfl(__hip_atomic_load(tmo, __ATOMIC_RELAXED, __HIP_MEMORY_SCOPE_AGENT)) == call || rt() - t0 > DEADLINE_TICKS) {
            __hip_atomic_store(tmo, call, __ATOMIC_RELAXED, __HIP_MEMORY_SCOPE_AGENT); dead = true; break; }
        }
        if (base >= (unsigned)(NSTEP - POLLEND)) __builtin_amdgcn_s_sleep(POLLFAST); else __builtin_amdgcn_s_sleep(POLLSLEEP);
      }
    }
    lds_st(xaddr, INF);
    STAMP(6);
    if (rfl(__hip_atomic_load(tmo, __ATOMIC_RELAXED, __HIP_MEMORY_SCOPE_AGENT)) == call) dead = true;
    lds_st(lb + L_MISC + 4, dead ? 2u : 1u);
  } else {
    if (pusher) {
      char* const peer = p.ws[(rank + 1 + (b - RAW0) % (NR - 1)) & (NR - 1)];
      sleep_until(tstart + 100 + rblk * RAWGAP);
#pragma unroll
      for (int m = 0; m < 4; ++m) st_sys(peer + xo + m * 1024, pv[m]);
      wait_vm0();
      if (lane == 0) __hip_atomic_fetch_add((unsigned*)(peer + f2o), 1u, __ATOMIC_RELAXED, __HIP_MEMORY_SCOPE_SYSTEM);
      STAMP(10);
    }
    if (wave == 7) {
      const int ostep = 2 * (ppair >> 1) + (bi & 1), ohalf = ppair & 1;
      sleep_until(tstart + OWN0 + ostep * OWNGAP);
      const char* xp = (const char*)p.x + ((size_t)(ohalf * 256 + (lane & 15)) * KD + ostep * 32 + (lane >> 4) * 8) * 2;
      u32x4 ov[16];
#pragma unroll
      for (int m = 0; m < 16; ++m) ov[m] = *(const u32x4*)(xp + (size_t)m * 16 * KD * 2);
      char* d = my + XT_OFF + par * XT_BYTES + (size_t)(ostep * 64 + rank * NRG + ohalf * 4) * 4096 + lane * 16;
#pragma unroll
      for (int m = 0; m < 16; ++m) st_sys(d + m * 1024, ov[m]);
      wait_vm0();
      if (lane == 0) __hip_atomic_fetch_add((unsigned*)(my + F2_OFF) + (rank * 2 + ohalf) * NSTEP + ostep, 4u, __ATOMIC_RELAXED, __HIP_MEMORY_SCOPE_SYSTEM);
    }
    if (b < PW * (NR - 1)) {
      Pull q;
      q.s = (rank + 1 + b % (NR - 1)) & (NR - 1);
      q.pr = __builtin_amdgcn_make_buffer_rsrc((void*)(p.ws[q.s] + PK_OFF + par * PK_BYTES), 0, (unsigned)PK_BYTES, 0x00020000);
      q.xw = __builtin_amdgcn_make_buffer_rsrc((void*)(my + XT_OFF + par * XT_BYTES), 0, (unsigned)XT_BYTES, 0x00020000);
      q.f1 = my + F1_OFF + par * F1_BYTES + (size_t)q.s * NUNIT * 128;
      q.my = my; q.tmo = tmo; q.call = call; q.lane = lane; q.tfree = tstart;
      constexpr unsigned B0 = 8 * NRAW, B1 = 4 * NUNIT - 4 * NTAIL;
      const unsigned wi = b / (NR - 1);
      if (wave == 5) {
        unsigned bases;
        bool ok = wait_f1(q.f1 + 2 * NRAW * 128, call, tmo, bases);
        q.t0 = (unsigned)rt() | 1u;
        lds_st(lb + L_MISC + 16, q.t0);
        for (unsigned k = B0 + wi * NB; ok && k < B1; k += PW * NB) ok = pull<NB>(q, k, k + PW * NB);
      } else {
        while ((q.t0 = rfl(lds_ld(lb + L_MISC + 16))) == 0u) __builtin_amdgcn_s_sleep(32);
        bool ok = true;
        for (unsigned k = B1 + wi; ok && k < 4u * NUNIT; k += PW) ok = pull<1>(q, k, k + PW);
      }
    }
  }
}

}

extern "C" void kernel_launch(int rank, int nranks, ncclComm_t comm, hipStream_t stream,
                              void* const* d_in, const int* in_sizes, int n_in,
                              void* d_out, int out_size, const int* in_dtypes, int out_dtype,
                              void* const* d_out_all, void* const* d_ws_all, size_t ws_bytes) {
  if (nranks != NR || (unsigned)rank >= (unsigned)NR || n_in != 2 || in_sizes[0] != ML * KD || in_sizes[1] != KD * NL ||
      out_size != MG * NL || ws_bytes < WS_NEED) { fprintf(stderr, "unexpected problem shape\n"); return; }
  static bool attr[NR] = {};
  if (!attr[rank]) { HIP_CHECK(hipFuncSetAttribute((const void*)ag_gemm, hipFuncAttributeMaxDynamicSharedMemorySize, LDS_BYTES)); attr[rank] = true; }
  Params p;
  p.x = d_in[0]; p.w = d_in[1]; p.out = d_out; p.rank = rank; p.pad = 0;
  for (int i = 0; i < NR; ++i) p.ws[i] = (char*)d_ws_all[i];
  ag_gemm<<<GRID, BLOCK, LDS_BYTES, stream>>>(p);
  HIP_CHECK(hipGetLastError());
}
```

```cpp
#include <hip/hip_runtime.h>
#include <rccl/rccl.h>
#include <cstdio>

#define HIP_CHECK(expr) do { hipError_t _e = (expr); if (_e != hipSuccess) { \
    fprintf(stderr, "HIP error %s at %s:%d\n", hipGetErrorString(_e), __FILE__, __LINE__); \
    return; } } while (0)

namespace {

#define LAS __attribute__((address_space(3)))
#define DI __device__ __forceinline__
typedef short    s16x4  __attribute__((ext_vector_type(4)));
typedef short    s16x8  __attribute__((ext_vector_type(8)));
typedef __bf16   bf16x8 __attribute__((ext_vector_type(8)));
typedef __bf16   bf16x2 __attribute__((ext_vector_type(2)));
typedef float    f32x2  __attribute__((ext_vector_type(2)));
typedef float    f32x4  __attribute__((ext_vector_type(4)));
typedef unsigned u32x4  __attribute__((ext_vector_type(4)));
typedef unsigned short u16x2 __attribute__((ext_vector_type(2)));

constexpr int NR = 8;
constexpr int KD = 4096, NL = 2048, ML = 512, MG = NR * ML;
constexpr int NSTEP = KD / 32;
constexpr int NRG = ML / 64;
constexpr int NUNIT = NSTEP * 2;
constexpr int UBYTES = 4 * 3072;
constexpr int GRID = 256, BLOCK = 512;

#define NSET 6
#define NGRP 3
constexpr int RING = NSET * NGRP;
#define WPF 3
constexpr int WN = 4;
static_assert(WPF < WN, "");
constexpr int INFL = 6;
constexpr int PW = 36;
#define TAU16 324
#define POLLSLEEP 8
#define POLLFAST 4
#define POLLEND 6
#define RSPACE 2
#define SBITS "sc0 sc1"
#define NB 2
#define POLLW 16
#define OWN0 100
#define OWNGAP 25
#define WGAP 25
#define WLEAD 32
#define JROT 0
#define WPRES 64
#define WPRED 4
#define PULLWAIT 1
#define XMARG 50
#define XMARG1 0
#define W0 4
#define WT0 500
#define PKGAP 0
#define TLBWARM 0
#define XAUX 17

constexpr int L_FREE = RING * 8192, L_XRDY = L_FREE + 1024, L_WFULL = L_FREE + 2048, L_MISC = L_FREE + 3072;
constexpr int LDS_BYTES = L_MISC + 64 + 256;
static_assert(LDS_BYTES <= 160 * 1024 && LDS_BYTES > 80 * 1024, "");
constexpr unsigned INF = 0x7fffffffu;

constexpr size_t XT_BYTES = (size_t)MG * KD * 2;
constexpr size_t XT_OFF = 0;
constexpr size_t PK_BYTES = (size_t)NUNIT * UBYTES;
constexpr size_t PK_OFF = XT_OFF + 2 * XT_BYTES;
constexpr size_t F1_BYTES = (size_t)NR * NUNIT * 128;
constexpr size_t F1_OFF = PK_OFF + 2 * PK_BYTES;
constexpr size_t F2_OFF = F1_OFF + 2 * F1_BYTES;
constexpr size_t F2_BYTES = (size_t)NR * NUNIT * 4;
constexpr size_t CT_OFF = F2_OFF + F2_BYTES;
constexpr size_t CC_OFF = CT_OFF + 4096;
constexpr size_t DBG_OFF = CC_OFF + GRID * 128;
constexpr size_t DBG_BYTES = 1 << 20;
constexpr size_t WS_NEED = DBG_OFF + DBG_BYTES;
constexpr unsigned long long DEADLINE_TICKS = 20000000ull;

struct Params { const void* x; const void* w; void* out; char* ws[NR]; int rank; int pad; };

DI unsigned rfl(unsigned v) { return __builtin_amdgcn_readfirstlane(v); }
DI unsigned long long rt() { return __builtin_amdgcn_s_memrealtime(); }
DI void st_sys(void* p, u32x4 v) { asm volatile("global_store_dwordx4 %0, %1, off sc0 sc1\n\ts_nop 1" :: "v"(p), "v"(v) : "memory"); }
DI void wait_vm0() { asm volatile("s_waitcnt vmcnt(0)" ::: "memory"); }
DI void lds_st(unsigned addr, unsigned v) { asm volatile("ds_write_b32 %0, %1" :: "v"(addr), "v"(v) : "memory"); }
DI unsigned lds_ld(unsigned addr) { unsigned v; asm volatile("ds_read_b32 %0, %1\n\ts_waitcnt lgkmcnt(0)" : "=&v"(v) : "v"(addr) : "memory"); return v; }
DI unsigned peek(const unsigned* p) { unsigned v; asm volatile("global_atomic_add %0, %1, %2, off sc0 sc1\n\ts_waitcnt vmcnt(0)" : "=&v"(v) : "v"(p), "v"(0u) : "memory"); return v; }
DI unsigned long long peek64(const void* p) { unsigned long long v; asm volatile("global_atomic_add_x2 %0, %1, %2, off sc0 sc1\n\ts_waitcnt vmcnt(0)" : "=&v"(v) : "v"(p), "v"(0ull) : "memory"); return v; }
DI void sleep_until(unsigned t) {
  for (int d; (d = (int)(t - (unsigned)rt())) > 0;) { if (d > 200) __builtin_amdgcn_s_sleep(48); else if (d > 40) __builtin_amdgcn_s_sleep(8); else __builtin_amdgcn_s_sleep(1); }
  asm volatile("" ::: "memory");
}
#define PIN() __builtin_amdgcn_sched_barrier(0)

#define STAMP(slot) do {} while (0)
#define STAMPC(slot) do {} while (0)

struct CW {
  f32x4 acc[4][8];
  u32x4 xs[NSET][4];
  s16x4 wf[WN][2];
  unsigned wa[4];
  unsigned ctl;
  unsigned xvo;
  unsigned freev;
  unsigned ready;
  unsigned need;
  unsigned xso;
  unsigned grp;
  __amdgpu_buffer_rsrc_t xr;
};

DI void wait_ready(CW& c) {
  asm volatile(
      "s_cmp_lt_u32 %0, %2\n\t"
      "s_cbranch_scc1 1f\n"
      "2:\n\t"
      ".subsection 1\n"
      "3:\n\t"
      "s_sleep 1\n"
      "1:\n\t"
      "ds_read_b32 v254, %1 offset:1024\n\t"
      "ds_read_b32 v255, %1 offset:2048\n\t"
      "s_waitcnt lgkmcnt(0)\n\t"
      "v_min_u32 v254, v254, v255\n\t"
      "s_nop 1\n\t"
      "v_readfirstlane_b32 %0, v254\n\t"
      "s_nop 3\n\t"
      "s_cmp_lt_u32 %0, %2\n\t"
      "s_cbranch_scc1 3b\n\t"
      "s_branch 2b\n\t"
      ".subsection 0"
      : "+s"(c.ready) : "v"(c.ctl), "s"(c.need) : "scc", "memory", "v254", "v255");
}

template <int SET, int M> DI void xload(CW& c) {
  c.xs[SET][M] = __builtin_amdgcn_raw_buffer_load_b128(c.xr, c.xvo + M * 1024, c.xso, XAUX);
}
template <int SL, int J, int T> DI void wread(CW& c) {
  c.wf[J % WN][T] = __builtin_amdgcn_ds_read_tr16_b64_v4i16((LAS s16x4*)((LAS char*)0 + c.wa[J & 3] + (SL * 8192 + (J >> 2) * 4096 + T * 1024)));
}
template <int S, int J, int M> DI void mma(CW& c) {
  const s16x8 w = __builtin_shufflevector(c.wf[J % WN][0], c.wf[J % WN][1], 0, 1, 2, 3, 4, 5, 6, 7);
  c.acc[M][J] = __builtin_amdgcn_mfma_f32_16x16x32_bf16(__builtin_bit_cast(bf16x8, w), __builtin_bit_cast(bf16x8, c.xs[S][M]), c.acc[M][J], 0, 0, 0);
}
DI void next_group(CW& c) {
  const bool wrap = (c.grp == NGRP - 1);
  const unsigned d = wrap ? (unsigned)(-(NGRP - 1) * NSET * 8192) : (unsigned)(NSET * 8192);
  c.grp = wrap ? 0u : c.grp + 1u;
#pragma unroll
  for (int i = 0; i < 4; ++i) c.wa[i] += d;
}
template <int S, int J> DI void frag(CW& c) {
  constexpr int JP = (J + WPF) & 7;
  constexpr bool NXT = (J + WPF) >= 8;
  constexpr int SL = NXT ? (S + 1) % NSET : S;
  constexpr int LS = (S + NSET - 1) % NSET;
  mma<S, J, 0>(c); PIN();
  wread<SL, JP, 0>(c); PIN();
  mma<S, J, 1>(c); PIN();
  wread<SL, JP, 1>(c); PIN();
  mma<S, J, 2>(c); PIN();
  if constexpr (J < 4) { xload<LS, J>(c); PIN(); }
  if constexpr (J == 7 - WPF) {
    c.freev += 1; lds_st(c.ctl, c.freev);
    if constexpr (S == NSET - 1) next_group(c);
    PIN();
  }
  if constexpr (J == 6) { c.need += 1; c.xso += 64 * 4096; asm volatile("" : "+s"(c.need), "+s"(c.xso)); PIN(); }
  if constexpr (J == 7) { wait_ready(c); PIN(); }
  mma<S, J, 3>(c); PIN();
}
template <int S> DI void step(CW& c) {
  frag<S, 0>(c); frag<S, 1>(c); frag<S, 2>(c); frag<S, 3>(c);
  frag<S, 4>(c); frag<S, 5>(c); frag<S, 6>(c); frag<S, 7>(c);
}
template <int S> DI void pro_x(CW& c) {
  c.need = S + 1; wait_ready(c); PIN();
  xload<S, 0>(c); xload<S, 1>(c); xload<S, 2>(c); xload<S, 3>(c); c.xso += 64 * 4096; PIN();
}

DI unsigned cvt_pk(float lo, float hi) { f32x2 v = {lo, hi}; return __builtin_bit_cast(unsigned, __builtin_convertvector(v, bf16x2)); }
DI float relu(float v) { return __builtin_elementwise_maximum(v, 0.0f); }

template <int S, int E> DI void steps(CW& c) { if constexpr (S < E) { step<S>(c); steps<S + 1, E>(c); } }
template <int S, int E> DI void pros(CW& c) { if constexpr (S < E) { pro_x<S>(c); pros<S + 1, E>(c); } }
template <int J, int E> DI void prow(CW& c) { if constexpr (J < E) { wread<0, J, 0>(c); wread<0, J, 1>(c); prow<J + 1, E>(c); } }

DI unsigned dpp_max(unsigned v, const int ctrl) {
  unsigned o;
  switch (ctrl) {
    case 1: o = __builtin_amdgcn_update_dpp(0u, v, 0x111, 0xf, 0xf, true); break;
    case 2: o = __builtin_amdgcn_update_dpp(0u, v, 0x112, 0xf, 0xf, true); break;
    case 4: o = __builtin_amdgcn_update_dpp(0u, v, 0x114, 0xf, 0xf, true); break;
    default: o = __builtin_amdgcn_update_dpp(0u, v, 0x118, 0xf, 0xf, true); break;
  }
  return o > v ? o : v;
}

DI bool wait_f1(const char* f, unsigned call, unsigned* tmo, unsigned& payload) {
  const unsigned long long t0 = rt();
  for (unsigned n = 1;; ++n) {
    unsigned long long v = 0;
    if (threadIdx.x % 64 == 0) v = peek64(f);
    if (rfl((unsigned)v) >= call) { payload = rfl((unsigned)(v >> 32)); return true; }
    if ((n & 63u) == 0u && (rfl(__hip_atomic_load(tmo, __ATOMIC_RELAXED, __HIP_MEMORY_SCOPE_AGENT)) == call || rt() - t0 > DEADLINE_TICKS)) {
      __hip_atomic_store(tmo, call, __ATOMIC_RELAXED, __HIP_MEMORY_SCOPE_AGENT); return false; }
    __builtin_amdgcn_s_sleep(4);
  }
}

struct Pull {
  __amdgpu_buffer_rsrc_t pr, xw; const char* f1; char* my; unsigned* tmo; unsigned* tick; unsigned call, t0, tfree, rot; int s, lane;
  unsigned lsum = 0, lmax = 0, lcnt = 0, rsum = 0, late = 0;
};
DI bool pull(Pull& q, unsigned j, unsigned& ticket) {
  constexpr int N = NB;
  const unsigned k = (JROT ? (j & ~(JROT - 1u)) | ((j + q.rot) & (JROT - 1u)) : j) * N, u = k >> 2, w0 = k & 3u;
  const unsigned off = j * N * TAU16 / 64;
  unsigned long long t0 = rt() - off + (j < (unsigned)PW ? XMARG1 : XMARG), old = 0, fv = 0;
  if (q.lane == 0)
    asm volatile("global_atomic_umax_x2 %0, %2, %3, off sc0\n\tglobal_atomic_add_x2 %1, %4, %5, off sc0 sc1\n\ts_waitcnt vmcnt(0)"
                 : "=&v"(old), "=&v"(fv) : "v"(q.tick + 32), "v"(t0), "v"(q.f1 + u * 128), "v"(0ull) : "memory");
  if (old > t0) t0 = old;
  const unsigned tgo = rfl((unsigned)t0) + off;
  unsigned bases = rfl((unsigned)(fv >> 32));
  if (rfl((unsigned)fv) < q.call && !wait_f1(q.f1 + u * 128, q.call, q.tmo, bases)) return false;
  bases >>= 8 * w0;
  sleep_until(tgo);
  u32x4 v[3 * N];
#pragma unroll
  for (int i = 0; i < 3 * N; ++i) {
    v[i] = __builtin_amdgcn_raw_buffer_load_b128(q.pr, q.lane * 16 + (i % 3) * 1024, (k + i / 3) * 3072, 17);
    if (RSPACE > 0 && i < 3 * N - 1) { PIN(); __builtin_amdgcn_s_sleep(RSPACE); PIN(); }
  }
  asm volatile("" :: "v"(v[3 * N - 1]));
  PIN();
  const unsigned d = ((u >> 1) * 64 + q.s * NRG + (k & 7u)) * 4096;
#pragma unroll
  for (int w = 0; w < N; ++w) {
    const unsigned base4 = ((bases >> (8 * w)) & 0xffu) * 0x01010101u;
#pragma unroll
    for (int m = 0; m < 4; ++m) {
      const unsigned l0 = v[3 * w + (m >> 1)][2 * (m & 1)], l1 = v[3 * w + (m >> 1)][2 * (m & 1) + 1], h = v[3 * w + 2][m];
      const unsigned h0 = (h & 0x87878787u) + base4, h1 = (__builtin_rotateright32(h, 4) & 0x87878787u) + base4;
      const u32x4 o = {__builtin_amdgcn_perm(h0, l0, 0x05010400u), __builtin_amdgcn_perm(h0, l0, 0x07030602u),
                       __builtin_amdgcn_perm(h1, l1, 0x05010400u), __builtin_amdgcn_perm(h1, l1, 0x07030602u)};
      asm volatile("buffer_store_dwordx4 %0, %1, %2, %3 offen offset:%4 " SBITS "\n\ts_nop 1"
                   :: "v"(o), "v"(h != 0u ? q.lane * 16u : 0x80000000u), "s"(q.xw), "s"(d + w * 4096), "i"(m * 1024) : "memory");
    }
  }
  unsigned t = 0;
  if (q.lane == 0) t = __hip_atomic_fetch_add(q.tick, 1u, __ATOMIC_RELAXED, __HIP_MEMORY_SCOPE_AGENT);
  wait_vm0();
  if (q.lane == 0) __hip_atomic_fetch_add((unsigned*)(q.my + F2_OFF) + (q.s * 2 + (u & 1)) * NSTEP + (u >> 1), (unsigned)N, __ATOMIC_RELAXED, __HIP_MEMORY_SCOPE_SYSTEM);
  ticket = rfl(t);
  return true;
}

DI void wfill(__amdgpu_buffer_rsrc_t wr, unsigned ldsaddr, unsigned vo, unsigned so) {
#pragma unroll
  for (int i = 0; i < 8; ++i)
    asm volatile("s_mov_b32 m0, %3\n\ts_nop 0\n\tbuffer_load_dwordx4 %0, %1, %2 offen lds"
                 :: "v"(vo), "s"(wr), "s"(so + (16 * ((i >> 1) & 1) + 4 * (i & 1)) * (NL * 2) + (i >> 2) * 128), "s"(ldsaddr + i * 1024) : "memory");
}

DI void wprefetch(__amdgpu_buffer_rsrc_t wr, unsigned lb, int lane, int chunk) {
#pragma unroll 4
  for (int i = 0; i < 65536 / (64 * WPRES); ++i)
    asm volatile("s_mov_b32 m0, %3\n\ts_nop 0\n\tbuffer_load_dword %0, %1, %2 offen lds"
                 :: "v"(lane * WPRES), "s"(wr), "s"(chunk * 65536 + i * 64 * WPRES), "s"(lb + L_MISC + 64) : "memory");
}

__global__ __launch_bounds__(BLOCK, 2) void ag_gemm(Params p) {
  extern __shared__ __attribute__((aligned(16))) char lds[];
  const int lane = threadIdx.x & 63;
  const int wave = rfl(threadIdx.x >> 6);
  const unsigned lb = (unsigned)(size_t)(LAS char*)lds;
  const int b = blockIdx.x, rank = p.rank & (NR - 1);
  char* const my = p.ws[rank];
  const int xcd = b & 7, bi = b >> 3;
  const int tm = (xcd >> 1) * 4 + (bi >> 3), tn = (xcd & 1) * 8 + (bi & 7);
  unsigned* const tmo = (unsigned*)(my + CT_OFF);
  const unsigned tstart = (unsigned)rt();
  if (wave == 0) STAMP(0);

  const int ppair = (bi >> 1) * 8 + xcd;
  const int pstep = 2 * (ppair >> 1) + (bi & 1), prg = (ppair & 1) * 4 + wave;
  u32x4 pv[4];
  if (wave < 4) {
    const char* xp = (const char*)p.x + ((size_t)(prg * 64 + (lane & 15)) * KD + pstep * 32 + (lane >> 4) * 8) * 2;
#pragma unroll
    for (int m = 0; m < 4; ++m) pv[m] = *(const u32x4*)(xp + (size_t)m * 16 * KD * 2);
  }
  const auto wr = __builtin_amdgcn_make_buffer_rsrc((void*)p.w, 0, KD * NL * 2, 0x00020000);
  const unsigned wvo = (8 * ((lane >> 3) & 1) + (lane >> 4)) * (NL * 2) + ((((lane >> 1) & 3) ^ (lane >> 4)) * 32) + (lane & 1) * 16;
  if (wave == 4) {
    __builtin_amdgcn_s_sleep(WPRED);
    for (int s = 0; s < W0; ++s) wfill(wr, lb + s * 8192, wvo, s * 32 * (NL * 2) + tn * 256);
  }
  if (WLEAD && wave == 7 && pstep + WLEAD >= NSTEP) { __builtin_amdgcn_s_sleep(WPRED); wprefetch(wr, lb, lane, (pstep + WLEAD - NSTEP) * 2 + (ppair & 1)); }
  if (wave == 6) {
    unsigned t = 0;
    if (lane == 0) t = __hip_atomic_fetch_add((unsigned*)(my + CC_OFF + b * 128), 1u, __ATOMIC_RELAXED, __HIP_MEMORY_SCOPE_AGENT);
    t = rfl(t) + 1;
    if (lane < 16) lds_st(lb + L_MISC + lane * 4, lane == 0 ? t : 0u);
  } else if (wave == 4 || wave == 5) {
#pragma unroll
    for (int i = 0; i < 6; ++i) lds_st(lb + L_FREE + (wave - 4) * 1536 + i * 256 + lane * 4, 0u);
  }
  asm volatile("s_waitcnt lgkmcnt(0)" ::: "memory");
  __builtin_amdgcn_s_barrier();
  asm volatile("" ::: "memory");
  const unsigned call = rfl(lds_ld(lb + L_MISC));
  const unsigned par = call & 1u;
  const size_t xo = XT_OFF + par * XT_BYTES + (size_t)(pstep * 64 + rank * NRG + prg) * 4096 + lane * 16;

  if (wave < 4) {
    {
      u16x2 mx = {0, 0};
#pragma unroll
      for (int m = 0; m < 4; ++m)
#pragma unroll
        for (int i = 0; i < 4; ++i) mx = __builtin_elementwise_max(mx, __builtin_bit_cast(u16x2, pv[m][i] & 0x7fff7fffu));
      unsigned v = mx[0] > mx[1] ? mx[0] : mx[1];
      v = dpp_max(v, 1); v = dpp_max(v, 2); v = dpp_max(v, 4); v = dpp_max(v, 8);
      const unsigned a0 = __builtin_amdgcn_readlane(v, 15), a1 = __builtin_amdgcn_readlane(v, 31), a2 = __builtin_amdgcn_readlane(v, 47), a3 = __builtin_amdgcn_readlane(v, 63);
      const unsigned top = max(max(a0, a1), max(a2, a3)) >> 8;
      const unsigned base = top > 7u ? top - 7u : 0u, base4 = base * 0x01010101u;
      u32x4 lo01, lo23, hh; unsigned esc = 0;
#pragma unroll
      for (int m = 0; m < 4; ++m) {
        const unsigned l0 = __builtin_amdgcn_perm(pv[m][1], pv[m][0], 0x06040200u), l1 = __builtin_amdgcn_perm(pv[m][3], pv[m][2], 0x06040200u);
        const unsigned h0 = __builtin_amdgcn_perm(pv[m][1], pv[m][0], 0x07050301u), h1 = __builtin_amdgcn_perm(pv[m][3], pv[m][2], 0x07050301u);
        const unsigned t0 = (h0 | 0x80808080u) - base4, t1 = (h1 | 0x80808080u) - base4;
        const unsigned c0 = (h0 & 0x80808080u) | (t0 & 0x07070707u), c1 = (h1 & 0x80808080u) | (t1 & 0x07070707u);
        unsigned h = c0 | __builtin_rotateleft32(c1, 4);
        const bool fits = ((t0 & 0xf8f8f8f8u) == 0x80808080u) && ((t1 & 0xf8f8f8f8u) == 0x80808080u) && h != 0u;
        if (!fits) { h = 0u; esc |= 1u << m; }
        hh[m] = h;
        if (m < 2) { lo01[2 * m] = l0; lo01[2 * m + 1] = l1; } else { lo23[2 * m - 4] = l0; lo23[2 * m - 3] = l1; }
      }
      if (wave == 0) STAMP(7);
      const int u = pstep * 2 + (prg >> 2);
      char* pk = my + PK_OFF + par * PK_BYTES + (size_t)u * UBYTES + wave * 3072 + lane * 16;
      if (PKGAP) sleep_until(tstart + 100 + pstep * PKGAP);
      st_sys(pk, lo01); st_sys(pk + 1024, lo23); st_sys(pk + 2048, hh);
      if (esc) {
        for (int i = 1; i < NR; ++i) {
          char* d = p.ws[(rank + i) & (NR - 1)] + xo;
#pragma unroll
          for (int m = 0; m < 4; ++m) if (esc >> m & 1u) st_sys(d + m * 1024, pv[m]);
        }
      }
      wait_vm0();
      unsigned old;
      asm volatile("s_mov_b64 exec, 1\n\tds_or_b32 %1, %2 offset:4\n\tds_add_rtn_u32 %0, %1, %3\n\ts_mov_b64 exec, -1\n\ts_waitcnt lgkmcnt(0)"
                   : "=&v"(old) : "v"(lb + L_MISC + 8), "v"(base << (8 * wave)), "v"(1u) : "memory");
      if (rfl(old) == 3u) {
        const unsigned bases = lds_ld(lb + L_MISC + 12);
        if (lane < NR && lane != rank)
          __hip_atomic_store((unsigned long long*)(p.ws[lane] + F1_OFF + par * F1_BYTES + (size_t)(rank * NUNIT + u) * 128),
                             ((unsigned long long)bases << 32) | call, __ATOMIC_RELAXED, __HIP_MEMORY_SCOPE_SYSTEM);
        STAMP(8);
      }
    }
    if (wave == 0) STAMP(1);
    CW c;
    const int g = lane >> 4, q = (lane >> 2) & 3, pp = lane & 3;
#pragma unroll
    for (int m = 0; m < 4; ++m)
#pragma unroll
      for (int j = 0; j < 8; ++j) c.acc[m][j] = (f32x4){0.f, 0.f, 0.f, 0.f};
#pragma unroll
    for (int i = 0; i < 4; ++i) c.wa[i] = lb + (g >> 1) * 2048 + q * 256 + (g & 1) * 128 + ((i ^ q) * 32) + pp * 8;
    c.ctl = lb + L_FREE + wave * 256 + lane * 4;
    c.freev = 0; c.ready = 0; c.grp = 0;
    c.xr = __builtin_amdgcn_make_buffer_rsrc((void*)(my + XT_OFF + par * XT_BYTES), 0, (unsigned)XT_BYTES, 0x00020000);
    c.xvo = lane * 16; c.xso = (tm * 4 + wave) * 4096;
    PIN();
    pros<0, NSET - 1>(c);
    prow<0, WPF>(c);
    c.need = NSET;
    wait_ready(c);
    PIN();
    if (wave == 0) { STAMP(2); STAMPC(16); }
#pragma unroll 1
    for (int it = 0; it < NSTEP / NSET; ++it) steps<0, NSET>(c);
    steps<0, NSTEP % NSET>(c);
    if (wave == 0) { STAMP(3); STAMPC(17); }
    unsigned verdict;
    while ((verdict = rfl(lds_ld(lb + L_MISC + 4))) == 0u) __builtin_amdgcn_s_sleep(1);
    int tz = threadIdx.x; asm volatile("" : "+v"(tz));
    const int l2 = tz & 63, g2 = l2 >> 4;
    char* op = (char*)p.out + ((size_t)(tm * 256 + wave * 64 + (l2 & 15)) * NL + tn * 128 + (g2 & 1) * 16 + (g2 >> 1) * 8) * 2;
#pragma unroll
    for (int m = 0; m < 4; ++m)
#pragma unroll
      for (int j = 0; j < 8; j += 2) {
        const f32x4 a = c.acc[m][j], d = c.acc[m][j + 1];
        const unsigned a0 = cvt_pk(relu(a[0]), relu(a[1])), a1 = cvt_pk(relu(a[2]), relu(a[3]));
        const unsigned d0 = cvt_pk(relu(d[0]), relu(d[1])), d1 = cvt_pk(relu(d[2]), relu(d[3]));
        const auto s0 = __builtin_amdgcn_permlane16_swap(a0, d0, false, false), s1 = __builtin_amdgcn_permlane16_swap(a1, d1, false, false);
        u32x4 o = {s0[0], s1[0], s0[1], s1[1]};
        if (verdict != 1u) o = (u32x4){0x7fc07fc0u, 0x7fc07fc0u, 0x7fc07fc0u, 0x7fc07fc0u};
        *(u32x4*)(op + (size_t)m * 16 * NL * 2 + j * 32) = o;
      }
  } else if (wave == 4) {
    const unsigned pubaddr = lb + L_WFULL + (lane & 3) * 256 + (lane >> 2) * 4;
    wait_vm0();
    int pub = W0, slot = W0; unsigned freec = 0;
    lds_st(pubaddr, pub + NSET - 2);
    for (int s = W0; s < NSTEP; ++s) {
      if (s >= RING && freec < (unsigned)(s - RING + 1)) {
        if (pub < s) { wait_vm0(); pub = s; lds_st(pubaddr, pub + NSET - 2); }
        for (;;) {
          const unsigned f = lds_ld(lb + L_FREE + (lane & 3) * 256);
          freec = min(min(__builtin_amdgcn_readlane(f, 0), __builtin_amdgcn_readlane(f, 1)), min(__builtin_amdgcn_readlane(f, 2), __builtin_amdgcn_readlane(f, 3)));
          if (freec >= (unsigned)(s - RING + 1)) break;
          __builtin_amdgcn_s_sleep(2);
        }
      }
      if (s < RING) {
        const unsigned tgo = tstart + WT0 + (s - W0) * WGAP;
        if ((int)(tgo - (unsigned)rt()) > 0) { if (pub < s) { wait_vm0(); pub = s; lds_st(pubaddr, pub + NSET - 2); } sleep_until(tgo); }
      }
      wfill(wr, lb + slot * 8192, wvo, s * 32 * (NL * 2) + tn * 256);
      slot = slot == RING - 1 ? 0 : slot + 1;
      if (s + 1 - pub > INFL) { asm volatile("s_waitcnt vmcnt(%0)" :: "i"(8 * INFL) : "memory"); pub = s + 1 - INFL; lds_st(pubaddr, pub + NSET - 2); }
    }
    wait_vm0(); lds_st(pubaddr, INF);
  } else if (wave == 6) {
    const unsigned xaddr = lb + L_XRDY + (lane & 3) * 256 + (lane >> 2) * 4;
    bool dead = false;
    {
      const unsigned* f2 = (const unsigned*)(my + F2_OFF) + tm * NSTEP;
      unsigned base = 0;
      const unsigned long long t0 = rt();
      for (unsigned n = 1;; ++n) {
        const unsigned idx = base + lane;
        unsigned v = 4u * call;
        if (idx < (unsigned)NSTEP && lane < POLLW) v = peek(f2 + idx);
        const unsigned nok = ~(unsigned)__ballot(v >= 4u * call);
        base += nok ? (unsigned)__builtin_ctz(nok) : (unsigned)POLLW;
        if (base >= (unsigned)NSTEP) {
          break; }
        lds_st(xaddr, base);
        if ((n & 63u) == 0u) {
          if (rfl(__hip_atomic_load(tmo, __ATOMIC_RELAXED, __HIP_MEMORY_SCOPE_AGENT)) == call || rt() - t0 > DEADLINE_TICKS) {
            __hip_atomic_store(tmo, call, __ATOMIC_RELAXED, __HIP_MEMORY_SCOPE_AGENT); dead = true; break; }
        }
        if (base >= (unsigned)(NSTEP - POLLEND)) __builtin_amdgcn_s_sleep(POLLFAST); else __builtin_amdgcn_s_sleep(POLLSLEEP);
      }
    }
    lds_st(xaddr, INF);
    STAMP(6);
    if (rfl(__hip_atomic_load(tmo, __ATOMIC_RELAXED, __HIP_MEMORY_SCOPE_AGENT)) == call) dead = true;
    lds_st(lb + L_MISC + 4, dead ? 2u : 1u);
  } else {
    if (wave == 7) {
      const int ostep = (pstep + NSTEP / 2) & (NSTEP - 1), ohalf = ppair & 1;
      sleep_until(tstart + OWN0 + ostep * OWNGAP);
      if (WLEAD && ostep + WLEAD < NSTEP) wprefetch(wr, lb, lane, (ostep + WLEAD) * 2 + ohalf);
      const char* xp = (const char*)p.x + ((size_t)(ohalf * 256 + (lane & 15)) * KD + ostep * 32 + (lane >> 4) * 8) * 2;
      u32x4 ov[16];
#pragma unroll
      for (int m = 0; m < 16; ++m) ov[m] = *(const u32x4*)(xp + (size_t)m * 16 * KD * 2);
      char* d = my + XT_OFF + par * XT_BYTES + (size_t)(ostep * 64 + rank * NRG + ohalf * 4) * 4096 + lane * 16;
#pragma unroll
      for (int m = 0; m < 16; ++m) st_sys(d + m * 1024, ov[m]);
      wait_vm0();
      if (lane == 0) __hip_atomic_fetch_add((unsigned*)(my + F2_OFF) + (rank * 2 + ohalf) * NSTEP + ostep, 4u, __ATOMIC_RELAXED, __HIP_MEMORY_SCOPE_SYSTEM);
    }
    if (wave == 5 && b < PW * (NR - 1)) {
      Pull q;
      const int qi = b % (NR - 1);
      q.s = (rank + 1 + qi) & (NR - 1);
      q.pr = __builtin_amdgcn_make_buffer_rsrc((void*)(p.ws[q.s] + PK_OFF + par * PK_BYTES), 0, (unsigned)PK_BYTES, 0x00020000);
      q.xw = __builtin_amdgcn_make_buffer_rsrc((void*)(my + XT_OFF + par * XT_BYTES), 0, (unsigned)XT_BYTES, 0x00020000);
      q.f1 = my + F1_OFF + par * F1_BYTES + (size_t)q.s * NUNIT * 128;
      q.my = my; q.tmo = tmo; q.call = call; q.lane = lane; q.tfree = tstart; q.rot = qi;
      q.tick = (unsigned*)(my + CT_OFF + (2 + 2 * qi) * 128);
      constexpr unsigned NJOB = 4 * NUNIT / NB;
      const unsigned tbase = (call - 1u) * (NJOB + PW);
      unsigned bases;
      while (rfl(lds_ld(lb + L_MISC + 8)) < 4u) __builtin_amdgcn_s_sleep(2);
      unsigned t = 0;
      if (lane == 0) t = __hip_atomic_fetch_add(q.tick, 1u, __ATOMIC_RELAXED, __HIP_MEMORY_SCOPE_AGENT);
      t = rfl(t);
      bool ok = wait_f1(q.f1, call, tmo, bases);
      while (ok && t - tbase < NJOB) ok = pull(q, t - tbase, t);
    }
  }
}

}

extern "C" void kernel_launch(int rank, int nranks, ncclComm_t comm, hipStream_t stream,
                              void* const* d_in, const int* in_sizes, int n_in,
                              void* d_out, int out_size, const int* in_dtypes, int out_dtype,
                              void* const* d_out_all, void* const* d_ws_all, size_t ws_bytes) {
  if (nranks != NR || (unsigned)rank >= (unsigned)NR || n_in != 2 || in_sizes[0] != ML * KD || in_sizes[1] != KD * NL ||
      out_size != MG * NL || ws_bytes < WS_NEED) { fprintf(stderr, "unexpected problem shape\n"); return; }
  static bool attr[NR] = {};
  if (!attr[rank]) { HIP_CHECK(hipFuncSetAttribute((const void*)ag_gemm, hipFuncAttributeMaxDynamicSharedMemorySize, LDS_BYTES)); attr[rank] = true; }
  Params p;
  p.x = d_in[0]; p.w = d_in[1]; p.out = d_out; p.rank = rank; p.pad = 0;
  for (int i = 0; i < NR; ++i) p.ws[i] = (char*)d_ws_all[i];
  ag_gemm<<<GRID, BLOCK, LDS_BYTES, stream>>>(p);
  HIP_CHECK(hipGetLastError());
}
```

```cpp
#include <hip/hip_runtime.h>
#include <rccl/rccl.h>
#include <cstdio>

#define HIP_CHECK(expr) do { hipError_t _e = (expr); if (_e != hipSuccess) { \
    fprintf(stderr, "HIP error %s at %s:%d\n", hipGetErrorString(_e), __FILE__, __LINE__); \
    return; } } while (0)

namespace {

#define LAS __attribute__((address_space(3)))
#define DI __device__ __forceinline__
typedef short    s16x4  __attribute__((ext_vector_type(4)));
typedef short    s16x8  __attribute__((ext_vector_type(8)));
typedef __bf16   bf16x8 __attribute__((ext_vector_type(8)));
typedef __bf16   bf16x2 __attribute__((ext_vector_type(2)));
typedef float    f32x2  __attribute__((ext_vector_type(2)));
typedef float    f32x4  __attribute__((ext_vector_type(4)));
typedef unsigned u32x4  __attribute__((ext_vector_type(4)));
typedef unsigned short u16x2 __attribute__((ext_vector_type(2)));

constexpr int NR = 8;
constexpr int KD = 4096, NL = 2048, ML = 512, MG = NR * ML;
constexpr int NSTEP = KD / 32;
constexpr int NRG = ML / 64;
constexpr int NUNIT = NSTEP * 2;
constexpr int UBYTES = 4 * 3072;
constexpr int GRID = 256, BLOCK = 512;

#define NSET 4
#define NGRP 4
constexpr int RING = NSET * NGRP;
#define WPF 3
constexpr int WN = 4;
static_assert(WPF < WN, "");
constexpr int INFL = 6;
constexpr int PW = 36;
#define TAU16 324
#define POLLSLEEP 8
#define POLLFAST 4
#define POLLEND 6
#define RSPACE 2
#define SBITS "sc0 sc1"
#define NB 2
#define POLLW 16
#define OWN0 350
#define OWNTAU16 240
#define WGAP 25
#define WPRED 4
#define PULLWAIT 1
#define XMARG 160
#define XMARG1 30
#define W0 4
#define WT0 500
#define OBITS "nt"
#define PFS 64
#define PF0 16
#define XAUX 17

constexpr int L_FREE = RING * 8192, L_XRDY = L_FREE + 1024, L_WFULL = L_FREE + 2048, L_MISC = L_FREE + 3072;
constexpr int L_EPI = L_MISC + 1024;
constexpr int LDS_BYTES = L_EPI + 4 * 4096;
static_assert(LDS_BYTES <= 160 * 1024 && LDS_BYTES > 80 * 1024, "");
constexpr unsigned INF = 0x7fffffffu;

constexpr size_t XT_BYTES = (size_t)MG * KD * 2;
constexpr size_t XT_OFF = 0;
constexpr unsigned EL_REL = NUNIT * UBYTES;
constexpr unsigned HB_REL = EL_REL + NUNIT * 4 * 64;
constexpr size_t PK_BYTES = HB_REL + NUNIT * 4 * 2048;
constexpr size_t PK_OFF = XT_OFF + 2 * XT_BYTES;
constexpr size_t F1_BYTES = (size_t)NR * NUNIT * 128;
constexpr size_t F1_OFF = PK_OFF + 2 * PK_BYTES;
constexpr size_t F2_OFF = F1_OFF + 2 * F1_BYTES;
constexpr size_t F2_BYTES = (size_t)NR * NUNIT * 4;
constexpr size_t CT_OFF = F2_OFF + F2_BYTES;
constexpr size_t CC_OFF = CT_OFF + 4096;
constexpr size_t DBG_OFF = CC_OFF + GRID * 128;
constexpr size_t DBG_BYTES = 1 << 20;
constexpr size_t WS_NEED = DBG_OFF + DBG_BYTES;
constexpr unsigned long long DEADLINE_TICKS = 20000000ull;

struct Params { const void* x; const void* w; void* out; char* ws[NR]; int rank; int pad; };

DI unsigned rfl(unsigned v) { return __builtin_amdgcn_readfirstlane(v); }
DI unsigned long long rt() { return __builtin_amdgcn_s_memrealtime(); }
DI void st_sys(void* p, u32x4 v) { asm volatile("global_store_dwordx4 %0, %1, off sc0 sc1\n\ts_nop 1" :: "v"(p), "v"(v) : "memory"); }
DI void wait_vm0() { asm volatile("s_waitcnt vmcnt(0)" ::: "memory"); }
DI void lds_st(unsigned addr, unsigned v) { asm volatile("ds_write_b32 %0, %1" :: "v"(addr), "v"(v) : "memory"); }
DI unsigned lds_ld(unsigned addr) { unsigned v; asm volatile("ds_read_b32 %0, %1\n\ts_waitcnt lgkmcnt(0)" : "=&v"(v) : "v"(addr) : "memory"); return v; }
DI unsigned peek(const unsigned* p) { unsigned v; asm volatile("global_atomic_add %0, %1, %2, off sc0 sc1\n\ts_waitcnt vmcnt(0)" : "=&v"(v) : "v"(p), "v"(0u) : "memory"); return v; }
DI unsigned long long peek64(const void* p) { unsigned long long v; asm volatile("global_atomic_add_x2 %0, %1, %2, off sc0 sc1\n\ts_waitcnt vmcnt(0)" : "=&v"(v) : "v"(p), "v"(0ull) : "memory"); return v; }
DI void sleep_until(unsigned t) {
  for (int d; (d = (int)(t - (unsigned)rt())) > 0;) { if (d > 200) __builtin_amdgcn_s_sleep(48); else if (d > 40) __builtin_amdgcn_s_sleep(8); else __builtin_amdgcn_s_sleep(1); }
  asm volatile("" ::: "memory");
}
#define PIN() __builtin_amdgcn_sched_barrier(0)

#define STAMP(slot) do {} while (0)
#define STAMPC(slot) do {} while (0)

struct CW {
  f32x4 acc[4][8];
  u32x4 xs[NSET][4];
  s16x4 wf[WN][2];
  unsigned wa[4];
  unsigned ctl;
  unsigned xvo;
  unsigned freev;
  unsigned ready;
  unsigned need;
  unsigned xso;
  unsigned grp;
  __amdgpu_buffer_rsrc_t xr;
};

DI void wait_ready(CW& c) {
  asm volatile(
      "s_cmp_lt_u32 %0, %2\n\t"
      "s_cbranch_scc1 1f\n"
      "2:\n\t"
      ".subsection 1\n"
      "3:\n\t"
      "s_sleep 1\n"
      "1:\n\t"
      "ds_read_b32 v254, %1 offset:1024\n\t"
      "ds_read_b32 v255, %1 offset:2048\n\t"
      "s_waitcnt lgkmcnt(0)\n\t"
      "v_min_u32 v254, v254, v255\n\t"
      "s_nop 1\n\t"
      "v_readfirstlane_b32 %0, v254\n\t"
      "s_nop 3\n\t"
      "s_cmp_lt_u32 %0, %2\n\t"
      "s_cbranch_scc1 3b\n\t"
      "s_branch 2b\n\t"
      ".subsection 0"
      : "+s"(c.ready) : "v"(c.ctl), "s"(c.need) : "scc", "memory", "v254", "v255");
}

template <int SET, int M> DI void xload(CW& c) {
  c.xs[SET][M] = __builtin_amdgcn_raw_buffer_load_b128(c.xr, c.xvo + M * 1024, c.xso, XAUX);
}
template <int SL, int J, int T> DI void wread(CW& c) {
  c.wf[J % WN][T] = __builtin_amdgcn_ds_read_tr16_b64_v4i16((LAS s16x4*)((LAS char*)0 + c.wa[J & 3] + (SL * 8192 + (J >> 2) * 4096 + T * 1024)));
}
template <int S, int J, int M> DI void mma(CW& c) {
  const s16x8 w = __builtin_shufflevector(c.wf[J % WN][0], c.wf[J % WN][1], 0, 1, 2, 3, 4, 5, 6, 7);
  c.acc[M][J] = __builtin_amdgcn_mfma_f32_16x16x32_bf16(__builtin_bit_cast(bf16x8, w), __builtin_bit_cast(bf16x8, c.xs[S][M]), c.acc[M][J], 0, 0, 0);
}
DI void next_group(CW& c) {
  const bool wrap = (c.grp == NGRP - 1);
  const unsigned d = wrap ? (unsigned)(-(NGRP - 1) * NSET * 8192) : (unsigned)(NSET * 8192);
  c.grp = wrap ? 0u : c.grp + 1u;
#pragma unroll
  for (int i = 0; i < 4; ++i) c.wa[i] += d;
}
template <int S, int J> DI void frag(CW& c) {
  constexpr int JP = (J + WPF) & 7;
  constexpr bool NXT = (J + WPF) >= 8;
  constexpr int SL = NXT ? (S + 1) % NSET : S;
  constexpr int LS = (S + NSET - 1) % NSET;
  mma<S, J, 0>(c); PIN();
  wread<SL, JP, 0>(c); PIN();
  mma<S, J, 1>(c); PIN();
  wread<SL, JP, 1>(c); PIN();
  mma<S, J, 2>(c); PIN();
  if constexpr (J < 4) { xload<LS, J>(c); PIN(); }
  if constexpr (J == 7 - WPF) {
    c.freev += 1; lds_st(c.ctl, c.freev);
    if constexpr (S == NSET - 1) next_group(c);
    PIN();
  }
  if constexpr (J == 6) { c.need += 1; c.xso += 64 * 4096; asm volatile("" : "+s"(c.need), "+s"(c.xso)); PIN(); }
  if constexpr (J == 7) { wait_ready(c); PIN(); }
  mma<S, J, 3>(c); PIN();
}
template <int S> DI void step(CW& c) {
  frag<S, 0>(c); frag<S, 1>(c); frag<S, 2>(c); frag<S, 3>(c);
  frag<S, 4>(c); frag<S, 5>(c); frag<S, 6>(c); frag<S, 7>(c);
}
template <int S> DI void pro_x(CW& c) {
  c.need = S + 1; wait_ready(c); PIN();
  xload<S, 0>(c); xload<S, 1>(c); xload<S, 2>(c); xload<S, 3>(c); c.xso += 64 * 4096; PIN();
}

DI unsigned cvt_pk(float lo, float hi) { f32x2 v = {lo, hi}; return __builtin_bit_cast(unsigned, __builtin_convertvector(v, bf16x2)); }
DI float relu(float v) { return __builtin_elementwise_maximum(v, 0.0f); }

template <int S, int E> DI void steps(CW& c) { if constexpr (S < E) { step<S>(c); steps<S + 1, E>(c); } }
template <int S, int E> DI void pros(CW& c) { if constexpr (S < E) { pro_x<S>(c); pros<S + 1, E>(c); } }
template <int J, int E> DI void prow(CW& c) { if constexpr (J < E) { wread<0, J, 0>(c); wread<0, J, 1>(c); prow<J + 1, E>(c); } }

DI unsigned dpp_max(unsigned v, const int ctrl) {
  unsigned o;
  switch (ctrl) {
    case 1: o = __builtin_amdgcn_update_dpp(0u, v, 0x111, 0xf, 0xf, true); break;
    case 2: o = __builtin_amdgcn_update_dpp(0u, v, 0x112, 0xf, 0xf, true); break;
    case 4: o = __builtin_amdgcn_update_dpp(0u, v, 0x114, 0xf, 0xf, true); break;
    default: o = __builtin_amdgcn_update_dpp(0u, v, 0x118, 0xf, 0xf, true); break;
  }
  return o > v ? o : v;
}

DI bool wait_f1(const char* f, unsigned call, unsigned* tmo, unsigned& payload) {
  const unsigned long long t0 = rt();
  for (unsigned n = 1;; ++n) {
    unsigned long long v = 0;
    if (threadIdx.x % 64 == 0) v = peek64(f);
    if (rfl((unsigned)v) >= call) { payload = rfl((unsigned)(v >> 32)); return true; }
    if ((n & 63u) == 0u && (rfl(__hip_atomic_load(tmo, __ATOMIC_RELAXED, __HIP_MEMORY_SCOPE_AGENT)) == call || rt() - t0 > DEADLINE_TICKS)) {
      __hip_atomic_store(tmo, call, __ATOMIC_RELAXED, __HIP_MEMORY_SCOPE_AGENT); return false; }
    __builtin_amdgcn_s_sleep(4);
  }
}

struct Pull {
  __amdgpu_buffer_rsrc_t pr, xw; const char* f1; char* my; unsigned* tmo; unsigned* tick; unsigned call, tbase, tau; int s, lane;
  unsigned j, tgo, fcall, info;
  unsigned t0 = 0, tlast = 0, lsum = 0, lmax = 0, lcnt = 0, rsum = 0, late = 0;
};
typedef unsigned u32x2 __attribute__((ext_vector_type(2)));
constexpr unsigned NJOB = 4 * NUNIT / NB;
DI bool pull(Pull& q) {
  constexpr int N = NB;
  const unsigned k = q.j * N, u = k >> 2, w0 = k & 3u;
  unsigned info = q.info;
  if (q.fcall < q.call && !wait_f1(q.f1 + u * 128, q.call, q.tmo, info)) return false;
  info >>= 8 * w0;
  sleep_until(q.tgo);
  unsigned tk;
  asm volatile("s_mov_b64 exec, 1\n\tglobal_atomic_add %0, %1, %2, off sc0\n\ts_mov_b64 exec, -1" : "=&v"(tk) : "v"(q.tick), "v"(1u) : "memory");
  PIN();
  u32x4 v[3 * N], hb[N]; u32x2 el[N];
#pragma unroll
  for (int w = 0; w < N; ++w) {
    const bool raw = ((info >> (8 * w)) & 0x7fu) == 0x7fu, list = (info >> (8 * w + 7)) & 1u;
    el[w] = __builtin_amdgcn_raw_buffer_load_b64(q.pr, list && q.lane < 8 ? q.lane * 8u : 0x80000000u, EL_REL + (k + w) * 64, 17);
    hb[w] = __builtin_amdgcn_raw_buffer_load_b128(q.pr, raw ? q.lane * 16u + 1024u : 0x80000000u, HB_REL + (k + w) * 2048, 17);
#pragma unroll
    for (int i = 0; i < 3; ++i) {
      v[3 * w + i] = __builtin_amdgcn_raw_buffer_load_b128(q.pr, q.lane * 16, i == 2 && raw ? HB_REL + (k + w) * 2048 : (k + w) * 3072 + i * 1024, 17);
      if (RSPACE > 0 && 3 * w + i < 3 * N - 1) { PIN(); __builtin_amdgcn_s_sleep(RSPACE); PIN(); }
    }
  }
  asm volatile("s_waitcnt vmcnt(0)" : "+v"(tk) : "v"(v[3 * N - 1]) : "memory");
  PIN();
  q.j = rfl(tk) - q.tbase;
  const unsigned jn = q.j < NJOB ? q.j : NJOB - 1, offn = jn * N * q.tau / 64;
  unsigned long long t0 = q.j < NJOB ? rt() - offn + XMARG : 0ull, old, fv;
  asm volatile("s_mov_b64 exec, 1\n\tglobal_atomic_umax_x2 %0, %2, %3, off sc0\n\tglobal_atomic_add_x2 %1, %4, %5, off sc0 sc1\n\ts_mov_b64 exec, -1"
               : "=&v"(old), "=&v"(fv) : "v"(q.tick + 32), "v"(t0), "v"(q.f1 + (jn * N >> 2) * 128), "v"(0ull) : "memory");
  const unsigned d = ((u >> 1) * 64 + q.s * NRG + (k & 7u)) * 4096;
#pragma unroll
  for (int w = 0; w < N; ++w) {
    const unsigned bi = (info >> (8 * w)) & 0xffu, base4 = (bi & 0x7fu) * 0x01010101u;
    unsigned h0[4], h1[4];
#pragma unroll
    for (int m = 0; m < 4; ++m) {
      const unsigned h = v[3 * w + 2][m];
      h0[m] = (h & 0x87878787u) + base4; h1[m] = (__builtin_rotateright32(h, 4) & 0x87878787u) + base4;
    }
    if (bi >= 0x7fu) {
      if (bi == 0x7fu) {
#pragma unroll
        for (int m = 0; m < 4; ++m) { const u32x4 t = m < 2 ? v[3 * w + 2] : hb[w]; h0[m] = t[2 * (m & 1)]; h1[m] = t[2 * (m & 1) + 1]; }
      } else {
        unsigned cnt = 0;
#pragma unroll
        for (int m = 0; m < 4; ++m) {
          const bool e = v[3 * w + 2][m] == 0u;
          const unsigned long long bal = __ballot(e);
          if (bal) {
            const unsigned idx = (cnt + __builtin_amdgcn_mbcnt_hi((unsigned)(bal >> 32), __builtin_amdgcn_mbcnt_lo((unsigned)bal, 0u))) * 4u;
            const unsigned a = __builtin_amdgcn_ds_bpermute(idx, el[w][0]), c = __builtin_amdgcn_ds_bpermute(idx, el[w][1]);
            if (e) { h0[m] = a; h1[m] = c; }
            cnt += __builtin_popcountll(bal);
          }
        }
      }
    }
#pragma unroll
    for (int m = 0; m < 4; ++m) {
      const unsigned l0 = v[3 * w + (m >> 1)][2 * (m & 1)], l1 = v[3 * w + (m >> 1)][2 * (m & 1) + 1];
      const u32x4 o = {__builtin_amdgcn_perm(h0[m], l0, 0x05010400u), __builtin_amdgcn_perm(h0[m], l0, 0x07030602u),
                       __builtin_amdgcn_perm(h1[m], l1, 0x05010400u), __builtin_amdgcn_perm(h1[m], l1, 0x07030602u)};
      asm volatile("buffer_store_dwordx4 %0, %1, %2, %3 offen offset:%4 " SBITS "\n\ts_nop 1"
                   :: "v"(o), "v"(q.lane * 16u), "s"(q.xw), "s"(d + w * 4096), "i"(m * 1024) : "memory");
    }
  }
  asm volatile("s_waitcnt vmcnt(0)" : "+v"(old), "+v"(fv) :: "memory");
  asm volatile("s_mov_b64 exec, 1\n\tglobal_atomic_add %0, %1, off sc1\n\ts_mov_b64 exec, -1"
               :: "v"((unsigned*)(q.my + F2_OFF) + (q.s * 2 + (u & 1)) * NSTEP + (u >> 1)), "v"((unsigned)N) : "memory");
  if (old > t0) t0 = old;
  q.tgo = rfl((unsigned)t0) + offn; q.fcall = rfl((unsigned)fv); q.info = rfl((unsigned)(fv >> 32));
  return true;
}

DI void wfill(__amdgpu_buffer_rsrc_t wr, unsigned ldsaddr, unsigned vo, unsigned so) {
#pragma unroll
  for (int i = 0; i < 8; ++i)
    asm volatile("s_mov_b32 m0, %3\n\ts_nop 0\n\tbuffer_load_dwordx4 %0, %1, %2 offen lds"
                 :: "v"(vo), "s"(wr), "s"(so + (16 * ((i >> 1) & 1) + 4 * (i & 1)) * (NL * 2) + (i >> 2) * 128), "s"(ldsaddr + i * 1024) : "memory");
}

DI void wtouch(const char* p) {
  unsigned t;
#define PF4(o) "s_load_dword %0, %1, " #o "+0\n\ts_load_dword %0, %1, " #o "+64\n\ts_load_dword %0, %1, " #o "+128\n\ts_load_dword %0, %1, " #o "+192\n\t" \
               "s_load_dword %0, %1, " #o "+256\n\ts_load_dword %0, %1, " #o "+320\n\ts_load_dword %0, %1, " #o "+384\n\ts_load_dword %0, %1, " #o "+448\n\t"
  asm volatile(PF4(0) PF4(131072) PF4(262144) PF4(393216) "s_waitcnt lgkmcnt(0)" : "=&s"(t) : "s"(p) : "memory");
}

__global__ __launch_bounds__(BLOCK, 2) void ag_gemm(Params p) {
  extern __shared__ __attribute__((aligned(16))) char lds[];
  const int lane = threadIdx.x & 63;
  const int wave = rfl(threadIdx.x >> 6);
  const unsigned lb = (unsigned)(size_t)(LAS char*)lds;
  const int b = blockIdx.x, rank = p.rank & (NR - 1);
  char* const my = p.ws[rank];
  const int xcd = b & 7, bi = b >> 3;
  const int tm = (xcd >> 1) * 4 + (bi >> 3), tn = (xcd & 1) * 8 + (bi & 7);
  unsigned* const tmo = (unsigned*)(my + CT_OFF);
  const unsigned tstart = (unsigned)rt();
  if (wave == 0) STAMP(0);

  const int ppair = (bi >> 1) * 8 + xcd;
  const int pstep = 2 * (ppair >> 1) + (bi & 1), prg = (ppair & 1) * 4 + wave;
  u32x4 pv[4];
  const char* const wp = (const char*)p.w + b * 512;
  if (wave < 4) {
    const char* xp = (const char*)p.x + ((size_t)(prg * 64 + (lane & 15)) * KD + pstep * 32 + (lane >> 4) * 8) * 2;
#pragma unroll
    for (int m = 0; m < 4; ++m) pv[m] = *(const u32x4*)(xp + (size_t)m * 16 * KD * 2);
  }
  const auto wr = __builtin_amdgcn_make_buffer_rsrc((void*)p.w, 0, KD * NL * 2, 0x00020000);
  const unsigned wvo = (8 * ((lane >> 3) & 1) + (lane >> 4)) * (NL * 2) + ((((lane >> 1) & 3) ^ (lane >> 4)) * 32) + (lane & 1) * 16;
  if (wave == 4) {
    __builtin_amdgcn_s_sleep(WPRED);
    for (int s = 0; s < W0; ++s) wfill(wr, lb + s * 8192, wvo, s * 32 * (NL * 2) + tn * 256);
  }
  if (wave == 6) {
    unsigned t = 0;
    if (lane == 0) t = __hip_atomic_fetch_add((unsigned*)(my + CC_OFF + b * 128), 1u, __ATOMIC_RELAXED, __HIP_MEMORY_SCOPE_AGENT);
    t = rfl(t) + 1;
    if (lane < 16) lds_st(lb + L_MISC + lane * 4, lane == 0 ? t : 0u);
  } else if (wave == 4 || wave == 5) {
#pragma unroll
    for (int i = 0; i < 6; ++i) lds_st(lb + L_FREE + (wave - 4) * 1536 + i * 256 + lane * 4, 0u);
  }
  asm volatile("s_waitcnt lgkmcnt(0)" ::: "memory");
  __builtin_amdgcn_s_barrier();
  asm volatile("" ::: "memory");
  const unsigned call = rfl(lds_ld(lb + L_MISC));
  const unsigned par = call & 1u;

  if (wave < 4) {
    {
      u16x2 mx = {0, 0};
#pragma unroll
      for (int m = 0; m < 4; ++m)
#pragma unroll
        for (int i = 0; i < 4; ++i) mx = __builtin_elementwise_max(mx, __builtin_bit_cast(u16x2, pv[m][i] & 0x7fff7fffu));
      unsigned v = mx[0] > mx[1] ? mx[0] : mx[1];
      v = dpp_max(v, 1); v = dpp_max(v, 2); v = dpp_max(v, 4); v = dpp_max(v, 8);
      const unsigned a0 = __builtin_amdgcn_readlane(v, 15), a1 = __builtin_amdgcn_readlane(v, 31), a2 = __builtin_amdgcn_readlane(v, 47), a3 = __builtin_amdgcn_readlane(v, 63);
      const unsigned top = max(max(a0, a1), max(a2, a3)) >> 8;
      const unsigned base = top > 7u ? top - 7u : 0u, base4 = base * 0x01010101u;
      u32x4 lo[2], hi[2], hh; unsigned cnt = 0, idx[4];
#pragma unroll
      for (int m = 0; m < 4; ++m) {
        const unsigned l0 = __builtin_amdgcn_perm(pv[m][1], pv[m][0], 0x06040200u), l1 = __builtin_amdgcn_perm(pv[m][3], pv[m][2], 0x06040200u);
        const unsigned h0 = __builtin_amdgcn_perm(pv[m][1], pv[m][0], 0x07050301u), h1 = __builtin_amdgcn_perm(pv[m][3], pv[m][2], 0x07050301u);
        const unsigned t0 = (h0 | 0x80808080u) - base4, t1 = (h1 | 0x80808080u) - base4;
        const unsigned c0 = (h0 & 0x80808080u) | (t0 & 0x07070707u), c1 = (h1 & 0x80808080u) | (t1 & 0x07070707u);
        unsigned h = c0 | __builtin_rotateleft32(c1, 4);
        if ((t0 & t1 & 0x80808080u) != 0x80808080u) h = 0u;
        const unsigned long long bal = __ballot(h == 0u);
        idx[m] = cnt + __builtin_amdgcn_mbcnt_hi((unsigned)(bal >> 32), __builtin_amdgcn_mbcnt_lo((unsigned)bal, 0u));
        cnt += __builtin_popcountll(bal);
        hh[m] = h;
        lo[m >> 1][2 * (m & 1)] = l0; lo[m >> 1][2 * (m & 1) + 1] = l1;
        hi[m >> 1][2 * (m & 1)] = h0; hi[m >> 1][2 * (m & 1) + 1] = h1;
      }
      if (wave == 0) STAMP(7);
      const int u = pstep * 2 + (prg >> 2), blk = u * 4 + wave;
      char* const pk = my + PK_OFF + par * PK_BYTES;
      st_sys(pk + blk * 3072 + lane * 16, lo[0]); st_sys(pk + blk * 3072 + 1024 + lane * 16, lo[1]);
      unsigned info = base;
      if (cnt > 8u) {
        info = 0x7fu;
        st_sys(pk + HB_REL + blk * 2048 + lane * 16, hi[0]); st_sys(pk + HB_REL + blk * 2048 + 1024 + lane * 16, hi[1]);
      } else {
        st_sys(pk + blk * 3072 + 2048 + lane * 16, hh);
        if (cnt) {
          info |= 0x80u;
#pragma unroll
          for (int m = 0; m < 4; ++m)
            if (hh[m] == 0u)
              __hip_atomic_store((unsigned long long*)(pk + EL_REL + blk * 64 + idx[m] * 8),
                                 ((unsigned long long)hi[m >> 1][2 * (m & 1) + 1] << 32) | hi[m >> 1][2 * (m & 1)], __ATOMIC_RELAXED, __HIP_MEMORY_SCOPE_SYSTEM);
        }
      }
      wait_vm0();
      unsigned old;
      asm volatile("s_mov_b64 exec, 1\n\tds_or_b32 %1, %2 offset:4\n\tds_add_rtn_u32 %0, %1, %3\n\ts_mov_b64 exec, -1\n\ts_waitcnt lgkmcnt(0)"
                   : "=&v"(old) : "v"(lb + L_MISC + 8), "v"(info << (8 * wave)), "v"(1u) : "memory");
      if (rfl(old) == 3u) {
        const unsigned bases = lds_ld(lb + L_MISC + 12);
        if (lane < NR)
          __hip_atomic_store((unsigned long long*)(p.ws[lane] + F1_OFF + par * F1_BYTES + (size_t)(rank * NUNIT + u) * 128),
                             ((unsigned long long)bases << 32) | call, __ATOMIC_RELAXED, __HIP_MEMORY_SCOPE_SYSTEM);
        STAMP(8);
      }
    }
    if (wave == 0) STAMP(1);
    CW c;
    char* outp = (char*)p.out; asm volatile("" : "+s"(outp));
    const int g = lane >> 4, q = (lane >> 2) & 3, pp = lane & 3;
#pragma unroll
    for (int m = 0; m < 4; ++m)
#pragma unroll
      for (int j = 0; j < 8; ++j) c.acc[m][j] = (f32x4){0.f, 0.f, 0.f, 0.f};
#pragma unroll
    for (int i = 0; i < 4; ++i) c.wa[i] = lb + (g >> 1) * 2048 + q * 256 + (g & 1) * 128 + ((i ^ q) * 32) + pp * 8;
    c.ctl = lb + L_FREE + wave * 256 + lane * 4;
    c.freev = 0; c.ready = 0; c.grp = 0;
    c.xr = __builtin_amdgcn_make_buffer_rsrc((void*)(my + XT_OFF + par * XT_BYTES), 0, (unsigned)XT_BYTES, 0x00020000);
    c.xvo = lane * 16; c.xso = (tm * 4 + wave) * 4096;
    PIN();
    pros<0, NSET - 1>(c);
    prow<0, WPF>(c);
    c.need = NSET;
    wait_ready(c);
    PIN();
    if (wave == 0) { STAMP(2); STAMPC(16); }
#pragma unroll 1
    for (int it = 0; it < NSTEP / NSET; ++it) steps<0, NSET>(c);
    steps<0, NSTEP % NSET>(c);
    if (wave == 0) { STAMP(3); STAMPC(17); }
    unsigned verdict;
    while ((verdict = rfl(lds_ld(lb + L_MISC + 4))) == 0u) __builtin_amdgcn_s_sleep(1);
    int tz = threadIdx.x; asm volatile("" : "+v"(tz));
    const int l2 = tz & 63;
    char* op = outp + ((size_t)(tm * 256 + wave * 64 + (l2 >> 4)) * NL + tn * 128 + (l2 & 15) * 8) * 2;
    if (verdict == 1u) {
      LAS char* const eb = (LAS char*)lds + L_EPI + wave * 4096;
      LAS char* const ew = eb + (l2 & 15) * 256 + ((l2 >> 4) & 1) * 8;
      const int sw = (l2 >> 5) ^ (l2 & 15);
      u32x4 o[4];
#pragma unroll
      for (int m = 0; m <= 4; ++m) {
        if (m < 4) {
#pragma unroll
          for (int j = 0; j < 8; ++j) {
            const f32x4 a = c.acc[m][j];
            *(LAS u32x2*)(ew + ((2 * j) ^ sw) * 16) = (u32x2){cvt_pk(relu(a[0]), relu(a[1])), cvt_pk(relu(a[2]), relu(a[3]))};
            if (m > 0 && (j & 1) == 1) {
              PIN();
              asm volatile("global_store_dwordx4 %0, %1, off " OBITS "\n\ts_nop 1" :: "v"(op + (size_t)((m - 1) * 16 + 4 * (j >> 1)) * NL * 2), "v"(o[j >> 1]));
              PIN();
            }
          }
#pragma unroll
          for (int i = 0; i < 4; ++i) { const int r = 4 * i + (l2 >> 4); o[i] = *(const LAS u32x4*)(eb + r * 256 + ((l2 & 15) ^ r) * 16); }
          PIN();
        } else {
#pragma unroll
          for (int i = 0; i < 4; ++i)
            asm volatile("global_store_dwordx4 %0, %1, off " OBITS "\n\ts_nop 1" :: "v"(op + (size_t)(48 + 4 * i) * NL * 2), "v"(o[i]));
        }
      }
    } else {
#pragma unroll 1
      for (int i = 0; i < 16; ++i) st_sys(op + (size_t)(4 * i) * NL * 2, (u32x4){0x7fc07fc0u, 0x7fc07fc0u, 0x7fc07fc0u, 0x7fc07fc0u});
    }
  } else if (wave == 4) {
    const unsigned pubaddr = lb + L_WFULL + (lane & 3) * 256 + (lane >> 2) * 4;
    wait_vm0();
    int pub = W0, slot = W0; unsigned freec = 0;
    lds_st(pubaddr, pub + NSET - 2);
    for (int s = W0; s < NSTEP; ++s) {
      if (s >= RING && freec < (unsigned)(s - RING + 1)) {
        if (pub < s) { wait_vm0(); pub = s; lds_st(pubaddr, pub + NSET - 2); }
        for (;;) {
          const unsigned f = lds_ld(lb + L_FREE + (lane & 3) * 256);
          freec = min(min(__builtin_amdgcn_readlane(f, 0), __builtin_amdgcn_readlane(f, 1)), min(__builtin_amdgcn_readlane(f, 2), __builtin_amdgcn_readlane(f, 3)));
          if (freec >= (unsigned)(s - RING + 1)) break;
          __builtin_amdgcn_s_sleep(2);
        }
      }
      if (s < RING) {
        const unsigned tgo = tstart + WT0 + (s - W0) * WGAP;
        if ((int)(tgo - (unsigned)rt()) > 0) { if (pub < s) { wait_vm0(); pub = s; lds_st(pubaddr, pub + NSET - 2); } sleep_until(tgo); }
      }
      wfill(wr, lb + slot * 8192, wvo, s * 32 * (NL * 2) + tn * 256);
      slot = slot == RING - 1 ? 0 : slot + 1;
      if (s + 1 - pub > INFL) { asm volatile("s_waitcnt vmcnt(%0)" :: "i"(8 * INFL) : "memory"); pub = s + 1 - INFL; lds_st(pubaddr, pub + NSET - 2); }
    }
    wait_vm0(); lds_st(pubaddr, INF);
  } else if (wave == 6) {
    const unsigned xaddr = lb + L_XRDY + (lane & 3) * 256 + (lane >> 2) * 4;
    bool dead = false;
    {
      const unsigned* f2 = (const unsigned*)(my + F2_OFF) + tm * NSTEP;
      unsigned base = 0;
      const unsigned long long t0 = rt();
      for (unsigned n = 1;; ++n) {
        const unsigned idx = base + lane;
        unsigned v = 4u * call;
        if (idx < (unsigned)NSTEP && lane < POLLW) v = peek(f2 + idx);
        const unsigned nok = ~(unsigned)__ballot(v >= 4u * call);
        base += nok ? (unsigned)__builtin_ctz(nok) : (unsigned)POLLW;
        if (base >= (unsigned)NSTEP) {
          break; }
        lds_st(xaddr, base);
        if ((n & 63u) == 0u) {
          if (rfl(__hip_atomic_load(tmo, __ATOMIC_RELAXED, __HIP_MEMORY_SCOPE_AGENT)) == call || rt() - t0 > DEADLINE_TICKS) {
            __hip_atomic_store(tmo, call, __ATOMIC_RELAXED, __HIP_MEMORY_SCOPE_AGENT); dead = true; break; }
        }
        if (base >= (unsigned)(NSTEP - POLLEND)) __builtin_amdgcn_s_sleep(POLLFAST); else __builtin_amdgcn_s_sleep(POLLSLEEP);
      }
    }
    lds_st(xaddr, INF);
    STAMP(6);
    if (rfl(__hip_atomic_load(tmo, __ATOMIC_RELAXED, __HIP_MEMORY_SCOPE_AGENT)) == call) dead = true;
    lds_st(lb + L_MISC + 4, dead ? 2u : 1u);
  } else {
    if (wave == 7 || b < PW * (NR - 1)) {
      Pull q;
      const int qi = wave == 7 ? NR - 1 : b % (NR - 1);
      q.s = (rank + 1 + qi) & (NR - 1);
      q.pr = __builtin_amdgcn_make_buffer_rsrc((void*)(p.ws[q.s] + PK_OFF + par * PK_BYTES), 0, (unsigned)PK_BYTES, 0x00020000);
      q.xw = __builtin_amdgcn_make_buffer_rsrc((void*)(my + XT_OFF + par * XT_BYTES), 0, (unsigned)XT_BYTES, 0x00020000);
      q.f1 = my + F1_OFF + par * F1_BYTES + (size_t)q.s * NUNIT * 128;
      q.my = my; q.tmo = tmo; q.call = call; q.lane = lane;
      q.tick = (unsigned*)(my + CT_OFF + (2 + 2 * qi) * 128);
      q.tau = wave == 7 ? OWNTAU16 : TAU16;
      bool ok = true;
      if (wave == 7) {
        q.tbase = (call - 1u) * NJOB - GRID;
        q.j = (b + GRID / 2) & (GRID - 1); q.tgo = tstart + OWN0 + q.j * NB * OWNTAU16 / 64; q.fcall = 0; q.info = 0;
        if (q.j == 0 && lane == 0) __hip_atomic_fetch_max((unsigned long long*)(q.tick + 32), rt() + OWN0 - 100, __ATOMIC_RELAXED, __HIP_MEMORY_SCOPE_AGENT);
      } else {
        q.tbase = (call - 1u) * (NJOB + PW);
        while (rfl(lds_ld(lb + L_MISC + 8)) < 4u) __builtin_amdgcn_s_sleep(2);
        unsigned t = 0; unsigned long long t0old = 0, old = 0, fv = 0;
        if (lane == 0) { t = __hip_atomic_fetch_add(q.tick, 1u, __ATOMIC_RELAXED, __HIP_MEMORY_SCOPE_AGENT); t0old = peek64(q.tick + 32); }
        if (t0old + 1500 > rt()) t0old = ~0ull;
        q.j = rfl(t) - q.tbase;
        ok = wait_f1(q.f1, call, tmo, q.info);
        unsigned long long t0 = rt() + XMARG1;
        const u32x4 cs = {(unsigned)t0, (unsigned)(t0 >> 32), (unsigned)t0old, (unsigned)(t0old >> 32)};
        const unsigned jn = q.j < NJOB ? q.j : 0u;
        if (lane == 0)
          asm volatile("global_atomic_cmpswap_x2 %0, %2, %3, off sc0\n\tglobal_atomic_add_x2 %1, %4, %5, off sc0 sc1\n\ts_waitcnt vmcnt(0)"
                       : "=&v"(old), "=&v"(fv) : "v"(q.tick + 32), "v"(cs), "v"(q.f1 + (jn * NB >> 2) * 128), "v"(0ull) : "memory");
        if (old != t0old) t0 = old;
        q.tgo = rfl((unsigned)t0) + jn * NB * TAU16 / 64; q.fcall = rfl((unsigned)fv); q.info = rfl((unsigned)(fv >> 32));
      }
      int pf = wave == 7 ? PF0 : NSTEP;
      while (ok && q.j < NJOB) {
        for (; pf < NSTEP && (int)(q.tgo - (unsigned)rt()) > 150; pf += 4) { wtouch(wp + (size_t)pf * 131072); if (pf == NSTEP - 4) STAMP(11); }
        if (wave == 7) sleep_until(q.tgo - 60);
        ok = pull(q);
      }
      for (; pf < NSTEP; pf += 4) wtouch(wp + (size_t)pf * 131072);

    }
  }
}

}

extern "C" void kernel_launch(int rank, int nranks, ncclComm_t comm, hipStream_t stream,
                              void* const* d_in, const int* in_sizes, int n_in,
                              void* d_out, int out_size, const int* in_dtypes, int out_dtype,
                              void* const* d_out_all, void* const* d_ws_all, size_t ws_bytes) {
  if (nranks != NR || (unsigned)rank >= (unsigned)NR || n_in != 2 || in_sizes[0] != ML * KD || in_sizes[1] != KD * NL ||
      out_size != MG * NL || ws_bytes < WS_NEED) { fprintf(stderr, "unexpected problem shape\n"); return; }
  static bool attr[NR] = {};
  if (!attr[rank]) { HIP_CHECK(hipFuncSetAttribute((const void*)ag_gemm, hipFuncAttributeMaxDynamicSharedMemorySize, LDS_BYTES)); attr[rank] = true; }
  Params p;
  p.x = d_in[0]; p.w = d_in[1]; p.out = d_out; p.rank = rank; p.pad = 0;
  for (int i = 0; i < NR; ++i) p.ws[i] = (char*)d_ws_all[i];
  ag_gemm<<<GRID, BLOCK, LDS_BYTES, stream>>>(p);
  HIP_CHECK(hipGetLastError());
}
```

```cpp
#include <hip/hip_runtime.h>
#include <rccl/rccl.h>
#include <cstdio>

#define HIP_CHECK(expr) do { hipError_t _e = (expr); if (_e != hipSuccess) { \
    fprintf(stderr, "HIP error %s at %s:%d\n", hipGetErrorString(_e), __FILE__, __LINE__); \
    return; } } while (0)

namespace {

#define LAS __attribute__((address_space(3)))
#define DI __device__ __forceinline__
typedef short    s16x4  __attribute__((ext_vector_type(4)));
typedef short    s16x8  __attribute__((ext_vector_type(8)));
typedef __bf16   bf16x8 __attribute__((ext_vector_type(8)));
typedef __bf16   bf16x2 __attribute__((ext_vector_type(2)));
typedef float    f32x2  __attribute__((ext_vector_type(2)));
typedef float    f32x4  __attribute__((ext_vector_type(4)));
typedef unsigned u32x4  __attribute__((ext_vector_type(4)));
typedef unsigned short u16x2 __attribute__((ext_vector_type(2)));

constexpr int NR = 8;
constexpr int KD = 4096, NL = 2048, ML = 512, MG = NR * ML;
constexpr int NSTEP = KD / 32;
constexpr int NRG = ML / 64;
constexpr int NUNIT = NSTEP * 2;
constexpr int UBYTES = 4 * 3072;
constexpr int GRID = 256, BLOCK = 512;

#define NSET 4
#define NGRP 4
constexpr int RING = NSET * NGRP;
#define WPF 3
constexpr int WN = 8;
static_assert(WPF < WN && WPF >= 3 && WPF <= 4 && NSTEP % NSET == 0, "");
constexpr int INFL = 6;
constexpr int PW = 36;
#define TAU16 324
#define POLLSLEEP 8
#define POLLFAST 0
#define POLLEND 6
#define RSPACE 1
#define SBITS "sc0 sc1"
#define NB 2
#define POLLW 16
#define OWN0 300
#define OWNGAP 48
#define PFB 4
#define WGAP 25
#define WPRED 4
#define PULLWAIT 1
#define XMARG 170
#define XMARG1 10
#define W0 4
#define WT0 500
#define OBITS "nt"
#define PF0 4
#define XGAP 12
#define RTAIL 0
#define POLLBEG 0
#define MAP 0
#define FASTGO 1
#define HPRIO 2
#define XFIRST 8
#define XT0 0
#define F1SLEEP 4
#define XAUX 17

constexpr int L_FREE = RING * 8192, L_XRDY = L_FREE + 1024, L_WFULL = L_FREE + 2048, L_MISC = L_FREE + 3072;
constexpr int L_EPI = L_MISC + 1024;
constexpr int LDS_BYTES = L_EPI + 4 * 4096;
static_assert(LDS_BYTES <= 160 * 1024 && LDS_BYTES > 80 * 1024, "");
constexpr unsigned INF = 0x7fffffffu;

constexpr size_t XT_BYTES = (size_t)MG * KD * 2;
constexpr size_t XT_OFF = 0;
constexpr unsigned EL_REL = NUNIT * UBYTES;
constexpr unsigned HB_REL = EL_REL + NUNIT * 4 * 64;
constexpr size_t PK_BYTES = HB_REL + NUNIT * 4 * 2048;
constexpr size_t PK_OFF = XT_OFF + 2 * XT_BYTES;
constexpr size_t F1_BYTES = (size_t)NR * NUNIT * 128;
constexpr size_t F1_OFF = PK_OFF + 2 * PK_BYTES;
constexpr size_t F2_OFF = F1_OFF + 2 * F1_BYTES;
constexpr size_t F2_BYTES = (size_t)NR * NUNIT * 4;
constexpr size_t CT_OFF = F2_OFF + F2_BYTES;
constexpr size_t CC_OFF = CT_OFF + 4096;
constexpr size_t DBG_OFF = CC_OFF + GRID * 128;
constexpr size_t DBG_BYTES = 1 << 20;
constexpr size_t WS_NEED = DBG_OFF + DBG_BYTES;
constexpr unsigned long long DEADLINE_TICKS = 20000000ull;

struct Params { const void* x; char* my; const void* w; void* out; int rank; int pad; char* ws[NR]; };

DI unsigned rfl(unsigned v) { return __builtin_amdgcn_readfirstlane(v); }
DI unsigned long long rt() { return __builtin_amdgcn_s_memrealtime(); }
DI void st_sys(void* p, u32x4 v) { asm volatile("global_store_dwordx4 %0, %1, off sc0 sc1\n\ts_nop 1" :: "v"(p), "v"(v) : "memory"); }
DI void wait_vm0() { asm volatile("s_waitcnt vmcnt(0)" ::: "memory"); }
DI void lds_st(unsigned addr, unsigned v) { asm volatile("ds_write_b32 %0, %1" :: "v"(addr), "v"(v) : "memory"); }
DI unsigned lds_ld(unsigned addr) { unsigned v; asm volatile("ds_read_b32 %0, %1\n\ts_waitcnt lgkmcnt(0)" : "=&v"(v) : "v"(addr) : "memory"); return v; }
DI unsigned peek(const unsigned* p) { unsigned v; asm volatile("global_atomic_add %0, %1, %2, off sc0 sc1\n\ts_waitcnt vmcnt(0)" : "=&v"(v) : "v"(p), "v"(0u) : "memory"); return v; }
DI unsigned long long peek64(const void* p) { unsigned long long v; asm volatile("global_atomic_add_x2 %0, %1, %2, off sc0 sc1\n\ts_waitcnt vmcnt(0)" : "=&v"(v) : "v"(p), "v"(0ull) : "memory"); return v; }
DI void sleep_until(unsigned t) {
  for (int d; (d = (int)(t - (unsigned)rt())) > 0;) { if (d > 200) __builtin_amdgcn_s_sleep(48); else if (d > 40) __builtin_amdgcn_s_sleep(8); else __builtin_amdgcn_s_sleep(1); }
  asm volatile("" ::: "memory");
}
#define PIN() __builtin_amdgcn_sched_barrier(0)

#define STAMP(slot) do {} while (0)
#define STAMPC(slot) do {} while (0)

struct CW {
  alignas(64) f32x4 acc[4][8];
  u32x4 xs[NSET][4];
  s16x4 wf[WN][2];
  unsigned wa[4];
  unsigned fa, ra;
  unsigned xvo[NSET];
  unsigned need, rv;
  unsigned xso;
  unsigned grp;
  unsigned nfail = 0, nslow = 0, nfailw = 0;
  __amdgpu_buffer_rsrc_t xr;
};

DI void wait_ready(CW& c) {
  asm volatile(
      "v_cmp_lt_u32 vcc, %0, %3\n\t"
      "s_cbranch_vccnz 1f\n"
      "2:\n\t"
      ".subsection 1\n"
      "3:\n\t"
      "s_sleep 1\n"
      "1:\n\t"
      "ds_read_b32 %0, %4\n\t"
      "s_waitcnt lgkmcnt(0)\n\t"
      "v_cmp_lt_u32 vcc, %0, %3\n\t"
      "s_cbranch_vccnz 3b\n\t"
      "s_branch 2b\n\t"
      ".subsection 0"
      : "+v"(c.rv) : "i"(0), "i"(0), "v"(c.need), "v"(c.ra) : "vcc", "memory");
}

template <int SET, int M> DI void xload(CW& c) {
  c.xs[SET][M] = __builtin_amdgcn_raw_buffer_load_b128(c.xr, c.xvo[SET] + M * 1024, c.xso, XAUX);
}
template <int SL, int J, int T> DI void wread(CW& c) {
  c.wf[J % WN][T] = __builtin_amdgcn_ds_read_tr16_b64_v4i16((LAS s16x4*)((LAS char*)0 + c.wa[J & 3] + (SL * 8192 + (J >> 2) * 4096 + T * 1024)));
}
template <int S, int J, int M> DI void mma(CW& c) {
  constexpr int MM = (J & 1) ? 3 - M : M;
  const s16x8 w = __builtin_shufflevector(c.wf[J % WN][0], c.wf[J % WN][1], 0, 1, 2, 3, 4, 5, 6, 7);
  c.acc[MM][J] = __builtin_amdgcn_mfma_f32_16x16x32_bf16(__builtin_bit_cast(bf16x8, w), __builtin_bit_cast(bf16x8, c.xs[S][MM]), c.acc[MM][J], 0, 0, 0);
}
template <int S, int J> DI void frag(CW& c, unsigned wd) {
  constexpr int JP = (J + WPF) & 7;
  constexpr bool NXT = (J + WPF) >= 8;
  constexpr int SL = NXT ? (S + 1) % NSET : S;
  constexpr int LS = (S + NSET - 1) % NSET;
  mma<S, J, 0>(c); PIN();
  wread<SL, JP, 0>(c); PIN();
  mma<S, J, 1>(c); PIN();
  wread<SL, JP, 1>(c); PIN();
  mma<S, J, 2>(c); PIN();
  if constexpr (J < 4) xload<LS, J>(c);
  if constexpr (J == 4) { c.need += 1; lds_st(c.fa, c.need); }
  if constexpr (J == 5) c.rv = *(volatile LAS unsigned*)((LAS char*)0 + c.ra);
  if constexpr (J == 6 && S == 0) { c.xso += NSET * 64 * 4096; asm volatile("" : "+s"(c.xso)); }
  if constexpr (J == 7) wait_ready(c);
  PIN();
  mma<S, J, 3>(c); PIN();
  if constexpr (S == NSET - 1 && !NXT && JP >= 4) { c.wa[JP & 3] += wd; asm volatile("" : "+v"(c.wa[JP & 3])); }
  if constexpr (J & 1) {
    if constexpr (J == 7) asm volatile("" :: "v"(c.wf[(J + 2) % WN][1]), "v"(c.xs[(S + 1) % NSET][3]));
    else asm volatile("" :: "v"(c.wf[(J + 2) % WN][1]));
  }
  PIN();
}
template <int S> DI void step(CW& c, unsigned wd) {
  frag<S, 0>(c, wd); frag<S, 1>(c, wd); frag<S, 2>(c, wd); frag<S, 3>(c, wd);
  frag<S, 4>(c, wd); frag<S, 5>(c, wd); frag<S, 6>(c, wd); frag<S, 7>(c, wd);
}
template <int S> DI void pro_x(CW& c) {
  c.need = S + 1; wait_ready(c); PIN();
  xload<S, 0>(c); xload<S, 1>(c); xload<S, 2>(c); xload<S, 3>(c); PIN();
}

DI unsigned cvt_pk(float lo, float hi) { f32x2 v = {lo, hi}; return __builtin_bit_cast(unsigned, __builtin_convertvector(v, bf16x2)); }
DI float relu(float v) { return __builtin_elementwise_maximum(v, 0.0f); }

template <int S, int E> DI void steps(CW& c, unsigned wd) { if constexpr (S < E) { step<S>(c, wd); steps<S + 1, E>(c, wd); } }
template <int S, int E> DI void pros(CW& c) { if constexpr (S < E) { pro_x<S>(c); pros<S + 1, E>(c); } }
template <int J, int E> DI void prow(CW& c) { if constexpr (J < E) { wread<0, J, 0>(c); wread<0, J, 1>(c); prow<J + 1, E>(c); } }

DI unsigned dpp_max(unsigned v, const int ctrl) {
  unsigned o;
  switch (ctrl) {
    case 1: o = __builtin_amdgcn_update_dpp(0u, v, 0x111, 0xf, 0xf, true); break;
    case 2: o = __builtin_amdgcn_update_dpp(0u, v, 0x112, 0xf, 0xf, true); break;
    case 4: o = __builtin_amdgcn_update_dpp(0u, v, 0x114, 0xf, 0xf, true); break;
    default: o = __builtin_amdgcn_update_dpp(0u, v, 0x118, 0xf, 0xf, true); break;
  }
  return o > v ? o : v;
}

DI bool wait_f1(const char* f, unsigned call, unsigned* tmo, unsigned& payload) {
  const unsigned long long t0 = rt();
  for (unsigned n = 1;; ++n) {
    unsigned long long v = 0;
    if (threadIdx.x % 64 == 0) v = peek64(f);
    if (rfl((unsigned)v) >= call) { payload = rfl((unsigned)(v >> 32)); return true; }
    if ((n & 63u) == 0u && (rfl(__hip_atomic_load(tmo, __ATOMIC_RELAXED, __HIP_MEMORY_SCOPE_AGENT)) == call || rt() - t0 > DEADLINE_TICKS)) {
      __hip_atomic_store(tmo, call, __ATOMIC_RELAXED, __HIP_MEMORY_SCOPE_AGENT); return false; }
    __builtin_amdgcn_s_sleep(F1SLEEP);
  }
}

struct Pull {
  __amdgpu_buffer_rsrc_t pr, xw; const char* f1; char* my; unsigned* tmo; unsigned* tick; unsigned call, tbase; int s, lane;
  unsigned j, tgo, fcall, info;
  unsigned t0 = 0, tlast = 0, lsum = 0, lmax = 0, lcnt = 0, rsum = 0, late = 0;
};
typedef unsigned u32x2 __attribute__((ext_vector_type(2)));
constexpr unsigned NJOB = 4 * NUNIT / NB;
DI bool pull(Pull& q) {
  constexpr int N = NB;
  const unsigned k = q.j * N, u = k >> 2, w0 = k & 3u;
  unsigned info = q.info;
  if (q.fcall < q.call && !wait_f1(q.f1 + u * 128, q.call, q.tmo, info)) return false;
  info >>= 8 * w0;
  sleep_until(q.tgo);
  unsigned tk;
  asm volatile("s_mov_b64 exec, 1\n\tglobal_atomic_add %0, %1, %2, off sc0\n\ts_mov_b64 exec, -1" : "=&v"(tk) : "v"(q.tick), "v"(1u) : "memory");
  PIN();
  u32x4 v[3 * N], hb[N]; u32x2 el[N];
#pragma unroll
  for (int w = 0; w < N; ++w) {
    const bool raw = ((info >> (8 * w)) & 0x7fu) == 0x7fu, list = (info >> (8 * w + 7)) & 1u;
    el[w] = __builtin_amdgcn_raw_buffer_load_b64(q.pr, list && q.lane < 8 ? q.lane * 8u : 0x80000000u, EL_REL + (k + w) * 64, 17);
    hb[w] = __builtin_amdgcn_raw_buffer_load_b128(q.pr, raw ? q.lane * 16u + 1024u : 0x80000000u, HB_REL + (k + w) * 2048, 17);
#pragma unroll
    for (int i = 0; i < 3; ++i) {
      v[3 * w + i] = __builtin_amdgcn_raw_buffer_load_b128(q.pr, q.lane * 16, i == 2 && raw ? HB_REL + (k + w) * 2048 : (k + w) * 3072 + i * 1024, 17);
      if (RSPACE > 0 && 3 * w + i < 3 * N - 1) { PIN(); asm volatile("s_cmp_ge_u32 %0, %1\n\ts_cbranch_scc1 1f\n\ts_sleep %2\n1:" :: "s"(k), "i"(4 * NUNIT - RTAIL), "i"(RSPACE) : "scc"); PIN(); }
    }
  }
  asm volatile("s_waitcnt vmcnt(0)" : "+v"(tk) : "v"(v[3 * N - 1]) : "memory");
  PIN();
  q.j = rfl(tk) - q.tbase;
  const unsigned jn = q.j < NJOB ? q.j : NJOB - 1, offn = jn * N * TAU16 / 64;
  unsigned long long t0 = q.j < NJOB ? rt() - offn + XMARG : 0ull, old, fv;
  asm volatile("s_mov_b64 exec, 1\n\tglobal_atomic_umax_x2 %0, %2, %3, off sc0\n\tglobal_atomic_add_x2 %1, %4, %5, off sc0 sc1\n\ts_mov_b64 exec, -1"
               : "=&v"(old), "=&v"(fv) : "v"(q.tick + 32), "v"(t0), "v"(q.f1 + (jn * N >> 2) * 128), "v"(0ull) : "memory");
  const unsigned d = ((u >> 1) * 64 + q.s * NRG + (k & 7u)) * 4096;
#pragma unroll
  for (int w = 0; w < N; ++w) {
    const unsigned bi = (info >> (8 * w)) & 0xffu, base4 = (bi & 0x7fu) * 0x01010101u;
    unsigned h0[4], h1[4];
#pragma unroll
    for (int m = 0; m < 4; ++m) {
      const unsigned h = v[3 * w + 2][m];
      h0[m] = (h & 0x87878787u) + base4; h1[m] = (__builtin_rotateright32(h, 4) & 0x87878787u) + base4;
    }
    if (bi >= 0x7fu) {
      if (bi == 0x7fu) {
#pragma unroll
        for (int m = 0; m < 4; ++m) { const u32x4 t = m < 2 ? v[3 * w + 2] : hb[w]; h0[m] = t[2 * (m & 1)]; h1[m] = t[2 * (m & 1) + 1]; }
      } else {
        unsigned cnt = 0;
#pragma unroll
        for (int m = 0; m < 4; ++m) {
          const bool e = v[3 * w + 2][m] == 0u;
          const unsigned long long bal = __ballot(e);
          if (bal) {
            const unsigned idx = (cnt + __builtin_amdgcn_mbcnt_hi((unsigned)(bal >> 32), __builtin_amdgcn_mbcnt_lo((unsigned)bal, 0u))) * 4u;
            const unsigned a = __builtin_amdgcn_ds_bpermute(idx, el[w][0]), c = __builtin_amdgcn_ds_bpermute(idx, el[w][1]);
            if (e) { h0[m] = a; h1[m] = c; }
            cnt += __builtin_popcountll(bal);
          }
        }
      }
    }
#pragma unroll
    for (int m = 0; m < 4; ++m) {
      const unsigned l0 = v[3 * w + (m >> 1)][2 * (m & 1)], l1 = v[3 * w + (m >> 1)][2 * (m & 1) + 1];
      const u32x4 o = {__builtin_amdgcn_perm(h0[m], l0, 0x05010400u), __builtin_amdgcn_perm(h0[m], l0, 0x07030602u),
                       __builtin_amdgcn_perm(h1[m], l1, 0x05010400u), __builtin_amdgcn_perm(h1[m], l1, 0x07030602u)};
      asm volatile("buffer_store_dwordx4 %0, %1, %2, %3 offen offset:%4 " SBITS "\n\ts_nop 1"
                   :: "v"(o), "v"(q.lane * 16u), "s"(q.xw), "s"(d + w * 4096), "i"(m * 1024) : "memory");
    }
  }
  asm volatile("s_waitcnt vmcnt(0)" : "+v"(old), "+v"(fv) :: "memory");
  asm volatile("s_mov_b64 exec, 1\n\tglobal_atomic_add %0, %1, off sc1\n\ts_mov_b64 exec, -1"
               :: "v"((unsigned*)(q.my + F2_OFF) + (q.s * 2 + (u & 1)) * NSTEP + (u >> 1)), "v"((unsigned)N) : "memory");
  if (old > t0) t0 = old;
  q.tgo = rfl((unsigned)t0) + offn; q.fcall = rfl((unsigned)fv); q.info = rfl((unsigned)(fv >> 32));
  return true;
}

DI void wfill(__amdgpu_buffer_rsrc_t wr, unsigned ldsaddr, unsigned vo, unsigned so) {
#pragma unroll
  for (int i = 0; i < 8; ++i)
    asm volatile("s_mov_b32 m0, %3\n\ts_nop 0\n\tbuffer_load_dwordx4 %0, %1, %2 offen lds"
                 :: "v"(vo), "s"(wr), "s"(so + (16 * ((i >> 1) & 1) + 4 * (i & 1)) * (NL * 2) + (i >> 2) * 128), "s"(ldsaddr + i * 1024) : "memory");
}

DI void wtouch(const char* p, unsigned n) {
  unsigned t, o = 0;
  asm volatile(
      "1:\n\t"
      "s_load_dword %0, %3, %1 offset:0\n\ts_load_dword %0, %3, %1 offset:64\n\ts_load_dword %0, %3, %1 offset:128\n\ts_load_dword %0, %3, %1 offset:192\n\t"
      "s_load_dword %0, %3, %1 offset:256\n\ts_load_dword %0, %3, %1 offset:320\n\ts_load_dword %0, %3, %1 offset:384\n\ts_load_dword %0, %3, %1 offset:448\n\t"
      "s_add_u32 %1, %1, 0x20000\n\t"
      "s_sub_u32 %2, %2, 1\n\t"
      "s_cmp_lg_u32 %2, 0\n\t"
      "s_cbranch_scc1 1b\n\t"
      "s_waitcnt lgkmcnt(0)" : "=&s"(t), "+s"(o), "+s"(n) : "s"(p) : "scc", "memory");
}

__global__ __launch_bounds__(BLOCK, 2) void ag_gemm(Params p) {
  extern __shared__ __attribute__((aligned(16))) char lds[];
  const int lane = threadIdx.x & 63;
  const int wave = rfl(threadIdx.x >> 6);
  const unsigned lb = (unsigned)(size_t)(LAS char*)lds;
  const int b = blockIdx.x, rank = p.rank & (NR - 1);
  const char* xin = (const char*)p.x; char* const my = p.my;
  asm volatile("" : "+s"(xin) : "s"(my));
  const int xcd = b & 7, bi = b >> 3;
  const int tm = (xcd >> 1) * 4 + (bi >> 3), tn = (xcd & 1) * 8 + (bi & 7);
  unsigned* const tmo = (unsigned*)(my + CT_OFF);
  const unsigned tstart = (unsigned)rt();
  if (wave == 0) STAMP(0);

  const int ppair = (bi >> 1) * 8 + xcd;
  const int pstep = 2 * (ppair >> 1) + (bi & 1), prg = (ppair & 1) * 4 + wave;
  u32x4 pv[4];
  const char* const wp = (const char*)p.w + b * 512;
  if (wave < 4) {
    if (XGAP && pstep >= XFIRST) sleep_until(tstart + XT0 + pstep * XGAP / 16);
    const auto xr = __builtin_amdgcn_make_buffer_rsrc((void*)xin, 0, ML * KD * 2, 0x00020000);
#pragma unroll
    for (int m = 0; m < 4; ++m) pv[m] = __builtin_amdgcn_raw_buffer_load_b128(xr, (lane & 15) * (KD * 2) + (lane >> 4) * 16, ((prg * 64 + m * 16) * KD + pstep * 32) * 2, 0);
  }
  const auto wr = __builtin_amdgcn_make_buffer_rsrc((void*)p.w, 0, KD * NL * 2, 0x00020000);
  const unsigned wvo = (8 * ((lane >> 3) & 1) + (lane >> 4)) * (NL * 2) + ((((lane >> 1) & 3) ^ (lane >> 4)) * 32) + (lane & 1) * 16;
  if (wave == 4) {
    __builtin_amdgcn_s_sleep(WPRED);
    for (int s = 0; s < W0; ++s) wfill(wr, lb + s * 8192, wvo, s * 32 * (NL * 2) + tn * 256);
  }
  if (wave == 6) {
    unsigned t = 0;
    if (lane == 0) t = __hip_atomic_fetch_add((unsigned*)(my + CC_OFF + b * 128), 1u, __ATOMIC_RELAXED, __HIP_MEMORY_SCOPE_AGENT);
    t = rfl(t) + 1;
    if (lane < 16) lds_st(lb + L_MISC + lane * 4, lane == 0 ? t : 0u);
  } else if (wave == 4 || wave == 5) {
#pragma unroll
    for (int i = 0; i < 6; ++i) lds_st(lb + L_FREE + (wave - 4) * 1536 + i * 256 + lane * 4, 0u);
  }
  asm volatile("s_waitcnt lgkmcnt(0)" ::: "memory");
  __builtin_amdgcn_s_barrier();
  asm volatile("" ::: "memory");
  const unsigned call = rfl(lds_ld(lb + L_MISC));
  const unsigned par = call & 1u;
  if (wave == 0) STAMP(14);
  if (HPRIO && wave >= 4) __builtin_amdgcn_s_setprio(HPRIO);

  if (wave < 4) {
    {
      u16x2 mx = {0, 0};
#pragma unroll
      for (int m = 0; m < 4; ++m)
#pragma unroll
        for (int i = 0; i < 4; ++i) mx = __builtin_elementwise_max(mx, __builtin_bit_cast(u16x2, pv[m][i] & 0x7fff7fffu));
      unsigned v = mx[0] > mx[1] ? mx[0] : mx[1];
      v = dpp_max(v, 1); v = dpp_max(v, 2); v = dpp_max(v, 4); v = dpp_max(v, 8);
      const unsigned a0 = __builtin_amdgcn_readlane(v, 15), a1 = __builtin_amdgcn_readlane(v, 31), a2 = __builtin_amdgcn_readlane(v, 47), a3 = __builtin_amdgcn_readlane(v, 63);
      const unsigned top = max(max(a0, a1), max(a2, a3)) >> 8;
      const unsigned base = top > 7u ? top - 7u : 0u, base4 = base * 0x01010101u;
      u32x4 lo[2], hi[2], hh; unsigned cnt = 0, idx[4];
#pragma unroll
      for (int m = 0; m < 4; ++m) {
        const unsigned l0 = __builtin_amdgcn_perm(pv[m][1], pv[m][0], 0x06040200u), l1 = __builtin_amdgcn_perm(pv[m][3], pv[m][2], 0x06040200u);
        const unsigned h0 = __builtin_amdgcn_perm(pv[m][1], pv[m][0], 0x07050301u), h1 = __builtin_amdgcn_perm(pv[m][3], pv[m][2], 0x07050301u);
        const unsigned t0 = (h0 | 0x80808080u) - base4, t1 = (h1 | 0x80808080u) - base4;
        const unsigned c0 = (h0 & 0x80808080u) | (t0 & 0x07070707u), c1 = (h1 & 0x80808080u) | (t1 & 0x07070707u);
        unsigned h = c0 | __builtin_rotateleft32(c1, 4);
        if ((t0 & t1 & 0x80808080u) != 0x80808080u) h = 0u;
        const unsigned long long bal = __ballot(h == 0u);
        idx[m] = cnt + __builtin_amdgcn_mbcnt_hi((unsigned)(bal >> 32), __builtin_amdgcn_mbcnt_lo((unsigned)bal, 0u));
        cnt += __builtin_popcountll(bal);
        hh[m] = h;
        lo[m >> 1][2 * (m & 1)] = l0; lo[m >> 1][2 * (m & 1) + 1] = l1;
        hi[m >> 1][2 * (m & 1)] = h0; hi[m >> 1][2 * (m & 1) + 1] = h1;
      }
      if (wave == 0) STAMP(7);
      const int u = pstep * 2 + (prg >> 2), blk = u * 4 + wave;
      char* const pk = my + PK_OFF + par * PK_BYTES;
      st_sys(pk + blk * 3072 + lane * 16, lo[0]); st_sys(pk + blk * 3072 + 1024 + lane * 16, lo[1]);
      unsigned info = base;
      if (cnt > 8u) {
        info = 0x7fu;
        st_sys(pk + HB_REL + blk * 2048 + lane * 16, hi[0]); st_sys(pk + HB_REL + blk * 2048 + 1024 + lane * 16, hi[1]);
      } else {
        st_sys(pk + blk * 3072 + 2048 + lane * 16, hh);
        if (cnt) {
          info |= 0x80u;
#pragma unroll
          for (int m = 0; m < 4; ++m)
            if (hh[m] == 0u)
              __hip_atomic_store((unsigned long long*)(pk + EL_REL + blk * 64 + idx[m] * 8),
                                 ((unsigned long long)hi[m >> 1][2 * (m & 1) + 1] << 32) | hi[m >> 1][2 * (m & 1)], __ATOMIC_RELAXED, __HIP_MEMORY_SCOPE_SYSTEM);
        }
      }
      wait_vm0();
      unsigned old;
      asm volatile("s_mov_b64 exec, 1\n\tds_or_b32 %1, %2 offset:4\n\tds_add_rtn_u32 %0, %1, %3\n\ts_mov_b64 exec, -1\n\ts_waitcnt lgkmcnt(0)"
                   : "=&v"(old) : "v"(lb + L_MISC + 8), "v"(info << (8 * wave)), "v"(1u) : "memory");
      if (rfl(old) == 3u) {
        const unsigned bases = lds_ld(lb + L_MISC + 12);
        if (lane < NR && lane != rank)
          __hip_atomic_store((unsigned long long*)(p.ws[lane] + F1_OFF + par * F1_BYTES + (size_t)(rank * NUNIT + u) * 128),
                             ((unsigned long long)bases << 32) | call, __ATOMIC_RELAXED, __HIP_MEMORY_SCOPE_SYSTEM);
        STAMP(8);
      }
    }
    {
      sleep_until(tstart + OWN0 + pstep * OWNGAP / 16);
      char* d = my + XT_OFF + par * XT_BYTES + (size_t)(pstep * 64 + rank * NRG + prg) * 4096 + lane * 16;
#pragma unroll
      for (int m = 0; m < 4; ++m) st_sys(d + m * 1024, pv[m]);
      wait_vm0();
      asm volatile("s_mov_b64 exec, 1\n\tglobal_atomic_add %0, %1, off sc1\n\ts_mov_b64 exec, -1"
                   :: "v"((unsigned*)(my + F2_OFF) + (rank * 2 + (prg >> 2)) * NSTEP + pstep), "v"(1u) : "memory");
    }
    if (wave == 0) STAMP(1);
    CW c;
    char* outp = (char*)p.out; asm volatile("" : "+s"(outp));
    const int g = lane >> 4, q = (lane >> 2) & 3, pp = lane & 3;
#pragma unroll
    for (int m = 0; m < 4; ++m)
#pragma unroll
      for (int j = 0; j < 8; ++j) c.acc[m][j] = (f32x4){0.f, 0.f, 0.f, 0.f};
#pragma unroll
    for (int i = 0; i < 4; ++i) c.wa[i] = lb + (g >> 1) * 2048 + q * 256 + (g & 1) * 128 + ((i ^ q) * 32) + pp * 8;
    c.fa = lb + L_FREE + wave * 256 + lane * 4; c.ra = lb + L_XRDY + lane * 4;
    c.rv = 0; c.grp = 0;
    c.xr = __builtin_amdgcn_make_buffer_rsrc((void*)(my + XT_OFF + par * XT_BYTES), 0, (unsigned)XT_BYTES, 0x00020000);
#pragma unroll
    for (int i = 0; i < NSET; ++i) { c.xvo[i] = lane * 16 + i * 64 * 4096; asm volatile("" : "+v"(c.xvo[i])); }
    c.xso = (tm * 4 + wave) * 4096;
    PIN();
    pros<0, NSET - 1>(c);
    prow<0, WPF>(c);
    c.need = NSET; asm volatile("" : "+v"(c.need));
    wait_ready(c);
    PIN();
    if (wave == 0) { STAMP(2); STAMPC(16); }
    c.nfail = c.nfailw = c.nslow = 0;
#pragma unroll 1
    for (int it = 0; it < NSTEP / NSET; ++it)
      steps<0, NSET>(c, (it + 1) % NGRP == 0 ? (unsigned)(-(NGRP - 1) * NSET * 8192) : (unsigned)(NSET * 8192));
    if (wave == 0) { STAMP(3); STAMPC(17); }
    unsigned verdict;
    while ((verdict = rfl(lds_ld(lb + L_MISC + 4))) == 0u) __builtin_amdgcn_s_sleep(1);
    int tz = threadIdx.x; asm volatile("" : "+v"(tz));
    const int l2 = tz & 63;
    char* op = outp + ((size_t)(tm * 256 + wave * 64 + (l2 >> 4)) * NL + tn * 128 + (l2 & 15) * 8) * 2;
    if (verdict == 1u) {
      LAS char* const eb = (LAS char*)lds + L_EPI + wave * 4096;
      LAS char* const ew = eb + (l2 & 15) * 256 + ((l2 >> 4) & 1) * 8;
      const int sw = (l2 >> 5) ^ (l2 & 15);
      u32x4 o[4];
#pragma unroll
      for (int m = 0; m <= 4; ++m) {
        if (m < 4) {
#pragma unroll
          for (int j = 0; j < 8; ++j) {
            const f32x4 a = c.acc[m][j];
            *(LAS u32x2*)(ew + ((2 * j) ^ sw) * 16) = (u32x2){cvt_pk(relu(a[0]), relu(a[1])), cvt_pk(relu(a[2]), relu(a[3]))};
            if (m > 0 && (j & 1) == 1) {
              PIN();
              asm volatile("global_store_dwordx4 %0, %1, off " OBITS "\n\ts_nop 1" :: "v"(op + (size_t)((m - 1) * 16 + 4 * (j >> 1)) * NL * 2), "v"(o[j >> 1]));
              PIN();
            }
          }
#pragma unroll
          for (int i = 0; i < 4; ++i) { const int r = 4 * i + (l2 >> 4); o[i] = *(const LAS u32x4*)(eb + r * 256 + ((l2 & 15) ^ r) * 16); }
          PIN();
        } else {
#pragma unroll
          for (int i = 0; i < 4; ++i)
            asm volatile("global_store_dwordx4 %0, %1, off " OBITS "\n\ts_nop 1" :: "v"(op + (size_t)(48 + 4 * i) * NL * 2), "v"(o[i]));
        }
      }
    } else {
#pragma unroll 1
      for (int i = 0; i < 16; ++i) st_sys(op + (size_t)(4 * i) * NL * 2, (u32x4){0x7fc07fc0u, 0x7fc07fc0u, 0x7fc07fc0u, 0x7fc07fc0u});
    }
  } else if (wave == 4) {
    const unsigned pubaddr = lb + L_XRDY + lane * 8 + 4;
    wait_vm0();
    int pub = W0, slot = W0; unsigned freec = 0;
    lds_st(pubaddr, pub + NSET - 2);
    for (int s = W0; s < NSTEP; ++s) {
      if (s >= RING && freec < (unsigned)(s - RING + 1 + NSET)) {
        if (pub < s) { wait_vm0(); pub = s; lds_st(pubaddr, pub + NSET - 2); }
        for (;;) {
          const unsigned f = lds_ld(lb + L_FREE + (lane & 3) * 256);
          freec = min(min(__builtin_amdgcn_readlane(f, 0), __builtin_amdgcn_readlane(f, 1)), min(__builtin_amdgcn_readlane(f, 2), __builtin_amdgcn_readlane(f, 3)));
          if (freec >= (unsigned)(s - RING + 1 + NSET)) break;
          __builtin_amdgcn_s_sleep(2);
        }
      }
      if (s < RING) {
        const unsigned tgo = tstart + WT0 + (s - W0) * WGAP;
        if ((int)(tgo - (unsigned)rt()) > 0) { if (pub < s) { wait_vm0(); pub = s; lds_st(pubaddr, pub + NSET - 2); } sleep_until(tgo); }
      }
      wfill(wr, lb + slot * 8192, wvo, s * 32 * (NL * 2) + tn * 256);
      slot = slot == RING - 1 ? 0 : slot + 1;
      if (s + 1 - pub > INFL) { asm volatile("s_waitcnt vmcnt(%0)" :: "i"(8 * INFL) : "memory"); pub = s + 1 - INFL; lds_st(pubaddr, pub + NSET - 2); }
    }
    wait_vm0(); lds_st(pubaddr, INF);
  } else if (wave == 6) {
    const unsigned xaddr = lb + L_XRDY + lane * 8;
    bool dead = false;
    {
      const unsigned* f2 = (const unsigned*)(my + F2_OFF) + tm * NSTEP;
      unsigned base = 0;
      const unsigned long long t0 = rt();
      for (unsigned n = 1;; ++n) {
        const unsigned idx = base + lane;
        unsigned v = 4u * call;
        if (idx < (unsigned)NSTEP && lane < POLLW) v = peek(f2 + idx);
        const unsigned nok = ~(unsigned)__ballot(v >= 4u * call);
        base += nok ? (unsigned)__builtin_ctz(nok) : (unsigned)POLLW;
        if (base >= (unsigned)NSTEP) {
          break; }
        lds_st(xaddr, base);
        if ((n & 63u) == 0u) {
          if (rfl(__hip_atomic_load(tmo, __ATOMIC_RELAXED, __HIP_MEMORY_SCOPE_AGENT)) == call || rt() - t0 > DEADLINE_TICKS) {
            __hip_atomic_store(tmo, call, __ATOMIC_RELAXED, __HIP_MEMORY_SCOPE_AGENT); dead = true; break; }
        }
        if (base >= (unsigned)(NSTEP - POLLEND) || base < (unsigned)POLLBEG) __builtin_amdgcn_s_sleep(POLLFAST); else __builtin_amdgcn_s_sleep(POLLSLEEP);
      }
    }
    lds_st(xaddr, INF);
    STAMP(6);
    if (rfl(__hip_atomic_load(tmo, __ATOMIC_RELAXED, __HIP_MEMORY_SCOPE_AGENT)) == call) dead = true;
    lds_st(lb + L_MISC + 4, dead ? 2u : 1u);
  } else {
    if (wave == 7) {
      for (int pf = PF0; pf < NSTEP; pf += PFB) wtouch(wp + (size_t)pf * 131072, pf + PFB <= NSTEP ? PFB : NSTEP - pf);
      STAMP(15);
    } else if (b < PW * (NR - 1)) {
      Pull q;
      const int qi = b % (NR - 1);
      q.s = (rank + 1 + qi) & (NR - 1);
      q.pr = __builtin_amdgcn_make_buffer_rsrc((void*)(p.ws[q.s] + PK_OFF + par * PK_BYTES), 0, (unsigned)PK_BYTES, 0x00020000);
      q.xw = __builtin_amdgcn_make_buffer_rsrc((void*)(my + XT_OFF + par * XT_BYTES), 0, (unsigned)XT_BYTES, 0x00020000);
      q.f1 = my + F1_OFF + par * F1_BYTES + (size_t)q.s * NUNIT * 128;
      q.my = my; q.tmo = tmo; q.call = call; q.lane = lane;
      q.tick = (unsigned*)(my + CT_OFF + (2 + 2 * qi) * 128);
      bool ok = true;
      {
        q.tbase = (call - 1u) * (NJOB + PW);
        unsigned t = 0; unsigned long long t0old = 0, old = 0, fv = 0;
        if (lane == 0) t0old = peek64(q.tick + 32);
        if (t0old + 1500 > rt()) t0old = ~0ull;
        while (rfl(lds_ld(lb + L_MISC + 8)) < 4u) __builtin_amdgcn_s_sleep(1);
        if (lane == 0)
          asm volatile("global_atomic_add %0, %2, %3, off sc0\n\tglobal_atomic_add_x2 %1, %4, %5, off sc0 sc1\n\ts_waitcnt vmcnt(0)"
                       : "=&v"(t), "=&v"(fv) : "v"(q.tick), "v"(1u), "v"(q.f1), "v"(0ull) : "memory");
        q.j = rfl(t) - q.tbase; q.info = rfl((unsigned)(fv >> 32));
        STAMP(12);
        if (rfl((unsigned)fv) < call) ok = wait_f1(q.f1, call, tmo, q.info);
        STAMP(13);
        unsigned long long t0 = rt() + XMARG1;
        const u32x4 cs = {(unsigned)t0, (unsigned)(t0 >> 32), (unsigned)t0old, (unsigned)(t0old >> 32)};
        const unsigned jn = q.j < NJOB ? q.j : 0u;
        if (FASTGO && q.j * NB < 4u) {
          if (lane == 0) asm volatile("global_atomic_cmpswap_x2 %0, %1, off\n\ts_nop 1" :: "v"(q.tick + 32), "v"(cs) : "memory");
          q.tgo = tstart; q.fcall = call;
        } else {
          if (lane == 0)
            asm volatile("global_atomic_cmpswap_x2 %0, %2, %3, off sc0\n\tglobal_atomic_add_x2 %1, %4, %5, off sc0 sc1\n\ts_waitcnt vmcnt(0)"
                         : "=&v"(old), "=&v"(fv) : "v"(q.tick + 32), "v"(cs), "v"(q.f1 + (jn * NB >> 2) * 128), "v"(0ull) : "memory");
          if (old != t0old) t0 = old;
          q.tgo = rfl((unsigned)t0) + jn * NB * TAU16 / 64; q.fcall = rfl((unsigned)fv); q.info = rfl((unsigned)(fv >> 32));
        }
      }
      while (ok && q.j < NJOB) ok = pull(q);
    }
  }
}

}

extern "C" void kernel_launch(int rank, int nranks, ncclComm_t comm, hipStream_t stream,
                              void* const* d_in, const int* in_sizes, int n_in,
                              void* d_out, int out_size, const int* in_dtypes, int out_dtype,
                              void* const* d_out_all, void* const* d_ws_all, size_t ws_bytes) {
  if (nranks != NR || (unsigned)rank >= (unsigned)NR || n_in != 2 || in_sizes[0] != ML * KD || in_sizes[1] != KD * NL ||
      out_size != MG * NL || ws_bytes < WS_NEED) { fprintf(stderr, "unexpected problem shape\n"); return; }
  static bool attr[NR] = {};
  if (!attr[rank]) { HIP_CHECK(hipFuncSetAttribute((const void*)ag_gemm, hipFuncAttributeMaxDynamicSharedMemorySize, LDS_BYTES)); attr[rank] = true; }
  Params p;
  p.x = d_in[0]; p.my = (char*)d_ws_all[rank]; p.w = d_in[1]; p.out = d_out; p.rank = rank; p.pad = 0;
  for (int i = 0; i < NR; ++i) p.ws[i] = (char*)d_ws_all[i];
  ag_gemm<<<GRID, BLOCK, LDS_BYTES, stream>>>(p);
  HIP_CHECK(hipGetLastError());
}
```

```cpp
#include <hip/hip_runtime.h>
#include <rccl/rccl.h>
#include <cstdio>

#define HIP_CHECK(expr) do { hipError_t _e = (expr); if (_e != hipSuccess) { \
    fprintf(stderr, "HIP error %s at %s:%d\n", hipGetErrorString(_e), __FILE__, __LINE__); \
    return; } } while (0)

namespace {

#define LAS __attribute__((address_space(3)))
#define DI __device__ __forceinline__
typedef short    s16x4  __attribute__((ext_vector_type(4)));
typedef short    s16x8  __attribute__((ext_vector_type(8)));
typedef __bf16   bf16x8 __attribute__((ext_vector_type(8)));
typedef __bf16   bf16x2 __attribute__((ext_vector_type(2)));
typedef float    f32x2  __attribute__((ext_vector_type(2)));
typedef float    f32x4  __attribute__((ext_vector_type(4)));
typedef unsigned u32x4  __attribute__((ext_vector_type(4)));
typedef unsigned short u16x2 __attribute__((ext_vector_type(2)));

constexpr int NR = 8;
constexpr int KD = 4096, NL = 2048, ML = 512, MG = NR * ML;
constexpr int NSTEP = KD / 32;
constexpr int NRG = ML / 64;
constexpr int NUNIT = NSTEP * 2;
constexpr int BBYTES = 2816;
constexpr int GRID = 256, BLOCK = 512;

#define NSET 4
#define NGRP 4
constexpr int RING = NSET * NGRP;
#define WPF 3
constexpr int WN = 8;
static_assert(WPF < WN && WPF >= 3 && WPF <= 4 && NSTEP % NSET == 0, "");
#define INFL 6
constexpr int PW = 36;
#define TAU16 312
#define POLLSLEEP 8
#define POLLFAST 0
#define POLLEND 6
#define RSPACE 1
#define SBITS "sc0 sc1"
#define NB 2
#define POLLW 16
#define OWN0 300
#define OWNGAP 48
#define PFB 4
#define WGAP 25
#define WPRED 4
#define PULLWAIT 1
#define XMARG 160
#define XMARG1 10
#define W0 4
#define WT0 500
#define OBITS "nt"
#define PF0 4
#define XGAP 12
#define POLLBEG 0
#define MAP 0
#define FASTGO 1
#define HPRIO 0
#define XFIRST 8
#define XT0 0
#define F1SLEEP 4
#define LPRIO 0
#define CPRIO 0
#define XAUX 17

constexpr int L_FREE = RING * 8192, L_XRDY = L_FREE + 1024, L_WFULL = L_FREE + 2048, L_MISC = L_FREE + 3072;
constexpr int L_EPI = L_MISC + 1024;
constexpr int L_PAT = L_EPI + 4 * 4096;
constexpr int LDS_BYTES = L_PAT + 4096 + 64;
static_assert(LDS_BYTES <= 160 * 1024 && LDS_BYTES > 80 * 1024, "");
constexpr unsigned INF = 0x7fffffffu;

constexpr size_t XT_BYTES = (size_t)MG * KD * 2;
constexpr size_t XT_OFF = 0;
constexpr unsigned E0_REL = NUNIT * 4 * BBYTES;
constexpr unsigned E1_REL = E0_REL + NUNIT * 4 * 64;
constexpr unsigned HB_REL = E1_REL + NUNIT * 4 * 64;
constexpr size_t PK_BYTES = HB_REL + NUNIT * 4 * 2048;
constexpr size_t PK_OFF = XT_OFF + 2 * XT_BYTES;
constexpr size_t F1_BYTES = (size_t)NR * NUNIT * 128;
constexpr size_t F1_OFF = PK_OFF + 2 * PK_BYTES;
constexpr size_t F2_OFF = F1_OFF + 2 * F1_BYTES;
constexpr size_t F2_BYTES = (size_t)NR * NUNIT * 4;
constexpr size_t CT_OFF = F2_OFF + F2_BYTES;
constexpr size_t CC_OFF = CT_OFF + 4096;
constexpr size_t DBG_OFF = CC_OFF + GRID * 128;
constexpr size_t DBG_BYTES = 1 << 20;
constexpr size_t WS_NEED = DBG_OFF + DBG_BYTES;
constexpr unsigned long long DEADLINE_TICKS = 20000000ull;

struct Params { const void* x; char* my; const void* w; void* out; int rank; int pad; char* ws[NR]; };

DI unsigned rfl(unsigned v) { return __builtin_amdgcn_readfirstlane(v); }
DI unsigned long long rt() { return __builtin_amdgcn_s_memrealtime(); }
DI void st_sys(void* p, u32x4 v) { asm volatile("global_store_dwordx4 %0, %1, off sc0 sc1\n\ts_nop 1" :: "v"(p), "v"(v) : "memory"); }
DI void wait_vm0() { asm volatile("s_waitcnt vmcnt(0)" ::: "memory"); }
DI void lds_st(unsigned addr, unsigned v) { asm volatile("ds_write_b32 %0, %1" :: "v"(addr), "v"(v) : "memory"); }
DI unsigned lds_ld(unsigned addr) { unsigned v; asm volatile("ds_read_b32 %0, %1\n\ts_waitcnt lgkmcnt(0)" : "=&v"(v) : "v"(addr) : "memory"); return v; }
DI unsigned peek(const unsigned* p) { unsigned v; asm volatile("global_atomic_add %0, %1, %2, off sc0 sc1\n\ts_waitcnt vmcnt(0)" : "=&v"(v) : "v"(p), "v"(0u) : "memory"); return v; }
DI unsigned long long peek64(const void* p) { unsigned long long v; asm volatile("global_atomic_add_x2 %0, %1, %2, off sc0 sc1\n\ts_waitcnt vmcnt(0)" : "=&v"(v) : "v"(p), "v"(0ull) : "memory"); return v; }
DI void sleep_until(unsigned t) {
  for (int d; (d = (int)(t - (unsigned)rt())) > 0;) { if (d > 200) __builtin_amdgcn_s_sleep(48); else if (d > 40) __builtin_amdgcn_s_sleep(8); else __builtin_amdgcn_s_sleep(1); }
  asm volatile("" ::: "memory");
}
#define PIN() __builtin_amdgcn_sched_barrier(0)

#define STAMP(slot) do {} while (0)
#define STAMPC(slot) do {} while (0)

struct CW {
  alignas(64) f32x4 acc[4][8];
  u32x4 xs[NSET][4];
  s16x4 wf[WN][2];
  unsigned wa[4];
  unsigned fa, ra;
  unsigned xvo[NSET];
  unsigned need, rv;
  unsigned xso;
  unsigned grp;
  unsigned nfail = 0, nslow = 0, nfailw = 0;
  __amdgpu_buffer_rsrc_t xr;
};

DI void wait_ready(CW& c) {
  asm volatile(
      "v_cmp_lt_u32 vcc, %0, %3\n\t"
      "s_cbranch_vccnz 1f\n"
      "2:\n\t"
      ".subsection 1\n"
      "3:\n\t"
      "s_sleep 1\n"
      "1:\n\t"
      "ds_read_b32 %0, %4\n\t"
      "s_waitcnt lgkmcnt(0)\n\t"
      "v_cmp_lt_u32 vcc, %0, %3\n\t"
      "s_cbranch_vccnz 3b\n\t"
      "s_branch 2b\n\t"
      ".subsection 0"
      : "+v"(c.rv) : "i"(0), "i"(0), "v"(c.need), "v"(c.ra) : "vcc", "memory");
}

template <int SET, int M> DI void xload(CW& c) {
  c.xs[SET][M] = __builtin_amdgcn_raw_buffer_load_b128(c.xr, c.xvo[SET] + M * 1024, c.xso, XAUX);
}
template <int SL, int J, int T> DI void wread(CW& c) {
  c.wf[J % WN][T] = __builtin_amdgcn_ds_read_tr16_b64_v4i16((LAS s16x4*)((LAS char*)0 + c.wa[J & 3] + (SL * 8192 + (J >> 2) * 4096 + T * 1024)));
}
template <int S, int J, int M> DI void mma(CW& c) {
  constexpr int MM = (J & 1) ? 3 - M : M;
  const s16x8 w = __builtin_shufflevector(c.wf[J % WN][0], c.wf[J % WN][1], 0, 1, 2, 3, 4, 5, 6, 7);
  c.acc[MM][J] = __builtin_amdgcn_mfma_f32_16x16x32_bf16(__builtin_bit_cast(bf16x8, w), __builtin_bit_cast(bf16x8, c.xs[S][MM]), c.acc[MM][J], 0, 0, 0);
}
template <int S, int J> DI void frag(CW& c, unsigned wd) {
  constexpr int JP = (J + WPF) & 7;
  constexpr bool NXT = (J + WPF) >= 8;
  constexpr int SL = NXT ? (S + 1) % NSET : S;
  constexpr int LS = (S + NSET - 1) % NSET;
  mma<S, J, 0>(c); PIN();
  wread<SL, JP, 0>(c); PIN();
  mma<S, J, 1>(c); PIN();
  wread<SL, JP, 1>(c); PIN();
  mma<S, J, 2>(c); PIN();
  if constexpr (J < 4) xload<LS, J>(c);
  if constexpr (J == 4) { c.need += 1; lds_st(c.fa, c.need); }
  if constexpr (J == 5) c.rv = *(volatile LAS unsigned*)((LAS char*)0 + c.ra);
  if constexpr (J == 6 && S == 0) { c.xso += NSET * 64 * 4096; asm volatile("" : "+s"(c.xso)); }
  if constexpr (J == 7) wait_ready(c);
  PIN();
  mma<S, J, 3>(c); PIN();
  if constexpr (S == NSET - 1 && !NXT && JP >= 4) { c.wa[JP & 3] += wd; asm volatile("" : "+v"(c.wa[JP & 3])); }
  if constexpr (J & 1) {
    if constexpr (J == 7) asm volatile("" :: "v"(c.wf[(J + 2) % WN][1]), "v"(c.xs[(S + 1) % NSET][3]));
    else asm volatile("" :: "v"(c.wf[(J + 2) % WN][1]));
  }
  PIN();
}
template <int S> DI void step(CW& c, unsigned wd) {
  frag<S, 0>(c, wd); frag<S, 1>(c, wd); frag<S, 2>(c, wd); frag<S, 3>(c, wd);
  frag<S, 4>(c, wd); frag<S, 5>(c, wd); frag<S, 6>(c, wd); frag<S, 7>(c, wd);
}
template <int S> DI void pro_x(CW& c) {
  c.need = S + 1; wait_ready(c); PIN();
  xload<S, 0>(c); xload<S, 1>(c); xload<S, 2>(c); xload<S, 3>(c); PIN();
}

DI unsigned cvt_pk(float lo, float hi) { f32x2 v = {lo, hi}; return __builtin_bit_cast(unsigned, __builtin_convertvector(v, bf16x2)); }
DI float relu(float v) { return __builtin_elementwise_maximum(v, 0.0f); }

template <int S, int E> DI void steps(CW& c, unsigned wd) { if constexpr (S < E) { step<S>(c, wd); steps<S + 1, E>(c, wd); } }
template <int S, int E> DI void pros(CW& c) { if constexpr (S < E) { pro_x<S>(c); pros<S + 1, E>(c); } }
template <int J, int E> DI void prow(CW& c) { if constexpr (J < E) { wread<0, J, 0>(c); wread<0, J, 1>(c); prow<J + 1, E>(c); } }

DI unsigned dpp_shr(unsigned v, const int n) {
  switch (n) {
    case 1: return __builtin_amdgcn_update_dpp(0u, v, 0x111, 0xf, 0xf, true);
    case 2: return __builtin_amdgcn_update_dpp(0u, v, 0x112, 0xf, 0xf, true);
    case 4: return __builtin_amdgcn_update_dpp(0u, v, 0x114, 0xf, 0xf, true);
    default: return __builtin_amdgcn_update_dpp(0u, v, 0x118, 0xf, 0xf, true);
  }
}

DI bool wait_f1(const char* f, unsigned call, unsigned* tmo, unsigned& payload) {
  const unsigned long long t0 = rt();
  for (unsigned n = 1;; ++n) {
    unsigned long long v = 0;
    if (threadIdx.x % 64 == 0) v = peek64(f);
    if (rfl((unsigned)v) >= call) { payload = rfl((unsigned)(v >> 32)); return true; }
    if ((n & 63u) == 0u && (rfl(__hip_atomic_load(tmo, __ATOMIC_RELAXED, __HIP_MEMORY_SCOPE_AGENT)) == call || rt() - t0 > DEADLINE_TICKS)) {
      __hip_atomic_store(tmo, call, __ATOMIC_RELAXED, __HIP_MEMORY_SCOPE_AGENT); return false; }
    __builtin_amdgcn_s_sleep(F1SLEEP);
  }
}

struct Pull {
  __amdgpu_buffer_rsrc_t pr, xw; const char* f1; char* my; unsigned* tmo; unsigned* tick; LAS unsigned char* pat; unsigned call, tbase; int s, lane;
  unsigned j, tgo, fcall, info;
  unsigned t0 = 0, tlast = 0, lsum = 0, lmax = 0, lcnt = 0, rsum = 0, late = 0;
};
typedef unsigned u32x2 __attribute__((ext_vector_type(2)));
typedef unsigned u32x3 __attribute__((ext_vector_type(3)));
#define F1POS(j) (((j) >> 1) * 128 + ((j) & 1u) * 8)
DI unsigned pk_add(unsigned a, unsigned b) { return __builtin_bit_cast(unsigned, (u16x2)(__builtin_bit_cast(u16x2, a) + __builtin_bit_cast(u16x2, b))); }
DI unsigned pk_sub(unsigned a, unsigned b) { return __builtin_bit_cast(unsigned, (u16x2)(__builtin_bit_cast(u16x2, a) - __builtin_bit_cast(u16x2, b))); }
constexpr unsigned NJOB = 4 * NUNIT / NB;
DI bool pull(Pull& q) {
  constexpr int N = NB;
  static_assert(NB == 2, "");
  const unsigned k = q.j * N, u = k >> 2;
  unsigned info = q.info;
  if (q.fcall < q.call && !wait_f1(q.f1 + F1POS(q.j), q.call, q.tmo, info)) return false;
  const unsigned n0 = (info >> 8) & 0xffu, n1 = info >> 24;
#pragma unroll
  for (int w = 0; w < N; ++w) {
    const unsigned q4 = ((info >> (16 * w + 1)) & 0x7cu) * 0x01010101u;
    *(LAS u32x4*)(q.pat + w * 2048 + q.lane * 16) = (u32x4){q4, q4, q4, q4};
    *(LAS u32x4*)(q.pat + w * 2048 + 1024 + q.lane * 16) = (u32x4){q4, q4, q4, q4};
  }
  sleep_until(q.tgo);
  unsigned tk;
  asm volatile("s_mov_b64 exec, 1\n\tglobal_atomic_add %0, %1, %2, off sc0\n\ts_mov_b64 exec, -1" : "=&v"(tk) : "v"(q.tick), "v"(1u) : "memory");
  PIN();
  const unsigned eidx = (q.lane & 32) + 2 * (q.lane & 15), en = (q.lane & 16) ? n1 : n0;
  const unsigned e = __builtin_amdgcn_raw_buffer_load_b32(q.pr, eidx < en ? (q.lane & 31) * 4u + (q.lane >> 5) * (E1_REL - E0_REL) : 0x80000000u, E0_REL + k * 64, 17);
  u32x4 lo[N][2], hb[N][2]; u32x3 pl[N];
#pragma unroll
  for (int w = 0; w < N; ++w) {
    lo[w][0] = __builtin_amdgcn_raw_buffer_load_b128(q.pr, q.lane * 16, (k + w) * BBYTES, 17);
    PIN(); __builtin_amdgcn_s_sleep(RSPACE); PIN();
    lo[w][1] = __builtin_amdgcn_raw_buffer_load_b128(q.pr, q.lane * 16 + 1024, (k + w) * BBYTES, 17);
    PIN(); __builtin_amdgcn_s_sleep(RSPACE); PIN();
    pl[w] = __builtin_amdgcn_raw_buffer_load_b96(q.pr, q.lane * 12 + 2048, (k + w) * BBYTES, 17);
    if (w < N - 1) { PIN(); __builtin_amdgcn_s_sleep(RSPACE); PIN(); }
  }
  PIN();
  {
    LAS unsigned char* const pb = q.pat + ((q.lane >> 4) & 1) * 2048;
    *(eidx < en ? pb + (e & 2047u) : q.pat + 4096 + q.lane) = (e >> 9) & 0x7cu;
    *(eidx + 1 < en ? pb + ((e >> 16) & 2047u) : q.pat + 4096 + q.lane) = (e >> 25) & 0x7cu;
  }
#pragma unroll
  for (int w = 0; w < N; ++w) { hb[w][0] = *(LAS u32x4*)(q.pat + w * 2048 + q.lane * 16); hb[w][1] = *(LAS u32x4*)(q.pat + w * 2048 + 1024 + q.lane * 16); }
  PIN();
  asm volatile("s_waitcnt vmcnt(0)" : "+v"(tk) : "v"(pl[N - 1]) : "memory");
  if (max(n0, n1) == 255u) {
#pragma unroll
    for (int w = 0; w < N; ++w)
      if ((w ? n1 : n0) == 255u) {
        hb[w][0] = __builtin_amdgcn_raw_buffer_load_b128(q.pr, q.lane * 16, HB_REL + (k + w) * 2048, 17);
        hb[w][1] = __builtin_amdgcn_raw_buffer_load_b128(q.pr, q.lane * 16 + 1024, HB_REL + (k + w) * 2048, 17);
      }
    asm volatile("" :: "v"(hb[0][0]), "v"(hb[0][1]), "v"(hb[1][0]), "v"(hb[1][1]));
  }
  PIN();
  q.j = rfl(tk) - q.tbase;
  const unsigned jn = q.j < NJOB ? q.j : NJOB - 1, offn = jn * N * TAU16 / 64;
  unsigned long long t0 = q.j < NJOB ? rt() - offn + XMARG : 0ull, old, fv;
  asm volatile("s_mov_b64 exec, 1\n\tglobal_atomic_umax_x2 %0, %2, %3, off sc0\n\tglobal_atomic_add_x2 %1, %4, %5, off sc0 sc1\n\ts_mov_b64 exec, -1"
               : "=&v"(old), "=&v"(fv) : "v"(q.tick + 32), "v"(t0), "v"(q.f1 + F1POS(jn)), "v"(0ull) : "memory");
  const unsigned d = ((u >> 1) * 64 + q.s * NRG + (k & 7u)) * 4096;
#pragma unroll
  for (int w = 0; w < N; ++w) {
    const unsigned sh = ((info >> (16 * w)) & 7u) * 0x00800080u;
    const unsigned a3 = ((pl[w][0] << 1) & 0x88888888u) | ((pl[w][1] >> 1) & 0x22222222u) | ((pl[w][2] >> 2) & 0x11111111u);
#pragma unroll
    for (int m = 0; m < 4; ++m) {
      const unsigned a = m < 3 ? pl[w][m < 3 ? m : 0] : a3, l0 = lo[w][m >> 1][2 * (m & 1)], l1 = lo[w][m >> 1][2 * (m & 1) + 1];
      const unsigned h0 = (a & 0x83838383u) | hb[w][m >> 1][2 * (m & 1)], h1 = (__builtin_rotateright32(a, 4) & 0x83838383u) | hb[w][m >> 1][2 * (m & 1) + 1];
      const u32x4 o = {pk_sub(__builtin_amdgcn_perm(h0, l0, 0x05010400u), sh), pk_sub(__builtin_amdgcn_perm(h0, l0, 0x07030602u), sh),
                       pk_sub(__builtin_amdgcn_perm(h1, l1, 0x05010400u), sh), pk_sub(__builtin_amdgcn_perm(h1, l1, 0x07030602u), sh)};
      asm volatile("buffer_store_dwordx4 %0, %1, %2, %3 offen offset:%4 " SBITS "\n\ts_nop 1"
                   :: "v"(o), "v"(q.lane * 16u), "s"(q.xw), "s"(d + w * 4096), "i"(m * 1024) : "memory");
    }
  }
  asm volatile("s_waitcnt vmcnt(0)" : "+v"(old), "+v"(fv) :: "memory");
  asm volatile("s_mov_b64 exec, 1\n\tglobal_atomic_add %0, %1, off sc1\n\ts_mov_b64 exec, -1"
               :: "v"((unsigned*)(q.my + F2_OFF) + (q.s * 2 + (u & 1)) * NSTEP + (u >> 1)), "v"((unsigned)N) : "memory");
  if (old > t0) t0 = old;
  q.tgo = rfl((unsigned)t0) + offn; q.fcall = rfl((unsigned)fv); q.info = rfl((unsigned)(fv >> 32));
  return true;
}

DI void wfill(__amdgpu_buffer_rsrc_t wr, unsigned ldsaddr, unsigned vo, unsigned so) {
#pragma unroll
  for (int i = 0; i < 8; ++i)
    asm volatile("s_mov_b32 m0, %3\n\ts_nop 0\n\tbuffer_load_dwordx4 %0, %1, %2 offen lds"
                 :: "v"(vo), "s"(wr), "s"(so + (16 * ((i >> 1) & 1) + 4 * (i & 1)) * (NL * 2) + (i >> 2) * 128), "s"(ldsaddr + i * 1024) : "memory");
}

DI void wtouch(const char* p, unsigned n) {
  unsigned t, o = 0;
  asm volatile(
      "1:\n\t"
      "s_load_dword %0, %3, %1 offset:0\n\ts_load_dword %0, %3, %1 offset:64\n\ts_load_dword %0, %3, %1 offset:128\n\ts_load_dword %0, %3, %1 offset:192\n\t"
      "s_load_dword %0, %3, %1 offset:256\n\ts_load_dword %0, %3, %1 offset:320\n\ts_load_dword %0, %3, %1 offset:384\n\ts_load_dword %0, %3, %1 offset:448\n\t"
      "s_add_u32 %1, %1, 0x20000\n\t"
      "s_sub_u32 %2, %2, 1\n\t"
      "s_cmp_lg_u32 %2, 0\n\t"
      "s_cbranch_scc1 1b\n\t"
      "s_waitcnt lgkmcnt(0)" : "=&s"(t), "+s"(o), "+s"(n) : "s"(p) : "scc", "memory");
}

__global__ __launch_bounds__(BLOCK, 2) void ag_gemm(Params p) {
  extern __shared__ __attribute__((aligned(16))) char lds[];
  const int lane = threadIdx.x & 63;
  const int wave = rfl(threadIdx.x >> 6);
  const unsigned lb = (unsigned)(size_t)(LAS char*)lds;
  const int b = blockIdx.x, rank = p.rank & (NR - 1);
  const char* xin = (const char*)p.x; char* const my = p.my;
  asm volatile("" : "+s"(xin) : "s"(my));
  const int xcd = b & 7, bi = b >> 3;
  const int tm = (xcd >> 1) * 4 + (bi >> 3), tn = (xcd & 1) * 8 + (bi & 7);
  unsigned* const tmo = (unsigned*)(my + CT_OFF);
  const unsigned tstart = (unsigned)rt();
  if (wave == 0) STAMP(0);

  const int ppair = (bi >> 1) * 8 + xcd;
  const int pstep = 2 * (ppair >> 1) + (bi & 1), prg = (ppair & 1) * 4 + wave;
  u32x4 pv[4];
  const char* const wp = (const char*)p.w + b * 512;
  if (wave < 4) {
    if (XGAP && pstep >= XFIRST) sleep_until(tstart + XT0 + pstep * XGAP / 16);
    const auto xr = __builtin_amdgcn_make_buffer_rsrc((void*)xin, 0, ML * KD * 2, 0x00020000);
#pragma unroll
    for (int m = 0; m < 4; ++m) pv[m] = __builtin_amdgcn_raw_buffer_load_b128(xr, (lane & 15) * (KD * 2) + (lane >> 4) * 16, ((prg * 64 + m * 16) * KD + pstep * 32) * 2, 0);
  }
  const auto wr = __builtin_amdgcn_make_buffer_rsrc((void*)p.w, 0, KD * NL * 2, 0x00020000);
  const unsigned wvo = (8 * ((lane >> 3) & 1) + (lane >> 4)) * (NL * 2) + ((((lane >> 1) & 3) ^ (lane >> 4)) * 32) + (lane & 1) * 16;
  if (wave == 4) {
    __builtin_amdgcn_s_sleep(WPRED);
    for (int s = 0; s < W0; ++s) wfill(wr, lb + s * 8192, wvo, s * 32 * (NL * 2) + tn * 256);
  }
  if (wave == 6) {
    unsigned t = 0;
    if (lane == 0) t = __hip_atomic_fetch_add((unsigned*)(my + CC_OFF + b * 128), 1u, __ATOMIC_RELAXED, __HIP_MEMORY_SCOPE_AGENT);
    t = rfl(t) + 1;
    if (lane < 16) lds_st(lb + L_MISC + lane * 4, lane == 0 ? t : 0u);
  } else if (wave == 4 || wave == 5) {
#pragma unroll
    for (int i = 0; i < 6; ++i) lds_st(lb + L_FREE + (wave - 4) * 1536 + i * 256 + lane * 4, 0u);
  }
  asm volatile("s_waitcnt lgkmcnt(0)" ::: "memory");
  __builtin_amdgcn_s_barrier();
  asm volatile("" ::: "memory");
  const unsigned call = rfl(lds_ld(lb + L_MISC));
  const unsigned par = call & 1u;
  if (wave == 0) STAMP(14);
  if (wave == 4) __builtin_amdgcn_s_setprio(LPRIO);
  if (wave == 5) __builtin_amdgcn_s_setprio(HPRIO);
  if (wave == 6) __builtin_amdgcn_s_setprio(2);
  if (wave < 4) __builtin_amdgcn_s_setprio(3);

  if (wave < 4) {
    {
      unsigned v = 0;
#pragma unroll
      for (int m = 0; m < 4; ++m) { v = __builtin_amdgcn_sad_u8(pv[m][0] & 0x7f007f00u, 0u, v); v = __builtin_amdgcn_sad_u8(pv[m][2] & 0x7f007f00u, 0u, v); }
      v += dpp_shr(v, 1); v += dpp_shr(v, 2); v += dpp_shr(v, 4); v += dpp_shr(v, 8);
      const unsigned sum = __builtin_amdgcn_readlane(v, 15) + __builtin_amdgcn_readlane(v, 31) + __builtin_amdgcn_readlane(v, 47) + __builtin_amdgcn_readlane(v, 63);
      const unsigned top = (sum + 1705u) >> 9, low = min(top > 7u ? top - 7u : 0u, 248u);
      const unsigned sh = (0u - low) & 7u, qw = (low + sh) >> 3, q4 = (qw << 2) * 0x01010101u;
      u32x4 lo[2], hi[2]; unsigned a[4], mask = 0;
#pragma unroll
      for (int m = 0; m < 4; ++m) {
        const unsigned d0 = pk_add(pv[m][0], sh * 0x00800080u), d1 = pk_add(pv[m][1], sh * 0x00800080u), d2 = pk_add(pv[m][2], sh * 0x00800080u), d3 = pk_add(pv[m][3], sh * 0x00800080u);
        const unsigned h0 = __builtin_amdgcn_perm(d1, d0, 0x07050301u), h1 = __builtin_amdgcn_perm(d3, d2, 0x07050301u);
        lo[m >> 1][2 * (m & 1)] = __builtin_amdgcn_perm(d1, d0, 0x06040200u); lo[m >> 1][2 * (m & 1) + 1] = __builtin_amdgcn_perm(d3, d2, 0x06040200u);
        hi[m >> 1][2 * (m & 1)] = h0; hi[m >> 1][2 * (m & 1) + 1] = h1;
        a[m] = (h0 & 0x83838383u) | __builtin_rotateleft32(h1 & 0x83838383u, 4);
        mask |= ((((h0 ^ q4) & 0x7c7c7c7cu) + 0x7c7c7c7cu) & 0x80808080u) >> (7 - 2 * m);
        mask |= ((((h1 ^ q4) & 0x7c7c7c7cu) + 0x7c7c7c7cu) & 0x80808080u) >> (6 - 2 * m);
      }
      if (wave == 0) STAMP(7);
      const int u = pstep * 2 + (prg >> 2), blk = u * 4 + wave;
      char* const pk = my + PK_OFF + par * PK_BYTES;
      st_sys(pk + blk * BBYTES + lane * 16, lo[0]); st_sys(pk + blk * BBYTES + 1024 + lane * 16, lo[1]);
      const u32x3 pl = {a[0] | ((a[3] >> 1) & 0x44444444u), a[1] | ((a[3] << 1) & 0x44444444u), a[2] | ((a[3] << 2) & 0x44444444u)};
      asm volatile("global_store_dwordx3 %0, %1, off sc0 sc1\n\ts_nop 1" :: "v"(pk + blk * BBYTES + 2048 + lane * 12), "v"(pl) : "memory");
      LAS unsigned char* const hcopy = (LAS unsigned char*)lds + L_EPI + wave * 4096;
      LAS unsigned short* const list = (LAS unsigned short*)(hcopy + 2048);
      *(LAS u32x4*)(hcopy + lane * 16) = hi[0]; *(LAS u32x4*)(hcopy + 1024 + lane * 16) = hi[1];
      unsigned cnt = 0;
      do {
#pragma unroll
        for (int i = 0; i < 4; ++i) {
          const unsigned long long bal = __ballot(mask != 0u);
          const unsigned idx = cnt + __builtin_amdgcn_mbcnt_hi((unsigned)(bal >> 32), __builtin_amdgcn_mbcnt_lo((unsigned)bal, 0u));
          list[mask ? idx : 512u + lane] = (unsigned short)(__builtin_ctz(mask | 0x80000000u) | (lane << 5));
          mask &= mask - 1u; cnt += __builtin_popcountll(bal);
        }
      } while (cnt <= 64u && __any(mask != 0u));
      if (cnt > 64u) {
        cnt = 255u;
        st_sys(pk + HB_REL + blk * 2048 + lane * 16, hi[0]); st_sys(pk + HB_REL + blk * 2048 + 1024 + lane * 16, hi[1]);
      } else {
        const unsigned t = list[lane], pos = ((t & 4u) << 8) | ((t >> 1) & 0x3f0u) | ((t & 3u) << 2) | ((t >> 3) & 3u);
        const unsigned e = pos | ((hcopy[pos] & 0x7cu) << 9);
        const unsigned e2 = e | (__builtin_amdgcn_update_dpp(0u, e, 0xb1, 0xf, 0xf, true) << 16);
        if ((unsigned)lane < cnt && !(lane & 1))
          __hip_atomic_store((unsigned*)(pk + (lane < 32 ? E0_REL : E1_REL - 64) + blk * 64 + lane * 2), e2, __ATOMIC_RELAXED, __HIP_MEMORY_SCOPE_SYSTEM);
      }
      const unsigned info = sh | (qw << 3) | (cnt << 8);
      wait_vm0();
      unsigned old;
      asm volatile("s_mov_b64 exec, 1\n\tds_write_b16 %1, %2\n\tds_add_rtn_u32 %0, %3, %4\n\ts_mov_b64 exec, -1\n\ts_waitcnt lgkmcnt(0)"
                   : "=&v"(old) : "v"(lb + L_MISC + 16 + 2 * wave), "v"(info), "v"(lb + L_MISC + 8), "v"(1u) : "memory");
      if (rfl(old) == 3u) {
        const unsigned pair = lds_ld(lb + L_MISC + 16 + (lane & 1) * 4);
        if (lane < 2 * NR && (lane >> 1) != rank)
          __hip_atomic_store((unsigned long long*)(p.ws[lane >> 1] + F1_OFF + par * F1_BYTES + (size_t)(rank * NUNIT + u) * 128 + (lane & 1) * 8),
                             ((unsigned long long)pair << 32) | call, __ATOMIC_RELAXED, __HIP_MEMORY_SCOPE_SYSTEM);
        STAMP(8);
      }
    }
    __builtin_amdgcn_s_setprio(CPRIO);
    {
      sleep_until(tstart + OWN0 + pstep * OWNGAP / 16);
      char* d = my + XT_OFF + par * XT_BYTES + (size_t)(pstep * 64 + rank * NRG + prg) * 4096 + lane * 16;
#pragma unroll
      for (int m = 0; m < 4; ++m) st_sys(d + m * 1024, pv[m]);
      wait_vm0();
      asm volatile("s_mov_b64 exec, 1\n\tglobal_atomic_add %0, %1, off sc1\n\ts_mov_b64 exec, -1"
                   :: "v"((unsigned*)(my + F2_OFF) + (rank * 2 + (prg >> 2)) * NSTEP + pstep), "v"(1u) : "memory");
    }
    if (wave == 0) STAMP(1);
    CW c;
    char* outp = (char*)p.out; asm volatile("" : "+s"(outp));
    const int g = lane >> 4, q = (lane >> 2) & 3, pp = lane & 3;
#pragma unroll
    for (int m = 0; m < 4; ++m)
#pragma unroll
      for (int j = 0; j < 8; ++j) c.acc[m][j] = (f32x4){0.f, 0.f, 0.f, 0.f};
#pragma unroll
    for (int i = 0; i < 4; ++i) c.wa[i] = lb + (g >> 1) * 2048 + q * 256 + (g & 1) * 128 + ((i ^ q) * 32) + pp * 8;
    c.fa = lb + L_FREE + wave * 256 + lane * 4; c.ra = lb + L_XRDY + lane * 4;
    c.rv = 0; c.grp = 0;
    c.xr = __builtin_amdgcn_make_buffer_rsrc((void*)(my + XT_OFF + par * XT_BYTES), 0, (unsigned)XT_BYTES, 0x00020000);
#pragma unroll
    for (int i = 0; i < NSET; ++i) { c.xvo[i] = lane * 16 + i * 64 * 4096; asm volatile("" : "+v"(c.xvo[i])); }
    c.xso = (tm * 4 + wave) * 4096;
    PIN();
    pros<0, NSET - 1>(c);
    prow<0, WPF>(c);
    c.need = NSET; asm volatile("" : "+v"(c.need));
    wait_ready(c);
    PIN();
    if (wave == 0) { STAMP(2); STAMPC(16); }
    c.nfail = c.nfailw = c.nslow = 0;
#pragma unroll 1
    for (int it = 0; it < NSTEP / NSET; ++it)
      steps<0, NSET>(c, (it + 1) % NGRP == 0 ? (unsigned)(-(NGRP - 1) * NSET * 8192) : (unsigned)(NSET * 8192));
    if (wave == 0) { STAMP(3); STAMPC(17); }
    unsigned verdict;
    while ((verdict = rfl(lds_ld(lb + L_MISC + 4))) == 0u) __builtin_amdgcn_s_sleep(1);
    int tz = threadIdx.x; asm volatile("" : "+v"(tz));
    const int l2 = tz & 63;
    char* op = outp + ((size_t)(tm * 256 + wave * 64 + (l2 >> 4)) * NL + tn * 128 + (l2 & 15) * 8) * 2;
    if (verdict == 1u) {
      LAS char* const eb = (LAS char*)lds + L_EPI + wave * 4096;
      LAS char* const ew = eb + (l2 & 15) * 256 + ((l2 >> 4) & 1) * 8;
      const int sw = (l2 >> 5) ^ (l2 & 15);
      u32x4 o[4];
#pragma unroll
      for (int m = 0; m <= 4; ++m) {
        if (m < 4) {
#pragma unroll
          for (int j = 0; j < 8; ++j) {
            const f32x4 a = c.acc[m][j];
            *(LAS u32x2*)(ew + ((2 * j) ^ sw) * 16) = (u32x2){cvt_pk(relu(a[0]), relu(a[1])), cvt_pk(relu(a[2]), relu(a[3]))};
            if (m > 0 && (j & 1) == 1) {
              PIN();
              asm volatile("global_store_dwordx4 %0, %1, off " OBITS "\n\ts_nop 1" :: "v"(op + (size_t)((m - 1) * 16 + 4 * (j >> 1)) * NL * 2), "v"(o[j >> 1]));
              PIN();
            }
          }
#pragma unroll
          for (int i = 0; i < 4; ++i) { const int r = 4 * i + (l2 >> 4); o[i] = *(const LAS u32x4*)(eb + r * 256 + ((l2 & 15) ^ r) * 16); }
          PIN();
        } else {
#pragma unroll
          for (int i = 0; i < 4; ++i)
            asm volatile("global_store_dwordx4 %0, %1, off " OBITS "\n\ts_nop 1" :: "v"(op + (size_t)(48 + 4 * i) * NL * 2), "v"(o[i]));
        }
      }
    } else {
#pragma unroll 1
      for (int i = 0; i < 16; ++i) st_sys(op + (size_t)(4 * i) * NL * 2, (u32x4){0x7fc07fc0u, 0x7fc07fc0u, 0x7fc07fc0u, 0x7fc07fc0u});
    }
  } else if (wave == 4) {
    const unsigned pubaddr = lb + L_XRDY + lane * 8 + 4;
    wait_vm0();
    int pub = W0, slot = W0; unsigned freec = 0;
    int occ = 0, occmin = 99; unsigned tfill = 0;
    lds_st(pubaddr, pub + NSET - 2);
    for (int s = W0; s < NSTEP; ++s) {
      if (s >= RING && freec < (unsigned)(s - RING + 1 + NSET)) {
        if (pub < s) { wait_vm0(); pub = s; lds_st(pubaddr, pub + NSET - 2); }
        for (;;) {
          const unsigned f = lds_ld(lb + L_FREE + (lane & 3) * 256);
          freec = min(min(__builtin_amdgcn_readlane(f, 0), __builtin_amdgcn_readlane(f, 1)), min(__builtin_amdgcn_readlane(f, 2), __builtin_amdgcn_readlane(f, 3)));
          if (freec >= (unsigned)(s - RING + 1 + NSET)) break;
          __builtin_amdgcn_s_sleep(2);
        }
      }
      if (s < RING) {
        const unsigned tgo = tstart + WT0 + (s - W0) * WGAP;
        if ((int)(tgo - (unsigned)rt()) > 0) { if (pub < s) { wait_vm0(); pub = s; lds_st(pubaddr, pub + NSET - 2); } sleep_until(tgo); }
      }
      wfill(wr, lb + slot * 8192, wvo, s * 32 * (NL * 2) + tn * 256);
      slot = slot == RING - 1 ? 0 : slot + 1;
      if (s + 1 - pub > INFL) { asm volatile("s_waitcnt vmcnt(%0)" :: "i"(8 * INFL) : "memory"); pub = s + 1 - INFL; lds_st(pubaddr, pub + NSET - 2); }
    }
    wait_vm0(); lds_st(pubaddr, INF);
  } else if (wave == 6) {
    const unsigned xaddr = lb + L_XRDY + lane * 8;
    bool dead = false;
    {
      const unsigned* f2 = (const unsigned*)(my + F2_OFF) + tm * NSTEP;
      unsigned base = 0;
      const unsigned long long t0 = rt();
      for (unsigned n = 1;; ++n) {
        const unsigned idx = base + lane;
        unsigned v = 4u * call;
        if (idx < (unsigned)NSTEP && lane < POLLW) v = peek(f2 + idx);
        const unsigned nok = ~(unsigned)__ballot(v >= 4u * call);
        base += nok ? (unsigned)__builtin_ctz(nok) : (unsigned)POLLW;
        if (base >= (unsigned)NSTEP) {
          break; }
        lds_st(xaddr, base);
        if ((n & 63u) == 0u) {
          if (rfl(__hip_atomic_load(tmo, __ATOMIC_RELAXED, __HIP_MEMORY_SCOPE_AGENT)) == call || rt() - t0 > DEADLINE_TICKS) {
            __hip_atomic_store(tmo, call, __ATOMIC_RELAXED, __HIP_MEMORY_SCOPE_AGENT); dead = true; break; }
        }
        if (base >= (unsigned)(NSTEP - POLLEND) || base < (unsigned)POLLBEG) __builtin_amdgcn_s_sleep(POLLFAST); else __builtin_amdgcn_s_sleep(POLLSLEEP);
      }
    }
    lds_st(xaddr, INF);
    STAMP(6);
    if (rfl(__hip_atomic_load(tmo, __ATOMIC_RELAXED, __HIP_MEMORY_SCOPE_AGENT)) == call) dead = true;
    lds_st(lb + L_MISC + 4, dead ? 2u : 1u);
  } else {
    if (wave == 7) {
      for (int pf = PF0; pf < NSTEP; pf += PFB) wtouch(wp + (size_t)pf * 131072, pf + PFB <= NSTEP ? PFB : NSTEP - pf);
      STAMP(15);
    } else if (b < PW * (NR - 1)) {
      Pull q;
      const int qi = b % (NR - 1);
      q.s = (rank + 1 + qi) & (NR - 1);
      q.pr = __builtin_amdgcn_make_buffer_rsrc((void*)(p.ws[q.s] + PK_OFF + par * PK_BYTES), 0, (unsigned)PK_BYTES, 0x00020000);
      q.xw = __builtin_amdgcn_make_buffer_rsrc((void*)(my + XT_OFF + par * XT_BYTES), 0, (unsigned)XT_BYTES, 0x00020000);
      q.f1 = my + F1_OFF + par * F1_BYTES + (size_t)q.s * NUNIT * 128;
      q.my = my; q.tmo = tmo; q.call = call; q.lane = lane; q.pat = (LAS unsigned char*)lds + L_PAT;
      q.tick = (unsigned*)(my + CT_OFF + (2 + 2 * qi) * 128);
      bool ok = true;
      {
        q.tbase = (call - 1u) * (NJOB + PW);
        unsigned t = 0; unsigned long long t0old = 0, old = 0, fv = 0;
        if (lane == 0) t0old = peek64(q.tick + 32);
        if (t0old + 1500 > rt()) t0old = ~0ull;
        while (rfl(lds_ld(lb + L_MISC + 8)) < 4u) __builtin_amdgcn_s_sleep(1);
        if (lane == 0)
          asm volatile("global_atomic_add %0, %2, %3, off sc0\n\tglobal_atomic_add_x2 %1, %4, %5, off sc0 sc1\n\ts_waitcnt vmcnt(0)"
                       : "=&v"(t), "=&v"(fv) : "v"(q.tick), "v"(1u), "v"(q.f1), "v"(0ull) : "memory");
        q.j = rfl(t) - q.tbase; q.info = rfl((unsigned)(fv >> 32));
        STAMP(12);
        if (rfl((unsigned)fv) < call) ok = wait_f1(q.f1, call, tmo, q.info);
        STAMP(13);
        unsigned long long t0 = rt() + XMARG1;
        const u32x4 cs = {(unsigned)t0, (unsigned)(t0 >> 32), (unsigned)t0old, (unsigned)(t0old >> 32)};
        const unsigned jn = q.j < NJOB ? q.j : 0u;
        if (FASTGO && q.j == 0u) {
          if (lane == 0) asm volatile("global_atomic_cmpswap_x2 %0, %1, off\n\ts_nop 1" :: "v"(q.tick + 32), "v"(cs) : "memory");
          q.tgo = tstart; q.fcall = call;
        } else {
          if (lane == 0)
            asm volatile("global_atomic_cmpswap_x2 %0, %2, %3, off sc0\n\tglobal_atomic_add_x2 %1, %4, %5, off sc0 sc1\n\ts_waitcnt vmcnt(0)"
                         : "=&v"(old), "=&v"(fv) : "v"(q.tick + 32), "v"(cs), "v"(q.f1 + F1POS(jn)), "v"(0ull) : "memory");
          if (old != t0old) t0 = old;
          q.tgo = rfl((unsigned)t0) + jn * NB * TAU16 / 64; q.fcall = rfl((unsigned)fv); q.info = rfl((unsigned)(fv >> 32));
        }
      }
      while (ok && q.j < NJOB) ok = pull(q);
    }
  }
}

}

extern "C" void kernel_launch(int rank, int nranks, ncclComm_t comm, hipStream_t stream,
                              void* const* d_in, const int* in_sizes, int n_in,
                              void* d_out, int out_size, const int* in_dtypes, int out_dtype,
                              void* const* d_out_all, void* const* d_ws_all, size_t ws_bytes) {
  if (nranks != NR || (unsigned)rank >= (unsigned)NR || n_in != 2 || in_sizes[0] != ML * KD || in_sizes[1] != KD * NL ||
      out_size != MG * NL || ws_bytes < WS_NEED) { fprintf(stderr, "unexpected problem shape\n"); return; }
  static bool attr[NR] = {};
  if (!attr[rank]) { HIP_CHECK(hipFuncSetAttribute((const void*)ag_gemm, hipFuncAttributeMaxDynamicSharedMemorySize, LDS_BYTES)); attr[rank] = true; }
  Params p;
  p.x = d_in[0]; p.my = (char*)d_ws_all[rank]; p.w = d_in[1]; p.out = d_out; p.rank = rank; p.pad = 0;
  for (int i = 0; i < NR; ++i) p.ws[i] = (char*)d_ws_all[i];
  ag_gemm<<<GRID, BLOCK, LDS_BYTES, stream>>>(p);
  HIP_CHECK(hipGetLastError());
}
```

```cpp
#include <hip/hip_runtime.h>
#include <rccl/rccl.h>
#include <cstdio>

#define HIP_CHECK(expr) do { hipError_t _e = (expr); if (_e != hipSuccess) { \
    fprintf(stderr, "HIP error %s at %s:%d\n", hipGetErrorString(_e), __FILE__, __LINE__); \
    return; } } while (0)

namespace {

#define LAS __attribute__((address_space(3)))
#define DI __device__ __forceinline__
typedef short    s16x4  __attribute__((ext_vector_type(4)));
typedef short    s16x8  __attribute__((ext_vector_type(8)));
typedef __bf16   bf16x8 __attribute__((ext_vector_type(8)));
typedef __bf16   bf16x2 __attribute__((ext_vector_type(2)));
typedef float    f32x2  __attribute__((ext_vector_type(2)));
typedef float    f32x4  __attribute__((ext_vector_type(4)));
typedef unsigned u32x4  __attribute__((ext_vector_type(4)));
typedef unsigned short u16x2 __attribute__((ext_vector_type(2)));

constexpr int NR = 8;
constexpr int KD = 4096, NL = 2048, ML = 512, MG = NR * ML;
constexpr int NSTEP = KD / 32;
constexpr int NRG = ML / 64;
constexpr int NUNIT = NSTEP * 2;
constexpr int BBYTES = 2816;
constexpr int GRID = 256, BLOCK = 512;

#define NSET 4
#define NGRP 4
constexpr int RING = NSET * NGRP;
#define WPF 3
constexpr int WN = 8;
static_assert((NSTEP - 4) % 4 == 0 && WPF < WN && WPF >= 3 && WPF <= 4, "");
#define INFL 6
constexpr int PW = 36;
#define TAU16 312
#define POLLSLEEP 8
#define POLLFAST 0
#define POLLEND 6
#define RSPACE 0
#define SBITS "sc0 sc1"
#define NB 2
#define POLLW 16
#define OWN0 300
#define OWNGAP 48
#define PFB 8
#define WGAP 25
#define WPRED 4
#define XMARG 160
#define XMARG1 25
#define W0 4
#define WT0 500
#define OBITS "nt"
#define PF0 4
#define XGAP 12
#define POLLBEG 0
#define MAP 0
#define FASTGO 8
#define HPRIO 0
#define XFIRST (8 + NRAW)
#define XT0 0
#define F1SLEEP 4
#define LPRIO 0
#define CPRIO 0
#define DMAX 3
#define LHI 150
#define TAILP 40
#define PFX2 0
#define LSMOOTH 0
#define LSLEEP 2
#define NRAW 2
#define RAWT0 150
#define RAWGAP 16
#define XAUX 17

constexpr int L_FREE = RING * 8192, L_XRDY = L_FREE + 1024, L_WFULL = L_FREE + 2048, L_MISC = L_FREE + 3072;
constexpr int L_EPI = L_MISC + 1024;
constexpr int L_PAT = L_EPI + 4 * 4096;
constexpr int LDS_BYTES = L_PAT + 4096 + 64;
static_assert(LDS_BYTES <= 160 * 1024 && LDS_BYTES > 80 * 1024, "");
constexpr unsigned INF = 0x7fffffffu;

constexpr size_t XT_BYTES = (size_t)MG * KD * 2;
constexpr size_t XT_OFF = 0;
constexpr unsigned E0_REL = NUNIT * 4 * BBYTES;
constexpr unsigned E1_REL = E0_REL + NUNIT * 4 * 64;
constexpr unsigned HB_REL = E1_REL + NUNIT * 4 * 64;
constexpr size_t PK_BYTES = HB_REL + NUNIT * 4 * 2048;
constexpr size_t PK_OFF = XT_OFF + 2 * XT_BYTES;
constexpr size_t F1_BYTES = (size_t)NR * NUNIT * 128;
constexpr size_t F1_OFF = PK_OFF + 2 * PK_BYTES;
constexpr size_t F2_OFF = F1_OFF + 2 * F1_BYTES;
constexpr size_t F2_BYTES = (size_t)NR * NUNIT * 4;
constexpr size_t CT_OFF = F2_OFF + F2_BYTES;
constexpr size_t CC_OFF = CT_OFF + 4096;
constexpr size_t DBG_OFF = CC_OFF + GRID * 128;
constexpr size_t DBG_BYTES = 1 << 20;
constexpr size_t WS_NEED = DBG_OFF + DBG_BYTES;
constexpr unsigned long long DEADLINE_TICKS = 20000000ull;

struct Params { const void* x; char* my; const void* w; void* out; int rank; int pad; char* ws[NR]; };

DI unsigned rfl(unsigned v) { return __builtin_amdgcn_readfirstlane(v); }
DI unsigned long long rt() { return __builtin_amdgcn_s_memrealtime(); }
DI void st_sys(void* p, u32x4 v) { asm volatile("global_store_dwordx4 %0, %1, off sc0 sc1\n\ts_nop 1" :: "v"(p), "v"(v) : "memory"); }
DI void wait_vm0() { asm volatile("s_waitcnt vmcnt(0)" ::: "memory"); }
DI void lds_st(unsigned addr, unsigned v) { asm volatile("ds_write_b32 %0, %1" :: "v"(addr), "v"(v) : "memory"); }
DI unsigned lds_ld(unsigned addr) { unsigned v; asm volatile("ds_read_b32 %0, %1\n\ts_waitcnt lgkmcnt(0)" : "=&v"(v) : "v"(addr) : "memory"); return v; }
DI unsigned peek(const unsigned* p) { unsigned v; asm volatile("global_atomic_add %0, %1, %2, off sc0 sc1\n\ts_waitcnt vmcnt(0)" : "=&v"(v) : "v"(p), "v"(0u) : "memory"); return v; }
DI unsigned long long peek64(const void* p) { unsigned long long v; asm volatile("global_atomic_add_x2 %0, %1, %2, off sc0 sc1\n\ts_waitcnt vmcnt(0)" : "=&v"(v) : "v"(p), "v"(0ull) : "memory"); return v; }
DI void sleep_until(unsigned t) {
  for (int d; (d = (int)(t - (unsigned)rt())) > 0;) { if (d > 200) __builtin_amdgcn_s_sleep(48); else if (d > 40) __builtin_amdgcn_s_sleep(8); else __builtin_amdgcn_s_sleep(1); }
  asm volatile("" ::: "memory");
}
#define PIN() __builtin_amdgcn_sched_barrier(0)

#define STAMP(slot) do {} while (0)
#define STAMPC(slot) do {} while (0)

struct CW {
  alignas(64) f32x4 acc[4][8];
  u32x4 xs[NSET][4];
  s16x4 wf[WN][2];
  unsigned wa[4];
  unsigned fa, ra;
  unsigned xvo[NSET];
  unsigned need, rv;
  unsigned xso;
  unsigned grp;
  unsigned nfail = 0, nslow = 0, nfailw = 0;
  __amdgpu_buffer_rsrc_t xr;
};

DI void wait_ready(CW& c) {
  asm volatile(
      "v_cmp_lt_u32 vcc, %0, %3\n\t"
      "s_cbranch_vccnz 1f\n"
      "2:\n\t"
      ".subsection 1\n"
      "3:\n\t"
      "s_sleep 1\n"
      "1:\n\t"
      "ds_read_b32 %0, %4\n\t"
      "s_waitcnt lgkmcnt(0)\n\t"
      "v_cmp_lt_u32 vcc, %0, %3\n\t"
      "s_cbranch_vccnz 3b\n\t"
      "s_branch 2b\n\t"
      ".subsection 0"
      : "+v"(c.rv) : "i"(0), "i"(0), "v"(c.need), "v"(c.ra) : "vcc", "memory");
}

template <int SET, int M> DI void xload(CW& c) {
  c.xs[SET][M] = __builtin_amdgcn_raw_buffer_load_b128(c.xr, c.xvo[SET] + M * 1024, c.xso, XAUX);
}
template <int SL, int J, int T> DI void wread(CW& c) {
  c.wf[J % WN][T] = __builtin_amdgcn_ds_read_tr16_b64_v4i16((LAS s16x4*)((LAS char*)0 + c.wa[J & 3] + (SL * 8192 + (J >> 2) * 4096 + T * 1024)));
}
template <int S, int J, int M> DI void mma(CW& c) {
  constexpr int MM = (J & 1) ? 3 - M : M;
  const s16x8 w = __builtin_shufflevector(c.wf[J % WN][0], c.wf[J % WN][1], 0, 1, 2, 3, 4, 5, 6, 7);
  c.acc[MM][J] = __builtin_amdgcn_mfma_f32_16x16x32_bf16(__builtin_bit_cast(bf16x8, w), __builtin_bit_cast(bf16x8, c.xs[S][MM]), c.acc[MM][J], 0, 0, 0);
}
template <int S, int J> DI void frag(CW& c, unsigned wd) {
  constexpr int JP = (J + WPF) & 7;
  constexpr bool NXT = (J + WPF) >= 8;
  constexpr int SL = NXT ? (S + 1) % NSET : S;
  constexpr int LS = (S + NSET - 1) % NSET;
  mma<S, J, 0>(c); PIN();
  wread<SL, JP, 0>(c); PIN();
  mma<S, J, 1>(c); PIN();
  wread<SL, JP, 1>(c); PIN();
  mma<S, J, 2>(c); PIN();
  if constexpr (J < 4) xload<LS, J>(c);
  if constexpr (J == 4) { c.need += 1; lds_st(c.fa, c.need); }
  if constexpr (J == 5) c.rv = *(volatile LAS unsigned*)((LAS char*)0 + c.ra);
  if constexpr (J == 6 && S == 0) { c.xso += NSET * 64 * 4096; asm volatile("" : "+s"(c.xso)); }
  if constexpr (J == 7) wait_ready(c);
  PIN();
  mma<S, J, 3>(c); PIN();
  if constexpr (S == NSET - 1 && !NXT && JP >= 4) { c.wa[JP & 3] += wd; asm volatile("" : "+v"(c.wa[JP & 3])); }
  if constexpr (J & 1) {
    if constexpr (J == 7) asm volatile("" :: "v"(c.wf[(J + 2) % WN][1]), "v"(c.xs[(S + 1) % NSET][3]));
    else asm volatile("" :: "v"(c.wf[(J + 2) % WN][1]));
  }
  PIN();
}
template <int S> DI void step(CW& c, unsigned wd) {
  frag<S, 0>(c, wd); frag<S, 1>(c, wd); frag<S, 2>(c, wd); frag<S, 3>(c, wd);
  frag<S, 4>(c, wd); frag<S, 5>(c, wd); frag<S, 6>(c, wd); frag<S, 7>(c, wd);
}
template <int S> DI void pro_x(CW& c) {
  c.need = S + 1; wait_ready(c); PIN();
  xload<S, 0>(c); xload<S, 1>(c); xload<S, 2>(c); xload<S, 3>(c); PIN();
}

DI unsigned cvt_pk(float lo, float hi) { f32x2 v = {lo, hi}; return __builtin_bit_cast(unsigned, __builtin_convertvector(v, bf16x2)); }
DI float relu(float v) { return __builtin_elementwise_maximum(v, 0.0f); }

template <int S, int E> DI void steps(CW& c, unsigned wd) { if constexpr (S < E) { step<S>(c, wd); steps<S + 1, E>(c, wd); } }
template <int S, int E> DI void pros(CW& c) { if constexpr (S < E) { pro_x<S>(c); pros<S + 1, E>(c); } }
template <int J, int E> DI void prow(CW& c) { if constexpr (J < E) { wread<0, J, 0>(c); wread<0, J, 1>(c); prow<J + 1, E>(c); } }

DI unsigned dpp_shr(unsigned v, const int n) {
  switch (n) {
    case 1: return __builtin_amdgcn_update_dpp(0u, v, 0x111, 0xf, 0xf, true);
    case 2: return __builtin_amdgcn_update_dpp(0u, v, 0x112, 0xf, 0xf, true);
    case 4: return __builtin_amdgcn_update_dpp(0u, v, 0x114, 0xf, 0xf, true);
    default: return __builtin_amdgcn_update_dpp(0u, v, 0x118, 0xf, 0xf, true);
  }
}

DI bool wait_f1(const char* f, unsigned call, unsigned* tmo, unsigned& payload) {
  const unsigned long long t0 = rt();
  for (unsigned n = 1;; ++n) {
    unsigned long long v = 0;
    if (threadIdx.x % 64 == 0) v = peek64(f);
    if (rfl((unsigned)v) >= call) { payload = rfl((unsigned)(v >> 32)); return true; }
    if ((n & 63u) == 0u && (rfl(__hip_atomic_load(tmo, __ATOMIC_RELAXED, __HIP_MEMORY_SCOPE_AGENT)) == call || rt() - t0 > DEADLINE_TICKS)) {
      __hip_atomic_store(tmo, call, __ATOMIC_RELAXED, __HIP_MEMORY_SCOPE_AGENT); return false; }
    __builtin_amdgcn_s_sleep(F1SLEEP);
  }
}

struct Pull {
  __amdgpu_buffer_rsrc_t pr, xw; const char* f1; char* my; unsigned* tmo; unsigned* tick; LAS unsigned char* pat; unsigned* f2; unsigned call, tbase; int s, lane;
  unsigned j, tgo, fcall, info;
  unsigned t0 = 0, tlast = 0, lsum = 0, lmax = 0, lcnt = 0, rsum = 0, late = 0;
};
typedef unsigned u32x2 __attribute__((ext_vector_type(2)));
typedef unsigned u32x3 __attribute__((ext_vector_type(3)));
#define F1POS(j) ((((j) + J0) >> 1) * 128 + ((j) & 1u) * 8)
DI unsigned pk_add(unsigned a, unsigned b) { return __builtin_bit_cast(unsigned, (u16x2)(__builtin_bit_cast(u16x2, a) + __builtin_bit_cast(u16x2, b))); }
DI unsigned pk_sub(unsigned a, unsigned b) { return __builtin_bit_cast(unsigned, (u16x2)(__builtin_bit_cast(u16x2, a) - __builtin_bit_cast(u16x2, b))); }
constexpr unsigned J0 = NRAW * 8 / NB, NJOB = 4 * NUNIT / NB - J0;
DI bool pull(Pull& q) {
  constexpr int N = NB;
  static_assert(NB == 2, "");
  const unsigned k = (q.j + J0) * N, u = k >> 2;
  unsigned info = q.info;
  if (q.fcall < q.call && !wait_f1(q.f1 + F1POS(q.j), q.call, q.tmo, info)) return false;
  const unsigned n0 = (info >> 8) & 0xffu, n1 = info >> 24;
#pragma unroll
  for (int w = 0; w < N; ++w) {
    const unsigned q4 = ((info >> (16 * w + 1)) & 0x7cu) * 0x01010101u;
    *(LAS u32x4*)(q.pat + w * 2048 + q.lane * 16) = (u32x4){q4, q4, q4, q4};
    *(LAS u32x4*)(q.pat + w * 2048 + 1024 + q.lane * 16) = (u32x4){q4, q4, q4, q4};
  }
  if (q.j >= NJOB - TAILP) __builtin_amdgcn_s_setprio(2);
  sleep_until(q.tgo);
  unsigned tk;
  asm volatile("s_mov_b64 exec, 1\n\tglobal_atomic_add %0, %1, %2, off sc0\n\ts_mov_b64 exec, -1" : "=&v"(tk) : "v"(q.tick), "v"(1u) : "memory");
  PIN();
  const unsigned eidx = (q.lane & 32) + 2 * (q.lane & 15), en = (q.lane & 16) ? n1 : n0;
  const unsigned e = __builtin_amdgcn_raw_buffer_load_b32(q.pr, eidx < en ? (q.lane & 31) * 4u + (q.lane >> 5) * (E1_REL - E0_REL) : 0x80000000u, E0_REL + k * 64, 17);
  u32x4 lo[N][2], hb[N][2]; u32x3 pl[N];
#pragma unroll
  for (int w = 0; w < N; ++w) {
    lo[w][0] = __builtin_amdgcn_raw_buffer_load_b128(q.pr, q.lane * 16, (k + w) * BBYTES, 17);
    PIN(); __builtin_amdgcn_s_sleep(RSPACE); PIN();
    lo[w][1] = __builtin_amdgcn_raw_buffer_load_b128(q.pr, q.lane * 16 + 1024, (k + w) * BBYTES, 17);
    PIN(); __builtin_amdgcn_s_sleep(RSPACE); PIN();
    pl[w] = __builtin_amdgcn_raw_buffer_load_b96(q.pr, q.lane * 12 + 2048, (k + w) * BBYTES, 17);
    if (w < N - 1) { PIN(); __builtin_amdgcn_s_sleep(RSPACE); PIN(); }
  }
  PIN();
  {
    LAS unsigned char* const pb = q.pat + ((q.lane >> 4) & 1) * 2048;
    *(eidx < en ? pb + (e & 2047u) : q.pat + 4096 + q.lane) = (e >> 9) & 0x7cu;
    *(eidx + 1 < en ? pb + ((e >> 16) & 2047u) : q.pat + 4096 + q.lane) = (e >> 25) & 0x7cu;
  }
#pragma unroll
  for (int w = 0; w < N; ++w) { hb[w][0] = *(LAS u32x4*)(q.pat + w * 2048 + q.lane * 16); hb[w][1] = *(LAS u32x4*)(q.pat + w * 2048 + 1024 + q.lane * 16); }
  PIN();
  asm volatile("s_waitcnt vmcnt(0)" : "+v"(tk) : "v"(pl[N - 1]) : "memory");
  if (max(n0, n1) == 255u) {
#pragma unroll
    for (int w = 0; w < N; ++w)
      if ((w ? n1 : n0) == 255u) {
        hb[w][0] = __builtin_amdgcn_raw_buffer_load_b128(q.pr, q.lane * 16, HB_REL + (k + w) * 2048, 17);
        hb[w][1] = __builtin_amdgcn_raw_buffer_load_b128(q.pr, q.lane * 16 + 1024, HB_REL + (k + w) * 2048, 17);
      }
    asm volatile("" :: "v"(hb[0][0]), "v"(hb[0][1]), "v"(hb[1][0]), "v"(hb[1][1]));
  }
  PIN();
  q.j = rfl(tk) - q.tbase;
  const unsigned jn = min(q.j, NJOB - 1), offn = jn * N * TAU16 / 64;
  const int over = (int)((unsigned)rt() - q.tgo) - LHI;
  unsigned long long t0 = over > 0 ? (over > DMAX ? DMAX : over) : 0, old, fv;
  asm volatile("s_mov_b64 exec, 1\n\tglobal_atomic_add_x2 %0, %2, %3, off sc0\n\t"
               "global_atomic_add_x2 %1, %4, %5, off sc0 sc1\n\ts_mov_b64 exec, -1"
               : "=&v"(old), "=&v"(fv) : "v"(q.tick + 32), "v"(t0), "v"(q.f1 + F1POS(jn)), "v"(0ull) : "memory");
  const unsigned d = ((u >> 1) * 64 + q.s * NRG + (k & 7u)) * 4096;
#pragma unroll
  for (int w = 0; w < N; ++w) {
    const unsigned sh = ((info >> (16 * w)) & 7u) * 0x00800080u;
    const unsigned a3 = ((pl[w][0] << 1) & 0x88888888u) | ((pl[w][1] >> 1) & 0x22222222u) | ((pl[w][2] >> 2) & 0x11111111u);
#pragma unroll
    for (int m = 0; m < 4; ++m) {
      const unsigned a = m < 3 ? pl[w][m < 3 ? m : 0] : a3, l0 = lo[w][m >> 1][2 * (m & 1)], l1 = lo[w][m >> 1][2 * (m & 1) + 1];
      const unsigned h0 = (a & 0x83838383u) | hb[w][m >> 1][2 * (m & 1)], h1 = (__builtin_rotateright32(a, 4) & 0x83838383u) | hb[w][m >> 1][2 * (m & 1) + 1];
      const u32x4 o = {pk_sub(__builtin_amdgcn_perm(h0, l0, 0x05010400u), sh), pk_sub(__builtin_amdgcn_perm(h0, l0, 0x07030602u), sh),
                       pk_sub(__builtin_amdgcn_perm(h1, l1, 0x05010400u), sh), pk_sub(__builtin_amdgcn_perm(h1, l1, 0x07030602u), sh)};
      asm volatile("buffer_store_dwordx4 %0, %1, %2, %3 offen offset:%4 " SBITS "\n\ts_nop 1"
                   :: "v"(o), "v"(q.lane * 16u), "s"(q.xw), "s"(d + w * 4096), "i"(m * 1024) : "memory");
    }
  }
  asm volatile("s_waitcnt vmcnt(0)" : "+v"(old), "+v"(fv) :: "memory");
  asm volatile("s_mov_b64 exec, 1\n\tglobal_atomic_add %0, %1, off sc1\n\ts_mov_b64 exec, -1"
               :: "v"(q.f2 + (u & 1) * NSTEP + (u >> 1)), "v"((unsigned)N) : "memory");
  t0 += old;
  q.tgo = rfl((unsigned)t0) + offn; q.fcall = rfl((unsigned)fv); q.info = rfl((unsigned)(fv >> 32));
  return true;
}

DI void wfill(__amdgpu_buffer_rsrc_t wr, unsigned ldsaddr, unsigned vo, unsigned so) {
#pragma unroll
  for (int i = 0; i < 8; ++i)
    asm volatile("s_mov_b32 m0, %3\n\ts_nop 0\n\tbuffer_load_dwordx4 %0, %1, %2 offen lds"
                 :: "v"(vo), "s"(wr), "s"(so + (16 * ((i >> 1) & 1) + 4 * (i & 1)) * (NL * 2) + (i >> 2) * 128), "s"(ldsaddr + i * 1024) : "memory");
}

DI void wtouch(const char* p, unsigned n) {
  unsigned o = 0;
  unsigned t;
  asm volatile(
      "1:\n\t"
      "s_load_dword %0, %3, %1 offset:0\n\ts_load_dword %0, %3, %1 offset:64\n\ts_load_dword %0, %3, %1 offset:128\n\ts_load_dword %0, %3, %1 offset:192\n\t"
      "s_load_dword %0, %3, %1 offset:256\n\ts_load_dword %0, %3, %1 offset:320\n\ts_load_dword %0, %3, %1 offset:384\n\ts_load_dword %0, %3, %1 offset:448\n\t"
      "s_add_u32 %1, %1, 0x20000\n\t"
      "s_sub_u32 %2, %2, 1\n\t"
      "s_cmp_lg_u32 %2, 0\n\t"
      "s_cbranch_scc1 1b\n\t"
      "s_waitcnt lgkmcnt(0)" : "=&s"(t), "+s"(o), "+s"(n) : "s"(p) : "scc", "memory");
}

__global__ __launch_bounds__(BLOCK, 2) void ag_gemm(Params p) {
  extern __shared__ __attribute__((aligned(16))) char lds[];
  const int lane = threadIdx.x & 63;
  const int wave = rfl(threadIdx.x >> 6);
  const unsigned lb = (unsigned)(size_t)(LAS char*)lds;
  const int b = blockIdx.x, rank = p.rank & (NR - 1);
  const char* xin = (const char*)p.x; char* const my = p.my;
  asm volatile("" : "+s"(xin) : "s"(my));
  const int xcd = b & 7, bi = b >> 3;
  const int tm = (xcd >> 1) * 4 + (bi >> 3), tn = (xcd & 1) * 8 + (bi & 7);
  unsigned* const tmo = (unsigned*)(my + CT_OFF);
  const unsigned tstart = (unsigned)rt();
  if (wave == 0) STAMP(0);

  const int ppair = (bi >> 1) * 8 + xcd;
  constexpr int RAW0 = GRID - NRAW * NRG * (NR - 1);
  const int rblk = (b - RAW0) / (NR - 1);
  const bool pusher = NRAW > 0 && wave == 6 && b >= RAW0;
  const int pstep = pusher ? rblk >> 3 : 2 * (ppair >> 1) + (bi & 1), prg = pusher ? rblk & 7 : (ppair & 1) * 4 + wave;
  u32x4 pv[4];
  const char* const wp = (const char*)p.w + b * 512;
  if (wave < 4 || pusher) {
    if (XGAP && pstep >= XFIRST) sleep_until(tstart + XT0 + pstep * XGAP / 16);
    const auto xr = __builtin_amdgcn_make_buffer_rsrc((void*)xin, 0, ML * KD * 2, 0x00020000);
#pragma unroll
    for (int m = 0; m < 4; ++m) pv[m] = __builtin_amdgcn_raw_buffer_load_b128(xr, (lane & 15) * (KD * 2) + (lane >> 4) * 16, ((prg * 64 + m * 16) * KD + pstep * 32) * 2, 0);
  }
  const auto wr = __builtin_amdgcn_make_buffer_rsrc((void*)p.w, 0, KD * NL * 2, 0x00020000);
  const unsigned wvo = (8 * ((lane >> 3) & 1) + (lane >> 4)) * (NL * 2) + ((((lane >> 1) & 3) ^ (lane >> 4)) * 32) + (lane & 1) * 16;
  if (wave == 4) {
    __builtin_amdgcn_s_sleep(WPRED);
    for (int s = 0; s < W0; ++s) wfill(wr, lb + s * 8192, wvo, s * 32 * (NL * 2) + tn * 256);
  }
  if (wave == 6) {
    unsigned t = 0;
    if (lane == 0) t = __hip_atomic_fetch_add((unsigned*)(my + CC_OFF + b * 128), 1u, __ATOMIC_RELAXED, __HIP_MEMORY_SCOPE_AGENT);
    t = rfl(t) + 1;
    if (lane < 16) lds_st(lb + L_MISC + lane * 4, lane == 0 ? t : 0u);
  } else if (wave == 4 || wave == 5) {
#pragma unroll
    for (int i = 0; i < 6; ++i) lds_st(lb + L_FREE + (wave - 4) * 1536 + i * 256 + lane * 4, 0u);
  }
  asm volatile("s_waitcnt lgkmcnt(0)" ::: "memory");
  __builtin_amdgcn_s_barrier();
  asm volatile("" ::: "memory");
  const unsigned call = rfl(lds_ld(lb + L_MISC));
  const unsigned par = call & 1u;
  if (wave == 0) STAMP(14);
  if (wave == 4) __builtin_amdgcn_s_setprio(LPRIO);
  if (wave == 5) __builtin_amdgcn_s_setprio(HPRIO);
  if (wave == 6) __builtin_amdgcn_s_setprio(2);
  if (wave < 4) __builtin_amdgcn_s_setprio(3);

  if (wave < 4) {
    if (pstep < NRAW) {
      asm volatile("s_mov_b64 exec, 1\n\tds_add_u32 %0, %1\n\ts_mov_b64 exec, -1" :: "v"(lb + L_MISC + 8), "v"(1u) : "memory");
    } else {
      unsigned v = 0;
#pragma unroll
      for (int m = 0; m < 4; ++m) { v = __builtin_amdgcn_sad_u8(pv[m][0] & 0x7f007f00u, 0u, v); v = __builtin_amdgcn_sad_u8(pv[m][2] & 0x7f007f00u, 0u, v); }
      v += dpp_shr(v, 1); v += dpp_shr(v, 2); v += dpp_shr(v, 4); v += dpp_shr(v, 8);
      const unsigned sum = __builtin_amdgcn_readlane(v, 15) + __builtin_amdgcn_readlane(v, 31) + __builtin_amdgcn_readlane(v, 47) + __builtin_amdgcn_readlane(v, 63);
      const unsigned top = (sum + 1705u) >> 9, low = min(top > 7u ? top - 7u : 0u, 248u);
      const unsigned sh = (0u - low) & 7u, qw = (low + sh) >> 3, q4 = (qw << 2) * 0x01010101u;
      u32x4 lo[2], hi[2]; unsigned a[4], mask = 0;
#pragma unroll
      for (int m = 0; m < 4; ++m) {
        const unsigned d0 = pk_add(pv[m][0], sh * 0x00800080u), d1 = pk_add(pv[m][1], sh * 0x00800080u), d2 = pk_add(pv[m][2], sh * 0x00800080u), d3 = pk_add(pv[m][3], sh * 0x00800080u);
        const unsigned h0 = __builtin_amdgcn_perm(d1, d0, 0x07050301u), h1 = __builtin_amdgcn_perm(d3, d2, 0x07050301u);
        lo[m >> 1][2 * (m & 1)] = __builtin_amdgcn_perm(d1, d0, 0x06040200u); lo[m >> 1][2 * (m & 1) + 1] = __builtin_amdgcn_perm(d3, d2, 0x06040200u);
        hi[m >> 1][2 * (m & 1)] = h0; hi[m >> 1][2 * (m & 1) + 1] = h1;
        a[m] = (h0 & 0x83838383u) | __builtin_rotateleft32(h1 & 0x83838383u, 4);
        mask |= ((((h0 ^ q4) & 0x7c7c7c7cu) + 0x7c7c7c7cu) & 0x80808080u) >> (7 - 2 * m);
        mask |= ((((h1 ^ q4) & 0x7c7c7c7cu) + 0x7c7c7c7cu) & 0x80808080u) >> (6 - 2 * m);
      }
      if (wave == 0) STAMP(7);
      const int u = pstep * 2 + (prg >> 2), blk = u * 4 + wave;
      char* const pk = my + PK_OFF + par * PK_BYTES;
      st_sys(pk + blk * BBYTES + lane * 16, lo[0]); st_sys(pk + blk * BBYTES + 1024 + lane * 16, lo[1]);
      const u32x3 pl = {a[0] | ((a[3] >> 1) & 0x44444444u), a[1] | ((a[3] << 1) & 0x44444444u), a[2] | ((a[3] << 2) & 0x44444444u)};
      asm volatile("global_store_dwordx3 %0, %1, off sc0 sc1\n\ts_nop 1" :: "v"(pk + blk * BBYTES + 2048 + lane * 12), "v"(pl) : "memory");
      LAS unsigned char* const hcopy = (LAS unsigned char*)lds + L_EPI + wave * 4096;
      LAS unsigned short* const list = (LAS unsigned short*)(hcopy + 2048);
      *(LAS u32x4*)(hcopy + lane * 16) = hi[0]; *(LAS u32x4*)(hcopy + 1024 + lane * 16) = hi[1];
      unsigned cnt = 0;
      do {
#pragma unroll
        for (int i = 0; i < 4; ++i) {
          const unsigned long long bal = __ballot(mask != 0u);
          const unsigned idx = cnt + __builtin_amdgcn_mbcnt_hi((unsigned)(bal >> 32), __builtin_amdgcn_mbcnt_lo((unsigned)bal, 0u));
          list[mask ? idx : 512u + lane] = (unsigned short)(__builtin_ctz(mask | 0x80000000u) | (lane << 5));
          mask &= mask - 1u; cnt += __builtin_popcountll(bal);
        }
      } while (cnt <= 64u && __any(mask != 0u));
      if (cnt > 64u) {
        cnt = 255u;
        st_sys(pk + HB_REL + blk * 2048 + lane * 16, hi[0]); st_sys(pk + HB_REL + blk * 2048 + 1024 + lane * 16, hi[1]);
      } else {
        const unsigned t = list[lane], pos = ((t & 4u) << 8) | ((t >> 1) & 0x3f0u) | ((t & 3u) << 2) | ((t >> 3) & 3u);
        const unsigned e = pos | ((hcopy[pos] & 0x7cu) << 9);
        const unsigned e2 = e | (__builtin_amdgcn_update_dpp(0u, e, 0xb1, 0xf, 0xf, true) << 16);
        if ((unsigned)lane < cnt && !(lane & 1))
          __hip_atomic_store((unsigned*)(pk + (lane < 32 ? E0_REL : E1_REL - 64) + blk * 64 + lane * 2), e2, __ATOMIC_RELAXED, __HIP_MEMORY_SCOPE_SYSTEM);
      }
      const unsigned info = sh | (qw << 3) | (cnt << 8);
      wait_vm0();
      unsigned old;
      asm volatile("s_mov_b64 exec, 1\n\tds_write_b16 %1, %2\n\tds_add_rtn_u32 %0, %3, %4\n\ts_mov_b64 exec, -1\n\ts_waitcnt lgkmcnt(0)"
                   : "=&v"(old) : "v"(lb + L_MISC + 16 + 2 * wave), "v"(info), "v"(lb + L_MISC + 8), "v"(1u) : "memory");
      if (rfl(old) == 3u) {
        const unsigned pair = lds_ld(lb + L_MISC + 16 + (lane & 1) * 4);
        if (lane < 2 * NR && (lane >> 1) != rank)
          __hip_atomic_store((unsigned long long*)(p.ws[lane >> 1] + F1_OFF + par * F1_BYTES + (size_t)(rank * NUNIT + u) * 128 + (lane & 1) * 8),
                             ((unsigned long long)pair << 32) | call, __ATOMIC_RELAXED, __HIP_MEMORY_SCOPE_SYSTEM);
        STAMP(8);
      }
    }
    __builtin_amdgcn_s_setprio(CPRIO);
    {
      sleep_until(tstart + OWN0 + pstep * OWNGAP / 16);
      char* d = my + XT_OFF + par * XT_BYTES + (size_t)(pstep * 64 + rank * NRG + prg) * 4096 + lane * 16;
#pragma unroll
      for (int m = 0; m < 4; ++m) st_sys(d + m * 1024, pv[m]);
      wait_vm0();
      asm volatile("s_mov_b64 exec, 1\n\tglobal_atomic_add %0, %1, off sc1\n\ts_mov_b64 exec, -1"
                   :: "v"((unsigned*)(my + F2_OFF) + (rank * 2 + (prg >> 2)) * NSTEP + pstep), "v"(1u) : "memory");
    }
    if (wave == 0) STAMP(1);
    CW c;
    char* outp = (char*)p.out; asm volatile("" : "+s"(outp));
    const int g = lane >> 4, q = (lane >> 2) & 3, pp = lane & 3;
#pragma unroll
    for (int m = 0; m < 4; ++m)
#pragma unroll
      for (int j = 0; j < 8; ++j) c.acc[m][j] = (f32x4){0.f, 0.f, 0.f, 0.f};
#pragma unroll
    for (int i = 0; i < 4; ++i) c.wa[i] = lb + (g >> 1) * 2048 + q * 256 + (g & 1) * 128 + ((i ^ q) * 32) + pp * 8;
    c.fa = lb + L_FREE + wave * 256 + lane * 4; c.ra = lb + L_XRDY + lane * 4;
    c.rv = 0; c.grp = 0;
    c.xr = __builtin_amdgcn_make_buffer_rsrc((void*)(my + XT_OFF + par * XT_BYTES), 0, (unsigned)XT_BYTES, 0x00020000);
#pragma unroll
    for (int i = 0; i < NSET; ++i) { c.xvo[i] = lane * 16 + i * 64 * 4096; asm volatile("" : "+v"(c.xvo[i])); }
    c.xso = (tm * 4 + wave) * 4096;
    PIN();
    pros<0, NSET - 1>(c);
    prow<0, WPF>(c);
    c.need = NSET; asm volatile("" : "+v"(c.need));
    wait_ready(c);
    PIN();
    if (wave == 0) { STAMP(2); STAMPC(16); }
    c.nfail = c.nfailw = c.nslow = 0;
#pragma unroll 1
    for (int it = 0; it < NSTEP / NSET; ++it)
      steps<0, NSET>(c, (it + 1) % NGRP == 0 ? (unsigned)(-(NGRP - 1) * NSET * 8192) : (unsigned)(NSET * 8192));
    steps<0, NSTEP % NSET>(c, 0u);
    if (wave == 0) { STAMP(3); STAMPC(17); }
    unsigned vd = *(volatile LAS unsigned*)((LAS char*)lds + L_MISC + 4);
    int tz = threadIdx.x; asm volatile("" : "+v"(tz));
    const int l2 = tz & 63;
    char* op = outp + ((size_t)(tm * 256 + wave * 64 + (l2 >> 4)) * NL + tn * 128 + (l2 & 15) * 8) * 2;
    const unsigned ovo = (l2 >> 4) * (NL * 2) + (l2 & 15) * 16;
    const auto orr = __builtin_amdgcn_make_buffer_rsrc((void*)(outp + ((size_t)(tm * 256 + wave * 64) * NL + tn * 128) * 2), 0, 64 * NL * 2, 0x00020000);
    LAS char* const eb = (LAS char*)lds + L_EPI + wave * 4096;
    const unsigned ew = (l2 & 15) * 256 + ((l2 >> 4) & 1) * 8 + (((l2 >> 5) ^ (l2 & 15)) * 16), er = (l2 >> 4) * 256 + (((l2 & 15) ^ (l2 >> 4)) * 16);
    unsigned verdict = rfl(vd);
    while (verdict == 0u) { __builtin_amdgcn_s_sleep(1); verdict = rfl(lds_ld(lb + L_MISC + 4)); }
    if (verdict == 1u) {
      u32x4 o[4];
#pragma unroll
      for (int m = 0; m <= 4; ++m) {
        u32x2 t[8];
#pragma unroll
        for (int h = 0; h < 4; ++h) {
          if (m < 4) {
            f32x4 a = c.acc[m][2 * h], d = c.acc[m][2 * h + 1];
#pragma unroll
            for (int i = 0; i < 4; ++i) { a[i] = relu(a[i]); d[i] = relu(d[i]); }
            PIN();
            t[2 * h] = (u32x2){cvt_pk(a[0], a[1]), cvt_pk(a[2], a[3])}; t[2 * h + 1] = (u32x2){cvt_pk(d[0], d[1]), cvt_pk(d[2], d[3])};
            PIN();
          }
          if (m > 0)
            asm volatile("buffer_store_dwordx4 %0, %1, %2, %3 offen " OBITS "\n\ts_nop 1" :: "v"(o[h]), "v"(ovo), "s"(orr), "s"(((m - 1) * 16 + 4 * h) * NL * 2));
          if (m < 4) { *(LAS u32x2*)(eb + (ew ^ (64 * h))) = t[2 * h]; *(LAS u32x2*)(eb + (ew ^ (64 * h + 32))) = t[2 * h + 1]; }
          PIN();
        }
        if (m < 4) {
#pragma unroll
          for (int i = 0; i < 4; ++i) o[i] = *(const LAS u32x4*)(eb + (er ^ (64 * i)) + 1024 * i);
          PIN();
        }
      }
    } else {
#pragma unroll 1
      for (int i = 0; i < 16; ++i) st_sys(op + (size_t)(4 * i) * NL * 2, (u32x4){0x7fc07fc0u, 0x7fc07fc0u, 0x7fc07fc0u, 0x7fc07fc0u});
    }
  } else if (wave == 4) {
    const unsigned pubaddr = lb + L_XRDY + lane * 8 + 4;
    wait_vm0();
    int pub = W0, slot = W0; unsigned freec = 0;
    int occ = 0, occmin = 99; unsigned tfill = 0;
    lds_st(pubaddr, pub + NSET - 2);
    for (int s = W0; s < NSTEP; ++s) {
      if (s >= RING && freec < (unsigned)(s - RING + 1 + NSET)) {
        if (!LSMOOTH && pub < s) { wait_vm0(); pub = s; lds_st(pubaddr, pub + NSET - 2); }
        for (;;) {
          const unsigned f = lds_ld(lb + L_FREE + (lane & 3) * 256);
          freec = min(min(__builtin_amdgcn_readlane(f, 0), __builtin_amdgcn_readlane(f, 1)), min(__builtin_amdgcn_readlane(f, 2), __builtin_amdgcn_readlane(f, 3)));
          if (freec >= (unsigned)(s - RING + 1 + NSET)) break;
          if (LSMOOTH && pub < s) {
            const unsigned fm = max(max(__builtin_amdgcn_readlane(f, 0), __builtin_amdgcn_readlane(f, 1)), max(__builtin_amdgcn_readlane(f, 2), __builtin_amdgcn_readlane(f, 3)));
            if (fm + 4u >= (unsigned)(pub + NSET)) { wait_vm0(); pub = s; lds_st(pubaddr, pub + NSET - 2); }
          }
          __builtin_amdgcn_s_sleep(LSLEEP);
        }
      }
      if (s < RING) {
        const unsigned tgo = tstart + WT0 + (s - W0) * WGAP;
        if ((int)(tgo - (unsigned)rt()) > 0) { if (pub < s) { wait_vm0(); pub = s; lds_st(pubaddr, pub + NSET - 2); } sleep_until(tgo); }
      }
      wfill(wr, lb + slot * 8192, wvo, s * 32 * (NL * 2) + tn * 256);
      slot = slot == RING - 1 ? 0 : slot + 1;
      if (s + 1 - pub > INFL) { asm volatile("s_waitcnt vmcnt(%0)" :: "i"(8 * INFL) : "memory"); pub = s + 1 - INFL; lds_st(pubaddr, pub + NSET - 2); }
    }
    wait_vm0(); lds_st(pubaddr, INF);
  } else if (wave == 6) {
    if (pusher) {
      char* const peer = p.ws[(rank + 1 + (b - RAW0) % (NR - 1)) & (NR - 1)];
      sleep_until(tstart + RAWT0 + rblk * RAWGAP / 4);
      char* d = peer + XT_OFF + par * XT_BYTES + (size_t)(pstep * 64 + rank * NRG + prg) * 4096 + lane * 16;
#pragma unroll
      for (int m = 0; m < 4; ++m) st_sys(d + m * 1024, pv[m]);
      wait_vm0();
      asm volatile("s_mov_b64 exec, 1\n\tglobal_atomic_add %0, %1, off sc1\n\ts_mov_b64 exec, -1"
                   :: "v"((unsigned*)(peer + F2_OFF) + (rank * 2 + (prg >> 2)) * NSTEP + pstep), "v"(1u) : "memory");
      STAMP(11);
    }
    const unsigned xaddr = lb + L_XRDY + lane * 8;
    bool dead = false;
    {
      const unsigned* f2 = (const unsigned*)(my + F2_OFF) + tm * NSTEP;
      unsigned base = 0;
      const unsigned long long t0 = rt();
      for (unsigned n = 1;; ++n) {
        const unsigned idx = base + lane;
        unsigned v = 4u * call;
        if (idx < (unsigned)NSTEP && lane < POLLW) v = peek(f2 + idx);
        const unsigned nok = ~(unsigned)__ballot(v >= 4u * call);
        base += nok ? (unsigned)__builtin_ctz(nok) : (unsigned)POLLW;
        if (base >= (unsigned)NSTEP) {
          break; }
        lds_st(xaddr, base);
        if ((n & 63u) == 0u) {
          if (rfl(__hip_atomic_load(tmo, __ATOMIC_RELAXED, __HIP_MEMORY_SCOPE_AGENT)) == call || rt() - t0 > DEADLINE_TICKS) {
            __hip_atomic_store(tmo, call, __ATOMIC_RELAXED, __HIP_MEMORY_SCOPE_AGENT); dead = true; break; }
        }
        if (base >= (unsigned)(NSTEP - POLLEND) || base < (unsigned)POLLBEG) __builtin_amdgcn_s_sleep(POLLFAST); else __builtin_amdgcn_s_sleep(POLLSLEEP);
      }
    }
    lds_st(xaddr, INF);
    STAMP(6);
    if (rfl(__hip_atomic_load(tmo, __ATOMIC_RELAXED, __HIP_MEMORY_SCOPE_AGENT)) == call) dead = true;
    lds_st(lb + L_MISC + 4, dead ? 2u : 1u);
  } else {
    if (wave == 7) {
      for (int pf = PF0; pf < NSTEP; pf += PFB) wtouch(wp + (size_t)pf * 131072, pf + PFB <= NSTEP ? PFB : NSTEP - pf);
      STAMP(15);
    } else if (b < PW * (NR - 1)) {
      Pull q;
      const int qi = b % (NR - 1);
      q.s = (rank + 1 + qi) & (NR - 1);
      q.my = my; q.tmo = tmo; q.call = call; q.lane = lane; q.pat = (LAS unsigned char*)lds + L_PAT;
      unsigned* const tick = (unsigned*)(my + CT_OFF + (2 + 2 * qi) * 128);
      unsigned long long t0old = 0;
      if (lane == 0) t0old = peek64(tick + 32);
      if (t0old + 1500 > rt()) t0old = ~0ull;
      bool ok = true;
      {
        q.pr = __builtin_amdgcn_make_buffer_rsrc((void*)(p.ws[q.s] + PK_OFF + par * PK_BYTES), 0, (unsigned)PK_BYTES, 0x00020000);
        q.xw = __builtin_amdgcn_make_buffer_rsrc((void*)(my + XT_OFF + par * XT_BYTES), 0, (unsigned)XT_BYTES, 0x00020000);
        q.f1 = my + F1_OFF + par * F1_BYTES + (size_t)q.s * NUNIT * 128;
        q.f2 = (unsigned*)(my + F2_OFF) + q.s * 2 * NSTEP;
        q.tick = tick;
        q.tbase = (call - 1u) * (NJOB + PW);
        unsigned t = 0; unsigned long long old = 0, fv = 0;
        while (rfl(lds_ld(lb + L_MISC + 8)) < 4u) __builtin_amdgcn_s_sleep(1);
        if (lane == 0)
          asm volatile("global_atomic_add %0, %2, %3, off sc0\n\tglobal_atomic_add_x2 %1, %4, %5, off sc0 sc1\n\ts_waitcnt vmcnt(0)"
                       : "=&v"(t), "=&v"(fv) : "v"(q.tick), "v"(1u), "v"(q.f1 + F1POS(0u)), "v"(0ull) : "memory");
        q.j = rfl(t) - q.tbase; q.info = rfl((unsigned)(fv >> 32)); q.fcall = rfl((unsigned)fv);
        STAMP(12);
        const unsigned jn = q.j < NJOB ? q.j : 0u;
        const bool lead = jn < FASTGO && (q.fcall < call || jn == 0u);
        if (q.fcall < call) ok = wait_f1(q.f1 + F1POS(lead ? jn : 0u), call, tmo, q.info);
        STAMP(13);
        unsigned long long t0 = rt() + XMARG1;
        const u32x4 cs = {(unsigned)t0, (unsigned)(t0 >> 32), (unsigned)t0old, (unsigned)(t0old >> 32)};
        if (lead) {
          if (lane == 0) asm volatile("global_atomic_cmpswap_x2 %0, %1, off\n\ts_nop 1" :: "v"(q.tick + 32), "v"(cs) : "memory");
          t0 -= XMARG1; q.fcall = call;
        } else {
          if (lane == 0)
            asm volatile("global_atomic_cmpswap_x2 %0, %2, %3, off sc0\n\tglobal_atomic_add_x2 %1, %4, %5, off sc0 sc1\n\ts_waitcnt vmcnt(0)"
                         : "=&v"(old), "=&v"(fv) : "v"(q.tick + 32), "v"(cs), "v"(q.f1 + F1POS(jn)), "v"(0ull) : "memory");
          if (old != t0old) t0 = old;
          q.fcall = rfl((unsigned)fv); q.info = rfl((unsigned)(fv >> 32));
        }
        q.tgo = rfl((unsigned)t0) + jn * NB * TAU16 / 64;
      }
      while (ok && q.j < NJOB) ok = pull(q);
    }
  }
}

}

extern "C" void kernel_launch(int rank, int nranks, ncclComm_t comm, hipStream_t stream,
                              void* const* d_in, const int* in_sizes, int n_in,
                              void* d_out, int out_size, const int* in_dtypes, int out_dtype,
                              void* const* d_out_all, void* const* d_ws_all, size_t ws_bytes) {
  if (nranks != NR || (unsigned)rank >= (unsigned)NR || n_in != 2 || in_sizes[0] != ML * KD || in_sizes[1] != KD * NL ||
      out_size != MG * NL || ws_bytes < WS_NEED) { fprintf(stderr, "unexpected problem shape\n"); return; }
  static bool attr[NR] = {};
  if (!attr[rank]) { HIP_CHECK(hipFuncSetAttribute((const void*)ag_gemm, hipFuncAttributeMaxDynamicSharedMemorySize, LDS_BYTES)); attr[rank] = true; }
  Params p;
  p.x = d_in[0]; p.my = (char*)d_ws_all[rank]; p.w = d_in[1]; p.out = d_out; p.rank = rank; p.pad = 0;
  for (int i = 0; i < NR; ++i) p.ws[i] = (char*)d_ws_all[i];
  ag_gemm<<<GRID, BLOCK, LDS_BYTES, stream>>>(p);
  HIP_CHECK(hipGetLastError());
}
```

```cpp
#include <hip/hip_runtime.h>
#include <rccl/rccl.h>
#include <cstdio>

#define HIP_CHECK(expr) do { hipError_t _e = (expr); if (_e != hipSuccess) { \
    fprintf(stderr, "HIP error %s at %s:%d\n", hipGetErrorString(_e), __FILE__, __LINE__); \
    return; } } while (0)

namespace {

#define LAS __attribute__((address_space(3)))
#define DI __device__ __forceinline__
#define PIN() __builtin_amdgcn_sched_barrier(0)
#define FENCE() asm volatile("" ::: "memory")
typedef short    s16x4  __attribute__((ext_vector_type(4)));
typedef short    s16x8  __attribute__((ext_vector_type(8)));
typedef __bf16   bf16x8 __attribute__((ext_vector_type(8)));
typedef __bf16   bf16x2 __attribute__((ext_vector_type(2)));
typedef float    f32x2  __attribute__((ext_vector_type(2)));
typedef float    f32x4  __attribute__((ext_vector_type(4)));
typedef unsigned u32x2  __attribute__((ext_vector_type(2)));
typedef unsigned u32x3  __attribute__((ext_vector_type(3)));
typedef unsigned u32x4  __attribute__((ext_vector_type(4)));
typedef unsigned short u16x2 __attribute__((ext_vector_type(2)));
typedef unsigned long long u64;

constexpr int NR = 8;
constexpr int KD = 4096, NL = 2048, ML = 512, MG = NR * ML;
constexpr int NSTEP = KD / 32;
constexpr int NRG = ML / 64;
constexpr int NUNIT = NSTEP * 2;
constexpr int BBYTES = 2816;
constexpr int GRID = 256, BLOCK = 512;
constexpr int RING = 16;
constexpr int INFL = 6;
constexpr int W0 = 4;
constexpr int PW = 36;
constexpr int NRAW = 2;
constexpr unsigned J0 = NRAW * 4, NJOB = NUNIT * 2 - J0;

#define TAU16 312
#define LHI 165
constexpr int DMAX = 3;
constexpr int T0MARG = 25, FASTGO = 8, TAILP = 40;
constexpr int XGAP = 12, XFIRST = 8 + NRAW;
constexpr int OWN0 = 300, OWNGAP = 48;
constexpr int RAWT0 = 150, RAWGAP = 16;
constexpr int WT0 = 500, WGAP = 25;
constexpr int PFB = 8;
constexpr int POLLW = 16, POLLEND = 6;

constexpr int L_FREE = RING * 8192;
constexpr int L_READY = L_FREE + 1024;
constexpr int L_MISC = L_FREE + 2048;
constexpr int L_EPI = L_MISC + 1024;
constexpr int L_PAT = L_EPI + 4 * 4096;
constexpr int LDS_BYTES = L_PAT + 4096 + 64;
static_assert(LDS_BYTES <= 160 * 1024 && LDS_BYTES > 80 * 1024, "one workgroup per CU");
constexpr unsigned INF = 0x7fffffffu;

constexpr size_t XT_BYTES = (size_t)MG * KD * 2;
constexpr size_t XT_OFF = 0;
constexpr unsigned E0_REL = NUNIT * 4 * BBYTES;
constexpr unsigned E1_REL = E0_REL + NUNIT * 4 * 64;
constexpr unsigned HB_REL = E1_REL + NUNIT * 4 * 64;
constexpr size_t PK_BYTES = HB_REL + NUNIT * 4 * 2048;
constexpr size_t PK_OFF = XT_OFF + 2 * XT_BYTES;
constexpr size_t F1_BYTES = (size_t)NR * NUNIT * 128;
constexpr size_t F1_OFF = PK_OFF + 2 * PK_BYTES;
constexpr size_t F2_OFF = F1_OFF + 2 * F1_BYTES;
constexpr size_t F2_BYTES = (size_t)NR * NUNIT * 4;
constexpr size_t CT_OFF = F2_OFF + F2_BYTES;
constexpr size_t CC_OFF = CT_OFF + 4096;
constexpr size_t WS_NEED = CC_OFF + GRID * 128;
constexpr u64 DEADLINE_TICKS = 20000000ull;
#define F1POS(j) ((((j) + J0) >> 1) * 128 + ((j) & 1u) * 8)

struct Params { const void* x; char* my; const void* w; void* out; int rank; int pad; char* ws[NR]; };

DI unsigned rfl(unsigned v) { return __builtin_amdgcn_readfirstlane(v); }
DI unsigned rl(unsigned v, int l) { return __builtin_amdgcn_readlane(v, l); }
DI u64 rt() { return __builtin_amdgcn_s_memrealtime(); }
DI void st_sys(void* p, u32x4 v) { asm volatile("global_store_dwordx4 %0, %1, off sc0 sc1\n\ts_nop 1" :: "v"(p), "v"(v) : "memory"); }
DI void wait_vm0() { asm volatile("s_waitcnt vmcnt(0)" ::: "memory"); }
DI void lds_st(unsigned addr, unsigned v) { asm volatile("ds_write_b32 %0, %1" :: "v"(addr), "v"(v) : "memory"); }
DI unsigned lds_ld(unsigned addr) { unsigned v; asm volatile("ds_read_b32 %0, %1\n\ts_waitcnt lgkmcnt(0)" : "=&v"(v) : "v"(addr) : "memory"); return v; }
DI void count_in(unsigned* p, unsigned n) { asm volatile("s_mov_b64 exec, 1\n\tglobal_atomic_add %0, %1, off sc1\n\ts_mov_b64 exec, -1" :: "v"(p), "v"(n) : "memory"); }
DI unsigned peek(const unsigned* p) { unsigned v; asm volatile("global_atomic_add %0, %1, %2, off sc0 sc1\n\ts_waitcnt vmcnt(0)" : "=&v"(v) : "v"(p), "v"(0u) : "memory"); return v; }
DI u64 peek64(const void* p) { u64 v; asm volatile("global_atomic_add_x2 %0, %1, %2, off sc0 sc1\n\ts_waitcnt vmcnt(0)" : "=&v"(v) : "v"(p), "v"(0ull) : "memory"); return v; }
DI void sleep_until(unsigned t) {
  for (int d; (d = (int)(t - (unsigned)rt())) > 0;) { if (d > 200) __builtin_amdgcn_s_sleep(48); else if (d > 40) __builtin_amdgcn_s_sleep(8); else __builtin_amdgcn_s_sleep(1); }
  FENCE();
}
DI bool timed_out(unsigned* tmo, unsigned call, u64 t0) {
  if (rfl(__hip_atomic_load(tmo, __ATOMIC_RELAXED, __HIP_MEMORY_SCOPE_AGENT)) != call && rt() - t0 <= DEADLINE_TICKS) return false;
  __hip_atomic_store(tmo, call, __ATOMIC_RELAXED, __HIP_MEMORY_SCOPE_AGENT);
  return true;
}
DI unsigned pk_add(unsigned a, unsigned b) { return __builtin_bit_cast(unsigned, (u16x2)(__builtin_bit_cast(u16x2, a) + __builtin_bit_cast(u16x2, b))); }
DI unsigned pk_sub(unsigned a, unsigned b) { return __builtin_bit_cast(unsigned, (u16x2)(__builtin_bit_cast(u16x2, a) - __builtin_bit_cast(u16x2, b))); }
DI unsigned cvt_pk(float lo, float hi) { f32x2 v = {lo, hi}; return __builtin_bit_cast(unsigned, __builtin_convertvector(v, bf16x2)); }
DI float relu(float v) { return __builtin_elementwise_maximum(v, 0.0f); }
template <int CTRL> DI unsigned dpp(unsigned v) { return __builtin_amdgcn_update_dpp(0u, v, CTRL, 0xf, 0xf, true); }

struct CW {
  f32x4 acc[4][8];
  u32x4 xs[4][4];
  s16x4 wf[8][2];
  unsigned wa[4];
  unsigned fa, ra;
  unsigned xvo[4];
  unsigned need, rv;
  unsigned xso;
  __amdgpu_buffer_rsrc_t xr;
};

DI void wait_ready(CW& c) {
  asm volatile(
      "v_cmp_lt_u32 vcc, %0, %1\n\t"
      "s_cbranch_vccnz 1f\n"
      "2:\n\t"
      ".subsection 1\n"
      "3:\n\t"
      "s_sleep 1\n"
      "1:\n\t"
      "ds_read_b32 %0, %2\n\t"
      "s_waitcnt lgkmcnt(0)\n\t"
      "v_cmp_lt_u32 vcc, %0, %1\n\t"
      "s_cbranch_vccnz 3b\n\t"
      "s_branch 2b\n\t"
      ".subsection 0"
      : "+v"(c.rv) : "v"(c.need), "v"(c.ra) : "vcc", "memory");
}
template <int SET, int M> DI void xload(CW& c) {
  c.xs[SET][M] = __builtin_amdgcn_raw_buffer_load_b128(c.xr, c.xvo[SET] + M * 1024, c.xso, 17);
}
template <int SET, int M> DI void xreload(CW& c) {
  asm volatile("" :: "v"(c.xs[SET][M])); xload<SET, M>(c);
}
template <int SLOT, int J, int KK, int T> DI void wread(CW& c) {
  c.wf[(2 * J + KK) & 7][T] = __builtin_amdgcn_ds_read_tr16_b64_v4i16((LAS s16x4*)((LAS char*)0 + c.wa[J & 3] + ((SLOT + KK) * 8192 + (J >> 2) * 4096 + T * 1024)));
}
template <int P, int J, int I> DI void mma(CW& c) {
  constexpr int M = (J & 1) ? 3 - (I >> 1) : (I >> 1), KK = ((I >> 1) ^ I) & 1, E = (2 * J + KK) & 7;
  const s16x8 w = __builtin_shufflevector(c.wf[E][0], c.wf[E][1], 0, 1, 2, 3, 4, 5, 6, 7);
  c.acc[M][J] = __builtin_amdgcn_mfma_f32_16x16x32_bf16(__builtin_bit_cast(bf16x8, w), __builtin_bit_cast(bf16x8, c.xs[2 * P + KK][M]), c.acc[M][J], 0, 0, 0);
}
template <int P, int J> DI void frag(CW& c, unsigned wd) {
  constexpr int JP = (J + 2) & 7, SLOT = J >= 6 ? 2 * (1 - P) : 2 * P, O = 2 * (1 - P);
  mma<P, J, 0>(c); PIN(); wread<SLOT, JP, 0, 0>(c); PIN();
  mma<P, J, 1>(c); PIN();
  if constexpr (J < 2) xreload<O, 2 * J>(c);
  if constexpr (J == 6) { c.need += 2; lds_st(c.fa, c.need); }
  PIN();
  mma<P, J, 2>(c); PIN(); wread<SLOT, JP, 0, 1>(c); PIN();
  mma<P, J, 3>(c); PIN();
  if constexpr (J < 2) xreload<O + 1, 2 * J>(c);
  if constexpr (J == 6) c.rv = *(volatile LAS unsigned*)((LAS char*)0 + c.ra);
  PIN();
  mma<P, J, 4>(c); PIN(); wread<SLOT, JP, 1, 0>(c); PIN();
  mma<P, J, 5>(c); PIN();
  if constexpr (J < 2) xreload<O, 2 * J + 1>(c);
  if constexpr (J == 3 && P == 0) { c.xso += 4 * 64 * 4096; asm volatile("" : "+s"(c.xso)); }
  if constexpr (J == 7) wait_ready(c);
  PIN();
  mma<P, J, 6>(c); PIN(); wread<SLOT, JP, 1, 1>(c); PIN();
  mma<P, J, 7>(c); PIN();
  if constexpr (J < 2) xreload<O + 1, 2 * J + 1>(c);
  if constexpr (P == 1 && J >= 2 && J < 6) { c.wa[JP & 3] += wd; asm volatile("" : "+v"(c.wa[JP & 3])); }
  if constexpr (J == 7) asm volatile("" :: "v"(c.wf[(2 * J + 3) & 7][1]), "v"(c.xs[O + 1][3]));
  else asm volatile("" :: "v"(c.wf[(2 * J + 3) & 7][1]));
  PIN();
}
template <int P> DI void dstep(CW& c, unsigned wd) {
  frag<P, 0>(c, wd); frag<P, 1>(c, wd); frag<P, 2>(c, wd); frag<P, 3>(c, wd);
  frag<P, 4>(c, wd); frag<P, 5>(c, wd); frag<P, 6>(c, wd); frag<P, 7>(c, wd);
}
template <int S> DI void first_x(CW& c) {
  c.need = S + 1; wait_ready(c); PIN();
  xload<S, 0>(c); xload<S, 1>(c); xload<S, 2>(c); xload<S, 3>(c); PIN();
}

DI bool wait_f1(const char* f, unsigned call, unsigned* tmo, unsigned& payload) {
  const u64 t0 = rt();
  for (unsigned n = 1;; ++n) {
    u64 v = 0;
    if (threadIdx.x % 64 == 0) v = peek64(f);
    if (rfl((unsigned)v) >= call) { payload = rfl((unsigned)(v >> 32)); return true; }
    if ((n & 63u) == 0u && timed_out(tmo, call, t0)) return false;
    __builtin_amdgcn_s_sleep(4);
  }
}

struct Pull {
  __amdgpu_buffer_rsrc_t pr, xw; const char* f1; unsigned* f2; unsigned* tmo; unsigned* tick; LAS unsigned char* pat;
  unsigned call, tbase; int s, lane;
  unsigned j, tgo, fcall, info;
};
DI bool pull(Pull& q) {
  const unsigned k = (q.j + J0) * 2, u = k >> 2;
  unsigned info = q.info;
  if (q.fcall < q.call && !wait_f1(q.f1 + F1POS(q.j), q.call, q.tmo, info)) return false;
  const unsigned n0 = (info >> 8) & 0xffu, n1 = info >> 24;
#pragma unroll
  for (int w = 0; w < 2; ++w) {
    const unsigned q4 = ((info >> (16 * w + 1)) & 0x7cu) * 0x01010101u;
    *(LAS u32x4*)(q.pat + w * 2048 + q.lane * 16) = (u32x4){q4, q4, q4, q4};
    *(LAS u32x4*)(q.pat + w * 2048 + 1024 + q.lane * 16) = (u32x4){q4, q4, q4, q4};
  }
  if (q.j >= NJOB - TAILP) __builtin_amdgcn_s_setprio(2);
  sleep_until(q.tgo);
  unsigned tk;
  asm volatile("s_mov_b64 exec, 1\n\tglobal_atomic_add %0, %1, %2, off sc0\n\ts_mov_b64 exec, -1" : "=&v"(tk) : "v"(q.tick), "v"(1u) : "memory");
  PIN();
  const unsigned eidx = (q.lane & 32) + 2 * (q.lane & 15), en = (q.lane & 16) ? n1 : n0;
  const unsigned e = __builtin_amdgcn_raw_buffer_load_b32(q.pr, eidx < en ? (q.lane & 31) * 4u + (q.lane >> 5) * (E1_REL - E0_REL) : 0x80000000u, E0_REL + k * 64, 17);
  u32x4 lo[2][2], hb[2][2]; u32x3 pl[2];
#pragma unroll
  for (int w = 0; w < 2; ++w) {
    PIN();
    lo[w][0] = __builtin_amdgcn_raw_buffer_load_b128(q.pr, q.lane * 16, (k + w) * BBYTES, 17);
    lo[w][1] = __builtin_amdgcn_raw_buffer_load_b128(q.pr, q.lane * 16 + 1024, (k + w) * BBYTES, 17);
    PIN();
    pl[w] = __builtin_amdgcn_raw_buffer_load_b96(q.pr, q.lane * 12 + 2048, (k + w) * BBYTES, 17);
  }
  PIN();
  {
    LAS unsigned char* const pb = q.pat + ((q.lane >> 4) & 1) * 2048;
    *(eidx < en ? pb + (e & 2047u) : q.pat + 4096 + q.lane) = (e >> 9) & 0x7cu;
    *(eidx + 1 < en ? pb + ((e >> 16) & 2047u) : q.pat + 4096 + q.lane) = (e >> 25) & 0x7cu;
  }
  FENCE();
#pragma unroll
  for (int w = 0; w < 2; ++w) { hb[w][0] = *(LAS u32x4*)(q.pat + w * 2048 + q.lane * 16); hb[w][1] = *(LAS u32x4*)(q.pat + w * 2048 + 1024 + q.lane * 16); }
  PIN();
  asm volatile("s_waitcnt vmcnt(0)" : "+v"(tk) : "v"(pl[1]), "v"(hb[1][1]) : "memory");
  if (max(n0, n1) == 255u) {
#pragma unroll
    for (int w = 0; w < 2; ++w)
      if ((w ? n1 : n0) == 255u) {
        hb[w][0] = __builtin_amdgcn_raw_buffer_load_b128(q.pr, q.lane * 16, HB_REL + (k + w) * 2048, 17);
        hb[w][1] = __builtin_amdgcn_raw_buffer_load_b128(q.pr, q.lane * 16 + 1024, HB_REL + (k + w) * 2048, 17);
      }
    asm volatile("" :: "v"(hb[0][0]), "v"(hb[0][1]), "v"(hb[1][0]), "v"(hb[1][1]));
  }
  PIN();
  q.j = rfl(tk) - q.tbase;
  const unsigned jn = min(q.j, NJOB - 1);
  const int over = (int)((unsigned)rt() - q.tgo) - LHI;
  const u64 dt = over > 0 ? (over > DMAX ? DMAX : over) : 0;
  u64 t0, fv;
  asm volatile("s_mov_b64 exec, 1\n\tglobal_atomic_add_x2 %0, %2, %3, off sc0\n\tglobal_atomic_add_x2 %1, %4, %5, off sc0 sc1\n\ts_mov_b64 exec, -1"
               : "=&v"(t0), "=&v"(fv) : "v"(q.tick + 32), "v"(dt), "v"(q.f1 + F1POS(jn)), "v"(0ull) : "memory");
  const unsigned d = ((u >> 1) * 64 + q.s * NRG + (k & 7u)) * 4096;
#pragma unroll
  for (int w = 0; w < 2; ++w) {
    const unsigned sh = ((info >> (16 * w)) & 7u) * 0x00800080u;
    const unsigned a3 = ((pl[w][0] << 1) & 0x88888888u) | ((pl[w][1] >> 1) & 0x22222222u) | ((pl[w][2] >> 2) & 0x11111111u);
#pragma unroll
    for (int m = 0; m < 4; ++m) {
      const unsigned a = m < 3 ? pl[w][m < 3 ? m : 0] : a3, l0 = lo[w][m >> 1][2 * (m & 1)], l1 = lo[w][m >> 1][2 * (m & 1) + 1];
      const unsigned h0 = (a & 0x83838383u) | hb[w][m >> 1][2 * (m & 1)], h1 = (__builtin_rotateright32(a, 4) & 0x83838383u) | hb[w][m >> 1][2 * (m & 1) + 1];
      const u32x4 o = {pk_sub(__builtin_amdgcn_perm(h0, l0, 0x05010400u), sh), pk_sub(__builtin_amdgcn_perm(h0, l0, 0x07030602u), sh),
                       pk_sub(__builtin_amdgcn_perm(h1, l1, 0x05010400u), sh), pk_sub(__builtin_amdgcn_perm(h1, l1, 0x07030602u), sh)};
      asm volatile("buffer_store_dwordx4 %0, %1, %2, %3 offen offset:%4 sc0 sc1\n\ts_nop 1"
                   :: "v"(o), "v"(q.lane * 16u), "s"(q.xw), "s"(d + w * 4096), "i"(m * 1024) : "memory");
    }
  }
  asm volatile("s_waitcnt vmcnt(0)" : "+v"(t0), "+v"(fv) :: "memory");
  count_in(q.f2 + (u & 1) * NSTEP + (u >> 1), 2u);
  q.tgo = rfl((unsigned)t0) + (unsigned)dt + jn * TAU16 / 32; q.fcall = rfl((unsigned)fv); q.info = rfl((unsigned)(fv >> 32));
  return true;
}

DI void wfill(__amdgpu_buffer_rsrc_t wr, unsigned ldsaddr, unsigned vo, unsigned so) {
#pragma unroll
  for (int i = 0; i < 8; ++i)
    asm volatile("s_mov_b32 m0, %3\n\ts_nop 0\n\tbuffer_load_dwordx4 %0, %1, %2 offen lds"
                 :: "v"(vo), "s"(wr), "s"(so + (16 * ((i >> 1) & 1) + 4 * (i & 1)) * (NL * 2) + (i >> 2) * 128), "s"(ldsaddr + i * 1024) : "memory");
}
DI void wtouch(const char* p, unsigned n) {
  unsigned t, o = 0;
  asm volatile(
      "1:\n\t"
      "s_load_dword %0, %3, %1 offset:0\n\ts_load_dword %0, %3, %1 offset:64\n\ts_load_dword %0, %3, %1 offset:128\n\ts_load_dword %0, %3, %1 offset:192\n\t"
      "s_load_dword %0, %3, %1 offset:256\n\ts_load_dword %0, %3, %1 offset:320\n\ts_load_dword %0, %3, %1 offset:384\n\ts_load_dword %0, %3, %1 offset:448\n\t"
      "s_add_u32 %1, %1, 0x20000\n\t"
      "s_sub_u32 %2, %2, 1\n\t"
      "s_cmp_lg_u32 %2, 0\n\t"
      "s_cbranch_scc1 1b\n\t"
      "s_waitcnt lgkmcnt(0)" : "=&s"(t), "+s"(o), "+s"(n) : "s"(p) : "scc", "memory");
}

DI void place_raw(char* ws, unsigned par, int rank, int step, int rg, int lane, const u32x4 (&v)[4]) {
  char* d = ws + XT_OFF + par * XT_BYTES + (size_t)(step * 64 + rank * NRG + rg) * 4096 + lane * 16;
#pragma unroll
  for (int m = 0; m < 4; ++m) st_sys(d + m * 1024, v[m]);
  wait_vm0();
  count_in((unsigned*)(ws + F2_OFF) + (rank * 2 + (rg >> 2)) * NSTEP + step, 1u);
}

__global__ __launch_bounds__(BLOCK, 2) void ag_gemm(Params p) {
  extern __shared__ __attribute__((aligned(16))) char lds[];
  const int lane = threadIdx.x & 63;
  const int wave = rfl(threadIdx.x >> 6);
  const unsigned lb = (unsigned)(size_t)(LAS char*)lds;
  const int b = blockIdx.x, rank = p.rank & (NR - 1);
  const char* xin = (const char*)p.x; char* const my = p.my;
  asm volatile("" : "+s"(xin) : "s"(my));
  const int xcd = b & 7, bi = b >> 3;
  const int tm = (xcd >> 1) * 4 + (bi >> 3), tn = (xcd & 1) * 8 + (bi & 7);
  unsigned* const tmo = (unsigned*)(my + CT_OFF);
  const unsigned tstart = (unsigned)rt();

  const int ppair = (bi >> 1) * 8 + xcd;
  constexpr int RAW0 = GRID - NRAW * NRG * (NR - 1);
  const int rblk = (b - RAW0) / (NR - 1);
  const bool pusher = wave == 6 && b >= RAW0;
  const int pstep = pusher ? rblk >> 3 : 2 * (ppair >> 1) + (bi & 1), prg = pusher ? rblk & 7 : (ppair & 1) * 4 + wave;
  u32x4 pv[4];
  if (wave < 4 || pusher) {
    if (pstep >= XFIRST) sleep_until(tstart + pstep * XGAP / 16);
    const auto xr = __builtin_amdgcn_make_buffer_rsrc((void*)xin, 0, ML * KD * 2, 0x00020000);
#pragma unroll
    for (int m = 0; m < 4; ++m) pv[m] = __builtin_amdgcn_raw_buffer_load_b128(xr, (lane & 15) * (KD * 2) + (lane >> 4) * 16, ((prg * 64 + m * 16) * KD + pstep * 32) * 2, 0);
  }
  const auto wr = __builtin_amdgcn_make_buffer_rsrc((void*)p.w, 0, KD * NL * 2, 0x00020000);
  const unsigned wvo = (8 * ((lane >> 3) & 1) + (lane >> 4)) * (NL * 2) + ((((lane >> 1) & 3) ^ (lane >> 4)) * 32) + (lane & 1) * 16;
  if (wave == 4) {
    __builtin_amdgcn_s_sleep(4);
    for (int s = 0; s < W0; ++s) wfill(wr, lb + s * 8192, wvo, s * 32 * (NL * 2) + tn * 256);
  }
  if (wave == 6) {
    unsigned t = 0;
    if (lane == 0) t = __hip_atomic_fetch_add((unsigned*)(my + CC_OFF + b * 128), 1u, __ATOMIC_RELAXED, __HIP_MEMORY_SCOPE_AGENT);
    t = rfl(t) + 1;
    if (lane < 16) lds_st(lb + L_MISC + lane * 4, lane == 0 ? t : 0u);
  } else if (wave == 5 || wave == 7) {
#pragma unroll
    for (int i = 0; i < 4; ++i) lds_st(lb + L_FREE + (wave - 5) * 512 + i * 256 + lane * 4, 0u);
  }
  asm volatile("s_waitcnt lgkmcnt(0)" ::: "memory");
  __builtin_amdgcn_s_barrier();
  FENCE();
  const unsigned call = rfl(lds_ld(lb + L_MISC));
  const unsigned par = call & 1u;

  if (wave < 4) {
    __builtin_amdgcn_s_setprio(3);
    if (pstep < NRAW) {
      asm volatile("s_mov_b64 exec, 1\n\tds_add_u32 %0, %1\n\ts_mov_b64 exec, -1" :: "v"(lb + L_MISC + 8), "v"(1u) : "memory");
    } else {
      unsigned v = 0;
#pragma unroll
      for (int m = 0; m < 4; ++m) { v = __builtin_amdgcn_sad_u8(pv[m][0] & 0x7f007f00u, 0u, v); v = __builtin_amdgcn_sad_u8(pv[m][2] & 0x7f007f00u, 0u, v); }
      v += dpp<0x111>(v); v += dpp<0x112>(v); v += dpp<0x114>(v); v += dpp<0x118>(v);
      const unsigned top = (rl(v, 15) + rl(v, 31) + rl(v, 47) + rl(v, 63) + 1705u) >> 9, low = min(top > 7u ? top - 7u : 0u, 248u);
      const unsigned sh = (0u - low) & 7u, qw = (low + sh) >> 3, q4 = (qw << 2) * 0x01010101u;
      u32x4 lo[2], hi[2]; unsigned a[4], mask = 0;
#pragma unroll
      for (int m = 0; m < 4; ++m) {
        const unsigned d0 = pk_add(pv[m][0], sh * 0x00800080u), d1 = pk_add(pv[m][1], sh * 0x00800080u), d2 = pk_add(pv[m][2], sh * 0x00800080u), d3 = pk_add(pv[m][3], sh * 0x00800080u);
        const unsigned h0 = __builtin_amdgcn_perm(d1, d0, 0x07050301u), h1 = __builtin_amdgcn_perm(d3, d2, 0x07050301u);
        lo[m >> 1][2 * (m & 1)] = __builtin_amdgcn_perm(d1, d0, 0x06040200u); lo[m >> 1][2 * (m & 1) + 1] = __builtin_amdgcn_perm(d3, d2, 0x06040200u);
        hi[m >> 1][2 * (m & 1)] = h0; hi[m >> 1][2 * (m & 1) + 1] = h1;
        a[m] = (h0 & 0x83838383u) | __builtin_rotateleft32(h1 & 0x83838383u, 4);
        mask |= ((((h0 ^ q4) & 0x7c7c7c7cu) + 0x7c7c7c7cu) & 0x80808080u) >> (7 - 2 * m);
        mask |= ((((h1 ^ q4) & 0x7c7c7c7cu) + 0x7c7c7c7cu) & 0x80808080u) >> (6 - 2 * m);
      }
      const int u = pstep * 2 + (prg >> 2), blk = u * 4 + wave;
      char* const pk = my + PK_OFF + par * PK_BYTES;
      st_sys(pk + blk * BBYTES + lane * 16, lo[0]); st_sys(pk + blk * BBYTES + 1024 + lane * 16, lo[1]);
      const u32x3 pl = {a[0] | ((a[3] >> 1) & 0x44444444u), a[1] | ((a[3] << 1) & 0x44444444u), a[2] | ((a[3] << 2) & 0x44444444u)};
      asm volatile("global_store_dwordx3 %0, %1, off sc0 sc1\n\ts_nop 1" :: "v"(pk + blk * BBYTES + 2048 + lane * 12), "v"(pl) : "memory");
      LAS unsigned char* const hcopy = (LAS unsigned char*)lds + L_EPI + wave * 4096;
      LAS unsigned short* const list = (LAS unsigned short*)(hcopy + 2048);
      *(LAS u32x4*)(hcopy + lane * 16) = hi[0]; *(LAS u32x4*)(hcopy + 1024 + lane * 16) = hi[1];
      unsigned cnt = 0;
      do {
#pragma unroll
        for (int i = 0; i < 4; ++i) {
          const u64 bal = __ballot(mask != 0u);
          const unsigned idx = cnt + __builtin_amdgcn_mbcnt_hi((unsigned)(bal >> 32), __builtin_amdgcn_mbcnt_lo((unsigned)bal, 0u));
          list[mask ? idx : 512u + lane] = (unsigned short)(__builtin_ctz(mask | 0x80000000u) | (lane << 5));
          mask &= mask - 1u; cnt += __builtin_popcountll(bal);
        }
      } while (cnt <= 64u && __any(mask != 0u));
      FENCE();
      if (cnt > 64u) {
        cnt = 255u;
        st_sys(pk + HB_REL + blk * 2048 + lane * 16, hi[0]); st_sys(pk + HB_REL + blk * 2048 + 1024 + lane * 16, hi[1]);
      } else {
        const unsigned t = list[lane], pos = ((t & 4u) << 8) | ((t >> 1) & 0x3f0u) | ((t & 3u) << 2) | ((t >> 3) & 3u);
        const unsigned e = pos | ((hcopy[pos] & 0x7cu) << 9);
        const unsigned e2 = e | (dpp<0xb1>(e) << 16);
        if ((unsigned)lane < cnt && !(lane & 1))
          __hip_atomic_store((unsigned*)(pk + (lane < 32 ? E0_REL : E1_REL - 64) + blk * 64 + lane * 2), e2, __ATOMIC_RELAXED, __HIP_MEMORY_SCOPE_SYSTEM);
      }
      wait_vm0();
      unsigned old;
      asm volatile("s_mov_b64 exec, 1\n\tds_write_b16 %1, %2\n\tds_add_rtn_u32 %0, %3, %4\n\ts_mov_b64 exec, -1\n\ts_waitcnt lgkmcnt(0)"
                   : "=&v"(old) : "v"(lb + L_MISC + 16 + 2 * wave), "v"(sh | (qw << 3) | (cnt << 8)), "v"(lb + L_MISC + 8), "v"(1u) : "memory");
      if (rfl(old) == 3u) {
        const unsigned pair = lds_ld(lb + L_MISC + 16 + (lane & 1) * 4);
        if (lane < 2 * NR && (lane >> 1) != rank)
          __hip_atomic_store((u64*)(p.ws[lane >> 1] + F1_OFF + par * F1_BYTES + (size_t)(rank * NUNIT + u) * 128 + (lane & 1) * 8),
                             ((u64)pair << 32) | call, __ATOMIC_RELAXED, __HIP_MEMORY_SCOPE_SYSTEM);
      }
    }
    __builtin_amdgcn_s_setprio(1);
    sleep_until(tstart + OWN0 + pstep * OWNGAP / 16);
    place_raw(my, par, rank, pstep, prg, lane, pv);

    CW c;
    char* outp = (char*)p.out; asm volatile("" : "+s"(outp));
    const int g = lane >> 4, q = (lane >> 2) & 3, pp = lane & 3;
#pragma unroll
    for (int m = 0; m < 4; ++m)
#pragma unroll
      for (int j = 0; j < 8; ++j) c.acc[m][j] = (f32x4){0.f, 0.f, 0.f, 0.f};
#pragma unroll
    for (int i = 0; i < 4; ++i) c.wa[i] = lb + (g >> 1) * 2048 + q * 256 + (g & 1) * 128 + ((i ^ q) * 32) + pp * 8;
    c.fa = lb + L_FREE + wave * 256 + lane * 4; c.ra = lb + L_READY + lane * 4;
    c.rv = 0;
    c.xr = __builtin_amdgcn_make_buffer_rsrc((void*)(my + XT_OFF + par * XT_BYTES), 0, (unsigned)XT_BYTES, 0x00020000);
#pragma unroll
    for (int i = 0; i < 4; ++i) { c.xvo[i] = lane * 16 + i * 64 * 4096; asm volatile("" : "+v"(c.xvo[i])); }
    c.xso = (tm * 4 + wave) * 4096;
    PIN();
    first_x<0>(c); first_x<1>(c);
    wread<0, 0, 0, 0>(c); wread<0, 0, 0, 1>(c); wread<0, 0, 1, 0>(c); wread<0, 0, 1, 1>(c);
#pragma unroll
    for (int m = 0; m < 4; ++m) { c.xs[2][m] = (u32x4){0, 0, 0, 0}; c.xs[3][m] = (u32x4){0, 0, 0, 0}; asm volatile("" : "+v"(c.xs[2][m]), "+v"(c.xs[3][m])); }
    wread<0, 1, 0, 0>(c); wread<0, 1, 0, 1>(c); wread<0, 1, 1, 0>(c); wread<0, 1, 1, 1>(c);
    c.need = 4; asm volatile("" : "+v"(c.need));
    wait_ready(c);
    asm volatile("" :: "v"(c.xs[1][3]), "v"(c.wf[1][1]));
    PIN();
#pragma unroll 1
    for (int it = 0; it < NSTEP / 4; ++it) {
      const unsigned wd = (it + 1) % (RING / 4) == 0 ? (unsigned)(-(RING - 4) * 8192) : (unsigned)(4 * 8192);
      dstep<0>(c, wd); dstep<1>(c, wd);
    }

    const unsigned vd = *(volatile LAS unsigned*)((LAS char*)lds + L_MISC + 4);
    int tz = threadIdx.x; asm volatile("" : "+v"(tz));
    const int l2 = tz & 63;
    const unsigned ovo = (l2 >> 4) * (NL * 2) + (l2 & 15) * 16;
    const auto orr = __builtin_amdgcn_make_buffer_rsrc((void*)(outp + ((size_t)(tm * 256 + wave * 64) * NL + tn * 128) * 2), 0, 63 * NL * 2 + 256, 0x00020000);
    LAS char* const eb = (LAS char*)lds + L_EPI + wave * 4096;
    const unsigned ew = (l2 & 15) * 256 + ((l2 >> 4) & 1) * 8 + (((l2 >> 5) ^ (l2 & 15)) * 16), er = (l2 >> 4) * 256 + (((l2 & 15) ^ (l2 >> 4)) * 16);
    unsigned verdict = rfl(vd);
    while (verdict == 0u) { __builtin_amdgcn_s_sleep(1); verdict = rfl(lds_ld(lb + L_MISC + 4)); }
    if (verdict == 1u) {
      u32x4 o[4];
#pragma unroll
      for (int m = 0; m <= 4; ++m) {
#pragma unroll
        for (int h = 0; h < 4; ++h) {
          u32x2 t0, t1;
          if (m < 4) {
            f32x4 a = c.acc[m][2 * h], d = c.acc[m][2 * h + 1];
#pragma unroll
            for (int i = 0; i < 4; ++i) { a[i] = relu(a[i]); d[i] = relu(d[i]); }
            PIN();
            t0 = (u32x2){cvt_pk(a[0], a[1]), cvt_pk(a[2], a[3])}; t1 = (u32x2){cvt_pk(d[0], d[1]), cvt_pk(d[2], d[3])};
            PIN();
          }
          if (m > 0)
            asm volatile("buffer_store_dwordx4 %0, %1, %2, %3 offen nt\n\ts_nop 1" :: "v"(o[h]), "v"(ovo), "s"(orr), "s"(((m - 1) * 16 + 4 * h) * NL * 2));
          if (m < 4) { *(LAS u32x2*)(eb + (ew ^ (64 * h))) = t0; *(LAS u32x2*)(eb + (ew ^ (64 * h + 32))) = t1; }
          PIN();
        }
        if (m < 4) {
          FENCE();
#pragma unroll
          for (int i = 0; i < 4; ++i) o[i] = *(const LAS u32x4*)(eb + (er ^ (64 * i)) + 1024 * i);
          FENCE();
          PIN();
        }
      }
    } else {
      const u32x4 nan4 = {0x7fc07fc0u, 0x7fc07fc0u, 0x7fc07fc0u, 0x7fc07fc0u};
#pragma unroll 1
      for (int i = 0; i < 16; ++i) asm volatile("buffer_store_dwordx4 %0, %1, %2, %3 offen\n\ts_nop 1" :: "v"(nan4), "v"(ovo), "s"(orr), "s"(4 * i * NL * 2));
    }
  } else if (wave == 4) {
    __builtin_amdgcn_s_setprio(1);
    const unsigned pubaddr = lb + L_READY + lane * 8 + 4;
    wait_vm0();
    int pub = W0, slot = W0; unsigned freec = 0;
    lds_st(pubaddr, pub);
    for (int s = W0; s < NSTEP; ++s) {
      while (s >= RING && freec < (unsigned)(s - RING + 5)) {
        const unsigned f = lds_ld(lb + L_FREE + (lane & 3) * 256);
        freec = min(min(rl(f, 0), rl(f, 1)), min(rl(f, 2), rl(f, 3)));
        if (freec >= (unsigned)(s - RING + 5)) break;
        if (pub < s && max(max(rl(f, 0), rl(f, 1)), max(rl(f, 2), rl(f, 3))) >= (unsigned)pub) { wait_vm0(); pub = s; lds_st(pubaddr, pub); }
        __builtin_amdgcn_s_sleep(2);
      }
      if (s < RING) {
        const unsigned tgo = tstart + WT0 + (s - W0) * WGAP;
        if ((int)(tgo - (unsigned)rt()) > 0) { if (pub < s) { wait_vm0(); pub = s; lds_st(pubaddr, pub); } sleep_until(tgo); }
      }
      wfill(wr, lb + slot * 8192, wvo, s * 32 * (NL * 2) + tn * 256);
      slot = (slot + 1) & (RING - 1);
      if (s + 1 - pub > INFL) { asm volatile("s_waitcnt vmcnt(%0)" :: "i"(8 * INFL) : "memory"); pub = s + 1 - INFL; lds_st(pubaddr, pub); }
    }
    wait_vm0(); lds_st(pubaddr, INF);
  } else if (wave == 6) {
    __builtin_amdgcn_s_setprio(2);
    if (pusher) {
      sleep_until(tstart + RAWT0 + rblk * RAWGAP / 4);
      place_raw(p.ws[(rank + 1 + (b - RAW0) % (NR - 1)) & (NR - 1)], par, rank, pstep, prg, lane, pv);
    }
    const unsigned xaddr = lb + L_READY + lane * 8;
    const unsigned* f2 = (const unsigned*)(my + F2_OFF) + tm * NSTEP;
    bool dead = false;
    unsigned base = 0;
    const u64 t0 = rt();
    for (unsigned n = 1;; ++n) {
      const unsigned idx = base + lane;
      unsigned v = 4u * call;
      if (idx < (unsigned)NSTEP && lane < POLLW) v = peek(f2 + idx);
      const unsigned nok = ~(unsigned)__ballot(v >= 4u * call);
      base += nok ? (unsigned)__builtin_ctz(nok) : (unsigned)POLLW;
      if (base >= (unsigned)NSTEP) break;
      lds_st(xaddr, base);
      if ((n & 63u) == 0u && timed_out(tmo, call, t0)) { dead = true; break; }
      if (base < (unsigned)(NSTEP - POLLEND)) __builtin_amdgcn_s_sleep(8);
    }
    lds_st(xaddr, INF);
    if (rfl(__hip_atomic_load(tmo, __ATOMIC_RELAXED, __HIP_MEMORY_SCOPE_AGENT)) == call) dead = true;
    lds_st(lb + L_MISC + 4, dead ? 2u : 1u);
  } else if (wave == 7) {
    for (int pf = W0; pf < NSTEP; pf += PFB) wtouch((const char*)p.w + b * 512 + (size_t)pf * 131072, pf + PFB <= NSTEP ? PFB : NSTEP - pf);
  } else if (b < PW * (NR - 1)) {
    Pull q;
    const int qi = b % (NR - 1);
    q.s = (rank + 1 + qi) & (NR - 1);
    q.tmo = tmo; q.call = call; q.lane = lane; q.pat = (LAS unsigned char*)lds + L_PAT;
    q.tick = (unsigned*)(my + CT_OFF + (2 + 2 * qi) * 128);
    q.pr = __builtin_amdgcn_make_buffer_rsrc((void*)(p.ws[q.s] + PK_OFF + par * PK_BYTES), 0, (unsigned)PK_BYTES, 0x00020000);
    q.xw = __builtin_amdgcn_make_buffer_rsrc((void*)(my + XT_OFF + par * XT_BYTES), 0, (unsigned)XT_BYTES, 0x00020000);
    q.f1 = my + F1_OFF + par * F1_BYTES + (size_t)q.s * NUNIT * 128;
    q.f2 = (unsigned*)(my + F2_OFF) + q.s * 2 * NSTEP;
    q.tbase = (call - 1u) * (NJOB + PW);
    u64 t0old = 0, old = 0, fv = 0; unsigned t = 0;
    if (lane == 0) t0old = peek64(q.tick + 32);
    if (t0old + 1500 > rt()) t0old = ~0ull;
    while (rfl(lds_ld(lb + L_MISC + 8)) < 4u) __builtin_amdgcn_s_sleep(1);
    if (lane == 0)
      asm volatile("global_atomic_add %0, %2, %3, off sc0\n\tglobal_atomic_add_x2 %1, %4, %5, off sc0 sc1\n\ts_waitcnt vmcnt(0)"
                   : "=&v"(t), "=&v"(fv) : "v"(q.tick), "v"(1u), "v"(q.f1 + F1POS(0u)), "v"(0ull) : "memory");
    q.j = rfl(t) - q.tbase; q.info = rfl((unsigned)(fv >> 32)); q.fcall = rfl((unsigned)fv);
    const unsigned jn = q.j < NJOB ? q.j : 0u;
    const bool lead = jn < (unsigned)FASTGO && (q.fcall < call || jn == 0u);
    bool ok = q.fcall >= call || wait_f1(q.f1 + F1POS(lead ? jn : 0u), call, tmo, q.info);
    u64 t0 = rt() + T0MARG;
    const u32x4 cs = {(unsigned)t0, (unsigned)(t0 >> 32), (unsigned)t0old, (unsigned)(t0old >> 32)};
    if (lead) {
      if (lane == 0) asm volatile("global_atomic_cmpswap_x2 %0, %1, off\n\ts_nop 1" :: "v"(q.tick + 32), "v"(cs) : "memory");
      t0 -= T0MARG; q.fcall = call;
    } else {
      if (lane == 0)
        asm volatile("global_atomic_cmpswap_x2 %0, %2, %3, off sc0\n\tglobal_atomic_add_x2 %1, %4, %5, off sc0 sc1\n\ts_waitcnt vmcnt(0)"
                     : "=&v"(old), "=&v"(fv) : "v"(q.tick + 32), "v"(cs), "v"(q.f1 + F1POS(jn)), "v"(0ull) : "memory");
      if (old != t0old) t0 = old;
      q.fcall = rfl((unsigned)fv); q.info = rfl((unsigned)(fv >> 32));
    }
    q.tgo = rfl((unsigned)t0) + jn * TAU16 / 32;
    while (ok && q.j < NJOB) ok = pull(q);
  }
}

}

extern "C" void kernel_launch(int rank, int nranks, ncclComm_t comm, hipStream_t stream,
                              void* const* d_in, const int* in_sizes, int n_in,
                              void* d_out, int out_size, const int* in_dtypes, int out_dtype,
                              void* const* d_out_all, void* const* d_ws_all, size_t ws_bytes) {
  if (nranks != NR || (unsigned)rank >= (unsigned)NR || n_in != 2 || in_sizes[0] != ML * KD || in_sizes[1] != KD * NL ||
      out_size != MG * NL || ws_bytes < WS_NEED + 65536) { fprintf(stderr, "unexpected problem shape\n"); return; }
  static bool attr_set = false;
  if (!attr_set) { HIP_CHECK(hipFuncSetAttribute((const void*)ag_gemm, hipFuncAttributeMaxDynamicSharedMemorySize, LDS_BYTES)); attr_set = true; }
  Params p;
  p.x = d_in[0]; p.my = (char*)d_ws_all[rank]; p.w = d_in[1]; p.out = d_out; p.rank = rank; p.pad = 0;
  for (int i = 0; i < NR; ++i) p.ws[i] = (char*)d_ws_all[i];
  ag_gemm<<<GRID, BLOCK, LDS_BYTES, stream>>>(p);
  HIP_CHECK(hipGetLastError());
}
```

```cpp
#include <hip/hip_runtime.h>
#include <rccl/rccl.h>
#include <cstdio>

#define HIP_CHECK(expr) do { hipError_t _e = (expr); if (_e != hipSuccess) { \
    fprintf(stderr, "HIP error %s at %s:%d\n", hipGetErrorString(_e), __FILE__, __LINE__); \
    return; } } while (0)

namespace {

#define LAS __attribute__((address_space(3)))
#define DI __device__ __forceinline__
#define PIN() __builtin_amdgcn_sched_barrier(0)
#define FENCE() asm volatile("" ::: "memory")
typedef short    s16x4  __attribute__((ext_vector_type(4)));
typedef short    s16x8  __attribute__((ext_vector_type(8)));
typedef __bf16   bf16x8 __attribute__((ext_vector_type(8)));
typedef __bf16   bf16x2 __attribute__((ext_vector_type(2)));
typedef float    f32x2  __attribute__((ext_vector_type(2)));
typedef float    f32x4  __attribute__((ext_vector_type(4)));
typedef unsigned u32x2  __attribute__((ext_vector_type(2)));
typedef unsigned u32x3  __attribute__((ext_vector_type(3)));
typedef unsigned u32x4  __attribute__((ext_vector_type(4)));
typedef unsigned short u16x2 __attribute__((ext_vector_type(2)));
typedef unsigned long long u64;

constexpr int NR = 8;
constexpr int KD = 4096, NL = 2048, ML = 512, MG = NR * ML;
constexpr int NSTEP = KD / 32;
constexpr int NRG = ML / 64;
constexpr int NUNIT = NSTEP * 2;
constexpr int BBYTES = 2816;
constexpr int GRID = 256, BLOCK = 512;
constexpr int RING = 16;
constexpr int INFL = 6;
constexpr int W0 = 4;
constexpr int PW = 36;
constexpr int NRAW = 2;
constexpr unsigned J0 = NRAW * 4, NJOB = NUNIT * 2 - J0;

#define TAU16 312
#define LHI 165
constexpr int DMAX = 3, EARLYP = 24;
constexpr int T0MARG = 25, FASTGO = 8, TAILP = 40;
constexpr int XGAP = 12, XFIRST = 8 + NRAW;
constexpr int OWN0 = 300, OWNGAP = 48;
constexpr int RAWT0 = 150, RAWGAP = 16;
constexpr int WT0 = 500, WGAP = 25;
constexpr int PFB = 8;
constexpr int POLLW = 16, POLLEND = 6;

constexpr int L_FREE = RING * 8192;
constexpr int L_READY = L_FREE + 1024;
constexpr int L_MISC = L_FREE + 2048;
constexpr int L_EPI = L_MISC + 1024;
constexpr int L_PAT = L_EPI + 4 * 4096;
constexpr int LDS_BYTES = L_PAT + 4096 + 64;
static_assert(LDS_BYTES <= 160 * 1024 && LDS_BYTES > 80 * 1024, "one workgroup per CU");
constexpr unsigned INF = 0x7fffffffu;

constexpr size_t XT_BYTES = (size_t)MG * KD * 2;
constexpr size_t XT_OFF = 0;
constexpr unsigned E0_REL = NUNIT * 4 * BBYTES;
constexpr unsigned E1_REL = E0_REL + NUNIT * 4 * 64;
constexpr unsigned HB_REL = E1_REL + NUNIT * 4 * 64;
constexpr size_t PK_BYTES = HB_REL + NUNIT * 4 * 2048;
constexpr size_t PK_OFF = XT_OFF + 2 * XT_BYTES;
constexpr size_t F1_BYTES = (size_t)NR * NUNIT * 128;
constexpr size_t F1_OFF = PK_OFF + 2 * PK_BYTES;
constexpr size_t F2_OFF = F1_OFF + 2 * F1_BYTES;
constexpr size_t F2_BYTES = (size_t)NR * NUNIT * 4;
constexpr size_t CT_OFF = F2_OFF + F2_BYTES;
constexpr size_t CC_OFF = CT_OFF + 4096;
constexpr size_t WS_NEED = CC_OFF + GRID * 128;
constexpr u64 DEADLINE_TICKS = 100000000ull;
#define F1POS(j) ((((j) + J0) >> 1) * 128 + ((j) & 1u) * 8)

struct Params { const void* x; char* my; const void* w; void* out; int rank; int pad; char* ws[NR]; };

DI unsigned rfl(unsigned v) { return __builtin_amdgcn_readfirstlane(v); }
DI unsigned rl(unsigned v, int l) { return __builtin_amdgcn_readlane(v, l); }
DI u64 rt() { return __builtin_amdgcn_s_memrealtime(); }
DI void st_sys(void* p, u32x4 v) { asm volatile("global_store_dwordx4 %0, %1, off sc0 sc1\n\ts_nop 1" :: "v"(p), "v"(v) : "memory"); }
DI void wait_vm0() { asm volatile("s_waitcnt vmcnt(0)" ::: "memory"); }
DI void lds_st(unsigned addr, unsigned v) { asm volatile("ds_write_b32 %0, %1" :: "v"(addr), "v"(v) : "memory"); }
DI unsigned lds_ld(unsigned addr) { unsigned v; asm volatile("ds_read_b32 %0, %1\n\ts_waitcnt lgkmcnt(0)" : "=&v"(v) : "v"(addr) : "memory"); return v; }
DI void count_in(unsigned* p, unsigned n) { asm volatile("s_mov_b64 exec, 1\n\tglobal_atomic_add %0, %1, off sc1\n\ts_mov_b64 exec, -1" :: "v"(p), "v"(n) : "memory"); }
DI unsigned peek(const unsigned* p) { unsigned v; asm volatile("global_atomic_add %0, %1, %2, off sc0 sc1\n\ts_waitcnt vmcnt(0)" : "=&v"(v) : "v"(p), "v"(0u) : "memory"); return v; }
DI u64 peek64(const void* p) { u64 v; asm volatile("global_atomic_add_x2 %0, %1, %2, off sc0 sc1\n\ts_waitcnt vmcnt(0)" : "=&v"(v) : "v"(p), "v"(0ull) : "memory"); return v; }
DI void sleep_until(unsigned t) {
  for (int d; (d = (int)(t - (unsigned)rt())) > 0;) { if (d > 200) __builtin_amdgcn_s_sleep(48); else if (d > 40) __builtin_amdgcn_s_sleep(8); else __builtin_amdgcn_s_sleep(1); }
  FENCE();
}
DI bool timed_out(unsigned* tmo, unsigned call, u64 t0) {
  if (rfl(__hip_atomic_load(tmo, __ATOMIC_RELAXED, __HIP_MEMORY_SCOPE_AGENT)) != call && rt() - t0 <= DEADLINE_TICKS) return false;
  __hip_atomic_store(tmo, call, __ATOMIC_RELAXED, __HIP_MEMORY_SCOPE_AGENT);
  return true;
}
DI unsigned pk_add(unsigned a, unsigned b) { return __builtin_bit_cast(unsigned, (u16x2)(__builtin_bit_cast(u16x2, a) + __builtin_bit_cast(u16x2, b))); }
DI unsigned pk_sub(unsigned a, unsigned b) { return __builtin_bit_cast(unsigned, (u16x2)(__builtin_bit_cast(u16x2, a) - __builtin_bit_cast(u16x2, b))); }
DI unsigned cvt_pk(float lo, float hi) { f32x2 v = {lo, hi}; return __builtin_bit_cast(unsigned, __builtin_convertvector(v, bf16x2)); }
DI float relu(float v) { return __builtin_elementwise_maximum(v, 0.0f); }
template <int CTRL> DI unsigned dpp(unsigned v) { return __builtin_amdgcn_update_dpp(0u, v, CTRL, 0xf, 0xf, true); }

struct CW {
  f32x4 acc[4][8];
  u32x4 xs[4][4];
  s16x4 wf[8][2];
  unsigned wa[4];
  unsigned fa, ra;
  unsigned xvo[4];
  unsigned need, rv;
  unsigned xso;
  __amdgpu_buffer_rsrc_t xr;
};

DI void wait_ready(CW& c) {
  asm volatile(
      "v_cmp_lt_u32 vcc, %0, %1\n\t"
      "s_cbranch_vccnz 1f\n"
      "2:\n\t"
      ".subsection 1\n"
      "3:\n\t"
      "s_sleep 1\n"
      "1:\n\t"
      "ds_read_b32 %0, %2\n\t"
      "s_waitcnt lgkmcnt(0)\n\t"
      "v_cmp_lt_u32 vcc, %0, %1\n\t"
      "s_cbranch_vccnz 3b\n\t"
      "s_branch 2b\n\t"
      ".subsection 0"
      : "+v"(c.rv) : "v"(c.need), "v"(c.ra) : "vcc", "memory");
}
template <int SET, int M> DI void xload(CW& c) {
  c.xs[SET][M] = __builtin_amdgcn_raw_buffer_load_b128(c.xr, c.xvo[SET] + M * 1024, c.xso, 17);
}
template <int SET, int M> DI void xreload(CW& c) {
  asm volatile("" :: "v"(c.xs[SET][M])); xload<SET, M>(c);
}
template <int SLOT, int J, int KK, int T> DI void wread(CW& c) {
  c.wf[(2 * J + KK) & 7][T] = __builtin_amdgcn_ds_read_tr16_b64_v4i16((LAS s16x4*)((LAS char*)0 + c.wa[J & 3] + ((SLOT + KK) * 8192 + (J >> 2) * 4096 + T * 1024)));
}
template <int P, int J, int I> DI void mma(CW& c) {
  constexpr int M = (J & 1) ? 3 - (I >> 1) : (I >> 1), KK = ((I >> 1) ^ I) & 1, E = (2 * J + KK) & 7;
  const s16x8 w = __builtin_shufflevector(c.wf[E][0], c.wf[E][1], 0, 1, 2, 3, 4, 5, 6, 7);
  c.acc[M][J] = __builtin_amdgcn_mfma_f32_16x16x32_bf16(__builtin_bit_cast(bf16x8, w), __builtin_bit_cast(bf16x8, c.xs[2 * P + KK][M]), c.acc[M][J], 0, 0, 0);
}
template <int P, int J> DI void frag(CW& c, unsigned wd) {
  constexpr int JP = (J + 2) & 7, SLOT = J >= 6 ? 2 * (1 - P) : 2 * P, O = 2 * (1 - P);
  mma<P, J, 0>(c); PIN(); wread<SLOT, JP, 0, 0>(c); PIN();
  mma<P, J, 1>(c); PIN();
  if constexpr (J < 2) xreload<O, 2 * J>(c);
  if constexpr (J == 6) { c.need += 2; lds_st(c.fa, c.need); }
  PIN();
  mma<P, J, 2>(c); PIN(); wread<SLOT, JP, 0, 1>(c); PIN();
  mma<P, J, 3>(c); PIN();
  if constexpr (J < 2) xreload<O + 1, 2 * J>(c);
  if constexpr (J == 6) c.rv = *(volatile LAS unsigned*)((LAS char*)0 + c.ra);
  PIN();
  mma<P, J, 4>(c); PIN(); wread<SLOT, JP, 1, 0>(c); PIN();
  mma<P, J, 5>(c); PIN();
  if constexpr (J < 2) xreload<O, 2 * J + 1>(c);
  if constexpr (J == 3 && P == 0) { c.xso += 4 * 64 * 4096; asm volatile("" : "+s"(c.xso)); }
  if constexpr (J == 7) wait_ready(c);
  PIN();
  mma<P, J, 6>(c); PIN(); wread<SLOT, JP, 1, 1>(c); PIN();
  mma<P, J, 7>(c); PIN();
  if constexpr (J < 2) xreload<O + 1, 2 * J + 1>(c);
  if constexpr (P == 1 && J >= 2 && J < 6) { c.wa[JP & 3] += wd; asm volatile("" : "+v"(c.wa[JP & 3])); }
  if constexpr (J == 7) asm volatile("" :: "v"(c.wf[(2 * J + 3) & 7][1]), "v"(c.xs[O + 1][3]));
  else asm volatile("" :: "v"(c.wf[(2 * J + 3) & 7][1]));
  PIN();
}
template <int P> DI void dstep(CW& c, unsigned wd) {
  frag<P, 0>(c, wd); frag<P, 1>(c, wd); frag<P, 2>(c, wd); frag<P, 3>(c, wd);
  frag<P, 4>(c, wd); frag<P, 5>(c, wd); frag<P, 6>(c, wd); frag<P, 7>(c, wd);
}
template <int S> DI void first_x(CW& c) {
  c.need = S + 1; wait_ready(c); PIN();
  xload<S, 0>(c); xload<S, 1>(c); xload<S, 2>(c); xload<S, 3>(c); PIN();
}

DI bool wait_f1(const char* f, unsigned call, unsigned* tmo, unsigned& payload) {
  const u64 t0 = rt();
  for (unsigned n = 1;; ++n) {
    u64 v = 0;
    if (threadIdx.x % 64 == 0) v = peek64(f);
    if (rfl((unsigned)v) >= call) { payload = rfl((unsigned)(v >> 32)); return true; }
    if ((n & 63u) == 0u && timed_out(tmo, call, t0)) return false;
    __builtin_amdgcn_s_sleep(4);
  }
}

struct Pull {
  __amdgpu_buffer_rsrc_t pr, xw; const char* f1; unsigned* f2; unsigned* tmo; unsigned* tick; LAS unsigned char* pat;
  unsigned call, tbase; int s, lane;
  unsigned j, tgo, fcall, info;
};
DI bool pull(Pull& q) {
  const unsigned k = (q.j + J0) * 2, u = k >> 2;
  unsigned info = q.info;
  if (q.fcall < q.call && !wait_f1(q.f1 + F1POS(q.j), q.call, q.tmo, info)) return false;
  const unsigned n0 = (info >> 8) & 0xffu, n1 = info >> 24;
#pragma unroll
  for (int w = 0; w < 2; ++w) {
    const unsigned q4 = ((info >> (16 * w + 1)) & 0x7cu) * 0x01010101u;
    *(LAS u32x4*)(q.pat + w * 2048 + q.lane * 16) = (u32x4){q4, q4, q4, q4};
    *(LAS u32x4*)(q.pat + w * 2048 + 1024 + q.lane * 16) = (u32x4){q4, q4, q4, q4};
  }
  if (q.j >= NJOB - TAILP) __builtin_amdgcn_s_setprio(2);
  sleep_until(q.tgo);
  unsigned tk;
  asm volatile("s_mov_b64 exec, 1\n\tglobal_atomic_add %0, %1, %2, off sc0\n\ts_mov_b64 exec, -1" : "=&v"(tk) : "v"(q.tick), "v"(1u) : "memory");
  PIN();
  const unsigned eidx = (q.lane & 32) + 2 * (q.lane & 15), en = (q.lane & 16) ? n1 : n0;
  const unsigned e = __builtin_amdgcn_raw_buffer_load_b32(q.pr, eidx < en ? (q.lane & 31) * 4u + (q.lane >> 5) * (E1_REL - E0_REL) : 0x80000000u, E0_REL + k * 64, 17);
  u32x4 lo[2][2], hb[2][2]; u32x3 pl[2];
#pragma unroll
  for (int w = 0; w < 2; ++w) {
    PIN();
    lo[w][0] = __builtin_amdgcn_raw_buffer_load_b128(q.pr, q.lane * 16, (k + w) * BBYTES, 17);
    lo[w][1] = __builtin_amdgcn_raw_buffer_load_b128(q.pr, q.lane * 16 + 1024, (k + w) * BBYTES, 17);
    PIN();
    pl[w] = __builtin_amdgcn_raw_buffer_load_b96(q.pr, q.lane * 12 + 2048, (k + w) * BBYTES, 17);
  }
  PIN();
  {
    LAS unsigned char* const pb = q.pat + ((q.lane >> 4) & 1) * 2048;
    *(eidx < en ? pb + (e & 2047u) : q.pat + 4096 + q.lane) = (e >> 9) & 0x7cu;
    *(eidx + 1 < en ? pb + ((e >> 16) & 2047u) : q.pat + 4096 + q.lane) = (e >> 25) & 0x7cu;
  }
  FENCE();
#pragma unroll
  for (int w = 0; w < 2; ++w) { hb[w][0] = *(LAS u32x4*)(q.pat + w * 2048 + q.lane * 16); hb[w][1] = *(LAS u32x4*)(q.pat + w * 2048 + 1024 + q.lane * 16); }
  PIN();
  asm volatile("s_waitcnt vmcnt(0)" : "+v"(tk) : "v"(pl[1]), "v"(hb[1][1]) : "memory");
  if (max(n0, n1) == 255u) {
#pragma unroll
    for (int w = 0; w < 2; ++w)
      if ((w ? n1 : n0) == 255u) {
        hb[w][0] = __builtin_amdgcn_raw_buffer_load_b128(q.pr, q.lane * 16, HB_REL + (k + w) * 2048, 17);
        hb[w][1] = __builtin_amdgcn_raw_buffer_load_b128(q.pr, q.lane * 16 + 1024, HB_REL + (k + w) * 2048, 17);
      }
    asm volatile("" :: "v"(hb[0][0]), "v"(hb[0][1]), "v"(hb[1][0]), "v"(hb[1][1]));
  }
  PIN();
  q.j = rfl(tk) - q.tbase;
  const unsigned jn = min(q.j, NJOB - 1);
  const int over = (int)((unsigned)rt() - q.tgo) - LHI;
  const u64 dt = over > 0 ? (over > DMAX ? DMAX : over) : 0;
  u64 t0, fv;
  asm volatile("s_mov_b64 exec, 1\n\tglobal_atomic_add_x2 %0, %2, %3, off sc0\n\tglobal_atomic_add_x2 %1, %4, %5, off sc0 sc1\n\ts_mov_b64 exec, -1"
               : "=&v"(t0), "=&v"(fv) : "v"(q.tick + 32), "v"(dt), "v"(q.f1 + F1POS(jn)), "v"(0ull) : "memory");
  const unsigned d = ((u >> 1) * 64 + q.s * NRG + (k & 7u)) * 4096;
#pragma unroll
  for (int w = 0; w < 2; ++w) {
    const unsigned sh = ((info >> (16 * w)) & 7u) * 0x00800080u;
    const unsigned a3 = ((pl[w][0] << 1) & 0x88888888u) | ((pl[w][1] >> 1) & 0x22222222u) | ((pl[w][2] >> 2) & 0x11111111u);
#pragma unroll
    for (int m = 0; m < 4; ++m) {
      const unsigned a = m < 3 ? pl[w][m < 3 ? m : 0] : a3, l0 = lo[w][m >> 1][2 * (m & 1)], l1 = lo[w][m >> 1][2 * (m & 1) + 1];
      const unsigned h0 = (a & 0x83838383u) | hb[w][m >> 1][2 * (m & 1)], h1 = (__builtin_rotateright32(a, 4) & 0x83838383u) | hb[w][m >> 1][2 * (m & 1) + 1];
      const u32x4 o = {pk_sub(__builtin_amdgcn_perm(h0, l0, 0x05010400u), sh), pk_sub(__builtin_amdgcn_perm(h0, l0, 0x07030602u), sh),
                       pk_sub(__builtin_amdgcn_perm(h1, l1, 0x05010400u), sh), pk_sub(__builtin_amdgcn_perm(h1, l1, 0x07030602u), sh)};
      asm volatile("buffer_store_dwordx4 %0, %1, %2, %3 offen offset:%4 sc0 sc1\n\ts_nop 1"
                   :: "v"(o), "v"(q.lane * 16u), "s"(q.xw), "s"(d + w * 4096), "i"(m * 1024) : "memory");
    }
  }
  asm volatile("s_waitcnt vmcnt(0)" : "+v"(t0), "+v"(fv) :: "memory");
  count_in(q.f2 + (u & 1) * NSTEP + (u >> 1), 2u);
  q.tgo = rfl((unsigned)t0) + (unsigned)dt + jn * TAU16 / 32; q.fcall = rfl((unsigned)fv); q.info = rfl((unsigned)(fv >> 32));
  return true;
}

DI void wfill(__amdgpu_buffer_rsrc_t wr, unsigned ldsaddr, unsigned vo, unsigned so) {
#pragma unroll
  for (int i = 0; i < 8; ++i)
    asm volatile("s_mov_b32 m0, %3\n\ts_nop 0\n\tbuffer_load_dwordx4 %0, %1, %2 offen lds"
                 :: "v"(vo), "s"(wr), "s"(so + (16 * ((i >> 1) & 1) + 4 * (i & 1)) * (NL * 2) + (i >> 2) * 128), "s"(ldsaddr + i * 1024) : "memory");
}
DI void wtouch(const char* p, unsigned n) {
  unsigned t, o = 0;
  asm volatile(
      "1:\n\t"
      "s_load_dword %0, %3, %1 offset:0\n\ts_load_dword %0, %3, %1 offset:64\n\ts_load_dword %0, %3, %1 offset:128\n\ts_load_dword %0, %3, %1 offset:192\n\t"
      "s_load_dword %0, %3, %1 offset:256\n\ts_load_dword %0, %3, %1 offset:320\n\ts_load_dword %0, %3, %1 offset:384\n\ts_load_dword %0, %3, %1 offset:448\n\t"
      "s_add_u32 %1, %1, 0x20000\n\t"
      "s_sub_u32 %2, %2, 1\n\t"
      "s_cmp_lg_u32 %2, 0\n\t"
      "s_cbranch_scc1 1b\n\t"
      "s_waitcnt lgkmcnt(0)" : "=&s"(t), "+s"(o), "+s"(n) : "s"(p) : "scc", "memory");
}

DI void place_raw(char* ws, unsigned par, int rank, int step, int rg, int lane, const u32x4 (&v)[4]) {
  char* d = ws + XT_OFF + par * XT_BYTES + (size_t)(step * 64 + rank * NRG + rg) * 4096 + lane * 16;
#pragma unroll
  for (int m = 0; m < 4; ++m) st_sys(d + m * 1024, v[m]);
  wait_vm0();
  count_in((unsigned*)(ws + F2_OFF) + (rank * 2 + (rg >> 2)) * NSTEP + step, 1u);
}

__global__ __launch_bounds__(BLOCK, 2) void ag_gemm(Params p) {
  extern __shared__ __attribute__((aligned(16))) char lds[];
  const int lane = threadIdx.x & 63;
  const int wave = rfl(threadIdx.x >> 6);
  const unsigned lb = (unsigned)(size_t)(LAS char*)lds;
  const int b = blockIdx.x, rank = p.rank & (NR - 1);
  const char* xin = (const char*)p.x; char* const my = p.my;
  asm volatile("" : "+s"(xin) : "s"(my), "s"(p.rank), "s"(p.w));
  const int xcd = b & 7, bi = b >> 3;
  const int tm = (xcd >> 1) * 4 + (bi >> 3), tn = (xcd & 1) * 8 + (bi & 7);
  unsigned* const tmo = (unsigned*)(my + CT_OFF);
  const unsigned tstart = (unsigned)rt();

  const int ppair = (bi >> 1) * 8 + xcd;
  constexpr int RAW0 = GRID - NRAW * NRG * (NR - 1);
  const int rblk = (b - RAW0) / (NR - 1);
  const bool pusher = wave == 6 && b >= RAW0;
  const int pstep = pusher ? rblk >> 3 : 2 * (ppair >> 1) + (bi & 1), prg = pusher ? rblk & 7 : (ppair & 1) * 4 + wave;
  u32x4 pv[4];
  char* peer = nullptr;
  if (wave < 4 || pusher) {
    if (pstep >= XFIRST) sleep_until(tstart + pstep * XGAP / 16);
    const auto xr = __builtin_amdgcn_make_buffer_rsrc((void*)xin, 0, ML * KD * 2, 0x00020000);
#pragma unroll
    for (int m = 0; m < 4; ++m) pv[m] = __builtin_amdgcn_raw_buffer_load_b128(xr, (lane & 15) * (KD * 2) + (lane >> 4) * 16, ((prg * 64 + m * 16) * KD + pstep * 32) * 2, 0);
    PIN();
    peer = p.ws[(pusher ? rank + 1 + (b - RAW0) % (NR - 1) : lane >> 1) & (NR - 1)];
  }
  const auto wr = __builtin_amdgcn_make_buffer_rsrc((void*)p.w, 0, KD * NL * 2, 0x00020000);
  const unsigned wvo = (8 * ((lane >> 3) & 1) + (lane >> 4)) * (NL * 2) + ((((lane >> 1) & 3) ^ (lane >> 4)) * 32) + (lane & 1) * 16;
  if (wave == 4) {
    __builtin_amdgcn_s_sleep(4);
    for (int s = 0; s < W0; ++s) wfill(wr, lb + s * 8192, wvo, s * 32 * (NL * 2) + tn * 256);
  }
  if (wave == 6) {
    unsigned t = 0;
    if (lane == 0) t = __hip_atomic_fetch_add((unsigned*)(my + CC_OFF + b * 128), 1u, __ATOMIC_RELAXED, __HIP_MEMORY_SCOPE_AGENT);
    t = rfl(t) + 1;
    if (lane < 16) lds_st(lb + L_MISC + lane * 4, lane == 0 ? t : 0u);
  } else if (wave == 5 || wave == 7) {
#pragma unroll
    for (int i = 0; i < 4; ++i) lds_st(lb + L_FREE + (wave - 5) * 512 + i * 256 + lane * 4, 0u);
  }
  asm volatile("s_waitcnt lgkmcnt(0)" ::: "memory");
  __builtin_amdgcn_s_barrier();
  FENCE();
  const unsigned call = rfl(lds_ld(lb + L_MISC));
  const unsigned par = call & 1u;

  if (wave < 4) {
    __builtin_amdgcn_s_setprio(3);
    if (pstep < NRAW) {
      asm volatile("s_mov_b64 exec, 1\n\tds_add_u32 %0, %1\n\ts_mov_b64 exec, -1" :: "v"(lb + L_MISC + 8), "v"(1u) : "memory");
    } else {
      unsigned v = 0;
#pragma unroll
      for (int m = 0; m < 4; ++m) { v = __builtin_amdgcn_sad_u8(pv[m][0] & 0x7f007f00u, 0u, v); v = __builtin_amdgcn_sad_u8(pv[m][2] & 0x7f007f00u, 0u, v); }
      v += dpp<0x111>(v); v += dpp<0x112>(v); v += dpp<0x114>(v); v += dpp<0x118>(v);
      const unsigned top = (rl(v, 15) + rl(v, 31) + rl(v, 47) + rl(v, 63) + 1705u) >> 9, low = min(top > 7u ? top - 7u : 0u, 248u);
      const unsigned sh = (0u - low) & 7u, qw = (low + sh) >> 3, q4 = (qw << 2) * 0x01010101u;
      u32x4 lo[2], hi[2]; unsigned a[4], mask = 0;
#pragma unroll
      for (int m = 0; m < 4; ++m) {
        const unsigned d0 = pk_add(pv[m][0], sh * 0x00800080u), d1 = pk_add(pv[m][1], sh * 0x00800080u), d2 = pk_add(pv[m][2], sh * 0x00800080u), d3 = pk_add(pv[m][3], sh * 0x00800080u);
        const unsigned h0 = __builtin_amdgcn_perm(d1, d0, 0x07050301u), h1 = __builtin_amdgcn_perm(d3, d2, 0x07050301u);
        lo[m >> 1][2 * (m & 1)] = __builtin_amdgcn_perm(d1, d0, 0x06040200u); lo[m >> 1][2 * (m & 1) + 1] = __builtin_amdgcn_perm(d3, d2, 0x06040200u);
        hi[m >> 1][2 * (m & 1)] = h0; hi[m >> 1][2 * (m & 1) + 1] = h1;
        a[m] = (h0 & 0x83838383u) | __builtin_rotateleft32(h1 & 0x83838383u, 4);
        mask |= ((((h0 ^ q4) & 0x7c7c7c7cu) + 0x7c7c7c7cu) & 0x80808080u) >> (7 - 2 * m);
        mask |= ((((h1 ^ q4) & 0x7c7c7c7cu) + 0x7c7c7c7cu) & 0x80808080u) >> (6 - 2 * m);
      }
      const int u = pstep * 2 + (prg >> 2), blk = u * 4 + wave;
      char* const pk = my + PK_OFF + par * PK_BYTES;
      char* fp = peer + F1_OFF + par * F1_BYTES + (size_t)(rank * NUNIT + u) * 128 + (lane & 1) * 8;
      asm volatile("" : "+v"(fp));
      st_sys(pk + blk * BBYTES + lane * 16, lo[0]); st_sys(pk + blk * BBYTES + 1024 + lane * 16, lo[1]);
      const u32x3 pl = {a[0] | ((a[3] >> 1) & 0x44444444u), a[1] | ((a[3] << 1) & 0x44444444u), a[2] | ((a[3] << 2) & 0x44444444u)};
      asm volatile("global_store_dwordx3 %0, %1, off sc0 sc1\n\ts_nop 1" :: "v"(pk + blk * BBYTES + 2048 + lane * 12), "v"(pl) : "memory");
      LAS unsigned char* const hcopy = (LAS unsigned char*)lds + L_EPI + wave * 4096;
      LAS unsigned short* const list = (LAS unsigned short*)(hcopy + 2048);
      *(LAS u32x4*)(hcopy + lane * 16) = hi[0]; *(LAS u32x4*)(hcopy + 1024 + lane * 16) = hi[1];
      unsigned cnt = 0;
      do {
#pragma unroll
        for (int i = 0; i < 4; ++i) {
          const u64 bal = __ballot(mask != 0u);
          const unsigned idx = cnt + __builtin_amdgcn_mbcnt_hi((unsigned)(bal >> 32), __builtin_amdgcn_mbcnt_lo((unsigned)bal, 0u));
          list[mask ? idx : 512u + lane] = (unsigned short)(__builtin_ctz(mask | 0x80000000u) | (lane << 5));
          mask &= mask - 1u; cnt += __builtin_popcountll(bal);
        }
      } while (cnt <= 64u && __any(mask != 0u));
      FENCE();
      if (cnt > 64u) {
        cnt = 255u;
        st_sys(pk + HB_REL + blk * 2048 + lane * 16, hi[0]); st_sys(pk + HB_REL + blk * 2048 + 1024 + lane * 16, hi[1]);
      } else {
        const unsigned t = list[lane], pos = ((t & 4u) << 8) | ((t >> 1) & 0x3f0u) | ((t & 3u) << 2) | ((t >> 3) & 3u);
        const unsigned e = pos | ((hcopy[pos] & 0x7cu) << 9);
        const unsigned e2 = e | (dpp<0xb1>(e) << 16);
        if ((unsigned)lane < cnt && !(lane & 1))
          __hip_atomic_store((unsigned*)(pk + (lane < 32 ? E0_REL : E1_REL - 64) + blk * 64 + lane * 2), e2, __ATOMIC_RELAXED, __HIP_MEMORY_SCOPE_SYSTEM);
      }
      wait_vm0();
      unsigned old;
      asm volatile("s_mov_b64 exec, 1\n\tds_write_b16 %1, %2\n\tds_add_rtn_u32 %0, %3, %4\n\ts_mov_b64 exec, -1\n\ts_waitcnt lgkmcnt(0)"
                   : "=&v"(old) : "v"(lb + L_MISC + 16 + 2 * wave), "v"(sh | (qw << 3) | (cnt << 8)), "v"(lb + L_MISC + 8), "v"(1u) : "memory");
      if (rfl(old) == 3u) {
        const unsigned pair = lds_ld(lb + L_MISC + 16 + (lane & 1) * 4);
        if (lane < 2 * NR && (lane >> 1) != rank)
          asm volatile("global_store_dwordx2 %0, %1, off sc0 sc1" :: "v"(fp), "v"(((u64)pair << 32) | call) : "memory");
      }
    }
    __builtin_amdgcn_s_setprio(1);
    sleep_until(tstart + OWN0 + pstep * OWNGAP / 16);
    place_raw(my, par, rank, pstep, prg, lane, pv);

    CW c;
    char* outp = (char*)p.out; asm volatile("" : "+s"(outp));
    const int g = lane >> 4, q = (lane >> 2) & 3, pp = lane & 3;
#pragma unroll
    for (int m = 0; m < 4; ++m)
#pragma unroll
      for (int j = 0; j < 8; ++j) c.acc[m][j] = (f32x4){0.f, 0.f, 0.f, 0.f};
#pragma unroll
    for (int i = 0; i < 4; ++i) c.wa[i] = lb + (g >> 1) * 2048 + q * 256 + (g & 1) * 128 + ((i ^ q) * 32) + pp * 8;
    c.fa = lb + L_FREE + wave * 256 + lane * 4; c.ra = lb + L_READY + lane * 4;
    c.rv = 0;
    c.xr = __builtin_amdgcn_make_buffer_rsrc((void*)(my + XT_OFF + par * XT_BYTES), 0, (unsigned)XT_BYTES, 0x00020000);
#pragma unroll
    for (int i = 0; i < 4; ++i) { c.xvo[i] = lane * 16 + i * 64 * 4096; asm volatile("" : "+v"(c.xvo[i])); }
    c.xso = (tm * 4 + wave) * 4096;
    PIN();
    first_x<0>(c); first_x<1>(c);
    wread<0, 0, 0, 0>(c); wread<0, 0, 0, 1>(c); wread<0, 0, 1, 0>(c); wread<0, 0, 1, 1>(c);
#pragma unroll
    for (int m = 0; m < 4; ++m) { c.xs[2][m] = (u32x4){0, 0, 0, 0}; c.xs[3][m] = (u32x4){0, 0, 0, 0}; asm volatile("" : "+v"(c.xs[2][m]), "+v"(c.xs[3][m])); }
    wread<0, 1, 0, 0>(c); wread<0, 1, 0, 1>(c); wread<0, 1, 1, 0>(c); wread<0, 1, 1, 1>(c);
    c.need = 4; asm volatile("" : "+v"(c.need));
    wait_ready(c);
    asm volatile("" :: "v"(c.xs[1][3]), "v"(c.wf[1][1]));
    PIN();
#pragma unroll 1
    for (int it = 0; it < NSTEP / 4; ++it) {
      const unsigned wd = (it + 1) % (RING / 4) == 0 ? (unsigned)(-(RING - 4) * 8192) : (unsigned)(4 * 8192);
      dstep<0>(c, wd); dstep<1>(c, wd);
    }

    const unsigned vd = *(volatile LAS unsigned*)((LAS char*)lds + L_MISC + 4);
    int tz = threadIdx.x; asm volatile("" : "+v"(tz));
    const int l2 = tz & 63;
    const unsigned ovo = (l2 >> 4) * (NL * 2) + (l2 & 15) * 16;
    const auto orr = __builtin_amdgcn_make_buffer_rsrc((void*)(outp + ((size_t)(tm * 256 + wave * 64) * NL + tn * 128) * 2), 0, 63 * NL * 2 + 256, 0x00020000);
    LAS char* const eb = (LAS char*)lds + L_EPI + wave * 4096;
    const unsigned ew = (l2 & 15) * 256 + ((l2 >> 4) & 1) * 8 + (((l2 >> 5) ^ (l2 & 15)) * 16), er = (l2 >> 4) * 256 + (((l2 & 15) ^ (l2 >> 4)) * 16);
    unsigned verdict = rfl(vd);
    while (verdict == 0u) { __builtin_amdgcn_s_sleep(1); verdict = rfl(lds_ld(lb + L_MISC + 4)); }
    if (verdict == 1u) {
      u32x4 o[4];
#pragma unroll
      for (int m = 0; m <= 4; ++m) {
#pragma unroll
        for (int h = 0; h < 4; ++h) {
          u32x2 t0, t1;
          if (m < 4) {
            f32x4 a = c.acc[m][2 * h], d = c.acc[m][2 * h + 1];
#pragma unroll
            for (int i = 0; i < 4; ++i) { a[i] = relu(a[i]); d[i] = relu(d[i]); }
            PIN();
            t0 = (u32x2){cvt_pk(a[0], a[1]), cvt_pk(a[2], a[3])}; t1 = (u32x2){cvt_pk(d[0], d[1]), cvt_pk(d[2], d[3])};
            PIN();
          }
          if (m > 0)
            asm volatile("buffer_store_dwordx4 %0, %1, %2, %3 offen nt\n\ts_nop 1" :: "v"(o[h]), "v"(ovo), "s"(orr), "s"(((m - 1) * 16 + 4 * h) * NL * 2));
          if (m < 4) { *(LAS u32x2*)(eb + (ew ^ (64 * h))) = t0; *(LAS u32x2*)(eb + (ew ^ (64 * h + 32))) = t1; }
          PIN();
        }
        if (m < 4) {
          FENCE();
#pragma unroll
          for (int i = 0; i < 4; ++i) o[i] = *(const LAS u32x4*)(eb + (er ^ (64 * i)) + 1024 * i);
          FENCE();
          PIN();
        }
      }
    } else {
      const u32x4 nan4 = {0x7fc07fc0u, 0x7fc07fc0u, 0x7fc07fc0u, 0x7fc07fc0u};
#pragma unroll 1
      for (int i = 0; i < 16; ++i) asm volatile("buffer_store_dwordx4 %0, %1, %2, %3 offen\n\ts_nop 1" :: "v"(nan4), "v"(ovo), "s"(orr), "s"(4 * i * NL * 2));
    }
  } else if (wave == 4) {
    __builtin_amdgcn_s_setprio(1);
    const unsigned pubaddr = lb + L_READY + lane * 8 + 4;
    wait_vm0();
    int pub = W0, slot = W0; unsigned freec = 0;
    lds_st(pubaddr, pub);
    for (int s = W0; s < NSTEP; ++s) {
      while (s >= RING && freec < (unsigned)(s - RING + 5)) {
        const unsigned f = lds_ld(lb + L_FREE + (lane & 3) * 256);
        freec = min(min(rl(f, 0), rl(f, 1)), min(rl(f, 2), rl(f, 3)));
        if (freec >= (unsigned)(s - RING + 5)) break;
        if (pub < s && max(max(rl(f, 0), rl(f, 1)), max(rl(f, 2), rl(f, 3))) >= (unsigned)pub) { wait_vm0(); pub = s; lds_st(pubaddr, pub); }
        __builtin_amdgcn_s_sleep(2);
      }
      if (s < RING) {
        const unsigned tgo = tstart + WT0 + (s - W0) * WGAP;
        if ((int)(tgo - (unsigned)rt()) > 0) { if (pub < s) { wait_vm0(); pub = s; lds_st(pubaddr, pub); } sleep_until(tgo); }
      }
      wfill(wr, lb + slot * 8192, wvo, s * 32 * (NL * 2) + tn * 256);
      slot = (slot + 1) & (RING - 1);
      if (s + 1 - pub > INFL) { asm volatile("s_waitcnt vmcnt(%0)" :: "i"(8 * INFL) : "memory"); pub = s + 1 - INFL; lds_st(pubaddr, pub); }
    }
    wait_vm0(); lds_st(pubaddr, INF);
  } else if (wave == 6) {
    __builtin_amdgcn_s_setprio(2);
    if (pusher) {
      sleep_until(tstart + RAWT0 + rblk * RAWGAP / 4);
      place_raw(peer, par, rank, pstep, prg, lane, pv);
    }
    const unsigned xaddr = lb + L_READY + lane * 8;
    const unsigned* f2 = (const unsigned*)(my + F2_OFF) + tm * NSTEP;
    bool dead = false;
    unsigned base = 0;
    const u64 t0 = rt();
    for (unsigned n = 1;; ++n) {
      const unsigned idx = base + lane;
      unsigned v = 4u * call;
      if (idx < (unsigned)NSTEP && lane < POLLW) v = peek(f2 + idx);
      const unsigned nok = ~(unsigned)__ballot(v >= 4u * call);
      base += nok ? (unsigned)__builtin_ctz(nok) : (unsigned)POLLW;
      if (base >= (unsigned)NSTEP) break;
      lds_st(xaddr, base);
      if ((n & 63u) == 0u && timed_out(tmo, call, t0)) { dead = true; break; }
      if (base < (unsigned)(NSTEP - POLLEND)) __builtin_amdgcn_s_sleep(8);
    }
    lds_st(xaddr, INF);
    if (rfl(__hip_atomic_load(tmo, __ATOMIC_RELAXED, __HIP_MEMORY_SCOPE_AGENT)) == call) dead = true;
    lds_st(lb + L_MISC + 4, dead ? 2u : 1u);
  } else if (wave == 7) {
    for (int pf = W0; pf < NSTEP; pf += PFB) wtouch((const char*)p.w + b * 512 + (size_t)pf * 131072, pf + PFB <= NSTEP ? PFB : NSTEP - pf);
  } else if (b < PW * (NR - 1)) {
    Pull q;
    const int qi = b % (NR - 1);
    q.s = (rank + 1 + qi) & (NR - 1);
    q.tmo = tmo; q.call = call; q.lane = lane; q.pat = (LAS unsigned char*)lds + L_PAT;
    q.tick = (unsigned*)(my + CT_OFF + (2 + 2 * qi) * 128);
    q.pr = __builtin_amdgcn_make_buffer_rsrc((void*)(p.ws[q.s] + PK_OFF + par * PK_BYTES), 0, (unsigned)PK_BYTES, 0x00020000);
    q.xw = __builtin_amdgcn_make_buffer_rsrc((void*)(my + XT_OFF + par * XT_BYTES), 0, (unsigned)XT_BYTES, 0x00020000);
    q.f1 = my + F1_OFF + par * F1_BYTES + (size_t)q.s * NUNIT * 128;
    q.f2 = (unsigned*)(my + F2_OFF) + q.s * 2 * NSTEP;
    q.tbase = (call - 1u) * (NJOB + PW);
    u64 t0old = 0, old = 0, fv = 0; unsigned t = 0;
    if (lane == 0)
      asm volatile("global_atomic_add_x2 %0, %2, %4, off sc0\n\tglobal_atomic_add_x2 %1, %3, %4, off sc0 sc1\n\ts_waitcnt vmcnt(0)"
                   : "=&v"(t0old), "=&v"(fv) : "v"(q.tick + 32), "v"(q.f1 + F1POS(0u)), "v"(0ull) : "memory");
    if (t0old + 1500 > rt()) t0old = ~0ull;
    if (rfl((unsigned)fv) < call || pstep < EARLYP)
      while (rfl(lds_ld(lb + L_MISC + 8)) < 4u) __builtin_amdgcn_s_sleep(1);
    if (lane == 0)
      asm volatile("global_atomic_add %0, %2, %3, off sc0\n\tglobal_atomic_add_x2 %1, %4, %5, off sc0 sc1\n\ts_waitcnt vmcnt(0)"
                   : "=&v"(t), "=&v"(fv) : "v"(q.tick), "v"(1u), "v"(q.f1 + F1POS(0u)), "v"(0ull) : "memory");
    q.j = rfl(t) - q.tbase; q.info = rfl((unsigned)(fv >> 32)); q.fcall = rfl((unsigned)fv);
    const unsigned jn = q.j < NJOB ? q.j : 0u;
    const bool lead = jn < (unsigned)FASTGO && (q.fcall < call || jn == 0u);
    bool ok = q.fcall >= call || wait_f1(q.f1 + F1POS(lead ? jn : 0u), call, tmo, q.info);
    u64 t0 = rt() + T0MARG;
    const u32x4 cs = {(unsigned)t0, (unsigned)(t0 >> 32), (unsigned)t0old, (unsigned)(t0old >> 32)};
    if (lead) {
      if (lane == 0) asm volatile("global_atomic_cmpswap_x2 %0, %1, off\n\ts_nop 1" :: "v"(q.tick + 32), "v"(cs) : "memory");
      t0 -= T0MARG; q.fcall = call;
    } else {
      if (lane == 0)
        asm volatile("global_atomic_cmpswap_x2 %0, %2, %3, off sc0\n\tglobal_atomic_add_x2 %1, %4, %5, off sc0 sc1\n\ts_waitcnt vmcnt(0)"
                     : "=&v"(old), "=&v"(fv) : "v"(q.tick + 32), "v"(cs), "v"(q.f1 + F1POS(jn)), "v"(0ull) : "memory");
      if (old != t0old) t0 = old;
      q.fcall = rfl((unsigned)fv); q.info = rfl((unsigned)(fv >> 32));
    }
    q.tgo = rfl((unsigned)t0) + jn * TAU16 / 32;
    while (ok && q.j < NJOB) ok = pull(q);
  }
}

}

extern "C" void kernel_launch(int rank, int nranks, ncclComm_t comm, hipStream_t stream,
                              void* const* d_in, const int* in_sizes, int n_in,
                              void* d_out, int out_size, const int* in_dtypes, int out_dtype,
                              void* const* d_out_all, void* const* d_ws_all, size_t ws_bytes) {
  if (nranks != NR || (unsigned)rank >= (unsigned)NR || n_in != 2 || in_sizes[0] != ML * KD || in_sizes[1] != KD * NL ||
      out_size != MG * NL || ws_bytes < WS_NEED + 65536) { fprintf(stderr, "unexpected problem shape\n"); return; }
  static bool attr_set = false;
  if (!attr_set) { HIP_CHECK(hipFuncSetAttribute((const void*)ag_gemm, hipFuncAttributeMaxDynamicSharedMemorySize, LDS_BYTES)); attr_set = true; }
  Params p;
  p.x = d_in[0]; p.my = (char*)d_ws_all[rank]; p.w = d_in[1]; p.out = d_out; p.rank = rank; p.pad = 0;
  for (int i = 0; i < NR; ++i) p.ws[i] = (char*)d_ws_all[i];
  ag_gemm<<<GRID, BLOCK, LDS_BYTES, stream>>>(p);
  HIP_CHECK(hipGetLastError());
}
```

```cpp
#include <hip/hip_runtime.h>
#include <rccl/rccl.h>
#include <cstdio>

#define HIP_CHECK(expr) do { hipError_t _e = (expr); if (_e != hipSuccess) { \
    fprintf(stderr, "HIP error %s at %s:%d\n", hipGetErrorString(_e), __FILE__, __LINE__); \
    return; } } while (0)

namespace {

#define LAS __attribute__((address_space(3)))
#define DI __device__ __forceinline__
#define PIN() __builtin_amdgcn_sched_barrier(0)
#define FENCE() asm volatile("" ::: "memory")
typedef short    s16x4  __attribute__((ext_vector_type(4)));
typedef short    s16x8  __attribute__((ext_vector_type(8)));
typedef __bf16   bf16x8 __attribute__((ext_vector_type(8)));
typedef __bf16   bf16x2 __attribute__((ext_vector_type(2)));
typedef float    f32x2  __attribute__((ext_vector_type(2)));
typedef float    f32x4  __attribute__((ext_vector_type(4)));
typedef unsigned u32x2  __attribute__((ext_vector_type(2)));
typedef unsigned u32x3  __attribute__((ext_vector_type(3)));
typedef unsigned u32x4  __attribute__((ext_vector_type(4)));
typedef unsigned short u16x2 __attribute__((ext_vector_type(2)));
typedef unsigned long long u64;

constexpr int NR = 8;
constexpr int KD = 4096, NL = 2048, ML = 512, MG = NR * ML;
constexpr int NSTEP = KD / 32;
constexpr int NRG = ML / 64;
constexpr int NUNIT = NSTEP * 2;
constexpr int BBYTES = 2816;
constexpr int GRID = 256, BLOCK = 512;
constexpr int RING = 16;
constexpr int INFL = 6;
constexpr int W0 = 4;
constexpr int PW = 36;
constexpr int NRAW = 2;
constexpr unsigned J0 = NRAW * 4, NJOB = NUNIT * 2 - J0;

constexpr unsigned TAU16 = 312;
constexpr int LHI = 165, DMAX = 3;
constexpr int ONTIME = 500, CATCH = 6, HLHI = 5;
constexpr int T0MARG = 40, FASTGO = 8, EARLYP = 24, TAILP = 40;
constexpr int XGAP = 12, XFIRST = 8 + NRAW;
constexpr int OWN0 = 300, OWNGAP = 48;
constexpr int RAWT0 = 150, RAWGAP = 16;
constexpr int WT0 = 500, WGAP = 25;
constexpr int PFB = 8;
constexpr int POLLW = 16, POLLEND = 6;

constexpr int L_FREE = RING * 8192;
constexpr int L_READY = L_FREE + 1024;
constexpr int L_MISC = L_FREE + 2048;
constexpr int L_EPI = L_MISC + 1024;
constexpr int L_PAT = L_EPI + 4 * 4096;
constexpr int LDS_BYTES = L_PAT + 4096 + 64;
static_assert(LDS_BYTES <= 160 * 1024 && LDS_BYTES > 80 * 1024, "one workgroup per CU");
constexpr unsigned INF = 0x7fffffffu;

constexpr size_t XT_BYTES = (size_t)MG * KD * 2;
constexpr size_t XT_OFF = 0;
constexpr unsigned E0_REL = NUNIT * 4 * BBYTES;
constexpr unsigned E1_REL = E0_REL + NUNIT * 4 * 64;
constexpr unsigned HB_REL = E1_REL + NUNIT * 4 * 64;
constexpr size_t PK_BYTES = HB_REL + NUNIT * 4 * 2048;
constexpr size_t PK_OFF = XT_OFF + 2 * XT_BYTES;
constexpr size_t F1_BYTES = (size_t)NR * NUNIT * 128;
constexpr size_t F1_OFF = PK_OFF + 2 * PK_BYTES;
constexpr size_t F2_OFF = F1_OFF + 2 * F1_BYTES;
constexpr size_t F2_BYTES = (size_t)NR * NUNIT * 4;
constexpr size_t CT_OFF = F2_OFF + F2_BYTES;
constexpr size_t CC_OFF = CT_OFF + 4096;
constexpr size_t WS_NEED = CC_OFF + GRID * 128;
constexpr u64 DEADLINE_TICKS = 100000000ull;
#define F1POS(j) ((((j) + J0) >> 1) * 128 + ((j) & 1u) * 8)

struct Params { const void* x; char* my; const void* w; void* out; int rank; int pad; char* ws[NR]; };

DI unsigned rfl(unsigned v) { return __builtin_amdgcn_readfirstlane(v); }
DI unsigned rl(unsigned v, int l) { return __builtin_amdgcn_readlane(v, l); }
DI u64 rt() { return __builtin_amdgcn_s_memrealtime(); }
DI void st_sys(void* p, u32x4 v) { asm volatile("global_store_dwordx4 %0, %1, off sc0 sc1\n\ts_nop 1" :: "v"(p), "v"(v) : "memory"); }
DI void wait_vm0() { asm volatile("s_waitcnt vmcnt(0)" ::: "memory"); }
DI void lds_st(unsigned addr, unsigned v) { asm volatile("ds_write_b32 %0, %1" :: "v"(addr), "v"(v) : "memory"); }
DI unsigned lds_ld(unsigned addr) { unsigned v; asm volatile("ds_read_b32 %0, %1\n\ts_waitcnt lgkmcnt(0)" : "=&v"(v) : "v"(addr) : "memory"); return v; }
DI void count_in(unsigned* p, unsigned n) { asm volatile("s_mov_b64 exec, 1\n\tglobal_atomic_add %0, %1, off sc1\n\ts_mov_b64 exec, -1" :: "v"(p), "v"(n) : "memory"); }
DI unsigned peek(const unsigned* p) { unsigned v; asm volatile("global_atomic_add %0, %1, %2, off sc0 sc1\n\ts_waitcnt vmcnt(0)" : "=&v"(v) : "v"(p), "v"(0u) : "memory"); return v; }
DI u64 peek64(const void* p) { u64 v; asm volatile("global_atomic_add_x2 %0, %1, %2, off sc0 sc1\n\ts_waitcnt vmcnt(0)" : "=&v"(v) : "v"(p), "v"(0ull) : "memory"); return v; }
DI void sleep_until(unsigned t) {
  for (int d; (d = (int)(t - (unsigned)rt())) > 0;) { if (d > 200) __builtin_amdgcn_s_sleep(48); else if (d > 40) __builtin_amdgcn_s_sleep(8); else __builtin_amdgcn_s_sleep(1); }
  FENCE();
}
DI bool timed_out(unsigned* tmo, unsigned call, u64 t0) {
  if (rfl(__hip_atomic_load(tmo, __ATOMIC_RELAXED, __HIP_MEMORY_SCOPE_AGENT)) != call && rt() - t0 <= DEADLINE_TICKS) return false;
  __hip_atomic_store(tmo, call, __ATOMIC_RELAXED, __HIP_MEMORY_SCOPE_AGENT);
  return true;
}
DI unsigned pk_add(unsigned a, unsigned b) { return __builtin_bit_cast(unsigned, (u16x2)(__builtin_bit_cast(u16x2, a) + __builtin_bit_cast(u16x2, b))); }
DI unsigned pk_sub(unsigned a, unsigned b) { return __builtin_bit_cast(unsigned, (u16x2)(__builtin_bit_cast(u16x2, a) - __builtin_bit_cast(u16x2, b))); }
DI unsigned cvt_pk(float lo, float hi) { f32x2 v = {lo, hi}; return __builtin_bit_cast(unsigned, __builtin_convertvector(v, bf16x2)); }
DI float relu(float v) { return __builtin_elementwise_maximum(v, 0.0f); }
template <int CTRL> DI unsigned dpp(unsigned v) { return __builtin_amdgcn_update_dpp(0u, v, CTRL, 0xf, 0xf, true); }

struct CW {
  f32x4 acc[4][8];
  u32x4 xs[4][4];
  s16x4 wf[8][2];
  unsigned wa[4];
  unsigned fa, ra;
  unsigned xvo[4];
  unsigned need, rv;
  unsigned xso;
  __amdgpu_buffer_rsrc_t xr;
};

DI void wait_ready(CW& c) {
  asm volatile(
      "v_cmp_lt_u32 vcc, %0, %1\n\t"
      "s_cbranch_vccnz 1f\n"
      "2:\n\t"
      ".subsection 1\n"
      "3:\n\t"
      "s_sleep 1\n"
      "1:\n\t"
      "ds_read_b32 %0, %2\n\t"
      "s_waitcnt lgkmcnt(0)\n\t"
      "v_cmp_lt_u32 vcc, %0, %1\n\t"
      "s_cbranch_vccnz 3b\n\t"
      "s_branch 2b\n\t"
      ".subsection 0"
      : "+v"(c.rv) : "v"(c.need), "v"(c.ra) : "vcc", "memory");
}
template <int SET, int M> DI void xload(CW& c) {
  c.xs[SET][M] = __builtin_amdgcn_raw_buffer_load_b128(c.xr, c.xvo[SET] + M * 1024, c.xso, 17);
}
template <int SET, int M> DI void xreload(CW& c) {
  asm volatile("" :: "v"(c.xs[SET][M])); xload<SET, M>(c);
}
template <int SLOT, int J, int KK, int T> DI void wread(CW& c) {
  c.wf[(2 * J + KK) & 7][T] = __builtin_amdgcn_ds_read_tr16_b64_v4i16((LAS s16x4*)((LAS char*)0 + c.wa[J & 3] + ((SLOT + KK) * 8192 + (J >> 2) * 4096 + T * 1024)));
}
template <int P, int J, int I> DI void mma(CW& c) {
  constexpr int M = (J & 1) ? 3 - (I >> 1) : (I >> 1), KK = ((I >> 1) ^ I) & 1, E = (2 * J + KK) & 7;
  const s16x8 w = __builtin_shufflevector(c.wf[E][0], c.wf[E][1], 0, 1, 2, 3, 4, 5, 6, 7);
  c.acc[M][J] = __builtin_amdgcn_mfma_f32_16x16x32_bf16(__builtin_bit_cast(bf16x8, w), __builtin_bit_cast(bf16x8, c.xs[2 * P + KK][M]), c.acc[M][J], 0, 0, 0);
}
template <int P, int J> DI void frag(CW& c, unsigned wd) {
  constexpr int JP = (J + 2) & 7, SLOT = J >= 6 ? 2 * (1 - P) : 2 * P, O = 2 * (1 - P);
  mma<P, J, 0>(c); PIN(); wread<SLOT, JP, 0, 0>(c); PIN();
  mma<P, J, 1>(c); PIN();
  if constexpr (J < 2) xreload<O, 2 * J>(c);
  if constexpr (J == 6) { c.need += 2; lds_st(c.fa, c.need); }
  PIN();
  mma<P, J, 2>(c); PIN(); wread<SLOT, JP, 0, 1>(c); PIN();
  mma<P, J, 3>(c); PIN();
  if constexpr (J < 2) xreload<O + 1, 2 * J>(c);
  if constexpr (J == 6) c.rv = *(volatile LAS unsigned*)((LAS char*)0 + c.ra);
  PIN();
  mma<P, J, 4>(c); PIN(); wread<SLOT, JP, 1, 0>(c); PIN();
  mma<P, J, 5>(c); PIN();
  if constexpr (J < 2) xreload<O, 2 * J + 1>(c);
  if constexpr (J == 3 && P == 0) { c.xso += 4 * 64 * 4096; asm volatile("" : "+s"(c.xso)); }
  if constexpr (J == 7) wait_ready(c);
  PIN();
  mma<P, J, 6>(c); PIN(); wread<SLOT, JP, 1, 1>(c); PIN();
  mma<P, J, 7>(c); PIN();
  if constexpr (J < 2) xreload<O + 1, 2 * J + 1>(c);
  if constexpr (P == 1 && J >= 2 && J < 6) { c.wa[JP & 3] += wd; asm volatile("" : "+v"(c.wa[JP & 3])); }
  if constexpr (J == 7) asm volatile("" :: "v"(c.wf[(2 * J + 3) & 7][1]), "v"(c.xs[O + 1][3]));
  else asm volatile("" :: "v"(c.wf[(2 * J + 3) & 7][1]));
  PIN();
}
template <int P> DI void dstep(CW& c, unsigned wd) {
  frag<P, 0>(c, wd); frag<P, 1>(c, wd); frag<P, 2>(c, wd); frag<P, 3>(c, wd);
  frag<P, 4>(c, wd); frag<P, 5>(c, wd); frag<P, 6>(c, wd); frag<P, 7>(c, wd);
}
template <int S> DI void first_x(CW& c) {
  c.need = S + 1; wait_ready(c); PIN();
  xload<S, 0>(c); xload<S, 1>(c); xload<S, 2>(c); xload<S, 3>(c); PIN();
}

DI bool wait_f1(const char* f, unsigned call, unsigned* tmo, unsigned& payload) {
  const u64 t0 = rt();
  for (unsigned n = 1;; ++n) {
    u64 v = 0;
    if (threadIdx.x % 64 == 0) v = peek64(f);
    if (rfl((unsigned)v) >= call) { payload = rfl((unsigned)(v >> 32)); return true; }
    if ((n & 63u) == 0u && timed_out(tmo, call, t0)) return false;
    __builtin_amdgcn_s_sleep(4);
  }
}

struct Pull {
  __amdgpu_buffer_rsrc_t pr, xw; const char* f1; unsigned* f2; unsigned* tmo; unsigned* tick; LAS unsigned char* pat;
  unsigned call, tbase, hurry; int s, lane;
  unsigned j, tgo, fcall, info;
};
DI bool pull(Pull& q) {
  const unsigned k = (q.j + J0) * 2, u = k >> 2;
  unsigned info = q.info;
  if (q.fcall < q.call && !wait_f1(q.f1 + F1POS(q.j), q.call, q.tmo, info)) return false;
  const unsigned n0 = (info >> 8) & 0xffu, n1 = info >> 24;
#pragma unroll
  for (int w = 0; w < 2; ++w) {
    const unsigned q4 = ((info >> (16 * w + 1)) & 0x7cu) * 0x01010101u;
    *(LAS u32x4*)(q.pat + w * 2048 + q.lane * 16) = (u32x4){q4, q4, q4, q4};
    *(LAS u32x4*)(q.pat + w * 2048 + 1024 + q.lane * 16) = (u32x4){q4, q4, q4, q4};
  }
  if (q.j >= NJOB - TAILP) __builtin_amdgcn_s_setprio(2);
  sleep_until(q.tgo);
  unsigned tk;
  asm volatile("s_mov_b64 exec, 1\n\tglobal_atomic_add %0, %1, %2, off sc0\n\ts_mov_b64 exec, -1" : "=&v"(tk) : "v"(q.tick), "v"(1u) : "memory");
  PIN();
  const unsigned eidx = (q.lane & 32) + 2 * (q.lane & 15), en = (q.lane & 16) ? n1 : n0;
  const unsigned e = __builtin_amdgcn_raw_buffer_load_b32(q.pr, eidx < en ? (q.lane & 31) * 4u + (q.lane >> 5) * (E1_REL - E0_REL) : 0x80000000u, E0_REL + k * 64, 17);
  u32x4 lo[2][2], hb[2][2]; u32x3 pl[2];
#pragma unroll
  for (int w = 0; w < 2; ++w) {
    PIN();
    lo[w][0] = __builtin_amdgcn_raw_buffer_load_b128(q.pr, q.lane * 16, (k + w) * BBYTES, 17);
    lo[w][1] = __builtin_amdgcn_raw_buffer_load_b128(q.pr, q.lane * 16 + 1024, (k + w) * BBYTES, 17);
    PIN();
    pl[w] = __builtin_amdgcn_raw_buffer_load_b96(q.pr, q.lane * 12 + 2048, (k + w) * BBYTES, 17);
  }
  PIN();
  {
    LAS unsigned char* const pb = q.pat + ((q.lane >> 4) & 1) * 2048;
    *(eidx < en ? pb + (e & 2047u) : q.pat + 4096 + q.lane) = (e >> 9) & 0x7cu;
    *(eidx + 1 < en ? pb + ((e >> 16) & 2047u) : q.pat + 4096 + q.lane) = (e >> 25) & 0x7cu;
  }
  FENCE();
#pragma unroll
  for (int w = 0; w < 2; ++w) { hb[w][0] = *(LAS u32x4*)(q.pat + w * 2048 + q.lane * 16); hb[w][1] = *(LAS u32x4*)(q.pat + w * 2048 + 1024 + q.lane * 16); }
  PIN();
  asm volatile("s_waitcnt vmcnt(0)" : "+v"(tk) : "v"(pl[1]), "v"(hb[1][1]) : "memory");
  if (max(n0, n1) == 255u) {
#pragma unroll
    for (int w = 0; w < 2; ++w)
      if ((w ? n1 : n0) == 255u) {
        hb[w][0] = __builtin_amdgcn_raw_buffer_load_b128(q.pr, q.lane * 16, HB_REL + (k + w) * 2048, 17);
        hb[w][1] = __builtin_amdgcn_raw_buffer_load_b128(q.pr, q.lane * 16 + 1024, HB_REL + (k + w) * 2048, 17);
      }
    asm volatile("" :: "v"(hb[0][0]), "v"(hb[0][1]), "v"(hb[1][0]), "v"(hb[1][1]));
  }
  PIN();
  q.j = rfl(tk) - q.tbase;
  const unsigned jn = min(q.j, NJOB - 1);
  const int over = (int)((unsigned)rt() - q.tgo) - LHI - (int)q.hurry * HLHI;
  const u64 dt = over > 0 ? (over > DMAX ? DMAX : over) : 0;
  u64 t0, fv;
  asm volatile("s_mov_b64 exec, 1\n\tglobal_atomic_add_x2 %0, %2, %3, off sc0\n\tglobal_atomic_add_x2 %1, %4, %5, off sc0 sc1\n\ts_mov_b64 exec, -1"
               : "=&v"(t0), "=&v"(fv) : "v"(q.tick + 32), "v"(dt), "v"(q.f1 + F1POS(jn)), "v"(0ull) : "memory");
  const unsigned d = ((u >> 1) * 64 + q.s * NRG + (k & 7u)) * 4096;
#pragma unroll
  for (int w = 0; w < 2; ++w) {
    const unsigned sh = ((info >> (16 * w)) & 7u) * 0x00800080u;
    const unsigned a3 = ((pl[w][0] << 1) & 0x88888888u) | ((pl[w][1] >> 1) & 0x22222222u) | ((pl[w][2] >> 2) & 0x11111111u);
#pragma unroll
    for (int m = 0; m < 4; ++m) {
      const unsigned a = m < 3 ? pl[w][m < 3 ? m : 0] : a3, l0 = lo[w][m >> 1][2 * (m & 1)], l1 = lo[w][m >> 1][2 * (m & 1) + 1];
      const unsigned h0 = (a & 0x83838383u) | hb[w][m >> 1][2 * (m & 1)], h1 = (__builtin_rotateright32(a, 4) & 0x83838383u) | hb[w][m >> 1][2 * (m & 1) + 1];
      const u32x4 o = {pk_sub(__builtin_amdgcn_perm(h0, l0, 0x05010400u), sh), pk_sub(__builtin_amdgcn_perm(h0, l0, 0x07030602u), sh),
                       pk_sub(__builtin_amdgcn_perm(h1, l1, 0x05010400u), sh), pk_sub(__builtin_amdgcn_perm(h1, l1, 0x07030602u), sh)};
      asm volatile("buffer_store_dwordx4 %0, %1, %2, %3 offen offset:%4 sc0 sc1\n\ts_nop 1"
                   :: "v"(o), "v"(q.lane * 16u), "s"(q.xw), "s"(d + w * 4096), "i"(m * 1024) : "memory");
    }
  }
  asm volatile("s_waitcnt vmcnt(0)" : "+v"(t0), "+v"(fv) :: "memory");
  count_in(q.f2 + (u & 1) * NSTEP + (u >> 1), 2u);
  q.hurry = rfl((unsigned)(t0 >> 32)) >> 24;
  q.tgo = rfl((unsigned)t0) + (unsigned)dt + jn * (TAU16 - q.hurry) / 32; q.fcall = rfl((unsigned)fv); q.info = rfl((unsigned)(fv >> 32));
  return true;
}

DI void wfill(__amdgpu_buffer_rsrc_t wr, unsigned ldsaddr, unsigned vo, unsigned so) {
#pragma unroll
  for (int i = 0; i < 8; ++i)
    asm volatile("s_mov_b32 m0, %3\n\ts_nop 0\n\tbuffer_load_dwordx4 %0, %1, %2 offen lds"
                 :: "v"(vo), "s"(wr), "s"(so + (16 * ((i >> 1) & 1) + 4 * (i & 1)) * (NL * 2) + (i >> 2) * 128), "s"(ldsaddr + i * 1024) : "memory");
}
DI void wtouch(const char* p, unsigned n) {
  unsigned t, o = 0;
  asm volatile(
      "1:\n\t"
      "s_load_dword %0, %3, %1 offset:0\n\ts_load_dword %0, %3, %1 offset:64\n\ts_load_dword %0, %3, %1 offset:128\n\ts_load_dword %0, %3, %1 offset:192\n\t"
      "s_load_dword %0, %3, %1 offset:256\n\ts_load_dword %0, %3, %1 offset:320\n\ts_load_dword %0, %3, %1 offset:384\n\ts_load_dword %0, %3, %1 offset:448\n\t"
      "s_add_u32 %1, %1, 0x20000\n\t"
      "s_sub_u32 %2, %2, 1\n\t"
      "s_cmp_lg_u32 %2, 0\n\t"
      "s_cbranch_scc1 1b\n\t"
      "s_waitcnt lgkmcnt(0)" : "=&s"(t), "+s"(o), "+s"(n) : "s"(p) : "scc", "memory");
}

DI void place_raw(char* ws, unsigned par, int rank, int step, int rg, int lane, const u32x4 (&v)[4]) {
  char* d = ws + XT_OFF + par * XT_BYTES + (size_t)(step * 64 + rank * NRG + rg) * 4096 + lane * 16;
#pragma unroll
  for (int m = 0; m < 4; ++m) st_sys(d + m * 1024, v[m]);
  wait_vm0();
  count_in((unsigned*)(ws + F2_OFF) + (rank * 2 + (rg >> 2)) * NSTEP + step, 1u);
}

__global__ __launch_bounds__(BLOCK, 2) void ag_gemm(Params p) {
  extern __shared__ __attribute__((aligned(16))) char lds[];
  const int lane = threadIdx.x & 63;
  const int wave = rfl(threadIdx.x >> 6);
  const unsigned lb = (unsigned)(size_t)(LAS char*)lds;
  const int b = blockIdx.x, rank = p.rank & (NR - 1);
  const char* xin = (const char*)p.x; char* const my = p.my;
  asm volatile("" : "+s"(xin) : "s"(my), "s"(p.rank), "s"(p.w));
  const int xcd = b & 7, bi = b >> 3;
  const int tm = (xcd >> 1) * 4 + (bi >> 3), tn = (xcd & 1) * 8 + (bi & 7);
  unsigned* const tmo = (unsigned*)(my + CT_OFF);
  const unsigned tstart = (unsigned)rt();

  const int ppair = (bi >> 1) * 8 + xcd;
  constexpr int RAW0 = GRID - NRAW * NRG * (NR - 1);
  const int rblk = (b - RAW0) / (NR - 1);
  const bool pusher = wave == 6 && b >= RAW0;
  const int pstep = pusher ? rblk >> 3 : 2 * (ppair >> 1) + (bi & 1), prg = pusher ? rblk & 7 : (ppair & 1) * 4 + wave;
  u32x4 pv[4];
  char* peer = nullptr;
  if (wave < 4 || pusher) {
    if (pstep >= XFIRST) sleep_until(tstart + pstep * XGAP / 16);
    const auto xr = __builtin_amdgcn_make_buffer_rsrc((void*)xin, 0, ML * KD * 2, 0x00020000);
#pragma unroll
    for (int m = 0; m < 4; ++m) pv[m] = __builtin_amdgcn_raw_buffer_load_b128(xr, (lane & 15) * (KD * 2) + (lane >> 4) * 16, ((prg * 64 + m * 16) * KD + pstep * 32) * 2, 0);
    PIN();
    peer = p.ws[(pusher ? rank + 1 + (b - RAW0) % (NR - 1) : lane >> 1) & (NR - 1)];
  }
  const auto wr = __builtin_amdgcn_make_buffer_rsrc((void*)p.w, 0, KD * NL * 2, 0x00020000);
  const unsigned wvo = (8 * ((lane >> 3) & 1) + (lane >> 4)) * (NL * 2) + ((((lane >> 1) & 3) ^ (lane >> 4)) * 32) + (lane & 1) * 16;
  if (wave == 4) {
    __builtin_amdgcn_s_sleep(4);
    for (int s = 0; s < W0; ++s) wfill(wr, lb + s * 8192, wvo, s * 32 * (NL * 2) + tn * 256);
  }
  if (wave == 6) {
    unsigned t = 0;
    if (lane == 0) t = __hip_atomic_fetch_add((unsigned*)(my + CC_OFF + b * 128), 1u, __ATOMIC_RELAXED, __HIP_MEMORY_SCOPE_AGENT);
    t = rfl(t) + 1;
    if (lane < 16) lds_st(lb + L_MISC + lane * 4, lane == 0 ? t : 0u);
  } else if (wave == 5 || wave == 7) {
#pragma unroll
    for (int i = 0; i < 4; ++i) lds_st(lb + L_FREE + (wave - 5) * 512 + i * 256 + lane * 4, 0u);
  }
  asm volatile("s_waitcnt lgkmcnt(0)" ::: "memory");
  __builtin_amdgcn_s_barrier();
  FENCE();
  const unsigned call = rfl(lds_ld(lb + L_MISC));
  const unsigned par = call & 1u;

  if (wave < 4) {
    __builtin_amdgcn_s_setprio(3);
    if (pstep < NRAW) {
      asm volatile("s_mov_b64 exec, 1\n\tds_add_u32 %0, %1\n\ts_mov_b64 exec, -1" :: "v"(lb + L_MISC + 8), "v"(1u) : "memory");
    } else {
      unsigned v = 0;
#pragma unroll
      for (int m = 0; m < 4; ++m) { v = __builtin_amdgcn_sad_u8(pv[m][0] & 0x7f007f00u, 0u, v); v = __builtin_amdgcn_sad_u8(pv[m][2] & 0x7f007f00u, 0u, v); }
      v += dpp<0x111>(v); v += dpp<0x112>(v); v += dpp<0x114>(v); v += dpp<0x118>(v);
      const unsigned top = (rl(v, 15) + rl(v, 31) + rl(v, 47) + rl(v, 63) + 1705u) >> 9, low = min(top > 7u ? top - 7u : 0u, 248u);
      const unsigned sh = (0u - low) & 7u, qw = (low + sh) >> 3, q4 = (qw << 2) * 0x01010101u;
      u32x4 lo[2], hi[2]; unsigned a[4], mask = 0;
#pragma unroll
      for (int m = 0; m < 4; ++m) {
        const unsigned d0 = pk_add(pv[m][0], sh * 0x00800080u), d1 = pk_add(pv[m][1], sh * 0x00800080u), d2 = pk_add(pv[m][2], sh * 0x00800080u), d3 = pk_add(pv[m][3], sh * 0x00800080u);
        const unsigned h0 = __builtin_amdgcn_perm(d1, d0, 0x07050301u), h1 = __builtin_amdgcn_perm(d3, d2, 0x07050301u);
        lo[m >> 1][2 * (m & 1)] = __builtin_amdgcn_perm(d1, d0, 0x06040200u); lo[m >> 1][2 * (m & 1) + 1] = __builtin_amdgcn_perm(d3, d2, 0x06040200u);
        hi[m >> 1][2 * (m & 1)] = h0; hi[m >> 1][2 * (m & 1) + 1] = h1;
        a[m] = (h0 & 0x83838383u) | __builtin_rotateleft32(h1 & 0x83838383u, 4);
        mask |= ((((h0 ^ q4) & 0x7c7c7c7cu) + 0x7c7c7c7cu) & 0x80808080u) >> (7 - 2 * m);
        mask |= ((((h1 ^ q4) & 0x7c7c7c7cu) + 0x7c7c7c7cu) & 0x80808080u) >> (6 - 2 * m);
      }
      const int u = pstep * 2 + (prg >> 2), blk = u * 4 + wave;
      char* const pk = my + PK_OFF + par * PK_BYTES;
      char* fp = peer + F1_OFF + par * F1_BYTES + (size_t)(rank * NUNIT + u) * 128 + (lane & 1) * 8;
      asm volatile("" : "+v"(fp));
      st_sys(pk + blk * BBYTES + lane * 16, lo[0]); st_sys(pk + blk * BBYTES + 1024 + lane * 16, lo[1]);
      const u32x3 pl = {a[0] | ((a[3] >> 1) & 0x44444444u), a[1] | ((a[3] << 1) & 0x44444444u), a[2] | ((a[3] << 2) & 0x44444444u)};
      asm volatile("global_store_dwordx3 %0, %1, off sc0 sc1\n\ts_nop 1" :: "v"(pk + blk * BBYTES + 2048 + lane * 12), "v"(pl) : "memory");
      LAS unsigned char* const hcopy = (LAS unsigned char*)lds + L_EPI + wave * 4096;
      LAS unsigned short* const list = (LAS unsigned short*)(hcopy + 2048);
      *(LAS u32x4*)(hcopy + lane * 16) = hi[0]; *(LAS u32x4*)(hcopy + 1024 + lane * 16) = hi[1];
      unsigned cnt = 0;
      do {
#pragma unroll
        for (int i = 0; i < 4; ++i) {
          const u64 bal = __ballot(mask != 0u);
          const unsigned idx = cnt + __builtin_amdgcn_mbcnt_hi((unsigned)(bal >> 32), __builtin_amdgcn_mbcnt_lo((unsigned)bal, 0u));
          list[mask ? idx : 512u + lane] = (unsigned short)(__builtin_ctz(mask | 0x80000000u) | (lane << 5));
          mask &= mask - 1u; cnt += __builtin_popcountll(bal);
        }
      } while (cnt <= 64u && __any(mask != 0u));
      FENCE();
      if (cnt > 64u) {
        cnt = 255u;
        st_sys(pk + HB_REL + blk * 2048 + lane * 16, hi[0]); st_sys(pk + HB_REL + blk * 2048 + 1024 + lane * 16, hi[1]);
      } else {
        const unsigned t = list[lane], pos = ((t & 4u) << 8) | ((t >> 1) & 0x3f0u) | ((t & 3u) << 2) | ((t >> 3) & 3u);
        const unsigned e = pos | ((hcopy[pos] & 0x7cu) << 9);
        const unsigned e2 = e | (dpp<0xb1>(e) << 16);
        if ((unsigned)lane < cnt && !(lane & 1))
          __hip_atomic_store((unsigned*)(pk + (lane < 32 ? E0_REL : E1_REL - 64) + blk * 64 + lane * 2), e2, __ATOMIC_RELAXED, __HIP_MEMORY_SCOPE_SYSTEM);
      }
      wait_vm0();
      unsigned old;
      asm volatile("s_mov_b64 exec, 1\n\tds_write_b16 %1, %2\n\tds_add_rtn_u32 %0, %3, %4\n\ts_mov_b64 exec, -1\n\ts_waitcnt lgkmcnt(0)"
                   : "=&v"(old) : "v"(lb + L_MISC + 16 + 2 * wave), "v"(sh | (qw << 3) | (cnt << 8)), "v"(lb + L_MISC + 8), "v"(1u) : "memory");
      if (rfl(old) == 3u) {
        const unsigned pair = lds_ld(lb + L_MISC + 16 + (lane & 1) * 4);
        if (lane < 2 * NR && (lane >> 1) != rank)
          asm volatile("global_store_dwordx2 %0, %1, off sc0 sc1" :: "v"(fp), "v"(((u64)pair << 32) | call) : "memory");
      }
    }
    __builtin_amdgcn_s_setprio(1);
    sleep_until(tstart + OWN0 + pstep * OWNGAP / 16);
    place_raw(my, par, rank, pstep, prg, lane, pv);

    CW c;
    char* outp = (char*)p.out; asm volatile("" : "+s"(outp));
    const int g = lane >> 4, q = (lane >> 2) & 3, pp = lane & 3;
#pragma unroll
    for (int m = 0; m < 4; ++m)
#pragma unroll
      for (int j = 0; j < 8; ++j) c.acc[m][j] = (f32x4){0.f, 0.f, 0.f, 0.f};
#pragma unroll
    for (int i = 0; i < 4; ++i) c.wa[i] = lb + (g >> 1) * 2048 + q * 256 + (g & 1) * 128 + ((i ^ q) * 32) + pp * 8;
    c.fa = lb + L_FREE + wave * 256 + lane * 4; c.ra = lb + L_READY + lane * 4;
    c.rv = 0;
    c.xr = __builtin_amdgcn_make_buffer_rsrc((void*)(my + XT_OFF + par * XT_BYTES), 0, (unsigned)XT_BYTES, 0x00020000);
#pragma unroll
    for (int i = 0; i < 4; ++i) { c.xvo[i] = lane * 16 + i * 64 * 4096; asm volatile("" : "+v"(c.xvo[i])); }
    c.xso = (tm * 4 + wave) * 4096;
    PIN();
    first_x<0>(c); first_x<1>(c);
    wread<0, 0, 0, 0>(c); wread<0, 0, 0, 1>(c); wread<0, 0, 1, 0>(c); wread<0, 0, 1, 1>(c);
#pragma unroll
    for (int m = 0; m < 4; ++m) { c.xs[2][m] = (u32x4){0, 0, 0, 0}; c.xs[3][m] = (u32x4){0, 0, 0, 0}; asm volatile("" : "+v"(c.xs[2][m]), "+v"(c.xs[3][m])); }
    wread<0, 1, 0, 0>(c); wread<0, 1, 0, 1>(c); wread<0, 1, 1, 0>(c); wread<0, 1, 1, 1>(c);
    c.need = 4; asm volatile("" : "+v"(c.need));
    wait_ready(c);
    asm volatile("" :: "v"(c.xs[1][3]), "v"(c.wf[1][1]));
    PIN();
#pragma unroll 1
    for (int it = 0; it < NSTEP / 4; ++it) {
      const unsigned wd = (it + 1) % (RING / 4) == 0 ? (unsigned)(-(RING - 4) * 8192) : (unsigned)(4 * 8192);
      dstep<0>(c, wd); dstep<1>(c, wd);
    }

    const unsigned vd = *(volatile LAS unsigned*)((LAS char*)lds + L_MISC + 4);
    int tz = threadIdx.x; asm volatile("" : "+v"(tz));
    const int l2 = tz & 63;
    const unsigned ovo = (l2 >> 4) * (NL * 2) + (l2 & 15) * 16;
    const auto orr = __builtin_amdgcn_make_buffer_rsrc((void*)(outp + ((size_t)(tm * 256 + wave * 64) * NL + tn * 128) * 2), 0, 63 * NL * 2 + 256, 0x00020000);
    LAS char* const eb = (LAS char*)lds + L_EPI + wave * 4096;
    const unsigned ew = (l2 & 15) * 256 + ((l2 >> 4) & 1) * 8 + (((l2 >> 5) ^ (l2 & 15)) * 16), er = (l2 >> 4) * 256 + (((l2 & 15) ^ (l2 >> 4)) * 16);
    unsigned verdict = rfl(vd);
    while (verdict == 0u) { __builtin_amdgcn_s_sleep(1); verdict = rfl(lds_ld(lb + L_MISC + 4)); }
    if (verdict == 1u) {
      u32x4 o[4];
#pragma unroll
      for (int m = 0; m <= 4; ++m) {
#pragma unroll
        for (int h = 0; h < 4; ++h) {
          u32x2 t0, t1;
          if (m < 4) {
            f32x4 a = c.acc[m][2 * h], d = c.acc[m][2 * h + 1];
#pragma unroll
            for (int i = 0; i < 4; ++i) { a[i] = relu(a[i]); d[i] = relu(d[i]); }
            PIN();
            t0 = (u32x2){cvt_pk(a[0], a[1]), cvt_pk(a[2], a[3])}; t1 = (u32x2){cvt_pk(d[0], d[1]), cvt_pk(d[2], d[3])};
            PIN();
          }
          if (m > 0)
            asm volatile("buffer_store_dwordx4 %0, %1, %2, %3 offen nt\n\ts_nop 1" :: "v"(o[h]), "v"(ovo), "s"(orr), "s"(((m - 1) * 16 + 4 * h) * NL * 2));
          if (m < 4) { *(LAS u32x2*)(eb + (ew ^ (64 * h))) = t0; *(LAS u32x2*)(eb + (ew ^ (64 * h + 32))) = t1; }
          PIN();
        }
        if (m < 4) {
          FENCE();
#pragma unroll
          for (int i = 0; i < 4; ++i) o[i] = *(const LAS u32x4*)(eb + (er ^ (64 * i)) + 1024 * i);
          FENCE();
          PIN();
        }
      }
    } else {
      const u32x4 nan4 = {0x7fc07fc0u, 0x7fc07fc0u, 0x7fc07fc0u, 0x7fc07fc0u};
#pragma unroll 1
      for (int i = 0; i < 16; ++i) asm volatile("buffer_store_dwordx4 %0, %1, %2, %3 offen\n\ts_nop 1" :: "v"(nan4), "v"(ovo), "s"(orr), "s"(4 * i * NL * 2));
    }
  } else if (wave == 4) {
    __builtin_amdgcn_s_setprio(1);
    const unsigned pubaddr = lb + L_READY + lane * 8 + 4;
    wait_vm0();
    int pub = W0, slot = W0; unsigned freec = 0;
    lds_st(pubaddr, pub);
    for (int s = W0; s < NSTEP; ++s) {
      while (s >= RING && freec < (unsigned)(s - RING + 5)) {
        const unsigned f = lds_ld(lb + L_FREE + (lane & 3) * 256);
        freec = min(min(rl(f, 0), rl(f, 1)), min(rl(f, 2), rl(f, 3)));
        if (freec >= (unsigned)(s - RING + 5)) break;
        if (pub < s && max(max(rl(f, 0), rl(f, 1)), max(rl(f, 2), rl(f, 3))) >= (unsigned)pub) { wait_vm0(); pub = s; lds_st(pubaddr, pub); }
        __builtin_amdgcn_s_sleep(2);
      }
      if (s < RING) {
        const unsigned tgo = tstart + WT0 + (s - W0) * WGAP;
        if ((int)(tgo - (unsigned)rt()) > 0) { if (pub < s) { wait_vm0(); pub = s; lds_st(pubaddr, pub); } sleep_until(tgo); }
      }
      wfill(wr, lb + slot * 8192, wvo, s * 32 * (NL * 2) + tn * 256);
      slot = (slot + 1) & (RING - 1);
      if (s + 1 - pub > INFL) { asm volatile("s_waitcnt vmcnt(%0)" :: "i"(8 * INFL) : "memory"); pub = s + 1 - INFL; lds_st(pubaddr, pub); }
    }
    wait_vm0(); lds_st(pubaddr, INF);
  } else if (wave == 6) {
    __builtin_amdgcn_s_setprio(2);
    if (pusher) {
      sleep_until(tstart + RAWT0 + rblk * RAWGAP / 4);
      place_raw(peer, par, rank, pstep, prg, lane, pv);
    }
    const unsigned xaddr = lb + L_READY + lane * 8;
    const unsigned* f2 = (const unsigned*)(my + F2_OFF) + tm * NSTEP;
    bool dead = false;
    unsigned base = 0;
    const u64 t0 = rt();
    for (unsigned n = 1;; ++n) {
      const unsigned idx = base + lane;
      unsigned v = 4u * call;
      if (idx < (unsigned)NSTEP && lane < POLLW) v = peek(f2 + idx);
      const unsigned nok = ~(unsigned)__ballot(v >= 4u * call);
      base += nok ? (unsigned)__builtin_ctz(nok) : (unsigned)POLLW;
      if (base >= (unsigned)NSTEP) break;
      lds_st(xaddr, base);
      if ((n & 63u) == 0u && timed_out(tmo, call, t0)) { dead = true; break; }
      if (base < (unsigned)(NSTEP - POLLEND)) __builtin_amdgcn_s_sleep(8);
    }
    lds_st(xaddr, INF);
    if (rfl(__hip_atomic_load(tmo, __ATOMIC_RELAXED, __HIP_MEMORY_SCOPE_AGENT)) == call) dead = true;
    lds_st(lb + L_MISC + 4, dead ? 2u : 1u);
  } else if (wave == 7) {
    for (int pf = W0; pf < NSTEP; pf += PFB) wtouch((const char*)p.w + b * 512 + (size_t)pf * 131072, pf + PFB <= NSTEP ? PFB : NSTEP - pf);
  } else if (b < PW * (NR - 1)) {
    Pull q;
    const int qi = b % (NR - 1);
    q.s = (rank + 1 + qi) & (NR - 1);
    q.tmo = tmo; q.call = call; q.lane = lane; q.pat = (LAS unsigned char*)lds + L_PAT;
    q.tick = (unsigned*)(my + CT_OFF + (2 + 2 * qi) * 128);
    q.pr = __builtin_amdgcn_make_buffer_rsrc((void*)(p.ws[q.s] + PK_OFF + par * PK_BYTES), 0, (unsigned)PK_BYTES, 0x00020000);
    q.xw = __builtin_amdgcn_make_buffer_rsrc((void*)(my + XT_OFF + par * XT_BYTES), 0, (unsigned)XT_BYTES, 0x00020000);
    q.f1 = my + F1_OFF + par * F1_BYTES + (size_t)q.s * NUNIT * 128;
    q.f2 = (unsigned*)(my + F2_OFF) + q.s * 2 * NSTEP;
    q.tbase = (call - 1u) * (NJOB + PW);
    u64 t0old = 0, old = 0, fv = 0; unsigned t = 0;
    if (lane == 0)
      asm volatile("global_atomic_add_x2 %0, %2, %4, off sc0\n\tglobal_atomic_add_x2 %1, %3, %4, off sc0 sc1\n\ts_waitcnt vmcnt(0)"
                   : "=&v"(t0old), "=&v"(fv) : "v"(q.tick + 32), "v"(q.f1 + F1POS(0u)), "v"(0ull) : "memory");
    if ((t0old & 0xffffffffffffffull) + 1500 > rt()) t0old = ~0ull;
    if (rfl((unsigned)fv) < call || pstep < EARLYP)
      while (rfl(lds_ld(lb + L_MISC + 8)) < 4u) __builtin_amdgcn_s_sleep(1);
    if (lane == 0)
      asm volatile("global_atomic_add %0, %2, %3, off sc0\n\tglobal_atomic_add_x2 %1, %4, %5, off sc0 sc1\n\ts_waitcnt vmcnt(0)"
                   : "=&v"(t), "=&v"(fv) : "v"(q.tick), "v"(1u), "v"(q.f1 + F1POS(0u)), "v"(0ull) : "memory");
    q.j = rfl(t) - q.tbase; q.info = rfl((unsigned)(fv >> 32)); q.fcall = rfl((unsigned)fv);
    const unsigned jn = q.j < NJOB ? q.j : 0u;
    const bool lead = jn < (unsigned)FASTGO && (q.fcall < call || jn == 0u);
    bool ok = q.fcall >= call || wait_f1(q.f1 + F1POS(lead ? jn : 0u), call, tmo, q.info);
    u64 t0 = rt() + T0MARG;
    {
      const int late = (int)((unsigned)t0 - tstart) - ONTIME;
      t0 |= (u64)(late > 0 ? min(late / 16, CATCH) : 0) << 56;
    }
    const u32x4 cs = {(unsigned)t0, (unsigned)(t0 >> 32), (unsigned)t0old, (unsigned)(t0old >> 32)};
    if (lead) {
      if (lane == 0) asm volatile("global_atomic_cmpswap_x2 %0, %1, off\n\ts_nop 1" :: "v"(q.tick + 32), "v"(cs) : "memory");
      t0 -= T0MARG; q.fcall = call;
    } else {
      if (lane == 0)
        asm volatile("global_atomic_cmpswap_x2 %0, %2, %3, off sc0\n\tglobal_atomic_add_x2 %1, %4, %5, off sc0 sc1\n\ts_waitcnt vmcnt(0)"
                     : "=&v"(old), "=&v"(fv) : "v"(q.tick + 32), "v"(cs), "v"(q.f1 + F1POS(jn)), "v"(0ull) : "memory");
      if (old != t0old) t0 = old;
      q.fcall = rfl((unsigned)fv); q.info = rfl((unsigned)(fv >> 32));
    }
    q.hurry = rfl((unsigned)(t0 >> 32)) >> 24;
    q.tgo = rfl((unsigned)t0) + jn * (TAU16 - q.hurry) / 32;
    while (ok && q.j < NJOB) ok = pull(q);
  }
}

}

extern "C" void kernel_launch(int rank, int nranks, ncclComm_t comm, hipStream_t stream,
                              void* const* d_in, const int* in_sizes, int n_in,
                              void* d_out, int out_size, const int* in_dtypes, int out_dtype,
                              void* const* d_out_all, void* const* d_ws_all, size_t ws_bytes) {
  if (nranks != NR || (unsigned)rank >= (unsigned)NR || n_in != 2 || in_sizes[0] != ML * KD || in_sizes[1] != KD * NL ||
      out_size != MG * NL || ws_bytes < WS_NEED) { fprintf(stderr, "unexpected problem shape\n"); return; }
  static bool attr_set = false;
  if (!attr_set) { HIP_CHECK(hipFuncSetAttribute((const void*)ag_gemm, hipFuncAttributeMaxDynamicSharedMemorySize, LDS_BYTES)); attr_set = true; }
  Params p;
  p.x = d_in[0]; p.my = (char*)d_ws_all[rank]; p.w = d_in[1]; p.out = d_out; p.rank = rank; p.pad = 0;
  for (int i = 0; i < NR; ++i) p.ws[i] = (char*)d_ws_all[i];
  ag_gemm<<<GRID, BLOCK, LDS_BYTES, stream>>>(p);
  HIP_CHECK(hipGetLastError());
}
```

```cpp
#include <hip/hip_runtime.h>
#include <rccl/rccl.h>
#include <cstdio>

#define HIP_CHECK(expr) do { hipError_t _e = (expr); if (_e != hipSuccess) { \
    fprintf(stderr, "HIP error %s at %s:%d\n", hipGetErrorString(_e), __FILE__, __LINE__); \
    return; } } while (0)

namespace {

#define LAS __attribute__((address_space(3)))
#define DI __device__ __forceinline__
#define PIN() __builtin_amdgcn_sched_barrier(0)
#define FENCE() asm volatile("" ::: "memory")
typedef short    s16x4  __attribute__((ext_vector_type(4)));
typedef short    s16x8  __attribute__((ext_vector_type(8)));
typedef __bf16   bf16x8 __attribute__((ext_vector_type(8)));
typedef __bf16   bf16x2 __attribute__((ext_vector_type(2)));
typedef float    f32x2  __attribute__((ext_vector_type(2)));
typedef float    f32x4  __attribute__((ext_vector_type(4)));
typedef unsigned u32x2  __attribute__((ext_vector_type(2)));
typedef unsigned u32x3  __attribute__((ext_vector_type(3)));
typedef unsigned u32x4  __attribute__((ext_vector_type(4)));
typedef unsigned short u16x2 __attribute__((ext_vector_type(2)));
typedef unsigned long long u64;

constexpr int NR = 8;
constexpr int KD = 4096, NL = 2048, ML = 512, MG = NR * ML;
constexpr int NSTEP = KD / 32;
constexpr int NRG = ML / 64;
constexpr int NUNIT = NSTEP * 2;
constexpr int BBYTES = 2816;
constexpr int GRID = 256, BLOCK = 512;
constexpr int RING = 16;
constexpr int INFL = 6;
constexpr int W0 = 4;
constexpr int PW = 36;
constexpr int NRAW = 2;
constexpr unsigned J0 = NRAW * 4, NJOB = NUNIT * 2 - J0;

constexpr unsigned TAU16 = 312;
constexpr int LHI = 165, DMAX = 3;
constexpr int ONTIME = 500, CATCH = 6, HLHI = 5;
constexpr int T0MARG = 40, FASTGO = 8, EARLYP = 24, TAILP = 40;
constexpr int XGAP = 12, XFIRST = 8 + NRAW;
constexpr int OWN0 = 300, OWNGAP = 48;
constexpr int RAWT0 = 150, RAWGAP = 16;
constexpr int WT0 = 500, WGAP = 25;
constexpr int PFB = 8;
constexpr int POLLW = 16, POLLEND = 6;

constexpr int L_FREE = RING * 8192;
constexpr int L_READY = L_FREE + 1024;
constexpr int L_MISC = L_FREE + 2048;
constexpr int L_EPI = L_MISC + 1024;
constexpr int L_PAT = L_EPI + 4 * 4096;
constexpr int LDS_BYTES = L_PAT + 4096 + 64;
static_assert(LDS_BYTES <= 160 * 1024 && LDS_BYTES > 80 * 1024, "one workgroup per CU");
constexpr unsigned INF = 0x7fffffffu;

constexpr size_t XT_BYTES = (size_t)MG * KD * 2;
constexpr size_t XT_OFF = 0;
constexpr unsigned E0_REL = NUNIT * 4 * BBYTES;
constexpr unsigned E1_REL = E0_REL + NUNIT * 4 * 64;
constexpr unsigned HB_REL = E1_REL + NUNIT * 4 * 64;
constexpr size_t PK_BYTES = HB_REL + NUNIT * 4 * 2048;
constexpr size_t PK_OFF = XT_OFF + 2 * XT_BYTES;
constexpr size_t F1_BYTES = (size_t)NR * NUNIT * 128;
constexpr size_t F1_OFF = PK_OFF + 2 * PK_BYTES;
constexpr size_t F2_OFF = F1_OFF + 2 * F1_BYTES;
constexpr size_t F2_BYTES = (size_t)NR * NUNIT * 4;
constexpr size_t CT_OFF = F2_OFF + F2_BYTES;
constexpr size_t CC_OFF = CT_OFF + 4096;
constexpr size_t WS_NEED = CC_OFF + GRID * 128;
constexpr u64 DEADLINE_TICKS = 100000000ull;
#define F1POS(j) ((((j) + J0) >> 1) * 128 + ((j) & 1u) * 8)

struct Params { const void* x; char* my; const void* w; void* out; int rank; int pad; char* ws[NR]; };

DI unsigned rfl(unsigned v) { return __builtin_amdgcn_readfirstlane(v); }
DI unsigned rl(unsigned v, int l) { return __builtin_amdgcn_readlane(v, l); }
DI u64 rt() { return __builtin_amdgcn_s_memrealtime(); }
DI void st_sys(void* p, u32x4 v) { asm volatile("global_store_dwordx4 %0, %1, off sc0 sc1\n\ts_nop 1" :: "v"(p), "v"(v) : "memory"); }
DI void wait_vm0() { asm volatile("s_waitcnt vmcnt(0)" ::: "memory"); }
DI void lds_st(unsigned addr, unsigned v) { asm volatile("ds_write_b32 %0, %1" :: "v"(addr), "v"(v) : "memory"); }
DI unsigned lds_ld(unsigned addr) { unsigned v; asm volatile("ds_read_b32 %0, %1\n\ts_waitcnt lgkmcnt(0)" : "=&v"(v) : "v"(addr) : "memory"); return v; }
DI void count_in(unsigned* p, unsigned n) { asm volatile("s_mov_b64 exec, 1\n\tglobal_atomic_add %0, %1, off sc1\n\ts_mov_b64 exec, -1" :: "v"(p), "v"(n) : "memory"); }
DI unsigned peek(const unsigned* p) { unsigned v; asm volatile("global_atomic_add %0, %1, %2, off sc0 sc1\n\ts_waitcnt vmcnt(0)" : "=&v"(v) : "v"(p), "v"(0u) : "memory"); return v; }
DI u64 peek64(const void* p) { u64 v; asm volatile("global_atomic_add_x2 %0, %1, %2, off sc0 sc1\n\ts_waitcnt vmcnt(0)" : "=&v"(v) : "v"(p), "v"(0ull) : "memory"); return v; }
DI void sleep_until(unsigned t) {
  for (int d; (d = (int)(t - (unsigned)rt())) > 0;) { if (d > 200) __builtin_amdgcn_s_sleep(48); else if (d > 40) __builtin_amdgcn_s_sleep(8); else __builtin_amdgcn_s_sleep(1); }
  FENCE();
}
DI bool timed_out(unsigned* tmo, unsigned call, u64 t0) {
  if (rfl(__hip_atomic_load(tmo, __ATOMIC_RELAXED, __HIP_MEMORY_SCOPE_AGENT)) != call && rt() - t0 <= DEADLINE_TICKS) return false;
  __hip_atomic_store(tmo, call, __ATOMIC_RELAXED, __HIP_MEMORY_SCOPE_AGENT);
  return true;
}
DI unsigned pk_add(unsigned a, unsigned b) { return __builtin_bit_cast(unsigned, (u16x2)(__builtin_bit_cast(u16x2, a) + __builtin_bit_cast(u16x2, b))); }
DI unsigned pk_sub(unsigned a, unsigned b) { return __builtin_bit_cast(unsigned, (u16x2)(__builtin_bit_cast(u16x2, a) - __builtin_bit_cast(u16x2, b))); }
DI unsigned cvt_pk(float lo, float hi) { f32x2 v = {lo, hi}; return __builtin_bit_cast(unsigned, __builtin_convertvector(v, bf16x2)); }
DI float relu(float v) { return __builtin_elementwise_maximum(v, 0.0f); }
template <int CTRL> DI unsigned dpp(unsigned v) { return __builtin_amdgcn_update_dpp(0u, v, CTRL, 0xf, 0xf, true); }

struct CW {
  f32x4 acc[4][8];
  u32x4 xs[4][4];
  s16x4 wf[8][2];
  unsigned wa[4];
  unsigned fa, ra;
  unsigned xvo[4];
  unsigned need, rv;
  unsigned xso;
  __amdgpu_buffer_rsrc_t xr;
};

DI void wait_ready(CW& c) {
  asm volatile(
      "v_cmp_lt_u32 vcc, %0, %1\n\t"
      "s_cbranch_vccnz 1f\n"
      "2:\n\t"
      ".subsection 1\n"
      "3:\n\t"
      "s_sleep 1\n"
      "1:\n\t"
      "ds_read_b32 %0, %2\n\t"
      "s_waitcnt lgkmcnt(0)\n\t"
      "v_cmp_lt_u32 vcc, %0, %1\n\t"
      "s_cbranch_vccnz 3b\n\t"
      "s_branch 2b\n\t"
      ".subsection 0"
      : "+v"(c.rv) : "v"(c.need), "v"(c.ra) : "vcc", "memory");
}
template <int SET, int M> DI void xload(CW& c) {
  c.xs[SET][M] = __builtin_amdgcn_raw_buffer_load_b128(c.xr, c.xvo[SET] + M * 1024, c.xso, 17);
}
template <int SET, int M> DI void xreload(CW& c) {
  asm volatile("" :: "v"(c.xs[SET][M])); xload<SET, M>(c);
}
template <int SLOT, int J, int KK, int T> DI void wread(CW& c) {
  c.wf[(2 * J + KK) & 7][T] = __builtin_amdgcn_ds_read_tr16_b64_v4i16((LAS s16x4*)((LAS char*)0 + c.wa[J & 3] + ((SLOT + KK) * 8192 + (J >> 2) * 4096 + T * 1024)));
}
template <int P, int J, int I> DI void mma(CW& c) {
  constexpr int M = (J & 1) ? 3 - (I >> 1) : (I >> 1), KK = ((I >> 1) ^ I) & 1, E = (2 * J + KK) & 7;
  const s16x8 w = __builtin_shufflevector(c.wf[E][0], c.wf[E][1], 0, 1, 2, 3, 4, 5, 6, 7);
  c.acc[M][J] = __builtin_amdgcn_mfma_f32_16x16x32_bf16(__builtin_bit_cast(bf16x8, w), __builtin_bit_cast(bf16x8, c.xs[2 * P + KK][M]), c.acc[M][J], 0, 0, 0);
}
template <int P, int J> DI void frag(CW& c, unsigned wd) {
  constexpr int JP = (J + 2) & 7, SLOT = J >= 6 ? 2 * (1 - P) : 2 * P, O = 2 * (1 - P);
  mma<P, J, 0>(c); PIN(); wread<SLOT, JP, 0, 0>(c); PIN();
  mma<P, J, 1>(c); PIN();
  if constexpr (J < 2) xreload<O, 2 * J>(c);
  if constexpr (J == 6) { c.need += 2; lds_st(c.fa, c.need); }
  PIN();
  mma<P, J, 2>(c); PIN(); wread<SLOT, JP, 0, 1>(c); PIN();
  mma<P, J, 3>(c); PIN();
  if constexpr (J < 2) xreload<O + 1, 2 * J>(c);
  if constexpr (J == 6) c.rv = *(volatile LAS unsigned*)((LAS char*)0 + c.ra);
  PIN();
  mma<P, J, 4>(c); PIN(); wread<SLOT, JP, 1, 0>(c); PIN();
  mma<P, J, 5>(c); PIN();
  if constexpr (J < 2) xreload<O, 2 * J + 1>(c);
  if constexpr (J == 3 && P == 0) { c.xso += 4 * 64 * 4096; asm volatile("" : "+s"(c.xso)); }
  if constexpr (J == 7) wait_ready(c);
  PIN();
  mma<P, J, 6>(c); PIN(); wread<SLOT, JP, 1, 1>(c); PIN();
  mma<P, J, 7>(c); PIN();
  if constexpr (J < 2) xreload<O + 1, 2 * J + 1>(c);
  if constexpr (P == 1 && J >= 2 && J < 6) { c.wa[JP & 3] += wd; asm volatile("" : "+v"(c.wa[JP & 3])); }
  if constexpr (J == 7) asm volatile("" :: "v"(c.wf[(2 * J + 3) & 7][1]), "v"(c.xs[O + 1][3]));
  else asm volatile("" :: "v"(c.wf[(2 * J + 3) & 7][1]));
  PIN();
}
template <int P> DI void dstep(CW& c, unsigned wd) {
  frag<P, 0>(c, wd); frag<P, 1>(c, wd); frag<P, 2>(c, wd); frag<P, 3>(c, wd);
  frag<P, 4>(c, wd); frag<P, 5>(c, wd); frag<P, 6>(c, wd); frag<P, 7>(c, wd);
}
template <int S> DI void first_x(CW& c) {
  c.need = S + 1; wait_ready(c); PIN();
  xload<S, 0>(c); xload<S, 1>(c); xload<S, 2>(c); xload<S, 3>(c); PIN();
}

DI bool wait_f1(const char* f, unsigned call, unsigned* tmo, unsigned& payload) {
  const u64 t0 = rt();
  for (unsigned n = 1;; ++n) {
    u64 v = 0;
    if (threadIdx.x % 64 == 0) v = peek64(f);
    if (rfl((unsigned)v) >= call) { payload = rfl((unsigned)(v >> 32)); return true; }
    if ((n & 63u) == 0u && timed_out(tmo, call, t0)) return false;
    __builtin_amdgcn_s_sleep(4);
  }
}

struct Pull {
  __amdgpu_buffer_rsrc_t pr, xw; const char* f1; unsigned* f2; unsigned* tmo; unsigned* tick; LAS unsigned char* pat;
  unsigned call, tbase, hurry; int s, lane;
  unsigned j, tgo, fcall, info;
};
DI bool pull(Pull& q) {
  const unsigned k = (q.j + J0) * 2, u = k >> 2;
  unsigned info = q.info;
  if (q.fcall < q.call && !wait_f1(q.f1 + F1POS(q.j), q.call, q.tmo, info)) return false;
  const unsigned n0 = (info >> 8) & 0xffu, n1 = info >> 24;
#pragma unroll
  for (int w = 0; w < 2; ++w) {
    const unsigned q4 = ((info >> (16 * w + 1)) & 0x7cu) * 0x01010101u;
    *(LAS u32x4*)(q.pat + w * 2048 + q.lane * 16) = (u32x4){q4, q4, q4, q4};
    *(LAS u32x4*)(q.pat + w * 2048 + 1024 + q.lane * 16) = (u32x4){q4, q4, q4, q4};
  }
  if (q.j >= NJOB - TAILP) __builtin_amdgcn_s_setprio(2);
  sleep_until(q.tgo);
  unsigned tk;
  asm volatile("s_mov_b64 exec, 1\n\tglobal_atomic_add %0, %1, %2, off sc0\n\ts_mov_b64 exec, -1" : "=&v"(tk) : "v"(q.tick), "v"(1u) : "memory");
  PIN();
  const unsigned eidx = (q.lane & 32) + 2 * (q.lane & 15), en = (q.lane & 16) ? n1 : n0;
  const unsigned e = __builtin_amdgcn_raw_buffer_load_b32(q.pr, eidx < en ? (q.lane & 31) * 4u + (q.lane >> 5) * (E1_REL - E0_REL) : 0x80000000u, E0_REL + k * 64, 17);
  u32x4 lo[2][2], hb[2][2]; u32x3 pl[2];
#pragma unroll
  for (int w = 0; w < 2; ++w) {
    PIN();
    lo[w][0] = __builtin_amdgcn_raw_buffer_load_b128(q.pr, q.lane * 16, (k + w) * BBYTES, 17);
    lo[w][1] = __builtin_amdgcn_raw_buffer_load_b128(q.pr, q.lane * 16 + 1024, (k + w) * BBYTES, 17);
    PIN();
    pl[w] = __builtin_amdgcn_raw_buffer_load_b96(q.pr, q.lane * 12 + 2048, (k + w) * BBYTES, 17);
  }
  PIN();
  {
    LAS unsigned char* const pb = q.pat + ((q.lane >> 4) & 1) * 2048;
    *(eidx < en ? pb + (e & 2047u) : q.pat + 4096 + q.lane) = (e >> 9) & 0x7cu;
    *(eidx + 1 < en ? pb + ((e >> 16) & 2047u) : q.pat + 4096 + q.lane) = (e >> 25) & 0x7cu;
  }
  FENCE();
#pragma unroll
  for (int w = 0; w < 2; ++w) { hb[w][0] = *(LAS u32x4*)(q.pat + w * 2048 + q.lane * 16); hb[w][1] = *(LAS u32x4*)(q.pat + w * 2048 + 1024 + q.lane * 16); }
  PIN();
  asm volatile("s_waitcnt vmcnt(0)" : "+v"(tk) : "v"(pl[1]), "v"(hb[1][1]) : "memory");
  if (max(n0, n1) == 255u) {
#pragma unroll
    for (int w = 0; w < 2; ++w)
      if ((w ? n1 : n0) == 255u) {
        hb[w][0] = __builtin_amdgcn_raw_buffer_load_b128(q.pr, q.lane * 16, HB_REL + (k + w) * 2048, 17);
        hb[w][1] = __builtin_amdgcn_raw_buffer_load_b128(q.pr, q.lane * 16 + 1024, HB_REL + (k + w) * 2048, 17);
      }
    asm volatile("" :: "v"(hb[0][0]), "v"(hb[0][1]), "v"(hb[1][0]), "v"(hb[1][1]));
  }
  PIN();
  q.j = rfl(tk) - q.tbase;
  const unsigned jn = min(q.j, NJOB - 1);
  const int over = (int)((unsigned)rt() - q.tgo) - LHI - (int)q.hurry * HLHI;
  const u64 dt = over > 0 ? (over > DMAX ? DMAX : over) : 0;
  u64 t0, fv;
  asm volatile("s_mov_b64 exec, 1\n\tglobal_atomic_add_x2 %0, %2, %3, off sc0\n\tglobal_atomic_add_x2 %1, %4, %5, off sc0 sc1\n\ts_mov_b64 exec, -1"
               : "=&v"(t0), "=&v"(fv) : "v"(q.tick + 32), "v"(dt), "v"(q.f1 + F1POS(jn)), "v"(0ull) : "memory");
  const unsigned d = ((u >> 1) * 64 + q.s * NRG + (k & 7u)) * 4096;
#pragma unroll
  for (int w = 0; w < 2; ++w) {
    const unsigned sh = ((info >> (16 * w)) & 7u) * 0x00800080u;
    const unsigned a3 = ((pl[w][0] << 1) & 0x88888888u) | ((pl[w][1] >> 1) & 0x22222222u) | ((pl[w][2] >> 2) & 0x11111111u);
#pragma unroll
    for (int m = 0; m < 4; ++m) {
      const unsigned a = m < 3 ? pl[w][m < 3 ? m : 0] : a3, l0 = lo[w][m >> 1][2 * (m & 1)], l1 = lo[w][m >> 1][2 * (m & 1) + 1];
      const unsigned h0 = (a & 0x83838383u) | hb[w][m >> 1][2 * (m & 1)], h1 = (__builtin_rotateright32(a, 4) & 0x83838383u) | hb[w][m >> 1][2 * (m & 1) + 1];
      const u32x4 o = {pk_sub(__builtin_amdgcn_perm(h0, l0, 0x05010400u), sh), pk_sub(__builtin_amdgcn_perm(h0, l0, 0x07030602u), sh),
                       pk_sub(__builtin_amdgcn_perm(h1, l1, 0x05010400u), sh), pk_sub(__builtin_amdgcn_perm(h1, l1, 0x07030602u), sh)};
      asm volatile("buffer_store_dwordx4 %0, %1, %2, %3 offen offset:%4 sc0 sc1\n\ts_nop 1"
                   :: "v"(o), "v"(q.lane * 16u), "s"(q.xw), "s"(d + w * 4096), "i"(m * 1024) : "memory");
    }
  }
  asm volatile("s_waitcnt vmcnt(0)" : "+v"(t0), "+v"(fv) :: "memory");
  count_in(q.f2 + (u & 1) * NSTEP + (u >> 1), 2u);
  q.hurry = rfl((unsigned)(t0 >> 32)) >> 24;
  q.tgo = rfl((unsigned)t0) + (unsigned)dt + jn * (TAU16 - q.hurry) / 32; q.fcall = rfl((unsigned)fv); q.info = rfl((unsigned)(fv >> 32));
  return true;
}

DI void wfill(__amdgpu_buffer_rsrc_t wr, unsigned ldsaddr, unsigned vo, unsigned so) {
#pragma unroll
  for (int i = 0; i < 8; ++i)
    asm volatile("s_mov_b32 m0, %3\n\ts_nop 0\n\tbuffer_load_dwordx4 %0, %1, %2 offen lds"
                 :: "v"(vo), "s"(wr), "s"(so + (16 * ((i >> 1) & 1) + 4 * (i & 1)) * (NL * 2) + (i >> 2) * 128), "s"(ldsaddr + i * 1024) : "memory");
}
DI void wtouch(const char* p, unsigned n) {
  unsigned t, o = 0;
  asm volatile(
      "1:\n\t"
      "s_load_dword %0, %3, %1 offset:0\n\ts_load_dword %0, %3, %1 offset:64\n\ts_load_dword %0, %3, %1 offset:128\n\ts_load_dword %0, %3, %1 offset:192\n\t"
      "s_load_dword %0, %3, %1 offset:256\n\ts_load_dword %0, %3, %1 offset:320\n\ts_load_dword %0, %3, %1 offset:384\n\ts_load_dword %0, %3, %1 offset:448\n\t"
      "s_add_u32 %1, %1, 0x20000\n\t"
      "s_sub_u32 %2, %2, 1\n\t"
      "s_cmp_lg_u32 %2, 0\n\t"
      "s_cbranch_scc1 1b\n\t"
      "s_waitcnt lgkmcnt(0)" : "=&s"(t), "+s"(o), "+s"(n) : "s"(p) : "scc", "memory");
}

DI void place_raw(char* ws, unsigned par, int rank, int step, int rg, int lane, const u32x4 (&v)[4]) {
  char* d = ws + XT_OFF + par * XT_BYTES + (size_t)(step * 64 + rank * NRG + rg) * 4096 + lane * 16;
#pragma unroll
  for (int m = 0; m < 4; ++m) st_sys(d + m * 1024, v[m]);
  wait_vm0();
  count_in((unsigned*)(ws + F2_OFF) + (rank * 2 + (rg >> 2)) * NSTEP + step, 1u);
}

__global__ __launch_bounds__(BLOCK, 2) void ag_gemm(Params p) {
  extern __shared__ __attribute__((aligned(16))) char lds[];
  const int lane = threadIdx.x & 63;
  const int wave = rfl(threadIdx.x >> 6);
  const unsigned lb = (unsigned)(size_t)(LAS char*)lds;
  const int b = blockIdx.x, rank = p.rank & (NR - 1);
  const char* xin = (const char*)p.x; char* const my = p.my;
  asm volatile("" : "+s"(xin) : "s"(my), "s"(p.rank), "s"(p.w));
  const int xcd = b & 7, bi = b >> 3;
  const int tm = (xcd >> 1) * 4 + (bi >> 3), tn = (xcd & 1) * 8 + (bi & 7);
  unsigned* const tmo = (unsigned*)(my + CT_OFF);
  const unsigned tstart = (unsigned)rt();

  const int ppair = (bi >> 1) * 8 + xcd;
  constexpr int RAW0 = GRID - NRAW * NRG * (NR - 1);
  const int rblk = (b - RAW0) / (NR - 1);
  const bool pusher = wave == 6 && b >= RAW0;
  const int pstep = pusher ? rblk >> 3 : 2 * (ppair >> 1) + (bi & 1), prg = pusher ? rblk & 7 : (ppair & 1) * 4 + wave;
  u32x4 pv[4];
  char* peer = nullptr;
  if (wave < 4 || pusher) {
    if (pstep >= XFIRST) sleep_until(tstart + pstep * XGAP / 16);
    const auto xr = __builtin_amdgcn_make_buffer_rsrc((void*)xin, 0, ML * KD * 2, 0x00020000);
#pragma unroll
    for (int m = 0; m < 4; ++m) pv[m] = __builtin_amdgcn_raw_buffer_load_b128(xr, (lane & 15) * (KD * 2) + (lane >> 4) * 16, ((prg * 64 + m * 16) * KD + pstep * 32) * 2, 0);
    PIN();
    peer = p.ws[(pusher ? rank + 1 + (b - RAW0) % (NR - 1) : lane >> 1) & (NR - 1)];
  }
  const auto wr = __builtin_amdgcn_make_buffer_rsrc((void*)p.w, 0, KD * NL * 2, 0x00020000);
  const unsigned wvo = (8 * ((lane >> 3) & 1) + (lane >> 4)) * (NL * 2) + ((((lane >> 1) & 3) ^ (lane >> 4)) * 32) + (lane & 1) * 16;
  if (wave == 4) {
    __builtin_amdgcn_s_sleep(4);
    for (int s = 0; s < W0; ++s) wfill(wr, lb + s * 8192, wvo, s * 32 * (NL * 2) + tn * 256);
  }
  if (wave == 5 && bi >= 8) {
    const int grp = (bi >> 3) - 1;
    size_t o = grp == 0 ? (lane < 2 ? F1_OFF + lane * F1_BYTES + (bi & 7) * (NUNIT * 128) : XT_OFF + (lane - 2) * XT_BYTES + 3 * 64 * 4096)
                        : PK_OFF + (grp - 1) * PK_BYTES + (lane < 3 ? lane * (1u << 20) : E0_REL);
    const char* a = p.ws[bi & 7] + o + (rank * 8 + xcd) * 128;
    if (lane < 4) asm volatile("s_mov_b32 m0, %1\n\ts_nop 0\n\tglobal_load_lds_dword %0, off sc0 sc1" :: "v"(a), "s"(lb + L_MISC + 256) : "memory");
  }
  if (wave == 6) {
    unsigned t = 0;
    if (lane == 0) t = __hip_atomic_fetch_add((unsigned*)(my + CC_OFF + b * 128), 1u, __ATOMIC_RELAXED, __HIP_MEMORY_SCOPE_AGENT);
    t = rfl(t) + 1;
    if (lane < 16) lds_st(lb + L_MISC + lane * 4, lane == 0 ? t : 0u);
  } else if (wave == 5 || wave == 7) {
#pragma unroll
    for (int i = 0; i < 4; ++i) lds_st(lb + L_FREE + (wave - 5) * 512 + i * 256 + lane * 4, 0u);
  }
  asm volatile("s_waitcnt lgkmcnt(0)" ::: "memory");
  __builtin_amdgcn_s_barrier();
  FENCE();
  const unsigned call = rfl(lds_ld(lb + L_MISC));
  const unsigned par = call & 1u;

  if (wave < 4) {
    __builtin_amdgcn_s_setprio(3);
    if (pstep < NRAW) {
      asm volatile("s_mov_b64 exec, 1\n\tds_add_u32 %0, %1\n\ts_mov_b64 exec, -1" :: "v"(lb + L_MISC + 8), "v"(1u) : "memory");
    } else {
      unsigned v = 0;
#pragma unroll
      for (int m = 0; m < 4; ++m) { v = __builtin_amdgcn_sad_u8(pv[m][0] & 0x7f007f00u, 0u, v); v = __builtin_amdgcn_sad_u8(pv[m][2] & 0x7f007f00u, 0u, v); }
      v += dpp<0x111>(v); v += dpp<0x112>(v); v += dpp<0x114>(v); v += dpp<0x118>(v);
      const unsigned top = (rl(v, 15) + rl(v, 31) + rl(v, 47) + rl(v, 63) + 1705u) >> 9, low = min(top > 7u ? top - 7u : 0u, 248u);
      const unsigned sh = (0u - low) & 7u, qw = (low + sh) >> 3, q4 = (qw << 2) * 0x01010101u;
      u32x4 lo[2], hi[2]; unsigned a[4], mask = 0;
#pragma unroll
      for (int m = 0; m < 4; ++m) {
        const unsigned d0 = pk_add(pv[m][0], sh * 0x00800080u), d1 = pk_add(pv[m][1], sh * 0x00800080u), d2 = pk_add(pv[m][2], sh * 0x00800080u), d3 = pk_add(pv[m][3], sh * 0x00800080u);
        const unsigned h0 = __builtin_amdgcn_perm(d1, d0, 0x07050301u), h1 = __builtin_amdgcn_perm(d3, d2, 0x07050301u);
        lo[m >> 1][2 * (m & 1)] = __builtin_amdgcn_perm(d1, d0, 0x06040200u); lo[m >> 1][2 * (m & 1) + 1] = __builtin_amdgcn_perm(d3, d2, 0x06040200u);
        hi[m >> 1][2 * (m & 1)] = h0; hi[m >> 1][2 * (m & 1) + 1] = h1;
        a[m] = (h0 & 0x83838383u) | __builtin_rotateleft32(h1 & 0x83838383u, 4);
        mask |= ((((h0 ^ q4) & 0x7c7c7c7cu) + 0x7c7c7c7cu) & 0x80808080u) >> (7 - 2 * m);
        mask |= ((((h1 ^ q4) & 0x7c7c7c7cu) + 0x7c7c7c7cu) & 0x80808080u) >> (6 - 2 * m);
      }
      const int u = pstep * 2 + (prg >> 2), blk = u * 4 + wave;
      char* const pk = my + PK_OFF + par * PK_BYTES;
      char* fp = peer + F1_OFF + par * F1_BYTES + (size_t)(rank * NUNIT + u) * 128 + (lane & 1) * 8;
      asm volatile("" : "+v"(fp));
      st_sys(pk + blk * BBYTES + lane * 16, lo[0]); st_sys(pk + blk * BBYTES + 1024 + lane * 16, lo[1]);
      const u32x3 pl = {a[0] | ((a[3] >> 1) & 0x44444444u), a[1] | ((a[3] << 1) & 0x44444444u), a[2] | ((a[3] << 2) & 0x44444444u)};
      asm volatile("global_store_dwordx3 %0, %1, off sc0 sc1\n\ts_nop 1" :: "v"(pk + blk * BBYTES + 2048 + lane * 12), "v"(pl) : "memory");
      LAS unsigned char* const hcopy = (LAS unsigned char*)lds + L_EPI + wave * 4096;
      LAS unsigned short* const list = (LAS unsigned short*)(hcopy + 2048);
      *(LAS u32x4*)(hcopy + lane * 16) = hi[0]; *(LAS u32x4*)(hcopy + 1024 + lane * 16) = hi[1];
      unsigned cnt = 0;
      do {
#pragma unroll
        for (int i = 0; i < 4; ++i) {
          const u64 bal = __ballot(mask != 0u);
          const unsigned idx = cnt + __builtin_amdgcn_mbcnt_hi((unsigned)(bal >> 32), __builtin_amdgcn_mbcnt_lo((unsigned)bal, 0u));
          list[mask ? idx : 512u + lane] = (unsigned short)(__builtin_ctz(mask | 0x80000000u) | (lane << 5));
          mask &= mask - 1u; cnt += __builtin_popcountll(bal);
        }
      } while (cnt <= 64u && __any(mask != 0u));
      FENCE();
      if (cnt > 64u) {
        cnt = 255u;
        st_sys(pk + HB_REL + blk * 2048 + lane * 16, hi[0]); st_sys(pk + HB_REL + blk * 2048 + 1024 + lane * 16, hi[1]);
      } else {
        const unsigned t = list[lane], pos = ((t & 4u) << 8) | ((t >> 1) & 0x3f0u) | ((t & 3u) << 2) | ((t >> 3) & 3u);
        const unsigned e = pos | ((hcopy[pos] & 0x7cu) << 9);
        const unsigned e2 = e | (dpp<0xb1>(e) << 16);
        if ((unsigned)lane < cnt && !(lane & 1))
          __hip_atomic_store((unsigned*)(pk + (lane < 32 ? E0_REL : E1_REL - 64) + blk * 64 + lane * 2), e2, __ATOMIC_RELAXED, __HIP_MEMORY_SCOPE_SYSTEM);
      }
      wait_vm0();
      unsigned old;
      asm volatile("s_mov_b64 exec, 1\n\tds_write_b16 %1, %2\n\tds_add_rtn_u32 %0, %3, %4\n\ts_mov_b64 exec, -1\n\ts_waitcnt lgkmcnt(0)"
                   : "=&v"(old) : "v"(lb + L_MISC + 16 + 2 * wave), "v"(sh | (qw << 3) | (cnt << 8)), "v"(lb + L_MISC + 8), "v"(1u) : "memory");
      if (rfl(old) == 3u) {
        const unsigned pair = lds_ld(lb + L_MISC + 16 + (lane & 1) * 4);
        if (lane < 2 * NR && (lane >> 1) != rank)
          asm volatile("global_store_dwordx2 %0, %1, off sc0 sc1" :: "v"(fp), "v"(((u64)pair << 32) | call) : "memory");
      }
    }
    __builtin_amdgcn_s_setprio(1);
    sleep_until(tstart + OWN0 + pstep * OWNGAP / 16);
    place_raw(my, par, rank, pstep, prg, lane, pv);

    CW c;
    char* outp = (char*)p.out; asm volatile("" : "+s"(outp));
    const int g = lane >> 4, q = (lane >> 2) & 3, pp = lane & 3;
#pragma unroll
    for (int m = 0; m < 4; ++m)
#pragma unroll
      for (int j = 0; j < 8; ++j) c.acc[m][j] = (f32x4){0.f, 0.f, 0.f, 0.f};
#pragma unroll
    for (int i = 0; i < 4; ++i) c.wa[i] = lb + (g >> 1) * 2048 + q * 256 + (g & 1) * 128 + ((i ^ q) * 32) + pp * 8;
    c.fa = lb + L_FREE + wave * 256 + lane * 4; c.ra = lb + L_READY + lane * 4;
    c.rv = 0;
    c.xr = __builtin_amdgcn_make_buffer_rsrc((void*)(my + XT_OFF + par * XT_BYTES), 0, (unsigned)XT_BYTES, 0x00020000);
#pragma unroll
    for (int i = 0; i < 4; ++i) { c.xvo[i] = lane * 16 + i * 64 * 4096; asm volatile("" : "+v"(c.xvo[i])); }
    c.xso = (tm * 4 + wave) * 4096;
    PIN();
    first_x<0>(c); first_x<1>(c);
    wread<0, 0, 0, 0>(c); wread<0, 0, 0, 1>(c); wread<0, 0, 1, 0>(c); wread<0, 0, 1, 1>(c);
#pragma unroll
    for (int m = 0; m < 4; ++m) { c.xs[2][m] = (u32x4){0, 0, 0, 0}; c.xs[3][m] = (u32x4){0, 0, 0, 0}; asm volatile("" : "+v"(c.xs[2][m]), "+v"(c.xs[3][m])); }
    wread<0, 1, 0, 0>(c); wread<0, 1, 0, 1>(c); wread<0, 1, 1, 0>(c); wread<0, 1, 1, 1>(c);
    c.need = 4; asm volatile("" : "+v"(c.need));
    wait_ready(c);
    asm volatile("" :: "v"(c.xs[1][3]), "v"(c.wf[1][1]));
    PIN();
#pragma unroll 1
    for (int it = 0; it < NSTEP / 4; ++it) {
      const unsigned wd = (it + 1) % (RING / 4) == 0 ? (unsigned)(-(RING - 4) * 8192) : (unsigned)(4 * 8192);
      dstep<0>(c, wd); dstep<1>(c, wd);
    }

    const unsigned vd = *(volatile LAS unsigned*)((LAS char*)lds + L_MISC + 4);
    int tz = threadIdx.x; asm volatile("" : "+v"(tz));
    const int l2 = tz & 63;
    const unsigned ovo = (l2 >> 4) * (NL * 2) + (l2 & 15) * 16;
    const auto orr = __builtin_amdgcn_make_buffer_rsrc((void*)(outp + ((size_t)(tm * 256 + wave * 64) * NL + tn * 128) * 2), 0, 63 * NL * 2 + 256, 0x00020000);
    LAS char* const eb = (LAS char*)lds + L_EPI + wave * 4096;
    const unsigned ew = (l2 & 15) * 256 + ((l2 >> 4) & 1) * 8 + (((l2 >> 5) ^ (l2 & 15)) * 16), er = (l2 >> 4) * 256 + (((l2 & 15) ^ (l2 >> 4)) * 16);
    unsigned verdict = rfl(vd);
    while (verdict == 0u) { __builtin_amdgcn_s_sleep(1); verdict = rfl(lds_ld(lb + L_MISC + 4)); }
    if (verdict == 1u) {
      u32x4 o[4];
#pragma unroll
      for (int m = 0; m <= 4; ++m) {
#pragma unroll
        for (int h = 0; h < 4; ++h) {
          u32x2 t0, t1;
          if (m < 4) {
            f32x4 a = c.acc[m][2 * h], d = c.acc[m][2 * h + 1];
#pragma unroll
            for (int i = 0; i < 4; ++i) { a[i] = relu(a[i]); d[i] = relu(d[i]); }
            PIN();
            t0 = (u32x2){cvt_pk(a[0], a[1]), cvt_pk(a[2], a[3])}; t1 = (u32x2){cvt_pk(d[0], d[1]), cvt_pk(d[2], d[3])};
            PIN();
          }
          if (m > 0)
            asm volatile("buffer_store_dwordx4 %0, %1, %2, %3 offen nt\n\ts_nop 1" :: "v"(o[h]), "v"(ovo), "s"(orr), "s"(((m - 1) * 16 + 4 * h) * NL * 2));
          if (m < 4) { *(LAS u32x2*)(eb + (ew ^ (64 * h))) = t0; *(LAS u32x2*)(eb + (ew ^ (64 * h + 32))) = t1; }
          PIN();
        }
        if (m < 4) {
          FENCE();
#pragma unroll
          for (int i = 0; i < 4; ++i) o[i] = *(const LAS u32x4*)(eb + (er ^ (64 * i)) + 1024 * i);
          FENCE();
          PIN();
        }
      }
    } else {
      const u32x4 nan4 = {0x7fc07fc0u, 0x7fc07fc0u, 0x7fc07fc0u, 0x7fc07fc0u};
#pragma unroll 1
      for (int i = 0; i < 16; ++i) asm volatile("buffer_store_dwordx4 %0, %1, %2, %3 offen\n\ts_nop 1" :: "v"(nan4), "v"(ovo), "s"(orr), "s"(4 * i * NL * 2));
    }
  } else if (wave == 4) {
    __builtin_amdgcn_s_setprio(1);
    const unsigned pubaddr = lb + L_READY + lane * 8 + 4;
    wait_vm0();
    int pub = W0, slot = W0; unsigned freec = 0;
    lds_st(pubaddr, pub);
    for (int s = W0; s < NSTEP; ++s) {
      while (s >= RING && freec < (unsigned)(s - RING + 5)) {
        const unsigned f = lds_ld(lb + L_FREE + (lane & 3) * 256);
        freec = min(min(rl(f, 0), rl(f, 1)), min(rl(f, 2), rl(f, 3)));
        if (freec >= (unsigned)(s - RING + 5)) break;
        if (pub < s && max(max(rl(f, 0), rl(f, 1)), max(rl(f, 2), rl(f, 3))) >= (unsigned)pub) { wait_vm0(); pub = s; lds_st(pubaddr, pub); }
        __builtin_amdgcn_s_sleep(2);
      }
      if (s < RING) {
        const unsigned tgo = tstart + WT0 + (s - W0) * WGAP;
        if ((int)(tgo - (unsigned)rt()) > 0) { if (pub < s) { wait_vm0(); pub = s; lds_st(pubaddr, pub); } sleep_until(tgo); }
      }
      wfill(wr, lb + slot * 8192, wvo, s * 32 * (NL * 2) + tn * 256);
      slot = (slot + 1) & (RING - 1);
      if (s + 1 - pub > INFL) { asm volatile("s_waitcnt vmcnt(%0)" :: "i"(8 * INFL) : "memory"); pub = s + 1 - INFL; lds_st(pubaddr, pub); }
    }
    wait_vm0(); lds_st(pubaddr, INF);
  } else if (wave == 6) {
    __builtin_amdgcn_s_setprio(2);
    if (pusher) {
      sleep_until(tstart + RAWT0 + rblk * RAWGAP / 4);
      place_raw(peer, par, rank, pstep, prg, lane, pv);
    }
    const unsigned xaddr = lb + L_READY + lane * 8;
    const unsigned* f2 = (const unsigned*)(my + F2_OFF) + tm * NSTEP;
    bool dead = false;
    unsigned base = 0;
    const u64 t0 = rt();
    for (unsigned n = 1;; ++n) {
      const unsigned idx = base + lane;
      unsigned v = 4u * call;
      if (idx < (unsigned)NSTEP && lane < POLLW) v = peek(f2 + idx);
      const unsigned nok = ~(unsigned)__ballot(v >= 4u * call);
      base += nok ? (unsigned)__builtin_ctz(nok) : (unsigned)POLLW;
      if (base >= (unsigned)NSTEP) break;
      lds_st(xaddr, base);
      if ((n & 63u) == 0u && timed_out(tmo, call, t0)) { dead = true; break; }
      if (base < (unsigned)(NSTEP - POLLEND)) __builtin_amdgcn_s_sleep(8);
    }
    lds_st(xaddr, INF);
    if (rfl(__hip_atomic_load(tmo, __ATOMIC_RELAXED, __HIP_MEMORY_SCOPE_AGENT)) == call) dead = true;
    lds_st(lb + L_MISC + 4, dead ? 2u : 1u);
  } else if (wave == 7) {
    for (int pf = W0; pf < NSTEP; pf += PFB) wtouch((const char*)p.w + b * 512 + (size_t)pf * 131072, pf + PFB <= NSTEP ? PFB : NSTEP - pf);
  } else if (b < PW * (NR - 1)) {
    Pull q;
    const int qi = b % (NR - 1);
    q.s = (rank + 1 + qi) & (NR - 1);
    q.tmo = tmo; q.call = call; q.lane = lane; q.pat = (LAS unsigned char*)lds + L_PAT;
    q.tick = (unsigned*)(my + CT_OFF + (2 + 2 * qi) * 128);
    q.pr = __builtin_amdgcn_make_buffer_rsrc((void*)(p.ws[q.s] + PK_OFF + par * PK_BYTES), 0, (unsigned)PK_BYTES, 0x00020000);
    q.xw = __builtin_amdgcn_make_buffer_rsrc((void*)(my + XT_OFF + par * XT_BYTES), 0, (unsigned)XT_BYTES, 0x00020000);
    q.f1 = my + F1_OFF + par * F1_BYTES + (size_t)q.s * NUNIT * 128;
    q.f2 = (unsigned*)(my + F2_OFF) + q.s * 2 * NSTEP;
    q.tbase = (call - 1u) * (NJOB + PW);
    u64 t0old = 0, old = 0, fv = 0; unsigned t = 0;
    if (lane == 0)
      asm volatile("global_atomic_add_x2 %0, %2, %4, off sc0\n\tglobal_atomic_add_x2 %1, %3, %4, off sc0 sc1\n\ts_waitcnt vmcnt(0)"
                   : "=&v"(t0old), "=&v"(fv) : "v"(q.tick + 32), "v"(q.f1 + F1POS(0u)), "v"(0ull) : "memory");
    if ((t0old & 0xffffffffffffffull) + 1500 > rt()) t0old = ~0ull;
    if (rfl((unsigned)fv) < call || pstep < EARLYP)
      while (rfl(lds_ld(lb + L_MISC + 8)) < 4u) __builtin_amdgcn_s_sleep(1);
    if (lane == 0)
      asm volatile("global_atomic_add %0, %2, %3, off sc0\n\tglobal_atomic_add_x2 %1, %4, %5, off sc0 sc1\n\ts_waitcnt vmcnt(0)"
                   : "=&v"(t), "=&v"(fv) : "v"(q.tick), "v"(1u), "v"(q.f1 + F1POS(0u)), "v"(0ull) : "memory");
    q.j = rfl(t) - q.tbase; q.info = rfl((unsigned)(fv >> 32)); q.fcall = rfl((unsigned)fv);
    const unsigned jn = q.j < NJOB ? q.j : 0u;
    const bool lead = jn < (unsigned)FASTGO && (q.fcall < call || jn == 0u);
    bool ok = q.fcall >= call || wait_f1(q.f1 + F1POS(lead ? jn : 0u), call, tmo, q.info);
    u64 t0 = rt() + T0MARG;
    {
      const int late = (int)((unsigned)t0 - tstart) - ONTIME;
      t0 |= (u64)(late > 0 ? min(late / 16, CATCH) : 0) << 56;
    }
    const u32x4 cs = {(unsigned)t0, (unsigned)(t0 >> 32), (unsigned)t0old, (unsigned)(t0old >> 32)};
    if (lead) {
      if (lane == 0) asm volatile("global_atomic_cmpswap_x2 %0, %1, off\n\ts_nop 1" :: "v"(q.tick + 32), "v"(cs) : "memory");
      t0 -= T0MARG; q.fcall = call;
    } else {
      if (lane == 0)
        asm volatile("global_atomic_cmpswap_x2 %0, %2, %3, off sc0\n\tglobal_atomic_add_x2 %1, %4, %5, off sc0 sc1\n\ts_waitcnt vmcnt(0)"
                     : "=&v"(old), "=&v"(fv) : "v"(q.tick + 32), "v"(cs), "v"(q.f1 + F1POS(jn)), "v"(0ull) : "memory");
      if (old != t0old) t0 = old;
      q.fcall = rfl((unsigned)fv); q.info = rfl((unsigned)(fv >> 32));
    }
    q.hurry = rfl((unsigned)(t0 >> 32)) >> 24;
    q.tgo = rfl((unsigned)t0) + jn * (TAU16 - q.hurry) / 32;
    while (ok && q.j < NJOB) ok = pull(q);
  }
}

}

extern "C" void kernel_launch(int rank, int nranks, ncclComm_t comm, hipStream_t stream,
                              void* const* d_in, const int* in_sizes, int n_in,
                              void* d_out, int out_size, const int* in_dtypes, int out_dtype,
                              void* const* d_out_all, void* const* d_ws_all, size_t ws_bytes) {
  if (nranks != NR || (unsigned)rank >= (unsigned)NR || n_in != 2 || in_sizes[0] != ML * KD || in_sizes[1] != KD * NL ||
      out_size != MG * NL || ws_bytes < WS_NEED) { fprintf(stderr, "unexpected problem shape\n"); return; }
  static bool attr_set = false;
  if (!attr_set) { HIP_CHECK(hipFuncSetAttribute((const void*)ag_gemm, hipFuncAttributeMaxDynamicSharedMemorySize, LDS_BYTES)); attr_set = true; }
  Params p;
  p.x = d_in[0]; p.my = (char*)d_ws_all[rank]; p.w = d_in[1]; p.out = d_out; p.rank = rank; p.pad = 0;
  for (int i = 0; i < NR; ++i) p.ws[i] = (char*)d_ws_all[i];
  ag_gemm<<<GRID, BLOCK, LDS_BYTES, stream>>>(p);
  HIP_CHECK(hipGetLastError());
}
```

```cpp
#include <hip/hip_runtime.h>
#include <rccl/rccl.h>
#include <cstdio>

#define HIP_CHECK(expr) do { hipError_t _e = (expr); if (_e != hipSuccess) { \
    fprintf(stderr, "HIP error %s at %s:%d\n", hipGetErrorString(_e), __FILE__, __LINE__); \
    return; } } while (0)

namespace {

#define LAS __attribute__((address_space(3)))
#define DI __device__ __forceinline__
#define PIN() __builtin_amdgcn_sched_barrier(0)
#define FENCE() asm volatile("" ::: "memory")
typedef short    s16x4  __attribute__((ext_vector_type(4)));
typedef short    s16x8  __attribute__((ext_vector_type(8)));
typedef __bf16   bf16x8 __attribute__((ext_vector_type(8)));
typedef __bf16   bf16x2 __attribute__((ext_vector_type(2)));
typedef float    f32x2  __attribute__((ext_vector_type(2)));
typedef float    f32x4  __attribute__((ext_vector_type(4)));
typedef unsigned u32x2  __attribute__((ext_vector_type(2)));
typedef unsigned u32x3  __attribute__((ext_vector_type(3)));
typedef unsigned u32x4  __attribute__((ext_vector_type(4)));
typedef unsigned short u16x2 __attribute__((ext_vector_type(2)));
typedef unsigned long long u64;

constexpr int NR = 8;
constexpr int KD = 4096, NL = 2048, ML = 512, MG = NR * ML;
constexpr int NSTEP = KD / 32;
constexpr int NRG = ML / 64;
constexpr int NUNIT = NSTEP * 2;
constexpr int BBYTES = 2816;
constexpr int GRID = 256, BLOCK = 512;
constexpr int RING = 16;
constexpr int INFL = 6;
constexpr int W0 = 4;
constexpr int PW = 36;
constexpr int NRAW = 2;
constexpr unsigned J0 = NRAW * 4, NJOB = NUNIT * 2 - J0;

constexpr unsigned TAU16 = 312;
constexpr int LHI = 165, DMAX = 3;
constexpr int ONTIME = 500, CATCH = 6, HLHI = 5;
constexpr int T0MARG = 40, FASTGO = 8, EARLYP = 24, TAILP = 40;
constexpr int XGAP = 12, XFIRST = 8 + NRAW;
constexpr int OWN0 = 300, OWNGAP = 48;
constexpr int RAWT0 = 150, RAWGAP = 16;
constexpr int WT0 = 500, WGAP = 25;
constexpr int PFB = 8;
constexpr int POLLW = 16, POLLEND = 6;

constexpr int L_FREE = RING * 8192;
constexpr int L_READY = L_FREE + 1024;
constexpr int L_MISC = L_FREE + 2048;
constexpr int L_EPI = L_MISC + 1024;
constexpr int L_PAT = L_EPI + 4 * 4096;
constexpr int LDS_BYTES = L_PAT + 4096 + 64;
static_assert(LDS_BYTES <= 160 * 1024 && LDS_BYTES > 80 * 1024, "one workgroup per CU");
constexpr unsigned INF = 0x7fffffffu;

constexpr size_t XT_BYTES = (size_t)MG * KD * 2;
constexpr size_t XT_OFF = 0;
constexpr unsigned E0_REL = NUNIT * 4 * BBYTES;
constexpr unsigned E1_REL = E0_REL + NUNIT * 4 * 64;
constexpr unsigned HB_REL = E1_REL + NUNIT * 4 * 64;
constexpr size_t PK_BYTES = HB_REL + NUNIT * 4 * 2048;
constexpr size_t PK_OFF = XT_OFF + 2 * XT_BYTES;
constexpr size_t F1_BYTES = (size_t)NR * NUNIT * 128;
constexpr size_t F1_OFF = PK_OFF + 2 * PK_BYTES;
constexpr size_t F2_OFF = F1_OFF + 2 * F1_BYTES;
constexpr size_t F2_BYTES = (size_t)NR * NUNIT * 4;
constexpr size_t CT_OFF = F2_OFF + F2_BYTES;
constexpr size_t CC_OFF = CT_OFF + 4096;
constexpr size_t WS_NEED = CC_OFF + GRID * 128;
constexpr u64 DEADLINE_TICKS = 100000000ull;
#define F1POS(j) ((((j) + J0) >> 1) * 128 + ((j) & 1u) * 8)

struct Params { const void* x; char* my; const void* w; void* out; int rank; int pad; char* ws[NR]; };

DI unsigned rfl(unsigned v) { return __builtin_amdgcn_readfirstlane(v); }
DI unsigned rl(unsigned v, int l) { return __builtin_amdgcn_readlane(v, l); }
DI u64 rt() { return __builtin_amdgcn_s_memrealtime(); }
DI void st_sys(void* p, u32x4 v) { asm volatile("global_store_dwordx4 %0, %1, off sc0 sc1\n\ts_nop 1" :: "v"(p), "v"(v) : "memory"); }
DI void wait_vm0() { asm volatile("s_waitcnt vmcnt(0)" ::: "memory"); }
DI void lds_st(unsigned addr, unsigned v) { asm volatile("ds_write_b32 %0, %1" :: "v"(addr), "v"(v) : "memory"); }
DI unsigned lds_ld(unsigned addr) { unsigned v; asm volatile("ds_read_b32 %0, %1\n\ts_waitcnt lgkmcnt(0)" : "=&v"(v) : "v"(addr) : "memory"); return v; }
DI void count_in(unsigned* p, unsigned n) { asm volatile("s_mov_b64 exec, 1\n\tglobal_atomic_add %0, %1, off sc1\n\ts_mov_b64 exec, -1" :: "v"(p), "v"(n) : "memory"); }
DI unsigned peek(const unsigned* p) { unsigned v; asm volatile("global_atomic_add %0, %1, %2, off sc0 sc1\n\ts_waitcnt vmcnt(0)" : "=&v"(v) : "v"(p), "v"(0u) : "memory"); return v; }
DI u64 peek64(const void* p) { u64 v; asm volatile("global_atomic_add_x2 %0, %1, %2, off sc0 sc1\n\ts_waitcnt vmcnt(0)" : "=&v"(v) : "v"(p), "v"(0ull) : "memory"); return v; }
DI void sleep_until(unsigned t) {
  for (int d; (d = (int)(t - (unsigned)rt())) > 0 && d < 30000;) { if (d > 200) __builtin_amdgcn_s_sleep(48); else if (d > 40) __builtin_amdgcn_s_sleep(8); else __builtin_amdgcn_s_sleep(1); }
  FENCE();
}
DI bool timed_out(unsigned* tmo, unsigned call, u64 t0) {
  if (rfl(__hip_atomic_load(tmo, __ATOMIC_RELAXED, __HIP_MEMORY_SCOPE_AGENT)) != call && rt() - t0 <= DEADLINE_TICKS) return false;
  __hip_atomic_store(tmo, call, __ATOMIC_RELAXED, __HIP_MEMORY_SCOPE_AGENT);
  return true;
}
DI unsigned pk_add(unsigned a, unsigned b) { return __builtin_bit_cast(unsigned, (u16x2)(__builtin_bit_cast(u16x2, a) + __builtin_bit_cast(u16x2, b))); }
DI unsigned pk_sub(unsigned a, unsigned b) { return __builtin_bit_cast(unsigned, (u16x2)(__builtin_bit_cast(u16x2, a) - __builtin_bit_cast(u16x2, b))); }
DI unsigned cvt_pk(float lo, float hi) { f32x2 v = {lo, hi}; return __builtin_bit_cast(unsigned, __builtin_convertvector(v, bf16x2)); }
DI float relu(float v) { return __builtin_elementwise_maximum(v, 0.0f); }
template <int CTRL> DI unsigned dpp(unsigned v) { return __builtin_amdgcn_update_dpp(0u, v, CTRL, 0xf, 0xf, true); }

struct CW {
  f32x4 acc[4][8];
  u32x4 xs[4][4];
  s16x4 wf[8][2];
  unsigned wa[4];
  unsigned fa, ra;
  unsigned xvo[4];
  unsigned need, rv;
  unsigned xso;
  __amdgpu_buffer_rsrc_t xr;
};

DI void wait_ready(CW& c) {
  asm volatile(
      "v_cmp_lt_u32 vcc, %0, %1\n\t"
      "s_cbranch_vccnz 1f\n"
      "2:\n\t"
      ".subsection 1\n"
      "3:\n\t"
      "s_sleep 1\n"
      "1:\n\t"
      "ds_read_b32 %0, %2\n\t"
      "s_waitcnt lgkmcnt(0)\n\t"
      "v_cmp_lt_u32 vcc, %0, %1\n\t"
      "s_cbranch_vccnz 3b\n\t"
      "s_branch 2b\n\t"
      ".subsection 0"
      : "+v"(c.rv) : "v"(c.need), "v"(c.ra) : "vcc", "memory");
}
template <int SET, int M> DI void xload(CW& c) {
  c.xs[SET][M] = __builtin_amdgcn_raw_buffer_load_b128(c.xr, c.xvo[SET] + M * 1024, c.xso, 17);
}
template <int SET, int M> DI void xreload(CW& c) {
  asm volatile("" :: "v"(c.xs[SET][M])); xload<SET, M>(c);
}
template <int SLOT, int J, int KK, int T> DI void wread(CW& c) {
  c.wf[(2 * J + KK) & 7][T] = __builtin_amdgcn_ds_read_tr16_b64_v4i16((LAS s16x4*)((LAS char*)0 + c.wa[J & 3] + ((SLOT + KK) * 8192 + (J >> 2) * 4096 + T * 1024)));
}
template <int P, int J, int I> DI void mma(CW& c) {
  constexpr int M = (J & 1) ? 3 - (I >> 1) : (I >> 1), KK = ((I >> 1) ^ I) & 1, E = (2 * J + KK) & 7;
  const s16x8 w = __builtin_shufflevector(c.wf[E][0], c.wf[E][1], 0, 1, 2, 3, 4, 5, 6, 7);
  c.acc[M][J] = __builtin_amdgcn_mfma_f32_16x16x32_bf16(__builtin_bit_cast(bf16x8, w), __builtin_bit_cast(bf16x8, c.xs[2 * P + KK][M]), c.acc[M][J], 0, 0, 0);
}
template <int P, int J> DI void frag(CW& c, unsigned wd) {
  constexpr int JP = (J + 2) & 7, SLOT = J >= 6 ? 2 * (1 - P) : 2 * P, O = 2 * (1 - P);
  mma<P, J, 0>(c); PIN(); wread<SLOT, JP, 0, 0>(c); PIN();
  mma<P, J, 1>(c); PIN();
  if constexpr (J < 2) xreload<O, 2 * J>(c);
  if constexpr (J == 6) { c.need += 2; lds_st(c.fa, c.need); }
  PIN();
  mma<P, J, 2>(c); PIN(); wread<SLOT, JP, 0, 1>(c); PIN();
  mma<P, J, 3>(c); PIN();
  if constexpr (J < 2) xreload<O + 1, 2 * J>(c);
  if constexpr (J == 6) c.rv = *(volatile LAS unsigned*)((LAS char*)0 + c.ra);
  PIN();
  mma<P, J, 4>(c); PIN(); wread<SLOT, JP, 1, 0>(c); PIN();
  mma<P, J, 5>(c); PIN();
  if constexpr (J < 2) xreload<O, 2 * J + 1>(c);
  if constexpr (J == 3 && P == 0) { c.xso += 4 * 64 * 4096; asm volatile("" : "+s"(c.xso)); }
  if constexpr (J == 7) wait_ready(c);
  PIN();
  mma<P, J, 6>(c); PIN(); wread<SLOT, JP, 1, 1>(c); PIN();
  mma<P, J, 7>(c); PIN();
  if constexpr (J < 2) xreload<O + 1, 2 * J + 1>(c);
  if constexpr (P == 1 && J >= 2 && J < 6) { c.wa[JP & 3] += wd; asm volatile("" : "+v"(c.wa[JP & 3])); }
  if constexpr (J == 7) asm volatile("" :: "v"(c.wf[(2 * J + 3) & 7][1]), "v"(c.xs[O + 1][3]));
  else asm volatile("" :: "v"(c.wf[(2 * J + 3) & 7][1]));
  PIN();
}
template <int P> DI void dstep(CW& c, unsigned wd) {
  frag<P, 0>(c, wd); frag<P, 1>(c, wd); frag<P, 2>(c, wd); frag<P, 3>(c, wd);
  frag<P, 4>(c, wd); frag<P, 5>(c, wd); frag<P, 6>(c, wd); frag<P, 7>(c, wd);
}
template <int S> DI void first_x(CW& c) {
  c.need = S + 1; wait_ready(c); PIN();
  xload<S, 0>(c); xload<S, 1>(c); xload<S, 2>(c); xload<S, 3>(c); PIN();
}

DI bool wait_f1(const char* f, unsigned call, unsigned* tmo, unsigned& payload) {
  const u64 t0 = rt();
  for (unsigned n = 1;; ++n) {
    u64 v = 0;
    if (threadIdx.x % 64 == 0) v = peek64(f);
    if (rfl((unsigned)v) >= call) { payload = rfl((unsigned)(v >> 32)); return true; }
    if ((n & 63u) == 0u && timed_out(tmo, call, t0)) return false;
    __builtin_amdgcn_s_sleep(4);
  }
}

struct Pull {
  __amdgpu_buffer_rsrc_t pr, xw; const char* f1; unsigned* f2; unsigned* tmo; unsigned* tick; LAS unsigned char* pat;
  unsigned call, tbase, hurry; int s, lane;
  unsigned j, tgo, fcall, info;
};
DI bool pull(Pull& q) {
  const unsigned k = (q.j + J0) * 2, u = k >> 2;
  unsigned info = q.info;
  if (q.fcall < q.call && !wait_f1(q.f1 + F1POS(q.j), q.call, q.tmo, info)) return false;
  const unsigned n0 = (info >> 8) & 0xffu, n1 = info >> 24;
#pragma unroll
  for (int w = 0; w < 2; ++w) {
    const unsigned q4 = ((info >> (16 * w + 1)) & 0x7cu) * 0x01010101u;
    *(LAS u32x4*)(q.pat + w * 2048 + q.lane * 16) = (u32x4){q4, q4, q4, q4};
    *(LAS u32x4*)(q.pat + w * 2048 + 1024 + q.lane * 16) = (u32x4){q4, q4, q4, q4};
  }
  if (q.j >= NJOB - TAILP) __builtin_amdgcn_s_setprio(2);
  sleep_until(q.tgo);
  unsigned tk;
  asm volatile("s_mov_b64 exec, 1\n\tglobal_atomic_add %0, %1, %2, off sc0\n\ts_mov_b64 exec, -1" : "=&v"(tk) : "v"(q.tick), "v"(1u) : "memory");
  PIN();
  const unsigned eidx = (q.lane & 32) + 2 * (q.lane & 15), en = (q.lane & 16) ? n1 : n0;
  const unsigned e = __builtin_amdgcn_raw_buffer_load_b32(q.pr, eidx < en ? (q.lane & 31) * 4u + (q.lane >> 5) * (E1_REL - E0_REL) : 0x80000000u, E0_REL + k * 64, 17);
  u32x4 lo[2][2], hb[2][2]; u32x3 pl[2];
#pragma unroll
  for (int w = 0; w < 2; ++w) {
    PIN();
    lo[w][0] = __builtin_amdgcn_raw_buffer_load_b128(q.pr, q.lane * 16, (k + w) * BBYTES, 17);
    lo[w][1] = __builtin_amdgcn_raw_buffer_load_b128(q.pr, q.lane * 16 + 1024, (k + w) * BBYTES, 17);
    PIN();
    pl[w] = __builtin_amdgcn_raw_buffer_load_b96(q.pr, q.lane * 12 + 2048, (k + w) * BBYTES, 17);
  }
  PIN();
  {
    LAS unsigned char* const pb = q.pat + ((q.lane >> 4) & 1) * 2048;
    *(eidx < en ? pb + (e & 2047u) : q.pat + 4096 + q.lane) = (e >> 9) & 0x7cu;
    *(eidx + 1 < en ? pb + ((e >> 16) & 2047u) : q.pat + 4096 + q.lane) = (e >> 25) & 0x7cu;
  }
  FENCE();
#pragma unroll
  for (int w = 0; w < 2; ++w) { hb[w][0] = *(LAS u32x4*)(q.pat + w * 2048 + q.lane * 16); hb[w][1] = *(LAS u32x4*)(q.pat + w * 2048 + 1024 + q.lane * 16); }
  PIN();
  asm volatile("s_waitcnt vmcnt(0)" : "+v"(tk) : "v"(pl[1]), "v"(hb[1][1]) : "memory");
  int extra = 0;
  if (max(n0, n1) == 255u) {
    extra = min(n0, n1) == 255u ? 8 : 4;
#pragma unroll
    for (int w = 0; w < 2; ++w)
      if ((w ? n1 : n0) == 255u) {
        hb[w][0] = __builtin_amdgcn_raw_buffer_load_b128(q.pr, q.lane * 16, HB_REL + (k + w) * 2048, 17);
        hb[w][1] = __builtin_amdgcn_raw_buffer_load_b128(q.pr, q.lane * 16 + 1024, HB_REL + (k + w) * 2048, 17);
      }
    asm volatile("" :: "v"(hb[0][0]), "v"(hb[0][1]), "v"(hb[1][0]), "v"(hb[1][1]));
  }
  PIN();
  q.j = rfl(tk) - q.tbase;
  const unsigned jn = min(q.j, NJOB - 1);
  const int over = (int)((unsigned)rt() - q.tgo) - LHI - (int)q.hurry * HLHI - extra * 16;
  const u64 dt = (over > 0 ? (over > DMAX ? DMAX : over) : 0) + extra;
  u64 t0, fv;
  asm volatile("s_mov_b64 exec, 1\n\tglobal_atomic_add_x2 %0, %2, %3, off sc0\n\tglobal_atomic_add_x2 %1, %4, %5, off sc0 sc1\n\ts_mov_b64 exec, -1"
               : "=&v"(t0), "=&v"(fv) : "v"(q.tick + 32), "v"(dt), "v"(q.f1 + F1POS(jn)), "v"(0ull) : "memory");
  const unsigned d = ((u >> 1) * 64 + q.s * NRG + (k & 7u)) * 4096;
#pragma unroll
  for (int w = 0; w < 2; ++w) {
    const unsigned sh = ((info >> (16 * w)) & 7u) * 0x00800080u;
    const unsigned a3 = ((pl[w][0] << 1) & 0x88888888u) | ((pl[w][1] >> 1) & 0x22222222u) | ((pl[w][2] >> 2) & 0x11111111u);
#pragma unroll
    for (int m = 0; m < 4; ++m) {
      const unsigned a = m < 3 ? pl[w][m < 3 ? m : 0] : a3, l0 = lo[w][m >> 1][2 * (m & 1)], l1 = lo[w][m >> 1][2 * (m & 1) + 1];
      const unsigned h0 = (a & 0x83838383u) | hb[w][m >> 1][2 * (m & 1)], h1 = (__builtin_rotateright32(a, 4) & 0x83838383u) | hb[w][m >> 1][2 * (m & 1) + 1];
      const u32x4 o = {pk_sub(__builtin_amdgcn_perm(h0, l0, 0x05010400u), sh), pk_sub(__builtin_amdgcn_perm(h0, l0, 0x07030602u), sh),
                       pk_sub(__builtin_amdgcn_perm(h1, l1, 0x05010400u), sh), pk_sub(__builtin_amdgcn_perm(h1, l1, 0x07030602u), sh)};
      asm volatile("buffer_store_dwordx4 %0, %1, %2, %3 offen offset:%4 sc0 sc1\n\ts_nop 1"
                   :: "v"(o), "v"(q.lane * 16u), "s"(q.xw), "s"(d + w * 4096), "i"(m * 1024) : "memory");
    }
  }
  asm volatile("s_waitcnt vmcnt(0)" : "+v"(t0), "+v"(fv) :: "memory");
  count_in(q.f2 + (u & 1) * NSTEP + (u >> 1), 2u);
  q.hurry = rfl((unsigned)(t0 >> 32)) >> 24;
  q.tgo = rfl((unsigned)t0) + (unsigned)dt + jn * (TAU16 - q.hurry) / 32; q.fcall = rfl((unsigned)fv); q.info = rfl((unsigned)(fv >> 32));
  return true;
}

DI void wfill(__amdgpu_buffer_rsrc_t wr, unsigned ldsaddr, unsigned vo, unsigned so) {
#pragma unroll
  for (int i = 0; i < 8; ++i)
    asm volatile("s_mov_b32 m0, %3\n\ts_nop 0\n\tbuffer_load_dwordx4 %0, %1, %2 offen lds"
                 :: "v"(vo), "s"(wr), "s"(so + (16 * ((i >> 1) & 1) + 4 * (i & 1)) * (NL * 2) + (i >> 2) * 128), "s"(ldsaddr + i * 1024) : "memory");
}
DI void wtouch(const char* p, unsigned n) {
  unsigned t, o = 0;
  asm volatile(
      "1:\n\t"
      "s_load_dword %0, %3, %1 offset:0\n\ts_load_dword %0, %3, %1 offset:64\n\ts_load_dword %0, %3, %1 offset:128\n\ts_load_dword %0, %3, %1 offset:192\n\t"
      "s_load_dword %0, %3, %1 offset:256\n\ts_load_dword %0, %3, %1 offset:320\n\ts_load_dword %0, %3, %1 offset:384\n\ts_load_dword %0, %3, %1 offset:448\n\t"
      "s_add_u32 %1, %1, 0x20000\n\t"
      "s_sub_u32 %2, %2, 1\n\t"
      "s_cmp_lg_u32 %2, 0\n\t"
      "s_cbranch_scc1 1b\n\t"
      "s_waitcnt lgkmcnt(0)" : "=&s"(t), "+s"(o), "+s"(n) : "s"(p) : "scc", "memory");
}

DI void place_raw(char* ws, unsigned par, int rank, int step, int rg, int lane, const u32x4 (&v)[4]) {
  char* d = ws + XT_OFF + par * XT_BYTES + (size_t)(step * 64 + rank * NRG + rg) * 4096 + lane * 16;
#pragma unroll
  for (int m = 0; m < 4; ++m) st_sys(d + m * 1024, v[m]);
  wait_vm0();
  count_in((unsigned*)(ws + F2_OFF) + (rank * 2 + (rg >> 2)) * NSTEP + step, 1u);
}

__global__ __launch_bounds__(BLOCK, 2) void ag_gemm(Params p) {
  extern __shared__ __attribute__((aligned(16))) char lds[];
  const int lane = threadIdx.x & 63;
  const int wave = rfl(threadIdx.x >> 6);
  const unsigned lb = (unsigned)(size_t)(LAS char*)lds;
  const int b = blockIdx.x, rank = p.rank & (NR - 1);
  const char* xin = (const char*)p.x; char* const my = p.my;
  asm volatile("" : "+s"(xin) : "s"(my), "s"(p.rank), "s"(p.w));
  const int xcd = b & 7, bi = b >> 3;
  const int tm = (xcd >> 1) * 4 + (bi >> 3), tn = (xcd & 1) * 8 + (bi & 7);
  unsigned* const tmo = (unsigned*)(my + CT_OFF);
  const unsigned tstart = (unsigned)rt();

  const int ppair = (bi >> 1) * 8 + xcd;
  constexpr int RAW0 = GRID - NRAW * NRG * (NR - 1);
  const int rblk = (b - RAW0) / (NR - 1);
  const bool pusher = wave == 6 && b >= RAW0;
  const int pstep = pusher ? rblk >> 3 : 2 * (ppair >> 1) + (bi & 1), prg = pusher ? rblk & 7 : (ppair & 1) * 4 + wave;
  u32x4 pv[4];
  char* peer = nullptr;
  if (wave < 4 || pusher) {
    if (pstep >= XFIRST) sleep_until(tstart + pstep * XGAP / 16);
    const auto xr = __builtin_amdgcn_make_buffer_rsrc((void*)xin, 0, ML * KD * 2, 0x00020000);
#pragma unroll
    for (int m = 0; m < 4; ++m) pv[m] = __builtin_amdgcn_raw_buffer_load_b128(xr, (lane & 15) * (KD * 2) + (lane >> 4) * 16, ((prg * 64 + m * 16) * KD + pstep * 32) * 2, 0);
    PIN();
    peer = p.ws[(pusher ? rank + 1 + (b - RAW0) % (NR - 1) : lane >> 1) & (NR - 1)];
  }
  const auto wr = __builtin_amdgcn_make_buffer_rsrc((void*)p.w, 0, KD * NL * 2, 0x00020000);
  const unsigned wvo = (8 * ((lane >> 3) & 1) + (lane >> 4)) * (NL * 2) + ((((lane >> 1) & 3) ^ (lane >> 4)) * 32) + (lane & 1) * 16;
  if (wave == 4) {
    __builtin_amdgcn_s_sleep(4);
    for (int s = 0; s < W0; ++s) wfill(wr, lb + s * 8192, wvo, s * 32 * (NL * 2) + tn * 256);
  }
  if (wave == 5 && bi >= 8) {
    const int grp = (bi >> 3) - 1;
    size_t o = grp == 0 ? (lane < 2 ? F1_OFF + lane * F1_BYTES + (bi & 7) * (NUNIT * 128) : XT_OFF + (lane - 2) * XT_BYTES + 3 * 64 * 4096)
                        : PK_OFF + (grp - 1) * PK_BYTES + (lane < 3 ? lane * (1u << 20) : E0_REL);
    const char* a = p.ws[bi & 7] + o + (rank * 8 + xcd) * 128;
    if (lane < 4) asm volatile("s_mov_b32 m0, %1\n\ts_nop 0\n\tglobal_load_lds_dword %0, off sc0 sc1" :: "v"(a), "s"(lb + L_MISC + 256) : "memory");
  }
  if (wave == 6) {
    unsigned t = 0;
    if (lane == 0) t = __hip_atomic_fetch_add((unsigned*)(my + CC_OFF + b * 128), 1u, __ATOMIC_RELAXED, __HIP_MEMORY_SCOPE_AGENT);
    t = rfl(t) + 1;
    if (lane < 16) lds_st(lb + L_MISC + lane * 4, lane == 0 ? t : 0u);
  } else if (wave == 5 || wave == 7) {
#pragma unroll
    for (int i = 0; i < 4; ++i) lds_st(lb + L_FREE + (wave - 5) * 512 + i * 256 + lane * 4, 0u);
  }
  asm volatile("s_waitcnt lgkmcnt(0)" ::: "memory");
  __builtin_amdgcn_s_barrier();
  FENCE();
  const unsigned call = rfl(lds_ld(lb + L_MISC));
  const unsigned par = call & 1u;

  if (wave < 4) {
    __builtin_amdgcn_s_setprio(3);
    if (pstep < NRAW) {
      asm volatile("s_mov_b64 exec, 1\n\tds_add_u32 %0, %1\n\ts_mov_b64 exec, -1" :: "v"(lb + L_MISC + 8), "v"(1u) : "memory");
    } else {
      unsigned v = 0;
#pragma unroll
      for (int m = 0; m < 4; ++m) { v = __builtin_amdgcn_sad_u8(pv[m][0] & 0x7f007f00u, 0u, v); v = __builtin_amdgcn_sad_u8(pv[m][2] & 0x7f007f00u, 0u, v); }
      v += dpp<0x111>(v); v += dpp<0x112>(v); v += dpp<0x114>(v); v += dpp<0x118>(v);
      const unsigned top = (rl(v, 15) + rl(v, 31) + rl(v, 47) + rl(v, 63) + 1705u) >> 9, low = min(top > 7u ? top - 7u : 0u, 248u);
      const unsigned sh = (0u - low) & 7u, qw = (low + sh) >> 3, q4 = (qw << 2) * 0x01010101u;
      u32x4 lo[2], hi[2]; unsigned a[4], mask = 0;
#pragma unroll
      for (int m = 0; m < 4; ++m) {
        const unsigned d0 = pk_add(pv[m][0], sh * 0x00800080u), d1 = pk_add(pv[m][1], sh * 0x00800080u), d2 = pk_add(pv[m][2], sh * 0x00800080u), d3 = pk_add(pv[m][3], sh * 0x00800080u);
        const unsigned h0 = __builtin_amdgcn_perm(d1, d0, 0x07050301u), h1 = __builtin_amdgcn_perm(d3, d2, 0x07050301u);
        lo[m >> 1][2 * (m & 1)] = __builtin_amdgcn_perm(d1, d0, 0x06040200u); lo[m >> 1][2 * (m & 1) + 1] = __builtin_amdgcn_perm(d3, d2, 0x06040200u);
        hi[m >> 1][2 * (m & 1)] = h0; hi[m >> 1][2 * (m & 1) + 1] = h1;
        a[m] = (h0 & 0x83838383u) | __builtin_rotateleft32(h1 & 0x83838383u, 4);
        mask |= ((((h0 ^ q4) & 0x7c7c7c7cu) + 0x7c7c7c7cu) & 0x80808080u) >> (7 - 2 * m);
        mask |= ((((h1 ^ q4) & 0x7c7c7c7cu) + 0x7c7c7c7cu) & 0x80808080u) >> (6 - 2 * m);
      }
      const int u = pstep * 2 + (prg >> 2), blk = u * 4 + wave;
      char* const pk = my + PK_OFF + par * PK_BYTES;
      char* fp = peer + F1_OFF + par * F1_BYTES + (size_t)(rank * NUNIT + u) * 128 + (lane & 1) * 8;
      asm volatile("" : "+v"(fp));
      st_sys(pk + blk * BBYTES + lane * 16, lo[0]); st_sys(pk + blk * BBYTES + 1024 + lane * 16, lo[1]);
      const u32x3 pl = {a[0] | ((a[3] >> 1) & 0x44444444u), a[1] | ((a[3] << 1) & 0x44444444u), a[2] | ((a[3] << 2) & 0x44444444u)};
      asm volatile("global_store_dwordx3 %0, %1, off sc0 sc1\n\ts_nop 1" :: "v"(pk + blk * BBYTES + 2048 + lane * 12), "v"(pl) : "memory");
      LAS unsigned char* const hcopy = (LAS unsigned char*)lds + L_EPI + wave * 4096;
      LAS unsigned short* const list = (LAS unsigned short*)(hcopy + 2048);
      *(LAS u32x4*)(hcopy + lane * 16) = hi[0]; *(LAS u32x4*)(hcopy + 1024 + lane * 16) = hi[1];
      unsigned cnt = 0;
      do {
#pragma unroll
        for (int i = 0; i < 4; ++i) {
          const u64 bal = __ballot(mask != 0u);
          const unsigned idx = cnt + __builtin_amdgcn_mbcnt_hi((unsigned)(bal >> 32), __builtin_amdgcn_mbcnt_lo((unsigned)bal, 0u));
          list[mask ? idx : 512u + lane] = (unsigned short)(__builtin_ctz(mask | 0x80000000u) | (lane << 5));
          mask &= mask - 1u; cnt += __builtin_popcountll(bal);
        }
      } while (cnt <= 64u && __any(mask != 0u));
      FENCE();
      if (cnt > 64u) {
        cnt = 255u;
        st_sys(pk + HB_REL + blk * 2048 + lane * 16, hi[0]); st_sys(pk + HB_REL + blk * 2048 + 1024 + lane * 16, hi[1]);
      } else {
        const unsigned t = list[lane], pos = ((t & 4u) << 8) | ((t >> 1) & 0x3f0u) | ((t & 3u) << 2) | ((t >> 3) & 3u);
        const unsigned e = pos | ((hcopy[pos] & 0x7cu) << 9);
        const unsigned e2 = e | (dpp<0xb1>(e) << 16);
        if ((unsigned)lane < cnt && !(lane & 1))
          __hip_atomic_store((unsigned*)(pk + (lane < 32 ? E0_REL : E1_REL - 64) + blk * 64 + lane * 2), e2, __ATOMIC_RELAXED, __HIP_MEMORY_SCOPE_SYSTEM);
      }
      wait_vm0();
      unsigned old;
      asm volatile("s_mov_b64 exec, 1\n\tds_write_b16 %1, %2\n\tds_add_rtn_u32 %0, %3, %4\n\ts_mov_b64 exec, -1\n\ts_waitcnt lgkmcnt(0)"
                   : "=&v"(old) : "v"(lb + L_MISC + 16 + 2 * wave), "v"(sh | (qw << 3) | (cnt << 8)), "v"(lb + L_MISC + 8), "v"(1u) : "memory");
      if (rfl(old) == 3u) {
        const unsigned pair = lds_ld(lb + L_MISC + 16 + (lane & 1) * 4);
        if (lane < 2 * NR && (lane >> 1) != rank)
          asm volatile("global_store_dwordx2 %0, %1, off sc0 sc1" :: "v"(fp), "v"(((u64)pair << 32) | call) : "memory");
      }
    }
    __builtin_amdgcn_s_setprio(1);
    sleep_until(tstart + OWN0 + pstep * OWNGAP / 16);
    place_raw(my, par, rank, pstep, prg, lane, pv);

    CW c;
    char* outp = (char*)p.out; asm volatile("" : "+s"(outp));
    const int g = lane >> 4, q = (lane >> 2) & 3, pp = lane & 3;
#pragma unroll
    for (int m = 0; m < 4; ++m)
#pragma unroll
      for (int j = 0; j < 8; ++j) c.acc[m][j] = (f32x4){0.f, 0.f, 0.f, 0.f};
#pragma unroll
    for (int i = 0; i < 4; ++i) c.wa[i] = lb + (g >> 1) * 2048 + q * 256 + (g & 1) * 128 + ((i ^ q) * 32) + pp * 8;
    c.fa = lb + L_FREE + wave * 256 + lane * 4; c.ra = lb + L_READY + lane * 4;
    c.rv = 0;
    c.xr = __builtin_amdgcn_make_buffer_rsrc((void*)(my + XT_OFF + par * XT_BYTES), 0, (unsigned)XT_BYTES, 0x00020000);
#pragma unroll
    for (int i = 0; i < 4; ++i) { c.xvo[i] = lane * 16 + i * 64 * 4096; asm volatile("" : "+v"(c.xvo[i])); }
    c.xso = (tm * 4 + wave) * 4096;
    PIN();
    first_x<0>(c); first_x<1>(c);
    wread<0, 0, 0, 0>(c); wread<0, 0, 0, 1>(c); wread<0, 0, 1, 0>(c); wread<0, 0, 1, 1>(c);
#pragma unroll
    for (int m = 0; m < 4; ++m) { c.xs[2][m] = (u32x4){0, 0, 0, 0}; c.xs[3][m] = (u32x4){0, 0, 0, 0}; asm volatile("" : "+v"(c.xs[2][m]), "+v"(c.xs[3][m])); }
    wread<0, 1, 0, 0>(c); wread<0, 1, 0, 1>(c); wread<0, 1, 1, 0>(c); wread<0, 1, 1, 1>(c);
    c.need = 4; asm volatile("" : "+v"(c.need));
    wait_ready(c);
    asm volatile("" :: "v"(c.xs[1][3]), "v"(c.wf[1][1]));
    PIN();
#pragma unroll 1
    for (int it = 0; it < NSTEP / 4; ++it) {
      const unsigned wd = (it + 1) % (RING / 4) == 0 ? (unsigned)(-(RING - 4) * 8192) : (unsigned)(4 * 8192);
      dstep<0>(c, wd); dstep<1>(c, wd);
    }

    const unsigned vd = *(volatile LAS unsigned*)((LAS char*)lds + L_MISC + 4);
    int tz = threadIdx.x; asm volatile("" : "+v"(tz));
    const int l2 = tz & 63;
    const unsigned ovo = (l2 >> 4) * (NL * 2) + (l2 & 15) * 16;
    const auto orr = __builtin_amdgcn_make_buffer_rsrc((void*)(outp + ((size_t)(tm * 256 + wave * 64) * NL + tn * 128) * 2), 0, 63 * NL * 2 + 256, 0x00020000);
    LAS char* const eb = (LAS char*)lds + L_EPI + wave * 4096;
    const unsigned ew = (l2 & 15) * 256 + ((l2 >> 4) & 1) * 8 + (((l2 >> 5) ^ (l2 & 15)) * 16), er = (l2 >> 4) * 256 + (((l2 & 15) ^ (l2 >> 4)) * 16);
    unsigned verdict = rfl(vd);
    while (verdict == 0u) { __builtin_amdgcn_s_sleep(1); verdict = rfl(lds_ld(lb + L_MISC + 4)); }
    if (verdict == 1u) {
      u32x4 o[4];
#pragma unroll
      for (int m = 0; m <= 4; ++m) {
#pragma unroll
        for (int h = 0; h < 4; ++h) {
          u32x2 t0, t1;
          if (m < 4) {
            f32x4 a = c.acc[m][2 * h], d = c.acc[m][2 * h + 1];
#pragma unroll
            for (int i = 0; i < 4; ++i) { a[i] = relu(a[i]); d[i] = relu(d[i]); }
            PIN();
            t0 = (u32x2){cvt_pk(a[0], a[1]), cvt_pk(a[2], a[3])}; t1 = (u32x2){cvt_pk(d[0], d[1]), cvt_pk(d[2], d[3])};
            PIN();
          }
          if (m > 0)
            asm volatile("buffer_store_dwordx4 %0, %1, %2, %3 offen nt\n\ts_nop 1" :: "v"(o[h]), "v"(ovo), "s"(orr), "s"(((m - 1) * 16 + 4 * h) * NL * 2));
          if (m < 4) { *(LAS u32x2*)(eb + (ew ^ (64 * h))) = t0; *(LAS u32x2*)(eb + (ew ^ (64 * h + 32))) = t1; }
          PIN();
        }
        if (m < 4) {
          FENCE();
#pragma unroll
          for (int i = 0; i < 4; ++i) o[i] = *(const LAS u32x4*)(eb + (er ^ (64 * i)) + 1024 * i);
          FENCE();
          PIN();
        }
      }
    } else {
      const u32x4 nan4 = {0x7fc07fc0u, 0x7fc07fc0u, 0x7fc07fc0u, 0x7fc07fc0u};
#pragma unroll 1
      for (int i = 0; i < 16; ++i) asm volatile("buffer_store_dwordx4 %0, %1, %2, %3 offen\n\ts_nop 1" :: "v"(nan4), "v"(ovo), "s"(orr), "s"(4 * i * NL * 2));
    }
  } else if (wave == 4) {
    __builtin_amdgcn_s_setprio(1);
    const unsigned pubaddr = lb + L_READY + lane * 8 + 4;
    wait_vm0();
    int pub = W0, slot = W0; unsigned freec = 0;
    lds_st(pubaddr, pub);
    for (int s = W0; s < NSTEP; ++s) {
      while (s >= RING && freec < (unsigned)(s - RING + 5)) {
        const unsigned f = lds_ld(lb + L_FREE + (lane & 3) * 256);
        freec = min(min(rl(f, 0), rl(f, 1)), min(rl(f, 2), rl(f, 3)));
        if (freec >= (unsigned)(s - RING + 5)) break;
        if (pub < s && max(max(rl(f, 0), rl(f, 1)), max(rl(f, 2), rl(f, 3))) >= (unsigned)pub) { wait_vm0(); pub = s; lds_st(pubaddr, pub); }
        __builtin_amdgcn_s_sleep(2);
      }
      if (s < RING) {
        const unsigned tgo = tstart + WT0 + (s - W0) * WGAP;
        if ((int)(tgo - (unsigned)rt()) > 0) { if (pub < s) { wait_vm0(); pub = s; lds_st(pubaddr, pub); } sleep_until(tgo); }
      }
      wfill(wr, lb + slot * 8192, wvo, s * 32 * (NL * 2) + tn * 256);
      slot = (slot + 1) & (RING - 1);
      if (s + 1 - pub > INFL) { asm volatile("s_waitcnt vmcnt(%0)" :: "i"(8 * INFL) : "memory"); pub = s + 1 - INFL; lds_st(pubaddr, pub); }
    }
    wait_vm0(); lds_st(pubaddr, INF);
  } else if (wave == 6) {
    __builtin_amdgcn_s_setprio(2);
    if (pusher) {
      sleep_until(tstart + RAWT0 + rblk * RAWGAP / 4);
      place_raw(peer, par, rank, pstep, prg, lane, pv);
    }
    const unsigned xaddr = lb + L_READY + lane * 8;
    const unsigned* f2 = (const unsigned*)(my + F2_OFF) + tm * NSTEP;
    bool dead = false;
    unsigned base = 0;
    const u64 t0 = rt();
    for (unsigned n = 1;; ++n) {
      const unsigned idx = base + lane;
      unsigned v = 4u * call;
      if (idx < (unsigned)NSTEP && lane < POLLW) v = peek(f2 + idx);
      const unsigned nok = ~(unsigned)__ballot(v >= 4u * call);
      base += nok ? (unsigned)__builtin_ctz(nok) : (unsigned)POLLW;
      if (base >= (unsigned)NSTEP) break;
      lds_st(xaddr, base);
      if ((n & 63u) == 0u && timed_out(tmo, call, t0)) { dead = true; break; }
      if (base < (unsigned)(NSTEP - POLLEND)) __builtin_amdgcn_s_sleep(8);
    }
    lds_st(xaddr, INF);
    if (rfl(__hip_atomic_load(tmo, __ATOMIC_RELAXED, __HIP_MEMORY_SCOPE_AGENT)) == call) dead = true;
    lds_st(lb + L_MISC + 4, dead ? 2u : 1u);
  } else if (wave == 7) {
    for (int pf = W0; pf < NSTEP; pf += PFB) wtouch((const char*)p.w + b * 512 + (size_t)pf * 131072, pf + PFB <= NSTEP ? PFB : NSTEP - pf);
  } else if (b < PW * (NR - 1)) {
    Pull q;
    const int qi = b % (NR - 1);
    q.s = (rank + 1 + qi) & (NR - 1);
    q.tmo = tmo; q.call = call; q.lane = lane; q.pat = (LAS unsigned char*)lds + L_PAT;
    q.tick = (unsigned*)(my + CT_OFF + (2 + 2 * qi) * 128);
    q.pr = __builtin_amdgcn_make_buffer_rsrc((void*)(p.ws[q.s] + PK_OFF + par * PK_BYTES), 0, (unsigned)PK_BYTES, 0x00020000);
    q.xw = __builtin_amdgcn_make_buffer_rsrc((void*)(my + XT_OFF + par * XT_BYTES), 0, (unsigned)XT_BYTES, 0x00020000);
    q.f1 = my + F1_OFF + par * F1_BYTES + (size_t)q.s * NUNIT * 128;
    q.f2 = (unsigned*)(my + F2_OFF) + q.s * 2 * NSTEP;
    q.tbase = (call - 1u) * (NJOB + PW);
    u64 t0old = 0, old = 0, fv = 0; unsigned t = 0;
    if (lane == 0)
      asm volatile("global_atomic_add_x2 %0, %2, %4, off sc0\n\tglobal_atomic_add_x2 %1, %3, %4, off sc0 sc1\n\ts_waitcnt vmcnt(0)"
                   : "=&v"(t0old), "=&v"(fv) : "v"(q.tick + 32), "v"(q.f1 + F1POS(0u)), "v"(0ull) : "memory");
    if ((t0old & 0xffffffffffffffull) + 1500 > rt()) t0old = ~0ull;
    if (rfl((unsigned)fv) < call || pstep < EARLYP)
      while (rfl(lds_ld(lb + L_MISC + 8)) < 4u) __builtin_amdgcn_s_sleep(1);
    if (lane == 0)
      asm volatile("global_atomic_add %0, %2, %3, off sc0\n\tglobal_atomic_add_x2 %1, %4, %5, off sc0 sc1\n\ts_waitcnt vmcnt(0)"
                   : "=&v"(t), "=&v"(fv) : "v"(q.tick), "v"(1u), "v"(q.f1 + F1POS(0u)), "v"(0ull) : "memory");
    q.j = rfl(t) - q.tbase; q.info = rfl((unsigned)(fv >> 32)); q.fcall = rfl((unsigned)fv);
    const unsigned jn = q.j < NJOB ? q.j : 0u;
    const bool lead = jn < (unsigned)FASTGO && (q.fcall < call || jn == 0u);
    bool ok = q.fcall >= call || wait_f1(q.f1 + F1POS(lead ? jn : 0u), call, tmo, q.info);
    u64 t0 = rt() + T0MARG;
    {
      const int late = (int)((unsigned)t0 - tstart) - ONTIME;
      t0 |= (u64)(late > 0 ? min(late / 16, CATCH) : 0) << 56;
    }
    const u32x4 cs = {(unsigned)t0, (unsigned)(t0 >> 32), (unsigned)t0old, (unsigned)(t0old >> 32)};
    if (lead) {
      if (lane == 0) asm volatile("global_atomic_cmpswap_x2 %0, %1, off\n\ts_nop 1" :: "v"(q.tick + 32), "v"(cs) : "memory");
      t0 -= T0MARG; q.fcall = call;
    } else {
      if (lane == 0)
        asm volatile("global_atomic_cmpswap_x2 %0, %2, %3, off sc0\n\tglobal_atomic_add_x2 %1, %4, %5, off sc0 sc1\n\ts_waitcnt vmcnt(0)"
                     : "=&v"(old), "=&v"(fv) : "v"(q.tick + 32), "v"(cs), "v"(q.f1 + F1POS(jn)), "v"(0ull) : "memory");
      if (old != t0old) t0 = old;
      q.fcall = rfl((unsigned)fv); q.info = rfl((unsigned)(fv >> 32));
    }
    q.hurry = rfl((unsigned)(t0 >> 32)) >> 24;
    q.tgo = rfl((unsigned)t0) + jn * (TAU16 - q.hurry) / 32;
    while (ok && q.j < NJOB) ok = pull(q);
  }
}

}

extern "C" void kernel_launch(int rank, int nranks, ncclComm_t comm, hipStream_t stream,
                              void* const* d_in, const int* in_sizes, int n_in,
                              void* d_out, int out_size, const int* in_dtypes, int out_dtype,
                              void* const* d_out_all, void* const* d_ws_all, size_t ws_bytes) {
  if (nranks != NR || (unsigned)rank >= (unsigned)NR || n_in != 2 || in_sizes[0] != ML * KD || in_sizes[1] != KD * NL ||
      out_size != MG * NL || ws_bytes < WS_NEED) { fprintf(stderr, "unexpected problem shape\n"); return; }
  static bool attr_set = false;
  if (!attr_set) { HIP_CHECK(hipFuncSetAttribute((const void*)ag_gemm, hipFuncAttributeMaxDynamicSharedMemorySize, LDS_BYTES)); attr_set = true; }
  Params p;
  p.x = d_in[0]; p.my = (char*)d_ws_all[rank]; p.w = d_in[1]; p.out = d_out; p.rank = rank; p.pad = 0;
  for (int i = 0; i < NR; ++i) p.ws[i] = (char*)d_ws_all[i];
  ag_gemm<<<GRID, BLOCK, LDS_BYTES, stream>>>(p);
  HIP_CHECK(hipGetLastError());
}
```
